```python
import numpy as np

kernel_name = "gpu_ag_gemm_m4096_k4096_n32768_bf16_none"


ABSMAX_RTOL = 0.008

COMPUTE_DTYPE = "bfloat16"
EPILOGUE = "none"
IN_QUANT = None
OUT_QUANT = None
IN_DTYPES = {"x": "bfloat16", "w_mat": "bfloat16"}
WORLD_SIZE = 8

def _to_bf16(x):
    u = np.ascontiguousarray(x, dtype=np.float32).view(np.uint32)
    return ((u + 0x7FFF + ((u >> 16) & 1)) & 0xFFFF0000).view(np.float32)

def setup_inputs():

    x = _to_bf16(np.random.randn(4096, 4096).astype(np.float32))
    w_mat = _to_bf16((np.random.randn(4096, 32768) / np.sqrt(4096)).astype(np.float32))
    return {"x": x, "w_mat": w_mat}

def _epilogue(y):
    return y

def reference(x, w_mat):
    y = (x @ w_mat).astype(np.float32)
    return _epilogue(y)
```

```cpp
#include <hip/hip_runtime.h>
#include <rccl/rccl.h>
#include <cstdio>

#define TAU 30
#define NOCOMM 0
#define AUXA 0
#define AUXO 0

namespace {
typedef short bf16x8 __attribute__((ext_vector_type(8)));
typedef short bf16x4 __attribute__((ext_vector_type(4)));
typedef float f32x4 __attribute__((ext_vector_type(4)));
typedef unsigned u32x4 __attribute__((ext_vector_type(4)));
#define LAS __attribute__((address_space(3)))
#define CAS __attribute__((address_space(4)))

constexpr int NR = 8;
constexpr int NT = 64;
constexpr int HT = 16384;
constexpr int LDS_A = 0, LDS_B = 4 * HT, LDS_P = 8 * HT, LDS_M = 9 * HT, LDS_BYTES = 9 * HT + 64;
constexpr unsigned ROWB = 8192;
constexpr size_t XG_PAR = 32ull << 20, XG_SRC = 4ull << 20, XG_STRIP = 2ull << 20, XG_KT = 32768;
constexpr size_t WS_FLAGS = 64ull << 20;
constexpr size_t WS_CTR = WS_FLAGS + 65536;
constexpr size_t WS_TMO = WS_CTR + 32768;
constexpr size_t WS_NEED = WS_TMO + 4096;
constexpr unsigned DEADLINE = 100000000u;

struct Args { const char* x; const char* w; char* out; char* ws[NR]; int rank; };

__device__ __forceinline__ unsigned cvt_pk_bf16(float lo, float hi) { unsigned r; asm volatile("v_cvt_pk_bf16_f32 %0, %1, %2" : "=v"(r) : "v"(lo), "v"(hi)); return r; }
__device__ __forceinline__ unsigned rfl(unsigned v) { return (unsigned)__builtin_amdgcn_readfirstlane((int)v); }
__device__ __forceinline__ unsigned now_sync() { unsigned long long t; asm volatile("s_memrealtime %0\n\ts_waitcnt lgkmcnt(0)" : "=s"(t) :: "memory"); return (unsigned)t; }
}

__global__ __launch_bounds__(512, 2) void ag_gemm_kernel(Args a) {
  extern __shared__ __attribute__((aligned(16))) char smem[];
  LAS char* lds = (LAS char*)smem;
  const CAS Args* kargs = (const CAS Args*)__builtin_amdgcn_kernarg_segment_ptr();
  const unsigned t_start = now_sync();
  const int tid = threadIdx.x, lane = tid & 63;
  const int wid = __builtin_amdgcn_readfirstlane(tid >> 6), wr = wid >> 2, wc = wid & 3;
  const int fr = lane & 15, fq = lane >> 4;
  const int b = blockIdx.x, rank = a.rank;
  const int xcd = b & 7, idx = b >> 3;
  const int pm = 4 * (xcd >> 1) + (idx & 3), pn = 8 * (xcd & 1) + (idx >> 2);
  const int src = pm >> 1, strip = pm & 1;
  const bool local = (src == rank);
  char* myws = a.ws[rank];

  const int rel = (src - rank - 1) & 7, oth = ((idx >> 2) << 2) | ((xcd & 1) << 1) | strip;
  const int j = local ? 224 + oth : oth * 7 + rel;
  const int jkt = j >> 2, jh = (j >> 1) & 1, js = j & 1;
  const unsigned push_at = (unsigned)j * TAU;

  unsigned voffX;
  { const int row = 8 * wid + (lane >> 3); voffX = (unsigned)row * ROWB + (unsigned)(((lane & 7) ^ ((row >> 1) & 7)) * 16); }
  const unsigned voffI = (unsigned)(wid * 1024 + lane * 16);
  const unsigned voffA = local ? voffX : voffI;
  unsigned voffB;
  { const int k = 4 * wid + (lane >> 4); const int f = (k & 3) | (((k >> 3) & 1) << 2); voffB = (unsigned)k * ROWB + (unsigned)(((lane & 15) ^ (2 * f)) * 16); }
  const unsigned ldsw = (unsigned)wid * 1024u;

#define GLDS(gptr, ldsoff, aux) __builtin_amdgcn_global_load_lds((const unsigned*)(gptr), (LAS unsigned*)(lds + (ldsoff)), 16, 0, aux)
#define WAIT_V(n) asm volatile("s_waitcnt vmcnt(" #n ")" ::: "memory")
#define WAIT_L(n) asm volatile("s_waitcnt lgkmcnt(" #n ")" ::: "memory")
#define BAR __builtin_amdgcn_s_barrier()
#define SCHED __builtin_amdgcn_sched_barrier(0)

  unsigned pv;
  { const char* cp = myws + WS_CTR + (size_t)b * 128; unsigned z = 0;
    asm volatile("global_load_dword %0, %1, %2 sc1" : "=v"(pv) : "v"(z), "s"(cp) : "memory"); }
  { const char* ps = a.x + (size_t)(js * 256 + jh * 128) * ROWB + (size_t)jkt * 128;
    GLDS(ps + voffX, LDS_P + ldsw, 0); GLDS(ps + 64 * ROWB + voffX, LDS_P + ldsw + 8192, 0); }
  const char* pB = a.w + (size_t)pn * 512;
  constexpr size_t iB = 32 * (size_t)ROWB, hB = 256, kB = 64 * (size_t)ROWB;
#define STAGE_B(bb, h, gp) do { GLDS((gp) + (h) * hB + voffB, LDS_B + ((bb) * 2 + (h)) * HT + ldsw, 0); GLDS((gp) + (h) * hB + iB + voffB, LDS_B + ((bb) * 2 + (h)) * HT + ldsw + 8192, 0); } while (0)
  STAGE_B(0, 0, pB); STAGE_B(0, 1, pB); STAGE_B(1, 0, pB + kB); STAGE_B(1, 1, pB + kB);
  if (tid == 0) { *(LAS unsigned*)(lds + LDS_M) = 0u; *(LAS unsigned*)(lds + LDS_M + 4) = 0u; }
  asm volatile("s_waitcnt vmcnt(8) lgkmcnt(0)" : "+v"(pv) :: "memory");
  const unsigned call = rfl(pv) + 1u;
  const unsigned par = call & 1u;
  BAR;

  const char* pA = local ? a.x + (size_t)strip * 256 * ROWB : myws + par * XG_PAR + (size_t)src * XG_SRC + (size_t)strip * XG_STRIP;
  const size_t iA = local ? 64 * (size_t)ROWB : 8192, hA = local ? 128 * (size_t)ROWB : HT, kA = local ? 128 : XG_KT;
#define STAGE_A(bb, h, gp) do { GLDS((gp) + (h) * hA + voffA, LDS_A + ((bb) * 2 + (h)) * HT + ldsw, AUXA); GLDS((gp) + (h) * hA + iA + voffA, LDS_A + ((bb) * 2 + (h)) * HT + ldsw + 8192, AUXA); } while (0)
  const char* flags = myws + WS_FLAGS + (size_t)pm * 1024;
  const size_t push_off = par * XG_PAR + (size_t)rank * XG_SRC + (size_t)js * XG_STRIP + (size_t)jkt * XG_KT + (size_t)jh * HT;
  const size_t flag_off = WS_FLAGS + (size_t)(rank * 2 + js) * 1024 + (size_t)(2 * jkt + jh) * 4;

  char* flagp = a.ws[0];
#pragma unroll
  for (int q = 1; q < NR; ++q) { char* c = a.ws[q]; asm volatile("" : "+s"(c)); if (lane == q) flagp = c; }
  flagp += flag_off;
  unsigned ready = (local || NOCOMM) ? 128u : 0u;
  unsigned pstate = NOCOMM ? 3u : 0u;
  bool polled = false;
  unsigned extra = 0;
  unsigned tnow = t_start;

#define PUSH_STORES() do { const CAS Args* ka_ = kargs; asm volatile("" : "+s"(ka_)); \
    int tz_ = threadIdx.x; asm volatile("" : "+v"(tz_)); const unsigned vi_ = (unsigned)tz_ * 16u; \
    const u32x4 v0_ = *(const LAS u32x4*)(lds + LDS_P + vi_), v1_ = *(const LAS u32x4*)(lds + LDS_P + vi_ + 8192); \
    _Pragma("unroll") for (int q_ = 1; q_ < NR; ++q_) { \
      auto rs_ = __builtin_amdgcn_make_buffer_rsrc(ka_->ws[(rank + q_) & 7] + push_off, 0, HT, 0x00020000); \
      __builtin_amdgcn_raw_buffer_store_b128(v0_, rs_, vi_, 0, 17); \
      __builtin_amdgcn_raw_buffer_store_b128(v1_, rs_, vi_, 8192, 17); } } while (0)
#define PUSH_SIGNAL() do { \
    unsigned c_ = (lane == 0) ? 1u : 0u; { const unsigned ad_ = (unsigned)(size_t)lds + LDS_M; \
      asm volatile("ds_add_rtn_u32 %0, %1, %0\n\ts_waitcnt lgkmcnt(0)" : "+v"(c_) : "v"(ad_) : "memory"); } \
    if (rfl(c_) == 7u) { \
      if (lane < NR && lane != rank) __hip_atomic_store((unsigned*)flagp, call, __ATOMIC_RELAXED, __HIP_MEMORY_SCOPE_SYSTEM); } } while (0)
#define TAKE_POLL() do { const unsigned long long m_ = ~__ballot(pv >= call); \
    ready += m_ ? (unsigned)__builtin_ctzll(m_) : 64u; } while (0)
#define SLOW_WAIT(need) do { unsigned spins_ = 0; \
    while (ready < (need)) { \
      const unsigned n_ = now_sync(); \
      if (pstate == 0u && n_ - t_start >= push_at) { PUSH_STORES(); pstate = 1u; } \
      { const char* fp_ = flags + ready * 4; const unsigned l4_ = (unsigned)lane * 4u; \
        asm volatile("global_load_dword %0, %1, %2 sc0 sc1\n\ts_waitcnt vmcnt(0)" : "=v"(pv) : "v"(l4_), "s"(fp_) : "memory"); } \
      if (pstate == 1u || pstate == 2u) { PUSH_SIGNAL(); pstate = 3u; } \
      TAKE_POLL(); \
      if (ready < (need)) { \
        bool dead_ = n_ - t_start > DEADLINE; \
        if ((++spins_ & 63u) == 0u) dead_ = dead_ || rfl(__hip_atomic_load((unsigned*)(myws + WS_TMO), __ATOMIC_RELAXED, __HIP_MEMORY_SCOPE_AGENT)) == call; \
        if (dead_) { if (lane == 0) { __hip_atomic_store((unsigned*)(myws + WS_TMO), call, __ATOMIC_RELAXED, __HIP_MEMORY_SCOPE_AGENT); *(LAS unsigned*)(lds + LDS_M + 4) = 1u; } ready = 128u; } \
        else __builtin_amdgcn_s_sleep(1); } } } while (0)

  SLOW_WAIT(1u);
  STAGE_A(0, 0, pA);
  SLOW_WAIT(2u);
  STAGE_A(0, 1, pA);
  SLOW_WAIT(3u);
  STAGE_A(1, 0, pA + kA);

  unsigned aoff0, aoff1, boff0, boff1; const unsigned lbase = (unsigned)(size_t)lds;
  { const unsigned rb = (unsigned)(wr * 64 + fr) * 128u; const unsigned s = (unsigned)(fr >> 1);
    aoff0 = lbase + LDS_A + rb + (((unsigned)fq ^ s) << 4); aoff1 = lbase + LDS_A + rb + (((unsigned)(4 + fq) ^ s) << 4);
    const unsigned q = (unsigned)(lane & 15) >> 2, p = (unsigned)lane & 3u, g = (unsigned)fq, f = q | ((g & 1u) << 2);
    const unsigned kb = (8u * g + q) * 256u + 8u * (p & 1u);
    boff0 = lbase + LDS_B + kb + ((((unsigned)(4 * wc) + (p >> 1)) ^ (2u * f)) << 4);
    boff1 = lbase + LDS_B + kb + ((((unsigned)(4 * wc + 2) + (p >> 1)) ^ (2u * f)) << 4); }

#define DSR128(dst, addr, off) asm volatile("ds_read_b128 %0, %1 offset:%2" : "=v"(dst) : "v"(addr), "i"(off) : "memory")
#define DSRTR(dst, addr, off) asm volatile("ds_read_b64_tr_b16 %0, %1 offset:%2" : "=v"(dst) : "v"(addr), "i"(off) : "memory")
#define LDA(bb, h) do { _Pragma("unroll") for (int m = 0; m < 4; ++m) { \
    DSR128(At[m][0], aoff0, ((bb) * 2 + (h)) * HT + m * 2048); DSR128(At[m][1], aoff1, ((bb) * 2 + (h)) * HT + m * 2048); } } while (0)
#define LDB(dst, bb, h) do { _Pragma("unroll") for (int kk = 0; kk < 2; ++kk) { \
    DSRTR(dst[0][kk][0], boff0, ((bb) * 2 + (h)) * HT + kk * 8192); DSRTR(dst[0][kk][1], boff0, ((bb) * 2 + (h)) * HT + kk * 8192 + 1024); \
    DSRTR(dst[1][kk][0], boff1, ((bb) * 2 + (h)) * HT + kk * 8192); DSRTR(dst[1][kk][1], boff1, ((bb) * 2 + (h)) * HT + kk * 8192 + 1024); } } while (0)
#define TIE_A "+v"(At[0][0]), "+v"(At[0][1]), "+v"(At[1][0]), "+v"(At[1][1]), "+v"(At[2][0]), "+v"(At[2][1]), "+v"(At[3][0]), "+v"(At[3][1])
#define TIE_B(B) "+v"(B[0][0][0]), "+v"(B[0][0][1]), "+v"(B[0][1][0]), "+v"(B[0][1][1]), "+v"(B[1][0][0]), "+v"(B[1][0][1]), "+v"(B[1][1][0]), "+v"(B[1][1][1])
#define TIE_C(ai, bj) "+v"(acc[ai][bj][0][0]), "+v"(acc[ai][bj][0][1]), "+v"(acc[ai][bj][1][0]), "+v"(acc[ai][bj][1][1]), "+v"(acc[ai][bj][2][0]), "+v"(acc[ai][bj][2][1]), "+v"(acc[ai][bj][3][0]), "+v"(acc[ai][bj][3][1])
#define WAIT_AB(B) asm volatile("s_waitcnt lgkmcnt(0)" : TIE_A, TIE_B(B) :: "memory")
#define WAIT_B(B) asm volatile("s_waitcnt lgkmcnt(0)" : TIE_B(B) :: "memory")
#define WAIT_AT() do { asm volatile("s_waitcnt lgkmcnt(0)" : TIE_A, "+s"(tt_) :: "memory"); tnow = (unsigned)tt_; } while (0)
#define MMA(ai, bj, Bt) do { asm volatile("s_setprio 1" : TIE_C(ai, bj) :: "memory"); \
    _Pragma("unroll") for (int m = 0; m < 4; ++m) _Pragma("unroll") for (int n = 0; n < 2; ++n) _Pragma("unroll") for (int k = 0; k < 2; ++k) \
      acc[ai][bj][m][n] = __builtin_amdgcn_mfma_f32_16x16x32_bf16(__builtin_shufflevector(Bt[n][k][0], Bt[n][k][1], 0, 1, 2, 3, 4, 5, 6, 7), At[m][k], acc[ai][bj][m][n], 0, 0, 0); \
    asm volatile("s_setprio 0" : TIE_C(ai, bj) :: "memory"); } while (0)

#define SLOT_CHECK(tau) do { \
    if (polled) { TAKE_POLL(); } \
    const unsigned need_ = min(2u * (unsigned)(tau) + 5u, 128u); \
    if (__builtin_expect(ready < need_, 0)) SLOW_WAIT(need_); } while (0)
#define SLOT_ISSUE(tau) do { \
    { const unsigned lim_ = min(2u * (unsigned)(tau) + 15u, 128u); polled = ready < lim_; const char* fp_ = flags + ready * 4; \
      asm volatile("s_cmp_lt_u32 %2, %3\n\ts_cbranch_scc0 1f\n\tv_mbcnt_lo_u32_b32 %0, -1, 0\n\tv_mbcnt_hi_u32_b32 %0, -1, %0\n\tv_lshlrev_b32 %0, 2, %0\n\t" \
                   "global_load_dword %0, %0, %1 sc0 sc1\n1:" : "+v"(pv) : "s"(fp_), "s"(ready), "s"(lim_) : "memory", "scc"); } \
    extra = 0; \
    if (__builtin_expect(pstate != 3u, 0)) { asm volatile("" : "+s"(pstate)); \
      if (pstate == 0u) { if (tnow - t_start >= push_at) { PUSH_STORES(); pstate = 1u; extra = 1u; } } \
      else if (pstate == 1u) pstate = 2u; \
      else { PUSH_SIGNAL(); pstate = 3u; } } } while (0)
#define WAIT_SLOT() asm volatile("s_cmp_eq_u32 %1, 0\n\ts_cbranch_scc1 1f\n\ts_waitcnt vmcnt(20)\n\ts_branch 2f\n1:\n\ts_waitcnt vmcnt(6)\n2:" : "+v"(pv) : "s"(extra) : "memory", "scc")
#define TIME_ISSUE() unsigned long long tt_; asm volatile("s_memrealtime %0" : "=s"(tt_) :: "memory")

  f32x4 acc[2][2][4][2];
#pragma unroll
  for (int i0 = 0; i0 < 2; ++i0)
#pragma unroll
    for (int i1 = 0; i1 < 2; ++i1)
#pragma unroll
      for (int m = 0; m < 4; ++m)
#pragma unroll
        for (int n = 0; n < 2; ++n) acc[i0][i1][m][n] = (f32x4){0.f, 0.f, 0.f, 0.f};
  bf16x8 At[4][2]; bf16x4 B0[2][2][2], B1[2][2][2];

  if (wr == 1) BAR;
  WAIT_V(2); BAR;
  BAR;

  for (int t = 0; t < NT; t += 2) {
    const bool last = (t == NT - 2);
    const char* a1 = pA + (size_t)(t + 1) * kA;
    const char* a2 = pA + (size_t)(last ? t : t + 2) * kA; const char* b2 = pB + (size_t)(last ? t : t + 2) * kB;
    const char* a3 = a2 + kA; const char* b3 = b2 + kB;
    SLOT_CHECK(t);
    LDB(B0, 0, 0); LDA(0, 0); STAGE_A(1, 1, a1);
    SLOT_ISSUE(t);
    WAIT_L(8); BAR; WAIT_AB(B0); MMA(0, 0, B0); BAR;
    LDB(B1, 0, 1); STAGE_B(0, 0, b2);
    BAR; WAIT_B(B1); MMA(0, 1, B1); BAR;
    { TIME_ISSUE(); LDA(0, 1); STAGE_A(0, 0, a2);
      BAR; WAIT_AT(); } MMA(1, 0, B0); BAR;
    STAGE_B(0, 1, b2);
    WAIT_SLOT(); BAR; MMA(1, 1, B1); BAR;
    SLOT_CHECK(t + 1);
    LDB(B0, 1, 0); LDA(1, 0); STAGE_A(0, 1, a2);
    SLOT_ISSUE(t + 1);
    WAIT_L(8); BAR; WAIT_AB(B0); MMA(0, 0, B0); BAR;
    LDB(B1, 1, 1); STAGE_B(1, 0, b3);
    BAR; WAIT_B(B1); MMA(0, 1, B1); BAR;
    { TIME_ISSUE(); LDA(1, 1); STAGE_A(1, 0, a3);
      BAR; WAIT_AT(); } MMA(1, 0, B0); BAR;
    STAGE_B(1, 1, b3);
    WAIT_SLOT(); BAR; MMA(1, 1, B1); BAR;
  }
  if (pstate != 3u) {
    if (pstate == 0u) PUSH_STORES();
    WAIT_V(0);
    PUSH_SIGNAL();
  }
  WAIT_L(0);
  if (wr == 0) BAR;
  BAR;
  const bool bad = *(volatile LAS unsigned*)(lds + LDS_M + 4) != 0u;
  { int tz = threadIdx.x; asm volatile("" : "+v"(tz));
    const int ln = tz & 63, r16 = ln & 15, q4 = ln >> 4;
    char* op = a.out + (size_t)(pm * 256 + wr * 64) * ROWB + (size_t)(pn * 256 + wc * 32) * 2;
    auto ro = __builtin_amdgcn_make_buffer_rsrc(op, 0, 192 * ROWB, 0x00020000);
    const unsigned vo = (unsigned)r16 * ROWB + (unsigned)((((q4 & 1) << 4) | ((q4 >> 1) << 3)) * 2);
#pragma unroll
    for (int ai = 0; ai < 2; ++ai)
#pragma unroll
      for (int m = 0; m < 4; ++m)
#pragma unroll
        for (int bj = 0; bj < 2; ++bj) {
          const f32x4 v0 = acc[ai][bj][m][0], v1 = acc[ai][bj][m][1];
          unsigned a0 = cvt_pk_bf16(v0[0], v0[1]), a1 = cvt_pk_bf16(v0[2], v0[3]), b0 = cvt_pk_bf16(v1[0], v1[1]), b1 = cvt_pk_bf16(v1[2], v1[3]);
          auto s0 = __builtin_amdgcn_permlane16_swap(a0, b0, false, false);
          auto s1 = __builtin_amdgcn_permlane16_swap(a1, b1, false, false);
          u32x4 w = {s0[0], s1[0], s0[1], s1[1]};
          if (bad) w = (u32x4){0x7fc07fc0u, 0x7fc07fc0u, 0x7fc07fc0u, 0x7fc07fc0u};
          __builtin_amdgcn_raw_buffer_store_b128(w, ro, vo + bj * 256, (ai * 128 + m * 16) * ROWB, AUXO);
        }
    if (tz == 0) __hip_atomic_store((unsigned*)(myws + WS_CTR + (size_t)b * 128), call, __ATOMIC_RELAXED, __HIP_MEMORY_SCOPE_AGENT);
  }
}

extern "C" void kernel_launch(int rank, int nranks, ncclComm_t comm, hipStream_t stream,
                              void* const* d_in, const int* in_sizes, int n_in,
                              void* d_out, int out_size, const int* in_dtypes, int out_dtype,
                              void* const* d_out_all, void* const* d_ws_all, size_t ws_bytes) {
  if (nranks != NR || (unsigned)rank >= (unsigned)NR || ws_bytes < WS_NEED || n_in < 2 ||
      in_sizes[0] != 512 * 4096 || in_sizes[1] != 4096 * 4096 || out_size != 4096 * 4096) {
    fprintf(stderr, "kernel_launch: unexpected shapes\n"); return;
  }
  static bool init = false;
  if (!init) {
    if (hipFuncSetAttribute((const void*)ag_gemm_kernel, hipFuncAttributeMaxDynamicSharedMemorySize, LDS_BYTES) != hipSuccess) {
      fprintf(stderr, "hipFuncSetAttribute failed\n"); return; }
    init = true;
  }
  Args a;
  a.x = (const char*)d_in[0]; a.w = (const char*)d_in[1]; a.out = (char*)d_out; a.rank = rank;
  for (int p = 0; p < NR; ++p) a.ws[p] = (char*)d_ws_all[p];
  ag_gemm_kernel<<<256, 512, LDS_BYTES, stream>>>(a);
}
```

```cpp
#include <hip/hip_runtime.h>
#include <rccl/rccl.h>
#include <cstdio>

#define D1 135
#define D2 147
#define ID1 2560
#define T0 150
#define NOCOMM 0
#define NOWAIT 0
#define SLP 6
#define PBITS sc0 sc1
#define STAMP 0
#define DEADLINE 100000000
#define STR_(x) #x
#define STR(x) STR_(x)

namespace {
typedef short bf16x8 __attribute__((ext_vector_type(8)));
typedef short bf16x4 __attribute__((ext_vector_type(4)));
typedef float f32x4 __attribute__((ext_vector_type(4)));
typedef unsigned u32x4 __attribute__((ext_vector_type(4)));
typedef unsigned long long u64;
#define LAS __attribute__((address_space(3)))

constexpr int NR = 8;
constexpr int NT = 64;
constexpr int HT = 16384;
constexpr int LDS_A = 0, LDS_B = 4 * HT, LDS_M = 8 * HT, LDS_BYTES = 8 * HT + 64;
constexpr unsigned ROWB = 8192;
constexpr size_t XG_PAR = 32ull << 20, XG_SRC = 4ull << 20;
constexpr unsigned XG_KT = 65536;
constexpr size_t WS_FLAGS = 64ull << 20;
constexpr size_t WS_CTR = WS_FLAGS + 524288;
constexpr size_t WS_TMO = WS_CTR + 32768;
constexpr size_t WS_STAMP = WS_TMO + 4096;
constexpr size_t WS_NEED = WS_STAMP + 65536;
constexpr int NEVER = 0x7fffffff;

struct Args { const char* x; const char* w; char* out; char* ws[NR]; int rank; };

__device__ __forceinline__ unsigned cvt_pk_bf16(float lo, float hi) { unsigned r; asm volatile("v_cvt_pk_bf16_f32 %0, %1, %2" : "=v"(r) : "v"(lo), "v"(hi)); return r; }
__device__ __forceinline__ unsigned rfl(unsigned v) { return (unsigned)__builtin_amdgcn_readfirstlane((int)v); }
__device__ __forceinline__ unsigned now_sync() { u64 t; asm volatile("s_memrealtime %0\n\ts_waitcnt lgkmcnt(0)" : "=s"(t) :: "memory"); return (unsigned)t; }
__device__ __forceinline__ unsigned off_x(int pc, int lane) { const int row = 8 * pc + (lane >> 3); return (unsigned)row * ROWB + (unsigned)(((lane & 7) ^ ((row >> 1) & 7)) * 16); }
__device__ __forceinline__ unsigned off_w(int pc, int lane) { const int k = 4 * pc + (lane >> 4); const int f = (k & 3) | (((k >> 3) & 1) << 2); return (unsigned)k * ROWB + (unsigned)(((lane & 15) ^ (2 * f)) * 16); }
}

#define GLDS(voff, sbase, ldsb, imm) asm volatile("s_add_u32 m0, %2, %3\n\ts_nop 0\n\tglobal_load_lds_dwordx4 %0, %1" :: "v"(voff), "s"(sbase), "s"(ldsb), "i"(imm) : "memory", "scc")
#define WAIT_V(n) asm volatile("s_waitcnt vmcnt(%0)" :: "i"(n) : "memory")
#define WAIT_L(n) asm volatile("s_waitcnt lgkmcnt(" #n ")" ::: "memory")
#define BAR __builtin_amdgcn_s_barrier()
#define DSR128(dst, addr, off) asm volatile("ds_read_b128 %0, %1 offset:%2" : "=v"(dst) : "v"(addr), "i"(off) : "memory")
#define DSRTR(dst, addr, off) asm volatile("ds_read_b64_tr_b16 %0, %1 offset:%2" : "=v"(dst) : "v"(addr), "i"(off) : "memory")
#define LDA(bb, h) do { _Pragma("unroll") for (int m = 0; m < 4; ++m) { \
    DSR128(At[m][0], aoff0, ((bb) * 2 + (h)) * HT + m * 2048); DSR128(At[m][1], aoff1, ((bb) * 2 + (h)) * HT + m * 2048); } } while (0)
#define LDB(dst, bb, h) do { _Pragma("unroll") for (int kk = 0; kk < 2; ++kk) { \
    DSRTR(dst[0][kk][0], boff0, ((bb) * 2 + (h)) * HT + kk * 8192); DSRTR(dst[0][kk][1], boff0, ((bb) * 2 + (h)) * HT + kk * 8192 + 1024); \
    DSRTR(dst[1][kk][0], boff1, ((bb) * 2 + (h)) * HT + kk * 8192); DSRTR(dst[1][kk][1], boff1, ((bb) * 2 + (h)) * HT + kk * 8192 + 1024); } } while (0)
#define TIE_A "+v"(At[0][0]), "+v"(At[0][1]), "+v"(At[1][0]), "+v"(At[1][1]), "+v"(At[2][0]), "+v"(At[2][1]), "+v"(At[3][0]), "+v"(At[3][1])
#define TIE_B(B) "+v"(B[0][0][0]), "+v"(B[0][0][1]), "+v"(B[0][1][0]), "+v"(B[0][1][1]), "+v"(B[1][0][0]), "+v"(B[1][0][1]), "+v"(B[1][1][0]), "+v"(B[1][1][1])
#define TIE_C(ai, bj) "+v"(acc[ai][bj][0][0]), "+v"(acc[ai][bj][0][1]), "+v"(acc[ai][bj][1][0]), "+v"(acc[ai][bj][1][1]), "+v"(acc[ai][bj][2][0]), "+v"(acc[ai][bj][2][1]), "+v"(acc[ai][bj][3][0]), "+v"(acc[ai][bj][3][1])
#define WAIT_AB(B) asm volatile("s_waitcnt lgkmcnt(0)" : TIE_A, TIE_B(B) :: "memory")
#define WAIT_B(B) asm volatile("s_waitcnt lgkmcnt(0)" : TIE_B(B) :: "memory")
#define WAIT_A() asm volatile("s_waitcnt lgkmcnt(0)" : TIE_A :: "memory")
#define BFRAG(Bt, n, k) __builtin_shufflevector(Bt[n][k][0], Bt[n][k][1], 0, 1, 2, 3, 4, 5, 6, 7)
#define MMA(ai, bj, Bt) do { asm volatile("s_setprio 1" : TIE_C(ai, bj) :: "memory"); \
    _Pragma("unroll") for (int m = 0; m < 4; ++m) _Pragma("unroll") for (int n = 0; n < 2; ++n) _Pragma("unroll") for (int k = 0; k < 2; ++k) \
      acc[ai][bj][m][n] = __builtin_amdgcn_mfma_f32_16x16x32_bf16(BFRAG(Bt, n, k), At[m][k], acc[ai][bj][m][n], 0, 0, 0); \
    asm volatile("s_setprio 0" : TIE_C(ai, bj) :: "memory"); } while (0)
#define MMAW(ai, bj, Bt, TEXT) asm volatile(TEXT : TIE_C(ai, bj) : "v"(At[0][0]), "v"(At[0][1]), "v"(At[1][0]), "v"(At[1][1]), "v"(At[2][0]), "v"(At[2][1]), "v"(At[3][0]), "v"(At[3][1]), \
    "v"(BFRAG(Bt, 0, 0)), "v"(BFRAG(Bt, 0, 1)), "v"(BFRAG(Bt, 1, 0)), "v"(BFRAG(Bt, 1, 1)) : "memory", "scc")

#define ROLE_SETUP() \
  int tid_ = threadIdx.x; asm volatile("" : "+v"(tid_)); const int lane = tid_ & 63, fr = lane & 15, fq = lane >> 4; \
  unsigned aoff0, aoff1, boff0, boff1; \
  { const unsigned rb = (unsigned)((wid >> 2) * 64 + fr) * 128u; const unsigned s = (unsigned)(fr >> 1); \
    aoff0 = lbase + LDS_A + rb + (((unsigned)fq ^ s) << 4); aoff1 = lbase + LDS_A + rb + (((unsigned)(4 + fq) ^ s) << 4); \
    const unsigned q = (unsigned)(lane & 15) >> 2, p = (unsigned)lane & 3u, g = (unsigned)fq, f = q | ((g & 1u) << 2); \
    const unsigned kb = (8u * g + q) * 256u + 8u * (p & 1u); \
    boff0 = lbase + LDS_B + kb + ((((unsigned)(4 * (wid & 3)) + (p >> 1)) ^ (2u * f)) << 4); \
    boff1 = lbase + LDS_B + kb + ((((unsigned)(4 * (wid & 3) + 2) + (p >> 1)) ^ (2u * f)) << 4); } \
  f32x4 acc[2][2][4][2]; \
  _Pragma("unroll") for (int i0 = 0; i0 < 2; ++i0) _Pragma("unroll") for (int i1 = 0; i1 < 2; ++i1) \
  _Pragma("unroll") for (int m = 0; m < 4; ++m) _Pragma("unroll") for (int n = 0; n < 2; ++n) acc[i0][i1][m][n] = (f32x4){0.f, 0.f, 0.f, 0.f}; \
  bf16x8 At[4][2]; bf16x4 B0[2][2][2], B1[2][2][2]

namespace {
__device__ __forceinline__ void epilogue(unsigned lbase, const f32x4 (&acc)[2][2][4][2], char* otile, int wid) {
  WAIT_L(0);
  if (wid < 4) BAR;
  BAR;
  unsigned badv; asm volatile("ds_read_b32 %0, %1\n\ts_waitcnt lgkmcnt(0)" : "=v"(badv) : "v"(lbase + LDS_M) : "memory");
  const bool bad = badv != 0u;
  int tz = threadIdx.x; asm volatile("" : "+v"(tz));
  const int ln = tz & 63, r16 = ln & 15, q4 = ln >> 4;
  char* op = otile + (size_t)((wid >> 2) * 64) * ROWB + (size_t)((wid & 3) * 64);
  auto ro = __builtin_amdgcn_make_buffer_rsrc(op, 0, 192 * ROWB, 0x00020000);
  const unsigned vo = (unsigned)r16 * ROWB + (unsigned)((((q4 & 1) << 4) | ((q4 >> 1) << 3)) * 2);
#pragma unroll
  for (int ai = 0; ai < 2; ++ai)
#pragma unroll
    for (int m = 0; m < 4; ++m)
#pragma unroll
      for (int bj = 0; bj < 2; ++bj) {
        const f32x4 v0 = acc[ai][bj][m][0], v1 = acc[ai][bj][m][1];
        unsigned a0 = cvt_pk_bf16(v0[0], v0[1]), a1 = cvt_pk_bf16(v0[2], v0[3]), b0 = cvt_pk_bf16(v1[0], v1[1]), b1 = cvt_pk_bf16(v1[2], v1[3]);
        auto s0 = __builtin_amdgcn_permlane16_swap(a0, b0, false, false);
        auto s1 = __builtin_amdgcn_permlane16_swap(a1, b1, false, false);
        u32x4 w = {s0[0], s1[0], s0[1], s1[1]};
        if (bad) w = (u32x4){0x7fc07fc0u, 0x7fc07fc0u, 0x7fc07fc0u, 0x7fc07fc0u};
        __builtin_amdgcn_raw_buffer_store_b128(w, ro, vo + bj * 256, (ai * 128 + m * 16) * ROWB, 16);
      }
}

template <int NG>
__device__ __forceinline__ void worker(unsigned lbase, const char* xs, const char* xg, const char* ctr, const char* pB, char* otile, bool local, int wid) {
  ROLE_SETUP();
  const int x3 = (wid - 1) * 8;
  const unsigned vA0 = local ? off_x(wid, lane) : (unsigned)(wid * 1024 + lane * 16), vB0 = off_w(wid, lane);
  const unsigned vA1 = vA0 + (local ? 64u * ROWB : 8192u), vB1 = vB0 + 32u * ROWB;
  const unsigned vA3 = local ? off_x(x3, lane) : (unsigned)(x3 * 1024 + lane * 16), vB3 = off_w(x3, lane);
  const unsigned ldsw = rfl(lbase + (unsigned)wid * 1024u), lds3 = rfl(lbase + (unsigned)x3 * 1024u);
  constexpr size_t hB = 256, kB = 64 * (size_t)ROWB;
  const size_t hA = local ? 128 * (size_t)ROWB : HT, kA = local ? 128 : XG_KT;
#define STAGE_B(bb, h, gp) do { const char* g_ = (gp) + (h) * hB; GLDS(vB0, g_, ldsw, LDS_B + ((bb) * 2 + (h)) * HT); GLDS(vB1, g_, ldsw, LDS_B + ((bb) * 2 + (h)) * HT + 8192); \
    if (NG == 3) GLDS(vB3, g_, lds3, LDS_B + ((bb) * 2 + (h)) * HT); } while (0)
#define STAGE_A(bb, h, gp) do { const char* g_ = (gp) + (h) * hA; GLDS(vA0, g_, ldsw, LDS_A + ((bb) * 2 + (h)) * HT); GLDS(vA1, g_, ldsw, LDS_A + ((bb) * 2 + (h)) * HT + 8192); \
    if (NG == 3) GLDS(vA3, g_, lds3, LDS_A + ((bb) * 2 + (h)) * HT); } while (0)
  unsigned cv; { unsigned z = 0; asm volatile("global_load_dword %0, %1, %2 sc1" : "=v"(cv) : "v"(z), "s"(ctr) : "memory"); }
  STAGE_B(0, 0, pB); STAGE_B(0, 1, pB); STAGE_B(1, 0, pB + kB);
  asm volatile("s_waitcnt vmcnt(%1)" : "+v"(cv) : "i"(3 * NG) : "memory");
  const char* pA = local ? xs : xg + ((rfl(cv) + 1u) & 1u) * XG_PAR;
  BAR; STAGE_A(0, 0, pA);
  BAR; STAGE_A(0, 1, pA);
  BAR; STAGE_A(1, 0, pA + kA); STAGE_B(1, 1, pB + kB);
  if (wid >= 4) BAR;
  WAIT_V(2 * NG); BAR;
  BAR;
  for (int t = 0; t < NT; t += 2) {
    const bool last = (t == NT - 2);
    const char* a1 = pA + (size_t)(t + 1) * kA;
    const char* a2 = pA + (size_t)(last ? t : t + 2) * kA; const char* b2 = pB + (size_t)(last ? t : t + 2) * kB;
    const char* a3 = a2 + kA; const char* b3 = b2 + kB;
    LDB(B0, 0, 0); LDA(0, 0); STAGE_A(1, 1, a1);
    WAIT_V(5 * NG); WAIT_L(8); BAR; WAIT_AB(B0); MMA(0, 0, B0); BAR;
    LDB(B1, 0, 1); STAGE_B(0, 0, b2);
    WAIT_V(5 * NG); BAR; WAIT_B(B1); MMA(0, 1, B1); BAR;
    LDA(0, 1); STAGE_A(0, 0, a2);
    BAR; WAIT_A(); MMA(1, 0, B0); BAR;
    STAGE_B(0, 1, b2);
    WAIT_V(5 * NG); BAR; MMA(1, 1, B1); BAR;
    LDB(B0, 1, 0); LDA(1, 0); STAGE_A(0, 1, a2);
    WAIT_V(5 * NG); WAIT_L(8); BAR; WAIT_AB(B0); MMA(0, 0, B0); BAR;
    LDB(B1, 1, 1); STAGE_B(1, 0, b3);
    WAIT_V(5 * NG); BAR; WAIT_B(B1); MMA(0, 1, B1); BAR;
    LDA(1, 1); STAGE_A(1, 0, a3);
    BAR; WAIT_A(); MMA(1, 0, B0); BAR;
    STAGE_B(1, 1, b3);
    WAIT_V(5 * NG); BAR; MMA(1, 1, B1); BAR;
  }
#undef STAGE_A
#undef STAGE_B
  epilogue(lbase, acc, otile, wid);
}
}

__global__ __launch_bounds__(512, 2) __attribute__((amdgpu_num_vgpr(112), amdgpu_num_sgpr(62))) void ag_gemm_kernel(Args a) {
  extern __shared__ __attribute__((aligned(16))) char smem[];
  LAS char* lds = (LAS char*)smem;
  asm volatile("v_mov_b32 v239, 0\n\tv_mov_b32 v236, 0" ::: "v224", "v255", "s56", "s101");
  const int wid = __builtin_amdgcn_readfirstlane(threadIdx.x >> 6);
  const int b = blockIdx.x, rank = a.rank;
  const int xcd = b & 7, idx = b >> 3;
  const int pm = 4 * (xcd >> 1) + (idx & 3), pn = 8 * (xcd & 1) + (idx >> 2);
  const int src = pm >> 1, strip = pm & 1;
  const bool local = (src == rank);
  char* myws = a.ws[rank];
  const char* ctr = myws + WS_CTR + (size_t)b * 128;

  const unsigned lbase = (unsigned)(size_t)lds;
  char* otile = a.out + (size_t)(pm * 256) * ROWB + (size_t)pn * 512;
  const char* xs = a.x + (size_t)strip * 256 * ROWB;
  const char* xg = myws + (size_t)src * XG_SRC + (size_t)strip * 2 * HT;
  const char* pB = a.w + (size_t)pn * 512;

  if (wid >= 3) worker<2>(lbase, xs, xg, ctr, pB, otile, local, wid);
  else if (wid >= 1) worker<3>(lbase, xs, xg, ctr, pB, otile, local, wid);
  else {
    const unsigned t_start = now_sync();
    ROLE_SETUP();
    const unsigned jj = (unsigned)((((((idx >> 1) ^ rank ^ 1) & 1) << 4) | ((idx >> 2) << 1) | (idx & 1)) * 8 + xcd);
    const unsigned vx0 = off_x(0, lane), vx1 = off_x(1, lane);
#define ID_NEXT(id) ((id) + 292u + ((((id) & 7u) + 5u) >> 3))
#define ID_SRC(id) (a.x + (size_t)((((id) << 14) & 0x3e0000u) | (((id) >> 1) & 0x1f80u)))
    unsigned cv; { unsigned z = 0; asm volatile("global_load_dword %0, %1, %2 sc1" : "=v"(cv) : "v"(z), "s"(ctr) : "memory"); }
    const unsigned idA = (jj / 7u) * 8u + jj % 7u, idB = ID_NEXT(idA);
    { const char* sa = ID_SRC(idA); const char* sb = ID_SRC(idB);
      asm volatile("global_load_dwordx4 v[240:243], %0, %2\n\tglobal_load_dwordx4 v[244:247], %1, %2\n\tglobal_load_dwordx4 v[248:251], %0, %3\n\tglobal_load_dwordx4 v[252:255], %1, %3"
                   :: "v"(vx0), "v"(vx1), "s"(sa), "s"(sb) : "memory"); }
    char* pw = a.ws[(rank + 1) & 7];
#pragma unroll
    for (int q = 1; q < 7; ++q) { char* c = a.ws[(rank + 1 + q) & 7]; asm volatile("" : "+s"(c)); if (lane == q) pw = c; }
    *(LAS unsigned*)(lds + LDS_M) = 0u;
    const unsigned foff = (unsigned)(src * 65536 + strip * 256);
    const char* flags = myws + WS_FLAGS + foff;
    const unsigned g = ((unsigned)lane >> 3) & 3u, gcode = (g >> 1) * 4u + (g & 1u);
    { const unsigned pbase = gcode * 128u + ((unsigned)lane & 7u) * 16u;
      asm volatile("v_mov_b32 v231, %0\n\tv_mov_b32 v230, %1\n\tv_mov_b32 v229, %2\n\tv_mov_b32 v228, %3\n\tv_mov_b32 v237, %3\n\tv_mov_b32 v227, %4\n\t"
                   "v_mov_b32 v226, %5\n\tv_mov_b32 v225, %6"
                   :: "v"((unsigned)lane * 16u), "v"(vx0), "v"(vx1), "v"(pbase), "v"(g),
                      "s"((unsigned)(WS_TMO - WS_FLAGS) - foff), "s"(lbase + LDS_M) : "memory"); }
    { const int due0 = T0 + (int)((jj * D1) >> 8);
      asm volatile("s_mov_b32 s64, 0\n\ts_mov_b32 s65, 0\n\ts_mov_b32 s66, 0x7fffffff\n\ts_mov_b32 s67, 0x7fffffff\n\ts_mov_b32 s71, 0\n\ts_mov_b32 s72, 0\n\t"
                   "s_mov_b32 s68, %0\n\ts_mov_b32 s63, %1\n\ts_mov_b32 s69, %2\n\ts_mov_b32 s70, %3\n\ts_mov_b32 s86, 0\n\ts_mov_b32 s87, %4\n\t"
                   "s_mov_b32 s89, %5\n\ts_mov_b64 s[90:91], %6\n\ts_mov_b64 s[92:93], %7\n\ts_mov_b32 s60, 0\n\ts_mov_b32 s77, 0\n\ts_mov_b32 s59, 0"
                   :: "s"(NOCOMM ? NEVER : due0), "s"(due0 + D1), "s"(idA), "s"(NOCOMM ? 0x4000u : idB), "s"((local || NOCOMM || NOWAIT) ? 0u : 128u),
                      "s"(t_start), "s"(a.x), "s"(flags) : "memory"); }
    asm volatile("s_waitcnt vmcnt(0)" : "+v"(cv) :: "memory");
    const unsigned call = rfl(cv) + 1u;
    { const unsigned xgo = (call & 1u) * (unsigned)XG_PAR + (unsigned)rank * (unsigned)XG_SRC; const u64 pd = (u64)(pw + xgo);
      asm volatile("v_mov_b32 v234, %0\n\tv_mov_b32 v235, %1\n\ts_mov_b32 s88, %2\n\ts_mov_b32 s61, %3\n\tv_mov_b32 v233, %4\n\tv_mov_b32 v224, %4\n\tv_mov_b32 v232, %5\n\ts_nop 1\n\t" "s_cmp_lg_u32 s87, 0\n\t" "s_cselect_b32 s58, -1, 0\n\t" "s_mov_b32 s75, 0\n\t" "s_mov_b32 s76, 0\n\t" "s_cmp_lt_u32 s70, " STR(ID1) "\n\t" "s_cselect_b32 s98, 0, (" STR(D2) "-" STR(D1) ")\n\t" "s_add_i32 s98, s98, s63\n\t" "s_addk_i32 s98, " STR(D1) "\n\t" "s_cmp_lg_u32 s75, 0\n\t" "s_cselect_b32 s66, 0x7fffffff, s66\n\t" "s_add_i32 s64, s64, 3\n\t" "s_cmp_lg_u32 s76, 0\n\t" "s_cselect_b32 s65, s64, s65\n\t" "s_cselect_b32 s68, s63, s68\n\t" "s_cselect_b32 s63, s98, s63\n\t" "s_cselect_b32 s69, s70, s69\n\t" "s_cselect_b32 s70, s62, s70\n\t" "s_cmp_eq_u32 s66, 0x7fffffff\n\t" "s_cselect_b32 s66, s67, s66\n\t" "s_cselect_b32 s71, s72, s71\n\t" "s_cselect_b32 s67, 0x7fffffff, s67\n\t" "s_and_b32 s98, s69, 7\n\t" "s_and_b32 s99, s69, 0x3ff8\n\t" "v_readlane_b32 s78, v234, s98\n\t" "v_readlane_b32 s79, v235, s98\n\t" "s_lshl_b32 s99, s99, 8\n\t" "s_and_b32 s98, s70, 7\n\t" "s_add_u32 s78, s78, s99\n\t" "s_addc_u32 s79, s79, 0\n\t" "s_cmp_ge_u32 s98, 3\n\t" "s_addc_u32 s62, s70, 292\n\t" "s_lshl_b32 s98, s62, 14\n\t" "s_lshr_b32 s99, s62, 1\n\t" "s_and_b32 s98, s98, 0x3e0000\n\t" "s_and_b32 s99, s99, 0x1f80\n\t" "s_or_b32 s98, s98, s99\n\t" "s_add_u32 s80, s90, s98\n\t" "s_addc_u32 s81, s91, 0\n\t" "s_add_i32 s64, s64, 2\n\t" "s_cmp_lg_u32 s75, 0\n\t" "s_cselect_b32 s68, 0x7fffffff, s68\n\t" "s_cselect_b32 s67, s64, s67\n\t" "s_cselect_b32 s72, s69, s72\n\t" "s_cmp_eq_u32 s66, 0x7fffffff\n\t" "s_cselect_b32 s66, s67, s66\n\t" "s_cselect_b32 s71, s72, s71\n\t" "s_cselect_b32 s67, 0x7fffffff, s67\n\t" "s_and_b32 s98, s71, 7\n\t" "s_lshl_b32 s99, s71, 1\n\t" "v_readlane_b32 s82, v234, s98\n\t" "v_readlane_b32 s83, v235, s98\n\t" "s_and_b32 s99, s99, 0x7ff0\n\t" "s_lshr_b32 s98, s71, 6\n\t" "s_add_i32 s98, s98, s88\n\t" "s_add_u32 s82, s82, s99\n\t" "s_addc_u32 s83, s83, 0\n\t" "v_mov_b32 v238, s98\n\t" "s_cmp_eq_u32 s68, 0x7fffffff\n\t" "s_cselect_b32 s74, s65, 0x7fffffff\n\t" "s_cmp_lt_u32 s70, 0x4000\n\t" "s_cselect_b32 s74, s74, 0x7fffffff\n\t" "s_mov_b32 s64, 0\n\t" "s_cmp_lt_u32 s86, s87\n\t" "s_cselect_b32 s84, s61, 0\n\t" "s_and_b32 s85, s86, 3\n\t" "s_lshl_b32 s85, s85, 3\n\t" "s_getreg_b32 s98, hwreg(HW_REG_IB_STS, 22, 2)\n\t" "s_getreg_b32 s99, hwreg(HW_REG_IB_STS, 0, 4)\n\t" "s_lshl4_add_u32 s98, s98, s99\n\t" "s_sub_i32 s73, s64, s98\n\t"
                   :: "v"((unsigned)pd), "v"((unsigned)(pd >> 32)), "s"(call * 256u + 1u), "s"(call * 256u + 1u + 2u * (unsigned)strip),
                      "v"(call * 256u + 1u + 2u * (unsigned)strip + gcode), "s"((unsigned)WS_FLAGS + (unsigned)rank * 65536u - xgo) : "memory", "scc"); }
    asm volatile("s_branch ag_slow_end\n\t" "ag_slow_sub:\n\t" "s_memrealtime s[96:97]\n\t" "s_add_i32 s60, s60, 1\n\t" "s_waitcnt lgkmcnt(0)\n\t" "s_sub_i32 s77, s96, s89\n\t" "s_getreg_b32 s98, hwreg(HW_REG_IB_STS, 22, 2)\n\t" "s_getreg_b32 s99, hwreg(HW_REG_IB_STS, 0, 4)\n\t" "s_lshl4_add_u32 s98, s98, s99\n\t" "s_sub_i32 s73, s64, s98\n\t" "s_cmp_ge_i32 s73, s66\n\t" "s_cselect_b64 exec, 1, 0\n\t" "global_store_dword v232, v238, s[82:83] sc0 sc1\n\t" "s_cselect_b32 s75, 1, 0\n\t" "s_cmp_ge_i32 s73, s74\n\t" "s_cselect_b64 exec, -1, 0\n\t" "v_mov_b64 v[240:241], v[248:249]\n\t" "v_mov_b64 v[242:243], v[250:251]\n\t" "v_mov_b64 v[244:245], v[252:253]\n\t" "v_mov_b64 v[246:247], v[254:255]\n\t" "global_load_dwordx4 v[248:251], v230, s[80:81]\n\t" "global_load_dwordx4 v[252:255], v229, s[80:81]\n\t" "s_mov_b64 exec, -1\n\t" "s_cselect_b32 s76, 2, 0\n\t" "s_cmp_lt_u32 s70, " STR(ID1) "\n\t" "s_cselect_b32 s98, 0, (" STR(D2) "-" STR(D1) ")\n\t" "s_add_i32 s98, s98, s63\n\t" "s_addk_i32 s98, " STR(D1) "\n\t" "s_cmp_lg_u32 s75, 0\n\t" "s_cselect_b32 s66, 0x7fffffff, s66\n\t" "s_add_i32 s64, s64, 3\n\t" "s_cmp_lg_u32 s76, 0\n\t" "s_cselect_b32 s65, s64, s65\n\t" "s_cselect_b32 s68, s63, s68\n\t" "s_cselect_b32 s63, s98, s63\n\t" "s_cselect_b32 s69, s70, s69\n\t" "s_cselect_b32 s70, s62, s70\n\t" "s_cmp_eq_u32 s66, 0x7fffffff\n\t" "s_cselect_b32 s66, s67, s66\n\t" "s_cselect_b32 s71, s72, s71\n\t" "s_cselect_b32 s67, 0x7fffffff, s67\n\t" "s_and_b32 s98, s69, 7\n\t" "s_and_b32 s99, s69, 0x3ff8\n\t" "v_readlane_b32 s78, v234, s98\n\t" "v_readlane_b32 s79, v235, s98\n\t" "s_lshl_b32 s99, s99, 8\n\t" "s_and_b32 s98, s70, 7\n\t" "s_add_u32 s78, s78, s99\n\t" "s_addc_u32 s79, s79, 0\n\t" "s_cmp_ge_u32 s98, 3\n\t" "s_addc_u32 s62, s70, 292\n\t" "s_lshl_b32 s98, s62, 14\n\t" "s_lshr_b32 s99, s62, 1\n\t" "s_and_b32 s98, s98, 0x3e0000\n\t" "s_and_b32 s99, s99, 0x1f80\n\t" "s_or_b32 s98, s98, s99\n\t" "s_add_u32 s80, s90, s98\n\t" "s_addc_u32 s81, s91, 0\n\t" "s_cmp_ge_i32 s77, s68\n\t" "s_cselect_b64 exec, -1, 0\n\t" "global_store_dwordx4 v231, v[240:243], s[78:79] sc0 sc1\n\t" "global_store_dwordx4 v231, v[244:247], s[78:79] offset:1024 sc0 sc1\n\t" "s_mov_b64 exec, -1\n\t" "s_cselect_b32 s75, 2, 0\n\t" "v_cmp_gt_u32 s[56:57], v224, v239\n\t" "s_mov_b32 exec_hi, 0\n\t" "s_and_b32 exec_lo, s56, s58\n\t" "global_load_dword v239, v237, s[92:93] " STR(PBITS) "\n\t" "s_mov_b64 exec, -1\n\t" "global_load_dword v236, v226, s[92:93] sc1\n\t" "s_add_i32 s64, s64, 2\n\t" "s_cmp_lg_u32 s75, 0\n\t" "s_cselect_b32 s68, 0x7fffffff, s68\n\t" "s_cselect_b32 s67, s64, s67\n\t" "s_cselect_b32 s72, s69, s72\n\t" "s_add_i32 s64, s64, 2\n\t" "s_cmp_eq_u32 s66, 0x7fffffff\n\t" "s_cselect_b32 s66, s67, s66\n\t" "s_cselect_b32 s71, s72, s71\n\t" "s_cselect_b32 s67, 0x7fffffff, s67\n\t" "s_and_b32 s98, s71, 7\n\t" "s_lshl_b32 s99, s71, 1\n\t" "v_readlane_b32 s82, v234, s98\n\t" "v_readlane_b32 s83, v235, s98\n\t" "s_and_b32 s99, s99, 0x7ff0\n\t" "s_lshr_b32 s98, s71, 6\n\t" "s_add_i32 s98, s98, s88\n\t" "s_add_u32 s82, s82, s99\n\t" "s_addc_u32 s83, s83, 0\n\t" "v_mov_b32 v238, s98\n\t" "s_cmp_eq_u32 s68, 0x7fffffff\n\t" "s_cselect_b32 s74, s65, 0x7fffffff\n\t" "s_cmp_lt_u32 s70, 0x4000\n\t" "s_cselect_b32 s74, s74, 0x7fffffff\n\t" "s_sleep " STR(SLP) "\n\t" "s_getreg_b32 s98, hwreg(HW_REG_IB_STS, 22, 2)\n\t" "s_getreg_b32 s99, hwreg(HW_REG_IB_STS, 0, 4)\n\t" "s_lshl4_add_u32 s98, s98, s99\n\t" "s_sub_i32 s73, s64, s98\n\t" "v_readfirstlane_b32 s98, v236\n\t" "s_cmp_gt_u32 s77, " STR(DEADLINE) "\n\t" "s_cselect_b32 s98, s88, s98\n\t" "s_cmp_eq_u32 s98, s88\n\t" "s_cbranch_scc1 ag_slow_dead\n\t" "s_setpc_b64 s[94:95]\n\t" "ag_slow_dead:\n\t" "s_mov_b32 s87, 0\n\t" "s_mov_b32 s58, 0\n\t" "s_mov_b32 s84, 0\n\t" "v_mov_b32 v224, s88\n\t" "global_store_dword v226, v224, s[92:93] sc1\n\t" "ds_write_b32 v225, v224\n\t" "s_add_i32 s64, s64, 1\n\t" "s_setpc_b64 s[94:95]\n\t" "ag_slow_end:\n\t" ::: "memory", "scc");
#define CHECK() asm volatile("s_getpc_b64 s[94:95]\n\t" "v_cmp_le_u32 s[100:101], s84, v239\n\t" "s_lshr_b32 s98, s100, s85\n\t" "s_andn2_b32 s98, 0xff, s98\n\t" "s_cbranch_scc1 ag_slow_sub\n\t" ::: "memory", "scc")
#define ACT_SP() asm volatile("s_cmp_ge_i32 s77, s68\n\t" "s_cselect_b64 exec, -1, 0\n\t" "global_store_dwordx4 v231, v[240:243], s[78:79] sc0 sc1\n\t" "global_store_dwordx4 v231, v[244:247], s[78:79] offset:1024 sc0 sc1\n\t" "s_mov_b64 exec, -1\n\t" "s_cselect_b32 s75, 2, 0\n\t" "v_cmp_gt_u32 s[56:57], v224, v239\n\t" "s_mov_b32 exec_hi, 0\n\t" "s_and_b32 exec_lo, s56, s58\n\t" "global_load_dword v239, v237, s[92:93] " STR(PBITS) "\n\t" "s_mov_b64 exec, -1\n\t" ::: "memory", "scc")
#define ACT_FL() asm volatile("s_cmp_ge_i32 s73, s66\n\t" "s_cselect_b64 exec, 1, 0\n\t" "global_store_dword v232, v238, s[82:83] sc0 sc1\n\t" "s_cselect_b32 s75, 1, 0\n\t" "s_cmp_ge_i32 s73, s74\n\t" "s_cselect_b64 exec, -1, 0\n\t" "v_mov_b64 v[240:241], v[248:249]\n\t" "v_mov_b64 v[242:243], v[250:251]\n\t" "v_mov_b64 v[244:245], v[252:253]\n\t" "v_mov_b64 v[246:247], v[254:255]\n\t" "global_load_dwordx4 v[248:251], v230, s[80:81]\n\t" "global_load_dwordx4 v[252:255], v229, s[80:81]\n\t" "s_mov_b64 exec, -1\n\t" "s_cselect_b32 s76, 2, 0\n\t" ::: "memory", "scc")
#define TIME_GET() asm volatile("s_memrealtime s[96:97]" ::: "memory")
#define MMA_S "s_setprio 1\n\t" "v_mfma_f32_16x16x32_bf16 %0, %16, %8, %0\n\t" "s_add_i32 s64, s64, 2\n\t" "s_cmp_lg_u32 s75, 0\n\t" "s_cselect_b32 s68, 0x7fffffff, s68\n\t" "v_mfma_f32_16x16x32_bf16 %1, %18, %8, %1\n\t" "s_cselect_b32 s67, s64, s67\n\t" "s_cselect_b32 s72, s69, s72\n\t" "v_mfma_f32_16x16x32_bf16 %2, %16, %10, %2\n\t" "s_add_i32 s64, s64, 1\n\t" "s_cmp_eq_u32 s66, 0x7fffffff\n\t" "v_mfma_f32_16x16x32_bf16 %3, %18, %10, %3\n\t" "s_cselect_b32 s66, s67, s66\n\t" "s_cselect_b32 s71, s72, s71\n\t" "v_mfma_f32_16x16x32_bf16 %4, %16, %12, %4\n\t" "s_cselect_b32 s67, 0x7fffffff, s67\n\t" "s_and_b32 s98, s71, 7\n\t" "v_mfma_f32_16x16x32_bf16 %5, %18, %12, %5\n\t" "s_lshl_b32 s99, s71, 1\n\t" "v_readlane_b32 s82, v234, s98\n\t" "v_mfma_f32_16x16x32_bf16 %6, %16, %14, %6\n\t" "v_readlane_b32 s83, v235, s98\n\t" "s_and_b32 s99, s99, 0x7ff0\n\t" "v_mfma_f32_16x16x32_bf16 %7, %18, %14, %7\n\t" "s_lshr_b32 s98, s71, 6\n\t" "s_add_i32 s98, s98, s88\n\t" "s_add_u32 s82, s82, s99\n\t" "v_mfma_f32_16x16x32_bf16 %0, %17, %9, %0\n\t" "s_addc_u32 s83, s83, 0\n\t" "v_mov_b32 v238, s98\n\t" "v_mfma_f32_16x16x32_bf16 %1, %19, %9, %1\n\t" "s_cmp_eq_u32 s68, 0x7fffffff\n\t" "s_cselect_b32 s74, s65, 0x7fffffff\n\t" "v_mfma_f32_16x16x32_bf16 %2, %17, %11, %2\n\t" "s_cmp_lt_u32 s70, 0x4000\n\t" "s_cselect_b32 s74, s74, 0x7fffffff\n\t" "v_mfma_f32_16x16x32_bf16 %3, %19, %11, %3\n\t" "s_cmp_lt_u32 s86, s87\n\t" "s_cselect_b32 s84, s61, 0\n\t" "v_mfma_f32_16x16x32_bf16 %4, %17, %13, %4\n\t" "s_and_b32 s85, s86, 3\n\t" "s_lshl_b32 s85, s85, 3\n\t" "v_mfma_f32_16x16x32_bf16 %5, %19, %13, %5\n\t" "s_getreg_b32 s98, hwreg(HW_REG_IB_STS, 22, 2)\n\t" "s_getreg_b32 s99, hwreg(HW_REG_IB_STS, 0, 4)\n\t" "v_mfma_f32_16x16x32_bf16 %6, %17, %15, %6\n\t" "s_lshl4_add_u32 s98, s98, s99\n\t" "s_sub_i32 s73, s64, s98\n\t" "v_mfma_f32_16x16x32_bf16 %7, %19, %15, %7\n\t" "s_setprio 0\n\t"
#define MMA_ST "s_setprio 1\n\t" "v_mfma_f32_16x16x32_bf16 %0, %16, %8, %0\n\t" "s_sub_i32 s77, s96, s89\n\t" "s_add_i32 s77, s77, 66\n\t" "s_add_i32 s64, s64, 2\n\t" "v_mfma_f32_16x16x32_bf16 %1, %18, %8, %1\n\t" "s_cmp_lg_u32 s75, 0\n\t" "s_cselect_b32 s68, 0x7fffffff, s68\n\t" "v_mfma_f32_16x16x32_bf16 %2, %16, %10, %2\n\t" "s_cselect_b32 s67, s64, s67\n\t" "s_cselect_b32 s72, s69, s72\n\t" "v_mfma_f32_16x16x32_bf16 %3, %18, %10, %3\n\t" "s_add_i32 s64, s64, 1\n\t" "s_cmp_eq_u32 s66, 0x7fffffff\n\t" "s_cselect_b32 s66, s67, s66\n\t" "v_mfma_f32_16x16x32_bf16 %4, %16, %12, %4\n\t" "s_cselect_b32 s71, s72, s71\n\t" "s_cselect_b32 s67, 0x7fffffff, s67\n\t" "v_mfma_f32_16x16x32_bf16 %5, %18, %12, %5\n\t" "s_and_b32 s98, s71, 7\n\t" "s_lshl_b32 s99, s71, 1\n\t" "v_mfma_f32_16x16x32_bf16 %6, %16, %14, %6\n\t" "v_readlane_b32 s82, v234, s98\n\t" "v_readlane_b32 s83, v235, s98\n\t" "v_mfma_f32_16x16x32_bf16 %7, %18, %14, %7\n\t" "s_and_b32 s99, s99, 0x7ff0\n\t" "s_lshr_b32 s98, s71, 6\n\t" "s_add_i32 s98, s98, s88\n\t" "v_mfma_f32_16x16x32_bf16 %0, %17, %9, %0\n\t" "s_add_u32 s82, s82, s99\n\t" "s_addc_u32 s83, s83, 0\n\t" "v_mfma_f32_16x16x32_bf16 %1, %19, %9, %1\n\t" "v_mov_b32 v238, s98\n\t" "s_cmp_eq_u32 s68, 0x7fffffff\n\t" "v_mfma_f32_16x16x32_bf16 %2, %17, %11, %2\n\t" "s_cselect_b32 s74, s65, 0x7fffffff\n\t" "s_cmp_lt_u32 s70, 0x4000\n\t" "v_mfma_f32_16x16x32_bf16 %3, %19, %11, %3\n\t" "s_cselect_b32 s74, s74, 0x7fffffff\n\t" "s_cmp_lt_u32 s86, s87\n\t" "s_cselect_b32 s84, s61, 0\n\t" "v_mfma_f32_16x16x32_bf16 %4, %17, %13, %4\n\t" "s_and_b32 s85, s86, 3\n\t" "s_lshl_b32 s85, s85, 3\n\t" "v_mfma_f32_16x16x32_bf16 %5, %19, %13, %5\n\t" "s_getreg_b32 s98, hwreg(HW_REG_IB_STS, 22, 2)\n\t" "s_getreg_b32 s99, hwreg(HW_REG_IB_STS, 0, 4)\n\t" "v_mfma_f32_16x16x32_bf16 %6, %17, %15, %6\n\t" "s_lshl4_add_u32 s98, s98, s99\n\t" "s_sub_i32 s73, s64, s98\n\t" "v_mfma_f32_16x16x32_bf16 %7, %19, %15, %7\n\t" "s_setprio 0\n\t"
#define MMA_F1T "s_setprio 1\n\t" "v_mfma_f32_16x16x32_bf16 %0, %16, %8, %0\n\t" "s_sub_i32 s77, s96, s89\n\t" "s_add_i32 s77, s77, 33\n\t" "s_cmp_lt_u32 s70, " STR(ID1) "\n\t" "v_mfma_f32_16x16x32_bf16 %1, %18, %8, %1\n\t" "s_cselect_b32 s98, 0, (" STR(D2) "-" STR(D1) ")\n\t" "s_add_i32 s98, s98, s63\n\t" "s_addk_i32 s98, " STR(D1) "\n\t" "v_mfma_f32_16x16x32_bf16 %2, %16, %10, %2\n\t" "s_cmp_lg_u32 s75, 0\n\t" "s_cselect_b32 s66, 0x7fffffff, s66\n\t" "s_add_i32 s64, s64, 3\n\t" "v_mfma_f32_16x16x32_bf16 %3, %18, %10, %3\n\t" "s_cmp_lg_u32 s76, 0\n\t" "s_cselect_b32 s65, s64, s65\n\t" "s_cselect_b32 s68, s63, s68\n\t" "v_mfma_f32_16x16x32_bf16 %4, %16, %12, %4\n\t" "s_cselect_b32 s63, s98, s63\n\t" "s_cselect_b32 s69, s70, s69\n\t" "s_cselect_b32 s70, s62, s70\n\t" "v_mfma_f32_16x16x32_bf16 %5, %18, %12, %5\n\t" "s_cmp_eq_u32 s66, 0x7fffffff\n\t" "s_cselect_b32 s66, s67, s66\n\t" "s_cselect_b32 s71, s72, s71\n\t" "v_mfma_f32_16x16x32_bf16 %6, %16, %14, %6\n\t" "s_cselect_b32 s67, 0x7fffffff, s67\n\t" "s_and_b32 s98, s69, 7\n\t" "s_and_b32 s99, s69, 0x3ff8\n\t" "v_mfma_f32_16x16x32_bf16 %7, %18, %14, %7\n\t" "v_readlane_b32 s78, v234, s98\n\t" "v_readlane_b32 s79, v235, s98\n\t" "v_mfma_f32_16x16x32_bf16 %0, %17, %9, %0\n\t" "s_lshl_b32 s99, s99, 8\n\t" "s_and_b32 s98, s70, 7\n\t" "s_add_u32 s78, s78, s99\n\t" "v_mfma_f32_16x16x32_bf16 %1, %19, %9, %1\n\t" "s_addc_u32 s79, s79, 0\n\t" "s_cmp_ge_u32 s98, 3\n\t" "s_addc_u32 s62, s70, 292\n\t" "v_mfma_f32_16x16x32_bf16 %2, %17, %11, %2\n\t" "s_lshl_b32 s98, s62, 14\n\t" "s_lshr_b32 s99, s62, 1\n\t" "s_and_b32 s98, s98, 0x3e0000\n\t" "v_mfma_f32_16x16x32_bf16 %3, %19, %11, %3\n\t" "s_and_b32 s99, s99, 0x1f80\n\t" "s_or_b32 s98, s98, s99\n\t" "s_add_u32 s80, s90, s98\n\t" "v_mfma_f32_16x16x32_bf16 %4, %17, %13, %4\n\t" "s_addc_u32 s81, s91, 0\n\t" "s_add_i32 s86, s86, 1\n\t" "s_add_i32 s61, s61, 1\n\t" "v_mfma_f32_16x16x32_bf16 %5, %19, %13, %5\n\t" "s_add_i32 s98, s86, 3\n\t" "v_sub_u32 v224, s98, v227\n\t" "v_lshrrev_b32 v224, 2, v224\n\t" "v_mfma_f32_16x16x32_bf16 %6, %17, %15, %6\n\t" "v_lshl_add_u32 v237, v224, 10, v228\n\t" "v_lshl_add_u32 v224, v224, 3, v233\n\t" "v_mfma_f32_16x16x32_bf16 %7, %19, %15, %7\n\t" "s_setprio 0\n\t"
#define MMA_F3 "s_setprio 1\n\t" "v_mfma_f32_16x16x32_bf16 %0, %16, %8, %0\n\t" "s_cmp_lt_u32 s70, " STR(ID1) "\n\t" "s_cselect_b32 s98, 0, (" STR(D2) "-" STR(D1) ")\n\t" "s_add_i32 s98, s98, s63\n\t" "v_mfma_f32_16x16x32_bf16 %1, %18, %8, %1\n\t" "s_addk_i32 s98, " STR(D1) "\n\t" "s_cmp_lg_u32 s75, 0\n\t" "s_cselect_b32 s66, 0x7fffffff, s66\n\t" "v_mfma_f32_16x16x32_bf16 %2, %16, %10, %2\n\t" "s_add_i32 s64, s64, 3\n\t" "s_cmp_lg_u32 s76, 0\n\t" "s_cselect_b32 s65, s64, s65\n\t" "v_mfma_f32_16x16x32_bf16 %3, %18, %10, %3\n\t" "s_cselect_b32 s68, s63, s68\n\t" "s_cselect_b32 s63, s98, s63\n\t" "v_mfma_f32_16x16x32_bf16 %4, %16, %12, %4\n\t" "s_cselect_b32 s69, s70, s69\n\t" "s_cselect_b32 s70, s62, s70\n\t" "s_cmp_eq_u32 s66, 0x7fffffff\n\t" "v_mfma_f32_16x16x32_bf16 %5, %18, %12, %5\n\t" "s_cselect_b32 s66, s67, s66\n\t" "s_cselect_b32 s71, s72, s71\n\t" "s_cselect_b32 s67, 0x7fffffff, s67\n\t" "v_mfma_f32_16x16x32_bf16 %6, %16, %14, %6\n\t" "s_and_b32 s98, s69, 7\n\t" "s_and_b32 s99, s69, 0x3ff8\n\t" "v_readlane_b32 s78, v234, s98\n\t" "v_mfma_f32_16x16x32_bf16 %7, %18, %14, %7\n\t" "v_readlane_b32 s79, v235, s98\n\t" "s_lshl_b32 s99, s99, 8\n\t" "v_mfma_f32_16x16x32_bf16 %0, %17, %9, %0\n\t" "s_and_b32 s98, s70, 7\n\t" "s_add_u32 s78, s78, s99\n\t" "s_addc_u32 s79, s79, 0\n\t" "v_mfma_f32_16x16x32_bf16 %1, %19, %9, %1\n\t" "s_cmp_ge_u32 s98, 3\n\t" "s_addc_u32 s62, s70, 292\n\t" "s_lshl_b32 s98, s62, 14\n\t" "v_mfma_f32_16x16x32_bf16 %2, %17, %11, %2\n\t" "s_lshr_b32 s99, s62, 1\n\t" "s_and_b32 s98, s98, 0x3e0000\n\t" "s_and_b32 s99, s99, 0x1f80\n\t" "v_mfma_f32_16x16x32_bf16 %3, %19, %11, %3\n\t" "s_or_b32 s98, s98, s99\n\t" "s_add_u32 s80, s90, s98\n\t" "v_mfma_f32_16x16x32_bf16 %4, %17, %13, %4\n\t" "s_addc_u32 s81, s91, 0\n\t" "s_add_i32 s86, s86, 1\n\t" "s_add_i32 s61, s61, 3\n\t" "v_mfma_f32_16x16x32_bf16 %5, %19, %13, %5\n\t" "s_add_i32 s98, s86, 3\n\t" "v_sub_u32 v224, s98, v227\n\t" "v_lshrrev_b32 v224, 2, v224\n\t" "v_mfma_f32_16x16x32_bf16 %6, %17, %15, %6\n\t" "v_lshl_add_u32 v237, v224, 10, v228\n\t" "v_lshl_add_u32 v224, v224, 3, v233\n\t" "v_mfma_f32_16x16x32_bf16 %7, %19, %15, %7\n\t" "s_setprio 0\n\t"
    CHECK(); BAR; asm volatile("s_add_i32 s86, s86, 1\n\t" "s_add_i32 s61, s61, 1\n\t" "s_add_i32 s98, s86, 3\n\t" "v_sub_u32 v224, s98, v227\n\t" "v_lshrrev_b32 v224, 2, v224\n\t" "v_lshl_add_u32 v237, v224, 10, v228\n\t" "v_lshl_add_u32 v224, v224, 3, v233\n\t" "s_cmp_lt_u32 s86, s87\n\t" "s_cselect_b32 s84, s61, 0\n\t" "s_and_b32 s85, s86, 3\n\t" "s_lshl_b32 s85, s85, 3\n\t" ::: "memory", "scc");
    CHECK(); BAR; asm volatile("s_add_i32 s86, s86, 1\n\t" "s_add_i32 s61, s61, 3\n\t" "s_add_i32 s98, s86, 3\n\t" "v_sub_u32 v224, s98, v227\n\t" "v_lshrrev_b32 v224, 2, v224\n\t" "v_lshl_add_u32 v237, v224, 10, v228\n\t" "v_lshl_add_u32 v224, v224, 3, v233\n\t" "s_cmp_lt_u32 s86, s87\n\t" "s_cselect_b32 s84, s61, 0\n\t" "s_and_b32 s85, s86, 3\n\t" "s_lshl_b32 s85, s85, 3\n\t" ::: "memory", "scc");
    CHECK(); BAR; asm volatile("s_add_i32 s86, s86, 1\n\t" "s_add_i32 s61, s61, 1\n\t" "s_add_i32 s98, s86, 3\n\t" "v_sub_u32 v224, s98, v227\n\t" "v_lshrrev_b32 v224, 2, v224\n\t" "v_lshl_add_u32 v237, v224, 10, v228\n\t" "v_lshl_add_u32 v224, v224, 3, v233\n\t" "s_cmp_lt_u32 s86, s87\n\t" "s_cselect_b32 s84, s61, 0\n\t" "s_and_b32 s85, s86, 3\n\t" "s_lshl_b32 s85, s85, 3\n\t" ::: "memory", "scc");
    CHECK(); asm volatile("s_cmp_ge_i32 s73, s66\n\t" "s_cselect_b64 exec, 1, 0\n\t" "global_store_dword v232, v238, s[82:83] sc0 sc1\n\t" "s_cselect_b32 s75, 1, 0\n\t" "s_cmp_ge_i32 s73, s74\n\t" "s_cselect_b64 exec, -1, 0\n\t" "v_mov_b64 v[240:241], v[248:249]\n\t" "v_mov_b64 v[242:243], v[250:251]\n\t" "v_mov_b64 v[244:245], v[252:253]\n\t" "v_mov_b64 v[246:247], v[254:255]\n\t" "global_load_dwordx4 v[248:251], v230, s[80:81]\n\t" "global_load_dwordx4 v[252:255], v229, s[80:81]\n\t" "s_mov_b64 exec, -1\n\t" "s_cselect_b32 s76, 2, 0\n\t" "s_cmp_lt_u32 s70, " STR(ID1) "\n\t" "s_cselect_b32 s98, 0, (" STR(D2) "-" STR(D1) ")\n\t" "s_add_i32 s98, s98, s63\n\t" "s_addk_i32 s98, " STR(D1) "\n\t" "s_cmp_lg_u32 s75, 0\n\t" "s_cselect_b32 s66, 0x7fffffff, s66\n\t" "s_add_i32 s64, s64, 3\n\t" "s_cmp_lg_u32 s76, 0\n\t" "s_cselect_b32 s65, s64, s65\n\t" "s_cselect_b32 s68, s63, s68\n\t" "s_cselect_b32 s63, s98, s63\n\t" "s_cselect_b32 s69, s70, s69\n\t" "s_cselect_b32 s70, s62, s70\n\t" "s_cmp_eq_u32 s66, 0x7fffffff\n\t" "s_cselect_b32 s66, s67, s66\n\t" "s_cselect_b32 s71, s72, s71\n\t" "s_cselect_b32 s67, 0x7fffffff, s67\n\t" "s_and_b32 s98, s69, 7\n\t" "s_and_b32 s99, s69, 0x3ff8\n\t" "v_readlane_b32 s78, v234, s98\n\t" "v_readlane_b32 s79, v235, s98\n\t" "s_lshl_b32 s99, s99, 8\n\t" "s_and_b32 s98, s70, 7\n\t" "s_add_u32 s78, s78, s99\n\t" "s_addc_u32 s79, s79, 0\n\t" "s_cmp_ge_u32 s98, 3\n\t" "s_addc_u32 s62, s70, 292\n\t" "s_lshl_b32 s98, s62, 14\n\t" "s_lshr_b32 s99, s62, 1\n\t" "s_and_b32 s98, s98, 0x3e0000\n\t" "s_and_b32 s99, s99, 0x1f80\n\t" "s_or_b32 s98, s98, s99\n\t" "s_add_u32 s80, s90, s98\n\t" "s_addc_u32 s81, s91, 0\n\t" "s_add_i32 s86, s86, 1\n\t" "s_add_i32 s61, s61, 3\n\t" "s_add_i32 s98, s86, 3\n\t" "v_sub_u32 v224, s98, v227\n\t" "v_lshrrev_b32 v224, 2, v224\n\t" "v_lshl_add_u32 v237, v224, 10, v228\n\t" "v_lshl_add_u32 v224, v224, 3, v233\n\t" "s_memrealtime s[96:97]\n\t" "s_waitcnt lgkmcnt(0)\n\t" "s_sub_i32 s77, s96, s89\n\t" "s_add_i32 s77, s77, 30\n\t" ::: "memory", "scc");
    BAR;
    BAR;
    for (int t = 0; t < NT; t += 2) {
      LDB(B0, 0, 0); LDA(0, 0); ACT_SP();
      WAIT_L(8); BAR; WAIT_AB(B0); MMAW(0, 0, B0, MMA_S); BAR;
      TIME_GET(); LDB(B1, 0, 1); CHECK(); ACT_FL();
      BAR; WAIT_B(B1); MMAW(0, 1, B1, MMA_F1T); BAR;
      TIME_GET(); LDA(0, 1); ACT_SP();
      BAR; WAIT_A(); MMAW(1, 0, B0, MMA_ST); BAR;
      CHECK(); ACT_FL();
      BAR; MMAW(1, 1, B1, MMA_F3); BAR;
      LDB(B0, 1, 0); LDA(1, 0); ACT_SP();
      WAIT_L(8); BAR; WAIT_AB(B0); MMAW(0, 0, B0, MMA_S); BAR;
      TIME_GET(); LDB(B1, 1, 1); CHECK(); ACT_FL();
      BAR; WAIT_B(B1); MMAW(0, 1, B1, MMA_F1T); BAR;
      TIME_GET(); LDA(1, 1); ACT_SP();
      BAR; WAIT_A(); MMAW(1, 0, B0, MMA_ST); BAR;
      CHECK(); ACT_FL();
      BAR; MMAW(1, 1, B1, MMA_F3); BAR;
    }
    asm volatile("ag_fin:\n\t" "s_mov_b32 s77, 0x7ffffffe\n\t" "s_cmp_ge_i32 s77, s68\n\t" "s_cselect_b64 exec, -1, 0\n\t" "global_store_dwordx4 v231, v[240:243], s[78:79] sc0 sc1\n\t" "global_store_dwordx4 v231, v[244:247], s[78:79] offset:1024 sc0 sc1\n\t" "s_mov_b64 exec, -1\n\t" "s_cselect_b32 s75, 2, 0\n\t" "s_add_i32 s64, s64, 2\n\t" "s_cmp_lg_u32 s75, 0\n\t" "s_cselect_b32 s68, 0x7fffffff, s68\n\t" "s_cselect_b32 s67, s64, s67\n\t" "s_cselect_b32 s72, s69, s72\n\t" "s_cmp_eq_u32 s66, 0x7fffffff\n\t" "s_cselect_b32 s66, s67, s66\n\t" "s_cselect_b32 s71, s72, s71\n\t" "s_cselect_b32 s67, 0x7fffffff, s67\n\t" "s_and_b32 s98, s71, 7\n\t" "s_lshl_b32 s99, s71, 1\n\t" "v_readlane_b32 s82, v234, s98\n\t" "v_readlane_b32 s83, v235, s98\n\t" "s_and_b32 s99, s99, 0x7ff0\n\t" "s_lshr_b32 s98, s71, 6\n\t" "s_add_i32 s98, s98, s88\n\t" "s_add_u32 s82, s82, s99\n\t" "s_addc_u32 s83, s83, 0\n\t" "v_mov_b32 v238, s98\n\t" "s_cmp_eq_u32 s68, 0x7fffffff\n\t" "s_cselect_b32 s74, s65, 0x7fffffff\n\t" "s_cmp_lt_u32 s70, 0x4000\n\t" "s_cselect_b32 s74, s74, 0x7fffffff\n\t" "s_waitcnt vmcnt(0)\n\t" "s_mov_b32 s73, s64\n\t" "s_cmp_ge_i32 s73, s66\n\t" "s_cselect_b64 exec, 1, 0\n\t" "global_store_dword v232, v238, s[82:83] sc0 sc1\n\t" "s_cselect_b32 s75, 1, 0\n\t" "s_cmp_ge_i32 s73, s74\n\t" "s_cselect_b64 exec, -1, 0\n\t" "v_mov_b64 v[240:241], v[248:249]\n\t" "v_mov_b64 v[242:243], v[250:251]\n\t" "v_mov_b64 v[244:245], v[252:253]\n\t" "v_mov_b64 v[246:247], v[254:255]\n\t" "global_load_dwordx4 v[248:251], v230, s[80:81]\n\t" "global_load_dwordx4 v[252:255], v229, s[80:81]\n\t" "s_mov_b64 exec, -1\n\t" "s_cselect_b32 s76, 2, 0\n\t" "s_cmp_lt_u32 s70, " STR(ID1) "\n\t" "s_cselect_b32 s98, 0, (" STR(D2) "-" STR(D1) ")\n\t" "s_add_i32 s98, s98, s63\n\t" "s_addk_i32 s98, " STR(D1) "\n\t" "s_cmp_lg_u32 s75, 0\n\t" "s_cselect_b32 s66, 0x7fffffff, s66\n\t" "s_add_i32 s64, s64, 3\n\t" "s_cmp_lg_u32 s76, 0\n\t" "s_cselect_b32 s65, s64, s65\n\t" "s_cselect_b32 s68, s63, s68\n\t" "s_cselect_b32 s63, s98, s63\n\t" "s_cselect_b32 s69, s70, s69\n\t" "s_cselect_b32 s70, s62, s70\n\t" "s_cmp_eq_u32 s66, 0x7fffffff\n\t" "s_cselect_b32 s66, s67, s66\n\t" "s_cselect_b32 s71, s72, s71\n\t" "s_cselect_b32 s67, 0x7fffffff, s67\n\t" "s_and_b32 s98, s69, 7\n\t" "s_and_b32 s99, s69, 0x3ff8\n\t" "v_readlane_b32 s78, v234, s98\n\t" "v_readlane_b32 s79, v235, s98\n\t" "s_lshl_b32 s99, s99, 8\n\t" "s_and_b32 s98, s70, 7\n\t" "s_add_u32 s78, s78, s99\n\t" "s_addc_u32 s79, s79, 0\n\t" "s_cmp_ge_u32 s98, 3\n\t" "s_addc_u32 s62, s70, 292\n\t" "s_lshl_b32 s98, s62, 14\n\t" "s_lshr_b32 s99, s62, 1\n\t" "s_and_b32 s98, s98, 0x3e0000\n\t" "s_and_b32 s99, s99, 0x1f80\n\t" "s_or_b32 s98, s98, s99\n\t" "s_add_u32 s80, s90, s98\n\t" "s_addc_u32 s81, s91, 0\n\t" "s_and_b32 s98, s66, s68\n\t" "s_cmp_lg_u32 s98, 0x7fffffff\n\t" "s_cbranch_scc1 ag_fin\n\t" "s_cmp_lt_u32 s70, 0x4000\n\t" "s_cbranch_scc1 ag_fin\n\t" ::: "memory", "scc");
    if (lane == 0) __hip_atomic_store((unsigned*)ctr, call, __ATOMIC_RELAXED, __HIP_MEMORY_SCOPE_AGENT);
    epilogue(lbase, acc, otile, wid);
  }
}

extern "C" void kernel_launch(int rank, int nranks, ncclComm_t comm, hipStream_t stream,
                              void* const* d_in, const int* in_sizes, int n_in,
                              void* d_out, int out_size, const int* in_dtypes, int out_dtype,
                              void* const* d_out_all, void* const* d_ws_all, size_t ws_bytes) {
  if (nranks != NR || (unsigned)rank >= (unsigned)NR || ws_bytes < WS_NEED || n_in < 2 ||
      in_sizes[0] != 512 * 4096 || in_sizes[1] != 4096 * 4096 || out_size != 4096 * 4096) {
    fprintf(stderr, "kernel_launch: unexpected shapes\n"); return;
  }
  static bool init = false;
  if (!init) {
    if (hipFuncSetAttribute((const void*)ag_gemm_kernel, hipFuncAttributeMaxDynamicSharedMemorySize, LDS_BYTES) != hipSuccess) {
      fprintf(stderr, "hipFuncSetAttribute failed\n"); return; }
    init = true;
  }
  Args a;
  a.x = (const char*)d_in[0]; a.w = (const char*)d_in[1]; a.out = (char*)d_out; a.rank = rank;
  for (int p = 0; p < NR; ++p) a.ws[p] = (char*)d_ws_all[p];
  ag_gemm_kernel<<<256, 512, LDS_BYTES, stream>>>(a);
}
```

```cpp
#include <hip/hip_runtime.h>
#include <rccl/rccl.h>
#include <cstdio>

#define D1 280
#define D2 300
#define ID1 1280
#define T0 100
#define NOCOMM 0
#define NOWAIT 0
#define SLP 3
#define AHEAD 10
#define LATEW 1
#define SFIRST 0
#define APOS 1
#define STAMP 0
#define DEADLINE 100000000
#define LDSBAD 131072
#define STR_(x) #x
#define STR(x) STR_(x)

namespace {
typedef short bf16x8 __attribute__((ext_vector_type(8)));
typedef short bf16x4 __attribute__((ext_vector_type(4)));
typedef float f32x4 __attribute__((ext_vector_type(4)));
typedef unsigned u32x4 __attribute__((ext_vector_type(4)));
typedef unsigned long long u64;
#define LAS __attribute__((address_space(3)))

constexpr int NR = 8;
constexpr int NT = 64;
constexpr int HT = 16384;
constexpr int LDS_A = 0, LDS_B = 4 * HT, LDS_M = 8 * HT, LDS_BYTES = 8 * HT + 64;
constexpr unsigned ROWB = 8192;
constexpr size_t XG_PAR = 32ull << 20, XG_SRC = 4ull << 20;
constexpr unsigned XG_KT = 65536;
constexpr size_t WS_FLAGS = 64ull << 20;
constexpr size_t WS_CTR = WS_FLAGS + 524288;
constexpr size_t WS_TMO = WS_CTR + 32768;
constexpr size_t WS_STAMP = WS_TMO + 4096;
constexpr size_t WS_NEED = WS_STAMP + 65536;
constexpr int NEVER = 0x7fffffff;

struct Args { const char* x; const char* w; char* out; char* ws[NR]; int rank; };

__device__ __forceinline__ unsigned cvt_pk_bf16(float lo, float hi) { unsigned r; asm volatile("v_cvt_pk_bf16_f32 %0, %1, %2" : "=v"(r) : "v"(lo), "v"(hi)); return r; }
__device__ __forceinline__ unsigned rfl(unsigned v) { return (unsigned)__builtin_amdgcn_readfirstlane((int)v); }
__device__ __forceinline__ unsigned now_sync() { u64 t; asm volatile("s_memrealtime %0\n\ts_waitcnt lgkmcnt(0)" : "=s"(t) :: "memory"); return (unsigned)t; }
__device__ __forceinline__ unsigned off_x(int pc, int lane) { const int row = 8 * pc + (lane >> 3); return (unsigned)row * ROWB + (unsigned)(((lane & 7) ^ ((row >> 1) & 7)) * 16); }
__device__ __forceinline__ unsigned off_w(int pc, int lane) { const int k = 4 * pc + (lane >> 4); const int f = (k & 3) | (((k >> 3) & 1) << 2); return (unsigned)k * ROWB + (unsigned)(((lane & 15) ^ (2 * f)) * 16); }
}

#define GLDS(voff, sbase, ldsb, imm) asm volatile("s_add_u32 m0, %2, %3\n\ts_nop 0\n\tglobal_load_lds_dwordx4 %0, %1" :: "v"(voff), "s"(sbase), "s"(ldsb), "i"(imm) : "memory", "scc")
#define WAIT_V(n) asm volatile("s_waitcnt vmcnt(%0)" :: "i"(n) : "memory")
#define WAIT_L(n) asm volatile("s_waitcnt lgkmcnt(" #n ")" ::: "memory")
#define BAR __builtin_amdgcn_s_barrier()
#define DSR128(dst, addr, off) asm volatile("ds_read_b128 %0, %1 offset:%2" : "=v"(dst) : "v"(addr), "i"(off) : "memory")
#define DSRTR(dst, addr, off) asm volatile("ds_read_b64_tr_b16 %0, %1 offset:%2" : "=v"(dst) : "v"(addr), "i"(off) : "memory")
#define LDA(bb, h) do { _Pragma("unroll") for (int m = 0; m < 4; ++m) { \
    DSR128(At[m][0], aoff0, ((bb) * 2 + (h)) * HT + m * 2048); DSR128(At[m][1], aoff1, ((bb) * 2 + (h)) * HT + m * 2048); } } while (0)
#define LDB(dst, bb, h) do { _Pragma("unroll") for (int kk = 0; kk < 2; ++kk) { \
    DSRTR(dst[0][kk][0], boff0, ((bb) * 2 + (h)) * HT + kk * 8192); DSRTR(dst[0][kk][1], boff0, ((bb) * 2 + (h)) * HT + kk * 8192 + 1024); \
    DSRTR(dst[1][kk][0], boff1, ((bb) * 2 + (h)) * HT + kk * 8192); DSRTR(dst[1][kk][1], boff1, ((bb) * 2 + (h)) * HT + kk * 8192 + 1024); } } while (0)
#define TIE_A "+v"(At[0][0]), "+v"(At[0][1]), "+v"(At[1][0]), "+v"(At[1][1]), "+v"(At[2][0]), "+v"(At[2][1]), "+v"(At[3][0]), "+v"(At[3][1])
#define TIE_B(B) "+v"(B[0][0][0]), "+v"(B[0][0][1]), "+v"(B[0][1][0]), "+v"(B[0][1][1]), "+v"(B[1][0][0]), "+v"(B[1][0][1]), "+v"(B[1][1][0]), "+v"(B[1][1][1])
#define TIE_C(ai, bj) "+v"(acc[ai][bj][0][0]), "+v"(acc[ai][bj][0][1]), "+v"(acc[ai][bj][1][0]), "+v"(acc[ai][bj][1][1]), "+v"(acc[ai][bj][2][0]), "+v"(acc[ai][bj][2][1]), "+v"(acc[ai][bj][3][0]), "+v"(acc[ai][bj][3][1])
#define WAIT_AB(B) asm volatile("s_waitcnt lgkmcnt(0)" : TIE_A, TIE_B(B) :: "memory")
#define WAIT_B(B) asm volatile("s_waitcnt lgkmcnt(0)" : TIE_B(B) :: "memory")
#define WAIT_A() asm volatile("s_waitcnt lgkmcnt(0)" : TIE_A :: "memory")
#define BFRAG(Bt, n, k) __builtin_shufflevector(Bt[n][k][0], Bt[n][k][1], 0, 1, 2, 3, 4, 5, 6, 7)
#define WAIT_ABB() asm volatile("s_waitcnt lgkmcnt(0)" : TIE_A, TIE_B(B0), TIE_B(B1) :: "memory")
#define MMAW(ai, bj, Bt, TEXT) asm volatile(TEXT : TIE_C(ai, bj) : "v"(At[0][0]), "v"(At[0][1]), "v"(At[1][0]), "v"(At[1][1]), "v"(At[2][0]), "v"(At[2][1]), "v"(At[3][0]), "v"(At[3][1]), \
    "v"(BFRAG(Bt, 0, 0)), "v"(BFRAG(Bt, 0, 1)), "v"(BFRAG(Bt, 1, 0)), "v"(BFRAG(Bt, 1, 1)) : "memory", "scc")

#define MMA2(ai) do { MMAW(ai, 0, B0, "s_setprio 1\n\t" "v_mfma_f32_16x16x32_bf16 %0, %16, %8, %0\n\t" "v_mfma_f32_16x16x32_bf16 %0, %17, %9, %0\n\t" "v_mfma_f32_16x16x32_bf16 %1, %18, %8, %1\n\t" "v_mfma_f32_16x16x32_bf16 %1, %19, %9, %1\n\t" "v_mfma_f32_16x16x32_bf16 %2, %16, %10, %2\n\t" "v_mfma_f32_16x16x32_bf16 %2, %17, %11, %2\n\t" "v_mfma_f32_16x16x32_bf16 %3, %18, %10, %3\n\t" "v_mfma_f32_16x16x32_bf16 %3, %19, %11, %3\n\t" "v_mfma_f32_16x16x32_bf16 %4, %16, %12, %4\n\t" "v_mfma_f32_16x16x32_bf16 %4, %17, %13, %4\n\t" "v_mfma_f32_16x16x32_bf16 %5, %18, %12, %5\n\t" "v_mfma_f32_16x16x32_bf16 %5, %19, %13, %5\n\t" "v_mfma_f32_16x16x32_bf16 %6, %16, %14, %6\n\t" "v_mfma_f32_16x16x32_bf16 %6, %17, %15, %6\n\t" "v_mfma_f32_16x16x32_bf16 %7, %18, %14, %7\n\t" "v_mfma_f32_16x16x32_bf16 %7, %19, %15, %7\n\t"); MMAW(ai, 1, B1, "v_mfma_f32_16x16x32_bf16 %0, %16, %8, %0\n\t" "v_mfma_f32_16x16x32_bf16 %0, %17, %9, %0\n\t" "v_mfma_f32_16x16x32_bf16 %1, %18, %8, %1\n\t" "v_mfma_f32_16x16x32_bf16 %1, %19, %9, %1\n\t" "v_mfma_f32_16x16x32_bf16 %2, %16, %10, %2\n\t" "v_mfma_f32_16x16x32_bf16 %2, %17, %11, %2\n\t" "v_mfma_f32_16x16x32_bf16 %3, %18, %10, %3\n\t" "v_mfma_f32_16x16x32_bf16 %3, %19, %11, %3\n\t" "v_mfma_f32_16x16x32_bf16 %4, %16, %12, %4\n\t" "v_mfma_f32_16x16x32_bf16 %4, %17, %13, %4\n\t" "v_mfma_f32_16x16x32_bf16 %5, %18, %12, %5\n\t" "v_mfma_f32_16x16x32_bf16 %5, %19, %13, %5\n\t" "v_mfma_f32_16x16x32_bf16 %6, %16, %14, %6\n\t" "v_mfma_f32_16x16x32_bf16 %6, %17, %15, %6\n\t" "v_mfma_f32_16x16x32_bf16 %7, %18, %14, %7\n\t" "v_mfma_f32_16x16x32_bf16 %7, %19, %15, %7\n\t" "s_setprio 0\n\t"); } while (0)
#define ROLE_SETUP() \
  int tid_ = threadIdx.x; asm volatile("" : "+v"(tid_)); const int lane = tid_ & 63, fr = lane & 15, fq = lane >> 4; \
  unsigned aoff0, aoff1, boff0, boff1; \
  { const unsigned rb = (unsigned)((wid >> 2) * 64 + fr) * 128u; const unsigned s = (unsigned)(fr >> 1); \
    aoff0 = lbase + LDS_A + rb + (((unsigned)fq ^ s) << 4); aoff1 = lbase + LDS_A + rb + (((unsigned)(4 + fq) ^ s) << 4); \
    const unsigned q = (unsigned)(lane & 15) >> 2, p = (unsigned)lane & 3u, g = (unsigned)fq, f = q | ((g & 1u) << 2); \
    const unsigned kb = (8u * g + q) * 256u + 8u * (p & 1u); \
    boff0 = lbase + LDS_B + kb + ((((unsigned)(4 * (wid & 3)) + (p >> 1)) ^ (2u * f)) << 4); \
    boff1 = lbase + LDS_B + kb + ((((unsigned)(4 * (wid & 3) + 2) + (p >> 1)) ^ (2u * f)) << 4); } \
  f32x4 acc[2][2][4][2]; \
  _Pragma("unroll") for (int i0 = 0; i0 < 2; ++i0) _Pragma("unroll") for (int i1 = 0; i1 < 2; ++i1) \
  _Pragma("unroll") for (int m = 0; m < 4; ++m) _Pragma("unroll") for (int n = 0; n < 2; ++n) acc[i0][i1][m][n] = (f32x4){0.f, 0.f, 0.f, 0.f}; \
  bf16x8 At[4][2]; bf16x4 B0[2][2][2], B1[2][2][2]

namespace {
__device__ __forceinline__ void epilogue(unsigned lbase, const f32x4 (&acc)[2][2][4][2], char* otile, int wid) {
  WAIT_L(0);
  if (wid < 4) BAR;
  BAR;
  unsigned badv; asm volatile("ds_read_b32 %0, %1\n\ts_waitcnt lgkmcnt(0)" : "=v"(badv) : "v"(lbase + LDS_M) : "memory");
  const bool bad = badv != 0u;
  int tz = threadIdx.x; asm volatile("" : "+v"(tz));
  const int ln = tz & 63, r16 = ln & 15, q4 = ln >> 4;
  char* op = otile + (size_t)((wid >> 2) * 64) * ROWB + (size_t)((wid & 3) * 64);
  auto ro = __builtin_amdgcn_make_buffer_rsrc(op, 0, 192 * ROWB, 0x00020000);
  const unsigned vo = (unsigned)r16 * ROWB + (unsigned)((((q4 & 1) << 4) | ((q4 >> 1) << 3)) * 2);
#pragma unroll
  for (int ai = 0; ai < 2; ++ai)
#pragma unroll
    for (int m = 0; m < 4; ++m)
#pragma unroll
      for (int bj = 0; bj < 2; ++bj) {
        const f32x4 v0 = acc[ai][bj][m][0], v1 = acc[ai][bj][m][1];
        unsigned a0 = cvt_pk_bf16(v0[0], v0[1]), a1 = cvt_pk_bf16(v0[2], v0[3]), b0 = cvt_pk_bf16(v1[0], v1[1]), b1 = cvt_pk_bf16(v1[2], v1[3]);
        auto s0 = __builtin_amdgcn_permlane16_swap(a0, b0, false, false);
        auto s1 = __builtin_amdgcn_permlane16_swap(a1, b1, false, false);
        u32x4 w = {s0[0], s1[0], s0[1], s1[1]};
        if (bad) w = (u32x4){0x7fc07fc0u, 0x7fc07fc0u, 0x7fc07fc0u, 0x7fc07fc0u};
        __builtin_amdgcn_raw_buffer_store_b128(w, ro, vo + bj * 256, (ai * 128 + m * 16) * ROWB, 16);
      }
}

template <int NG, bool G0>
__device__ __forceinline__ void worker(unsigned lbase, const char* xs, const char* xg, const char* ctr, const char* pB, char* otile, bool local, int wid) {
  ROLE_SETUP();
  const int x3 = (wid - 1) * 8;
  const unsigned vA0 = local ? off_x(wid, lane) : (unsigned)(wid * 1024 + lane * 16), vB0 = off_w(wid, lane);
  const unsigned vA1 = vA0 + (local ? 64u * ROWB : 8192u), vB1 = vB0 + 32u * ROWB;
  const unsigned vA3 = local ? off_x(x3, lane) : (unsigned)(x3 * 1024 + lane * 16), vB3 = off_w(x3, lane);
  const unsigned ldsw = rfl(lbase + (unsigned)wid * 1024u), lds3 = rfl(lbase + (unsigned)x3 * 1024u);
  constexpr size_t hB = 256, kB = 64 * (size_t)ROWB;
  const size_t hA = local ? 128 * (size_t)ROWB : HT, kA = local ? 128 : XG_KT;
#define STAGE_B(bb, h, gp) do { const char* g_ = (gp) + (h) * hB; GLDS(vB0, g_, ldsw, LDS_B + ((bb) * 2 + (h)) * HT); GLDS(vB1, g_, ldsw, LDS_B + ((bb) * 2 + (h)) * HT + 8192); \
    if (NG == 3) GLDS(vB3, g_, lds3, LDS_B + ((bb) * 2 + (h)) * HT); } while (0)
#define STAGE_A(bb, h, gp) do { const char* g_ = (gp) + (h) * hA; GLDS(vA0, g_, ldsw, LDS_A + ((bb) * 2 + (h)) * HT); GLDS(vA1, g_, ldsw, LDS_A + ((bb) * 2 + (h)) * HT + 8192); \
    if (NG == 3) GLDS(vA3, g_, lds3, LDS_A + ((bb) * 2 + (h)) * HT); } while (0)
  unsigned cv; { unsigned z = 0; asm volatile("global_load_dword %0, %1, %2 sc1" : "=v"(cv) : "v"(z), "s"(ctr) : "memory"); }
  STAGE_B(0, 0, pB); STAGE_B(0, 1, pB);
  asm volatile("s_waitcnt vmcnt(%1)" : "+v"(cv) : "i"(2 * NG) : "memory");
  const char* pA = local ? xs : xg + ((rfl(cv) + 1u) & 1u) * XG_PAR;
  BAR; STAGE_A(0, 0, pA);
  BAR; STAGE_A(0, 1, pA);
  BAR; STAGE_B(1, 0, pB + kB); STAGE_A(1, 0, pA + kA); STAGE_B(1, 1, pB + kB);
  if (wid >= 4) BAR;
  WAIT_V(4 * NG); BAR;
  BAR;
  for (int t = 0; t < NT; t += 2) {
    const bool last = (t == NT - 2);
    const char* a1 = pA + (size_t)(t + 1) * kA;
    const char* a2 = pA + (size_t)(last ? t : t + 2) * kA; const char* b2 = pB + (size_t)(last ? t : t + 2) * kB;
    const char* a3 = a2 + kA; const char* b3 = b2 + kB;
#define WV_EARLY do { if (!(G0 && LATEW)) WAIT_V(4 * NG); } while (0)
#define WV_LATE do { if (G0 && LATEW) WAIT_V(4 * NG); } while (0)
#define P1(bb, ST) do { if (SFIRST) ST; LDB(B0, bb, 0); LDA(bb, 0); LDB(B1, bb, 1); if (!SFIRST) ST; WV_EARLY; WAIT_ABB(); BAR; MMA2(0); WV_LATE; BAR; } while (0)
#define P2(bb, ST) do { if (SFIRST) { ST; } LDA(bb, 1); if (!SFIRST) { ST; } WV_EARLY; WAIT_A(); BAR; MMA2(1); WV_LATE; BAR; } while (0)
    P1(0, STAGE_A(1, 1, a1));
    P2(0, STAGE_B(0, 0, b2); STAGE_A(0, 0, a2); STAGE_B(0, 1, b2));
    P1(1, STAGE_A(0, 1, a2));
    P2(1, STAGE_B(1, 0, b3); STAGE_A(1, 0, a3); STAGE_B(1, 1, b3));
  }
#undef STAGE_A
#undef STAGE_B
  epilogue(lbase, acc, otile, wid);
}
}

__global__ __launch_bounds__(512, 2) __attribute__((amdgpu_num_vgpr(108), amdgpu_num_sgpr(66))) void ag_gemm_kernel(Args a) {
  extern __shared__ __attribute__((aligned(16))) char smem[];
  LAS char* lds = (LAS char*)smem;
  asm volatile("v_mov_b32 v239, 0\n\tv_mov_b32 v236, 0" ::: "v222", "v255", "s60", "s101");
  const int wid = __builtin_amdgcn_readfirstlane(threadIdx.x >> 6);
  const int b = blockIdx.x, rank = a.rank;
  const int xcd = b & 7, idx = b >> 3;
  const int pm = 4 * (xcd >> 1) + (idx & 3), pn = 8 * (xcd & 1) + (idx >> 2);
  const int src = pm >> 1, strip = pm & 1;
  const bool local = (src == rank);
  char* myws = a.ws[rank];
  const char* ctr = myws + WS_CTR + (size_t)b * 128;

  const unsigned lbase = (unsigned)(size_t)lds;
  char* otile = a.out + (size_t)(pm * 256) * ROWB + (size_t)pn * 512;
  const char* xs = a.x + (size_t)strip * 256 * ROWB;
  const char* xg = myws + (size_t)src * XG_SRC + (size_t)strip * 2 * HT;
  const char* pB = a.w + (size_t)pn * 512;

  if (wid >= 4) worker<2, false>(lbase, xs, xg, ctr, pB, otile, local, wid);
  else if (wid == 3) worker<2, true>(lbase, xs, xg, ctr, pB, otile, local, wid);
  else if (wid >= 1) worker<3, true>(lbase, xs, xg, ctr, pB, otile, local, wid);
  else {
    const unsigned t_start = now_sync();
    ROLE_SETUP();
    const unsigned jj = (unsigned)((((((idx >> 1) ^ rank) & 1) << 4) | ((idx >> 2) << 1) | (idx & 1)) * 8 + xcd);
    const unsigned vx0 = off_x(0, lane), vx1 = off_x(1, lane);
    unsigned cv; { unsigned z = 0; asm volatile("global_load_dword %0, %1, %2 sc1" : "=v"(cv) : "v"(z), "s"(ctr) : "memory"); }
    const unsigned idA = (jj / 7u) * 8u + jj % 7u;
    { const char* sa = a.x + (size_t)(((idA << 15) & 0x3c0000u) | (idA & 0x1f80u));
      asm volatile("v_mov_b32 v230, %0\n\tv_mov_b32 v229, %1\n\tv_add_u32 v225, 0x20000, %0\n\tv_add_u32 v223, 0x20000, %1\n\t"
                   "global_load_dwordx4 v[240:243], v230, %2\n\tglobal_load_dwordx4 v[244:247], v229, %2\n\tglobal_load_dwordx4 v[248:251], v225, %2\n\tglobal_load_dwordx4 v[252:255], v223, %2"
                   :: "v"(vx0), "v"(vx1), "s"(sa) : "memory"); }
    char* pw = a.ws[(rank + 1) & 7];
#pragma unroll
    for (int q = 1; q < 7; ++q) { char* c = a.ws[(rank + 1 + q) & 7]; asm volatile("" : "+s"(c)); if (lane == q) pw = c; }
    *(LAS unsigned*)(lds + LDS_M) = 0u;
    const unsigned foff = (unsigned)(src * 65536 + strip * 256);
    const char* flags = myws + WS_FLAGS + foff;
    const unsigned g = ((unsigned)lane >> 2) & 7u, gcode = (g >> 1) * 4u + (g & 1u);
    { const unsigned pbase = gcode * 128u + ((unsigned)lane & 3u) * 32u;
      asm volatile("v_mov_b32 v231, %0\n\tv_mov_b32 v228, %1\n\tv_mov_b32 v237, %1\n\tv_mov_b32 v227, %2\n\tv_mov_b32 v226, %3"
                   :: "v"((unsigned)lane * 16u), "v"(pbase), "v"(g), "s"((unsigned)(WS_TMO - WS_FLAGS) - foff) : "memory"); }
    { const int due0 = T0 + (int)((jj * D1) >> 8);
      asm volatile("s_mov_b32 s70, 0\n\ts_mov_b32 s71, 0\n\ts_mov_b32 s72, 0x7fffffff\n\ts_mov_b32 s75, 0\n\t"
                   "s_mov_b32 s73, %0\n\ts_mov_b32 s74, %1\n\ts_mov_b32 s86, 0\n\ts_sub_u32 s66, 0, %2\n\t"
                   "s_mov_b32 s89, %3\n\ts_mov_b64 s[90:91], %4\n\ts_mov_b64 s[92:93], %5\n\ts_mov_b32 s67, 0"
                   :: "s"(NOCOMM ? NEVER : due0), "s"(idA), "s"(rfl((local || NOCOMM || NOWAIT) ? 0u : 1u)),
                      "s"(t_start), "s"(a.x), "s"(flags) : "memory", "scc"); }
    asm volatile("" : TIE_C(0, 0), TIE_C(0, 1)); asm volatile("" : TIE_C(1, 0), TIE_C(1, 1));
    asm volatile("s_waitcnt vmcnt(0)" : "+v"(cv) :: "memory");
    const unsigned call = rfl(cv) + 1u;
    { const unsigned xgo = (call & 1u) * (unsigned)XG_PAR + (unsigned)rank * (unsigned)XG_SRC; const u64 pd = (u64)(pw + xgo);
      asm volatile("v_mov_b32 v234, %0\n\tv_mov_b32 v235, %1\n\ts_mov_b32 s88, %2\n\ts_mov_b32 s68, %3\n\tv_mov_b32 v233, %4\n\tv_mov_b32 v224, %4\n\tv_mov_b32 v232, %5\n\ts_nop 1\n\t" "s_mov_b32 s87, 0\n\t" "s_mov_b32 s63, 0\n\t" "s_mov_b32 s69, 0\n\t" "s_cmp_lg_u32 s63, 0\n\t" "s_cselect_b32 s72, 0x7fffffff, s72\n\t" "s_add_i32 s98, s70, 6\n\t" "s_add_i32 s70, s70, 10\n\t" "s_cmp_lt_u32 s74, " STR(ID1) "\n\t" "s_cselect_b32 s99, 0, (" STR(D2) "-" STR(D1) ")\n\t" "s_add_i32 s99, s99, s73\n\t" "s_addk_i32 s99, " STR(D1) "\n\t" "s_cmp_lt_u32 s69, 0x2000\n\t" "s_cselect_b32 s99, s99, 0x7fffffff\n\t" "s_cmp_lg_u32 s87, 0\n\t" "s_cselect_b32 s72, s98, s72\n\t" "s_cselect_b32 s75, s74, s75\n\t" "s_cselect_b32 s71, s70, s71\n\t" "s_cselect_b32 s73, s99, s73\n\t" "s_cselect_b32 s74, s69, s74\n\t" "s_and_b32 s98, s75, 7\n\t" "s_lshl_b32 s99, s75, 2\n\t" "v_readlane_b32 s82, v234, s98\n\t" "v_readlane_b32 s83, v235, s98\n\t" "s_and_b32 s99, s99, 0x7fe0\n\t" "s_lshr_b32 s98, s75, 5\n\t" "s_add_i32 s98, s98, s88\n\t" "s_add_u32 s82, s82, s99\n\t" "s_addc_u32 s83, s83, 0\n\t" "v_mov_b32 v238, s98\n\t" "s_mov_b32 s70, 0\n\t" "s_cmp_lt_u32 s86, 128\n\t" "s_cselect_b32 s84, s68, 0\n\t" "s_and_b32 s85, s86, 7\n\t" "s_and_b32 s84, s84, s66\n\t" "s_lshl_b32 s85, s85, 2\n\t" "s_memrealtime s[96:97]\n\t" "s_and_b32 s98, s74, 7\n\t" "s_and_b32 s99, s74, 0x1ff8\n\t" "v_readlane_b32 s78, v234, s98\n\t" "v_readlane_b32 s79, v235, s98\n\t" "s_lshl_b32 s99, s99, 9\n\t" "s_cmp_ge_u32 s98, 3\n\t" "s_addc_u32 s69, s74, 292\n\t" "s_add_u32 s78, s78, s99\n\t" "s_addc_u32 s79, s79, 0\n\t" "s_lshl_b32 s98, s69, 15\n\t" "s_and_b32 s99, s69, 0x1f80\n\t" "s_and_b32 s98, s98, 0x3c0000\n\t" "s_or_b32 s98, s98, s99\n\t" "s_add_u32 s80, s90, s98\n\t" "s_addc_u32 s81, s91, 0\n\t" "s_waitcnt lgkmcnt(0)\n\t" "s_sub_i32 s62, s96, s89\n\t" "s_getreg_b32 s98, hwreg(HW_REG_IB_STS, 22, 2)\n\t" "s_getreg_b32 s99, hwreg(HW_REG_IB_STS, 0, 4)\n\t" "s_lshl4_add_u32 s98, s98, s99\n\t" "s_sub_i32 s76, s70, s98\n\t" "s_cmp_ge_i32 s76, s71\n\t" "s_cselect_b32 s77, s73, 0x7fffffff\n\t"
                   :: "v"((unsigned)pd), "v"((unsigned)(pd >> 32)), "s"(call * 256u + 1u), "s"(call * 256u + 1u + 2u * (unsigned)strip),
                      "v"(call * 256u + 1u + 2u * (unsigned)strip + gcode), "s"((unsigned)WS_FLAGS + (unsigned)rank * 65536u - xgo) : "memory", "scc"); }
    asm volatile("s_branch ag_slow_end\n\t" "ag_tr_a:\n\t" "s_call_b64 s[94:95], ag_slow_sub\n\t" "s_branch ag_bk_a\n\t" "ag_tr_b:\n\t" "s_call_b64 s[94:95], ag_slow_sub\n\t" "s_branch ag_bk_b\n\t" "ag_tr_c:\n\t" "s_call_b64 s[94:95], ag_slow_sub\n\t" "s_branch ag_bk_c\n\t" "ag_tr_d:\n\t" "s_call_b64 s[94:95], ag_slow_sub\n\t" "s_branch ag_bk_d\n\t" "ag_tr_p0:\n\t" "s_call_b64 s[94:95], ag_slow_sub\n\t" "s_branch ag_bk_p0\n\t" "ag_tr_p1:\n\t" "s_call_b64 s[94:95], ag_slow_sub\n\t" "s_branch ag_bk_p1\n\t" "ag_tr_p2:\n\t" "s_call_b64 s[94:95], ag_slow_sub\n\t" "s_branch ag_bk_p2\n\t" "ag_tr_p3:\n\t" "s_call_b64 s[94:95], ag_slow_sub\n\t" "s_branch ag_bk_p3\n\t" "ag_slow_sub:\n\t" "s_memrealtime s[96:97]\n\t" "s_and_b32 s98, s74, 7\n\t" "s_and_b32 s99, s74, 0x1ff8\n\t" "v_readlane_b32 s78, v234, s98\n\t" "v_readlane_b32 s79, v235, s98\n\t" "s_lshl_b32 s99, s99, 9\n\t" "s_cmp_ge_u32 s98, 3\n\t" "s_addc_u32 s69, s74, 292\n\t" "s_add_u32 s78, s78, s99\n\t" "s_addc_u32 s79, s79, 0\n\t" "s_lshl_b32 s98, s69, 15\n\t" "s_and_b32 s99, s69, 0x1f80\n\t" "s_and_b32 s98, s98, 0x3c0000\n\t" "s_or_b32 s98, s98, s99\n\t" "s_add_u32 s80, s90, s98\n\t" "s_addc_u32 s81, s91, 0\n\t" "s_waitcnt lgkmcnt(0)\n\t" "s_sub_i32 s62, s96, s89\n\t" "s_getreg_b32 s98, hwreg(HW_REG_IB_STS, 22, 2)\n\t" "s_getreg_b32 s99, hwreg(HW_REG_IB_STS, 0, 4)\n\t" "s_lshl4_add_u32 s98, s98, s99\n\t" "s_sub_i32 s76, s70, s98\n\t" "s_cmp_ge_i32 s76, s71\n\t" "s_cselect_b32 s77, s73, 0x7fffffff\n\t" "ag_slow_loop:\n\t" "s_add_i32 s67, s67, 1\n\t" "v_cmp_gt_u32 s[64:65], v224, v239\n\t" "s_cmp_ge_i32 s76, s72\n\t" "s_cselect_b64 exec, 1, 0\n\t" "global_store_dword v232, v238, s[82:83] sc0 sc1\n\t" "s_cselect_b32 s63, 1, 0\n\t" "s_and_b32 exec_lo, s64, s66\n\t" "global_load_dword v239, v237, s[92:93] sc0 sc1\n\t" "s_cmp_ge_i32 s62, s77\n\t" "s_cselect_b64 exec, -1, 0\n\t" "global_store_dwordx4 v231, v[240:243], s[78:79] sc0 sc1\n\t" "global_store_dwordx4 v231, v[244:247], s[78:79] offset:1024 sc0 sc1\n\t" "global_store_dwordx4 v231, v[248:251], s[78:79] offset:2048 sc0 sc1\n\t" "global_store_dwordx4 v231, v[252:255], s[78:79] offset:3072 sc0 sc1\n\t" "global_load_dwordx4 v[240:243], v230, s[80:81]\n\t" "global_load_dwordx4 v[244:247], v229, s[80:81]\n\t" "global_load_dwordx4 v[248:251], v225, s[80:81]\n\t" "global_load_dwordx4 v[252:255], v223, s[80:81]\n\t" "s_mov_b64 exec, -1\n\t" "s_cselect_b32 s87, 1, 0\n\t" "global_load_dword v236, v226, s[92:93] sc1\n\t" "s_cmp_lg_u32 s63, 0\n\t" "s_cselect_b32 s72, 0x7fffffff, s72\n\t" "s_add_i32 s98, s70, 6\n\t" "s_add_i32 s70, s70, 10\n\t" "s_cmp_lt_u32 s74, " STR(ID1) "\n\t" "s_cselect_b32 s99, 0, (" STR(D2) "-" STR(D1) ")\n\t" "s_add_i32 s99, s99, s73\n\t" "s_addk_i32 s99, " STR(D1) "\n\t" "s_cmp_lt_u32 s69, 0x2000\n\t" "s_cselect_b32 s99, s99, 0x7fffffff\n\t" "s_cmp_lg_u32 s87, 0\n\t" "s_cselect_b32 s72, s98, s72\n\t" "s_cselect_b32 s75, s74, s75\n\t" "s_cselect_b32 s71, s70, s71\n\t" "s_cselect_b32 s73, s99, s73\n\t" "s_cselect_b32 s74, s69, s74\n\t" "s_add_i32 s70, s70, 1\n\t" "s_and_b32 s98, s75, 7\n\t" "s_lshl_b32 s99, s75, 2\n\t" "v_readlane_b32 s82, v234, s98\n\t" "v_readlane_b32 s83, v235, s98\n\t" "s_and_b32 s99, s99, 0x7fe0\n\t" "s_lshr_b32 s98, s75, 5\n\t" "s_add_i32 s98, s98, s88\n\t" "s_add_u32 s82, s82, s99\n\t" "s_addc_u32 s83, s83, 0\n\t" "v_mov_b32 v238, s98\n\t" "s_sleep " STR(SLP) "\n\t" "s_memrealtime s[96:97]\n\t" "s_and_b32 s98, s74, 7\n\t" "s_and_b32 s99, s74, 0x1ff8\n\t" "v_readlane_b32 s78, v234, s98\n\t" "v_readlane_b32 s79, v235, s98\n\t" "s_lshl_b32 s99, s99, 9\n\t" "s_cmp_ge_u32 s98, 3\n\t" "s_addc_u32 s69, s74, 292\n\t" "s_add_u32 s78, s78, s99\n\t" "s_addc_u32 s79, s79, 0\n\t" "s_lshl_b32 s98, s69, 15\n\t" "s_and_b32 s99, s69, 0x1f80\n\t" "s_and_b32 s98, s98, 0x3c0000\n\t" "s_or_b32 s98, s98, s99\n\t" "s_add_u32 s80, s90, s98\n\t" "s_addc_u32 s81, s91, 0\n\t" "s_waitcnt lgkmcnt(0)\n\t" "s_sub_i32 s62, s96, s89\n\t" "s_getreg_b32 s98, hwreg(HW_REG_IB_STS, 22, 2)\n\t" "s_getreg_b32 s99, hwreg(HW_REG_IB_STS, 0, 4)\n\t" "s_lshl4_add_u32 s98, s98, s99\n\t" "s_sub_i32 s76, s70, s98\n\t" "s_cmp_ge_i32 s76, s71\n\t" "s_cselect_b32 s77, s73, 0x7fffffff\n\t" "v_readfirstlane_b32 s98, v236\n\t" "s_cmp_gt_u32 s62, " STR(DEADLINE) "\n\t" "s_cselect_b32 s98, s88, s98\n\t" "s_cmp_eq_u32 s98, s88\n\t" "s_cbranch_scc1 ag_slow_dead\n\t" "v_cmp_le_u32 s[100:101], s84, v239\n\t" "s_lshr_b32 s98, s100, s85\n\t" "s_andn2_b32 s98, 15, s98\n\t" "s_cbranch_scc1 ag_slow_loop\n\t" "s_setpc_b64 s[94:95]\n\t" "ag_slow_dead:\n\t" "s_mov_b32 s66, 0\n\t" "s_mov_b32 s84, 0\n\t" "v_mov_b32 v224, s88\n\t" "v_mov_b32 v237, " STR(LDSBAD) "\n\t" "global_store_dword v226, v224, s[92:93] sc1\n\t" "ds_write_b32 v237, v224\n\t" "s_add_i32 s70, s70, 1\n\t" "s_waitcnt lgkmcnt(0)\n\t" "s_setpc_b64 s[94:95]\n\t" "ag_slow_end:\n\t" ::: "memory", "scc");
#define ENG(TEXT) asm volatile(TEXT ::: "memory", "scc")
#define PST(x)
    PST(st_pa);
    ENG("s_cmp_lg_u32 s66, 0\n\t" "s_cbranch_scc1 ag_pre_end\n\t" "s_cmp_eq_u32 s73, 0x7fffffff\n\t" "s_cbranch_scc1 ag_pre_end\n\t" "s_mov_b32 s61, s74\n\t" "ag_pre:\n\t" "s_call_b64 s[94:95], ag_slow_sub\n\t" "s_cmp_eq_u32 s74, s61\n\t" "s_cbranch_scc1 ag_pre\n\t" "s_cmp_lg_u32 s72, 0x7fffffff\n\t" "s_cbranch_scc1 ag_pre\n\t" "ag_pre_end:\n\t");
    ENG("v_cmp_le_u32 s[100:101], s84, v239\n\t" "s_lshr_b32 s98, s100, s85\n\t" "s_andn2_b32 s98, 15, s98\n\t" "s_cbranch_scc1 ag_tr_p0\n\t" "ag_bk_p0:\n\t"); PST(st_p0); BAR; ENG("s_add_i32 s86, s86, 1\n\t" "s_add_i32 s68, s68, 1\n\t" "s_add_i32 s98, s86, 7\n\t" "v_sub_u32 v224, s98, v227\n\t" "v_lshrrev_b32 v224, 3, v224\n\t" "v_lshl_add_u32 v237, v224, 11, v228\n\t" "v_lshl_add_u32 v224, v224, 4, v233\n\t" "s_cmp_lt_u32 s86, 128\n\t" "s_cselect_b32 s84, s68, 0\n\t" "s_and_b32 s85, s86, 7\n\t" "s_and_b32 s84, s84, s66\n\t" "s_lshl_b32 s85, s85, 2\n\t");
    ENG("v_cmp_le_u32 s[100:101], s84, v239\n\t" "s_lshr_b32 s98, s100, s85\n\t" "s_andn2_b32 s98, 15, s98\n\t" "s_cbranch_scc1 ag_tr_p1\n\t" "ag_bk_p1:\n\t"); PST(st_p1); BAR; ENG("s_add_i32 s86, s86, 1\n\t" "s_add_i32 s68, s68, 3\n\t" "s_add_i32 s98, s86, 7\n\t" "v_sub_u32 v224, s98, v227\n\t" "v_lshrrev_b32 v224, 3, v224\n\t" "v_lshl_add_u32 v237, v224, 11, v228\n\t" "v_lshl_add_u32 v224, v224, 4, v233\n\t" "s_cmp_lt_u32 s86, 128\n\t" "s_cselect_b32 s84, s68, 0\n\t" "s_and_b32 s85, s86, 7\n\t" "s_and_b32 s84, s84, s66\n\t" "s_lshl_b32 s85, s85, 2\n\t");
    ENG("v_cmp_le_u32 s[100:101], s84, v239\n\t" "s_lshr_b32 s98, s100, s85\n\t" "s_andn2_b32 s98, 15, s98\n\t" "s_cbranch_scc1 ag_tr_p2\n\t" "ag_bk_p2:\n\t"); PST(st_p2); BAR; ENG("s_add_i32 s86, s86, 1\n\t" "s_add_i32 s68, s68, 1\n\t" "s_add_i32 s98, s86, 7\n\t" "v_sub_u32 v224, s98, v227\n\t" "v_lshrrev_b32 v224, 3, v224\n\t" "v_lshl_add_u32 v237, v224, 11, v228\n\t" "v_lshl_add_u32 v224, v224, 4, v233\n\t" "s_cmp_lt_u32 s86, 128\n\t" "s_cselect_b32 s84, s68, 0\n\t" "s_and_b32 s85, s86, 7\n\t" "s_and_b32 s84, s84, s66\n\t" "s_lshl_b32 s85, s85, 2\n\t");
    BAR;
    ENG("v_cmp_le_u32 s[100:101], s84, v239\n\t" "s_lshr_b32 s98, s100, s85\n\t" "s_andn2_b32 s98, 15, s98\n\t" "s_cbranch_scc1 ag_tr_p3\n\t" "ag_bk_p3:\n\t"); PST(st_p3);
    BAR;
#define W0P1(bb) do { if (APOS == 0) ENG("v_cmp_gt_u32 s[64:65], v224, v239\n\t" "s_cmp_ge_i32 s76, s72\n\t" "s_cselect_b64 exec, 1, 0\n\t" "global_store_dword v232, v238, s[82:83] sc0 sc1\n\t" "s_cselect_b32 s63, 1, 0\n\t" "s_and_b32 exec_lo, s64, s66\n\t" "global_load_dword v239, v237, s[92:93] sc0 sc1\n\t" "s_cmp_ge_i32 s62, s77\n\t" "s_cselect_b64 exec, -1, 0\n\t" "global_store_dwordx4 v231, v[240:243], s[78:79] sc0 sc1\n\t" "global_store_dwordx4 v231, v[244:247], s[78:79] offset:1024 sc0 sc1\n\t" "global_store_dwordx4 v231, v[248:251], s[78:79] offset:2048 sc0 sc1\n\t" "global_store_dwordx4 v231, v[252:255], s[78:79] offset:3072 sc0 sc1\n\t" "global_load_dwordx4 v[240:243], v230, s[80:81]\n\t" "global_load_dwordx4 v[244:247], v229, s[80:81]\n\t" "global_load_dwordx4 v[248:251], v225, s[80:81]\n\t" "global_load_dwordx4 v[252:255], v223, s[80:81]\n\t" "s_mov_b64 exec, -1\n\t" "s_cselect_b32 s87, 1, 0\n\t"); LDB(B0, bb, 0); if (APOS == 1) ENG("v_cmp_gt_u32 s[64:65], v224, v239\n\t" "s_cmp_ge_i32 s76, s72\n\t" "s_cselect_b64 exec, 1, 0\n\t" "global_store_dword v232, v238, s[82:83] sc0 sc1\n\t" "s_cselect_b32 s63, 1, 0\n\t" "s_and_b32 exec_lo, s64, s66\n\t" "global_load_dword v239, v237, s[92:93] sc0 sc1\n\t" "s_cmp_ge_i32 s62, s77\n\t" "s_cselect_b64 exec, -1, 0\n\t" "global_store_dwordx4 v231, v[240:243], s[78:79] sc0 sc1\n\t" "global_store_dwordx4 v231, v[244:247], s[78:79] offset:1024 sc0 sc1\n\t" "global_store_dwordx4 v231, v[248:251], s[78:79] offset:2048 sc0 sc1\n\t" "global_store_dwordx4 v231, v[252:255], s[78:79] offset:3072 sc0 sc1\n\t" "global_load_dwordx4 v[240:243], v230, s[80:81]\n\t" "global_load_dwordx4 v[244:247], v229, s[80:81]\n\t" "global_load_dwordx4 v[248:251], v225, s[80:81]\n\t" "global_load_dwordx4 v[252:255], v223, s[80:81]\n\t" "s_mov_b64 exec, -1\n\t" "s_cselect_b32 s87, 1, 0\n\t"); LDA(bb, 0); if (APOS == 2) ENG("v_cmp_gt_u32 s[64:65], v224, v239\n\t" "s_cmp_ge_i32 s76, s72\n\t" "s_cselect_b64 exec, 1, 0\n\t" "global_store_dword v232, v238, s[82:83] sc0 sc1\n\t" "s_cselect_b32 s63, 1, 0\n\t" "s_and_b32 exec_lo, s64, s66\n\t" "global_load_dword v239, v237, s[92:93] sc0 sc1\n\t" "s_cmp_ge_i32 s62, s77\n\t" "s_cselect_b64 exec, -1, 0\n\t" "global_store_dwordx4 v231, v[240:243], s[78:79] sc0 sc1\n\t" "global_store_dwordx4 v231, v[244:247], s[78:79] offset:1024 sc0 sc1\n\t" "global_store_dwordx4 v231, v[248:251], s[78:79] offset:2048 sc0 sc1\n\t" "global_store_dwordx4 v231, v[252:255], s[78:79] offset:3072 sc0 sc1\n\t" "global_load_dwordx4 v[240:243], v230, s[80:81]\n\t" "global_load_dwordx4 v[244:247], v229, s[80:81]\n\t" "global_load_dwordx4 v[248:251], v225, s[80:81]\n\t" "global_load_dwordx4 v[252:255], v223, s[80:81]\n\t" "s_mov_b64 exec, -1\n\t" "s_cselect_b32 s87, 1, 0\n\t"); LDB(B1, bb, 1); if (APOS == 3) ENG("v_cmp_gt_u32 s[64:65], v224, v239\n\t" "s_cmp_ge_i32 s76, s72\n\t" "s_cselect_b64 exec, 1, 0\n\t" "global_store_dword v232, v238, s[82:83] sc0 sc1\n\t" "s_cselect_b32 s63, 1, 0\n\t" "s_and_b32 exec_lo, s64, s66\n\t" "global_load_dword v239, v237, s[92:93] sc0 sc1\n\t" "s_cmp_ge_i32 s62, s77\n\t" "s_cselect_b64 exec, -1, 0\n\t" "global_store_dwordx4 v231, v[240:243], s[78:79] sc0 sc1\n\t" "global_store_dwordx4 v231, v[244:247], s[78:79] offset:1024 sc0 sc1\n\t" "global_store_dwordx4 v231, v[248:251], s[78:79] offset:2048 sc0 sc1\n\t" "global_store_dwordx4 v231, v[252:255], s[78:79] offset:3072 sc0 sc1\n\t" "global_load_dwordx4 v[240:243], v230, s[80:81]\n\t" "global_load_dwordx4 v[244:247], v229, s[80:81]\n\t" "global_load_dwordx4 v[248:251], v225, s[80:81]\n\t" "global_load_dwordx4 v[252:255], v223, s[80:81]\n\t" "s_mov_b64 exec, -1\n\t" "s_cselect_b32 s87, 1, 0\n\t"); } while (0)
#pragma unroll 1
    for (int t = 0; t < NT; t += 2) {
      W0P1(0);
      WAIT_ABB(); BAR; MMAW(0, 0, B0, "s_setprio 1\n\t" "v_mfma_f32_16x16x32_bf16 %0, %16, %8, %0\n\t" "s_cmp_lg_u32 s63, 0\n\t" "s_cselect_b32 s72, 0x7fffffff, s72\n\t" "v_mfma_f32_16x16x32_bf16 %0, %17, %9, %0\n\t" "s_add_i32 s98, s70, 6\n\t" "v_mfma_f32_16x16x32_bf16 %1, %18, %8, %1\n\t" "s_add_i32 s70, s70, 10\n\t" "s_cmp_lt_u32 s74, " STR(ID1) "\n\t" "v_mfma_f32_16x16x32_bf16 %1, %19, %9, %1\n\t" "s_cselect_b32 s99, 0, (" STR(D2) "-" STR(D1) ")\n\t" "v_mfma_f32_16x16x32_bf16 %2, %16, %10, %2\n\t" "s_add_i32 s99, s99, s73\n\t" "s_addk_i32 s99, " STR(D1) "\n\t" "v_mfma_f32_16x16x32_bf16 %2, %17, %11, %2\n\t" "s_cmp_lt_u32 s69, 0x2000\n\t" "v_mfma_f32_16x16x32_bf16 %3, %18, %10, %3\n\t" "s_cselect_b32 s99, s99, 0x7fffffff\n\t" "s_cmp_lg_u32 s87, 0\n\t" "v_mfma_f32_16x16x32_bf16 %3, %19, %11, %3\n\t" "s_cselect_b32 s72, s98, s72\n\t" "v_mfma_f32_16x16x32_bf16 %4, %16, %12, %4\n\t" "s_cselect_b32 s75, s74, s75\n\t" "v_mfma_f32_16x16x32_bf16 %4, %17, %13, %4\n\t" "s_cselect_b32 s71, s70, s71\n\t" "s_cselect_b32 s73, s99, s73\n\t" "v_mfma_f32_16x16x32_bf16 %5, %18, %12, %5\n\t" "s_cselect_b32 s74, s69, s74\n\t" "v_mfma_f32_16x16x32_bf16 %5, %19, %13, %5\n\t" "s_add_i32 s86, s86, 1\n\t" "s_add_i32 s68, s68, 3\n\t" "v_mfma_f32_16x16x32_bf16 %6, %16, %14, %6\n\t" "s_add_i32 s98, s86, 7\n\t" "v_mfma_f32_16x16x32_bf16 %6, %17, %15, %6\n\t" "v_sub_u32 v224, s98, v227\n\t" "v_lshrrev_b32 v224, 3, v224\n\t" "v_mfma_f32_16x16x32_bf16 %7, %18, %14, %7\n\t" "v_lshl_add_u32 v237, v224, 11, v228\n\t" "v_mfma_f32_16x16x32_bf16 %7, %19, %15, %7\n\t" "v_lshl_add_u32 v224, v224, 4, v233\n\t"); MMAW(0, 1, B1, "v_mfma_f32_16x16x32_bf16 %0, %16, %8, %0\n\t" "s_cmp_lt_u32 s86, 128\n\t" "s_cselect_b32 s84, s68, 0\n\t" "v_mfma_f32_16x16x32_bf16 %0, %17, %9, %0\n\t" "s_and_b32 s85, s86, 7\n\t" "v_mfma_f32_16x16x32_bf16 %1, %18, %8, %1\n\t" "s_and_b32 s84, s84, s66\n\t" "s_lshl_b32 s85, s85, 2\n\t" "v_mfma_f32_16x16x32_bf16 %1, %19, %9, %1\n\t" "s_and_b32 s98, s75, 7\n\t" "v_mfma_f32_16x16x32_bf16 %2, %16, %10, %2\n\t" "s_lshl_b32 s99, s75, 2\n\t" "v_readlane_b32 s82, v234, s98\n\t" "v_mfma_f32_16x16x32_bf16 %2, %17, %11, %2\n\t" "v_readlane_b32 s83, v235, s98\n\t" "v_mfma_f32_16x16x32_bf16 %3, %18, %10, %3\n\t" "s_and_b32 s99, s99, 0x7fe0\n\t" "s_lshr_b32 s98, s75, 5\n\t" "v_mfma_f32_16x16x32_bf16 %3, %19, %11, %3\n\t" "s_add_i32 s98, s98, s88\n\t" "v_mfma_f32_16x16x32_bf16 %4, %16, %12, %4\n\t" "s_add_u32 s82, s82, s99\n\t" "s_addc_u32 s83, s83, 0\n\t" "v_mfma_f32_16x16x32_bf16 %4, %17, %13, %4\n\t" "v_mov_b32 v238, s98\n\t" "v_mfma_f32_16x16x32_bf16 %5, %18, %12, %5\n\t" "s_getreg_b32 s98, hwreg(HW_REG_IB_STS, 22, 2)\n\t" "s_getreg_b32 s99, hwreg(HW_REG_IB_STS, 0, 4)\n\t" "v_mfma_f32_16x16x32_bf16 %5, %19, %13, %5\n\t" "s_lshl4_add_u32 s98, s98, s99\n\t" "v_mfma_f32_16x16x32_bf16 %6, %16, %14, %6\n\t" "s_sub_i32 s76, s70, s98\n\t" "v_cmp_le_u32 s[100:101], s84, v239\n\t" "v_mfma_f32_16x16x32_bf16 %6, %17, %15, %6\n\t" "s_lshr_b32 s98, s100, s85\n\t" "v_mfma_f32_16x16x32_bf16 %7, %18, %14, %7\n\t" "s_andn2_b32 s98, 15, s98\n\t" "v_mfma_f32_16x16x32_bf16 %7, %19, %15, %7\n\t" "s_setprio 0\n\t" "s_cbranch_scc1 ag_tr_a\n\t" "ag_bk_a:\n\t"); BAR;
      LDA(0, 1); ENG("v_cmp_gt_u32 s[64:65], v224, v239\n\t" "s_cmp_ge_i32 s76, s72\n\t" "s_cselect_b64 exec, 1, 0\n\t" "global_store_dword v232, v238, s[82:83] sc0 sc1\n\t" "s_cselect_b32 s63, 1, 0\n\t" "s_and_b32 exec_lo, s64, s66\n\t" "global_load_dword v239, v237, s[92:93] sc0 sc1\n\t" "s_mov_b64 exec, -1\n\t");
      WAIT_A(); BAR; MMAW(1, 0, B0, "s_setprio 1\n\t" "v_mfma_f32_16x16x32_bf16 %0, %16, %8, %0\n\t" "s_cmp_lg_u32 s63, 0\n\t" "s_cselect_b32 s72, 0x7fffffff, s72\n\t" "v_mfma_f32_16x16x32_bf16 %0, %17, %9, %0\n\t" "s_add_i32 s70, s70, 2\n\t" "v_mfma_f32_16x16x32_bf16 %1, %18, %8, %1\n\t" "s_add_i32 s86, s86, 1\n\t" "v_mfma_f32_16x16x32_bf16 %1, %19, %9, %1\n\t" "s_add_i32 s68, s68, 1\n\t" "s_add_i32 s98, s86, 7\n\t" "v_mfma_f32_16x16x32_bf16 %2, %16, %10, %2\n\t" "v_sub_u32 v224, s98, v227\n\t" "v_mfma_f32_16x16x32_bf16 %2, %17, %11, %2\n\t" "v_lshrrev_b32 v224, 3, v224\n\t" "v_mfma_f32_16x16x32_bf16 %3, %18, %10, %3\n\t" "v_lshl_add_u32 v237, v224, 11, v228\n\t" "v_lshl_add_u32 v224, v224, 4, v233\n\t" "v_mfma_f32_16x16x32_bf16 %3, %19, %11, %3\n\t" "s_cmp_lt_u32 s86, 128\n\t" "v_mfma_f32_16x16x32_bf16 %4, %16, %12, %4\n\t" "s_cselect_b32 s84, s68, 0\n\t" "v_mfma_f32_16x16x32_bf16 %4, %17, %13, %4\n\t" "s_and_b32 s85, s86, 7\n\t" "s_and_b32 s84, s84, s66\n\t" "v_mfma_f32_16x16x32_bf16 %5, %18, %12, %5\n\t" "s_lshl_b32 s85, s85, 2\n\t" "v_mfma_f32_16x16x32_bf16 %5, %19, %13, %5\n\t" "s_and_b32 s98, s74, 7\n\t" "v_mfma_f32_16x16x32_bf16 %6, %16, %14, %6\n\t" "s_and_b32 s99, s74, 0x1ff8\n\t" "v_readlane_b32 s78, v234, s98\n\t" "v_mfma_f32_16x16x32_bf16 %6, %17, %15, %6\n\t" "v_readlane_b32 s79, v235, s98\n\t" "v_mfma_f32_16x16x32_bf16 %7, %18, %14, %7\n\t" "s_lshl_b32 s99, s99, 9\n\t" "v_mfma_f32_16x16x32_bf16 %7, %19, %15, %7\n\t" "s_memrealtime s[96:97]\n\t"); MMAW(1, 1, B1, "v_mfma_f32_16x16x32_bf16 %0, %16, %8, %0\n\t" "s_cmp_ge_u32 s98, 3\n\t" "s_addc_u32 s69, s74, 292\n\t" "v_mfma_f32_16x16x32_bf16 %0, %17, %9, %0\n\t" "s_add_u32 s78, s78, s99\n\t" "v_mfma_f32_16x16x32_bf16 %1, %18, %8, %1\n\t" "s_addc_u32 s79, s79, 0\n\t" "s_lshl_b32 s98, s69, 15\n\t" "v_mfma_f32_16x16x32_bf16 %1, %19, %9, %1\n\t" "s_and_b32 s99, s69, 0x1f80\n\t" "v_mfma_f32_16x16x32_bf16 %2, %16, %10, %2\n\t" "s_and_b32 s98, s98, 0x3c0000\n\t" "s_or_b32 s98, s98, s99\n\t" "v_mfma_f32_16x16x32_bf16 %2, %17, %11, %2\n\t" "s_add_u32 s80, s90, s98\n\t" "v_mfma_f32_16x16x32_bf16 %3, %18, %10, %3\n\t" "s_addc_u32 s81, s91, 0\n\t" "s_waitcnt lgkmcnt(0)\n\t" "v_mfma_f32_16x16x32_bf16 %3, %19, %11, %3\n\t" "s_sub_i32 s62, s96, s89\n\t" "v_mfma_f32_16x16x32_bf16 %4, %16, %12, %4\n\t" "s_add_i32 s62, s62, " STR(AHEAD) "\n\t" "s_getreg_b32 s98, hwreg(HW_REG_IB_STS, 22, 2)\n\t" "v_mfma_f32_16x16x32_bf16 %4, %17, %13, %4\n\t" "s_getreg_b32 s99, hwreg(HW_REG_IB_STS, 0, 4)\n\t" "v_mfma_f32_16x16x32_bf16 %5, %18, %12, %5\n\t" "s_lshl4_add_u32 s98, s98, s99\n\t" "s_sub_i32 s76, s70, s98\n\t" "v_mfma_f32_16x16x32_bf16 %5, %19, %13, %5\n\t" "s_cmp_ge_i32 s76, s71\n\t" "v_mfma_f32_16x16x32_bf16 %6, %16, %14, %6\n\t" "s_cselect_b32 s77, s73, 0x7fffffff\n\t" "v_cmp_le_u32 s[100:101], s84, v239\n\t" "v_mfma_f32_16x16x32_bf16 %6, %17, %15, %6\n\t" "s_lshr_b32 s98, s100, s85\n\t" "v_mfma_f32_16x16x32_bf16 %7, %18, %14, %7\n\t" "s_andn2_b32 s98, 15, s98\n\t" "v_mfma_f32_16x16x32_bf16 %7, %19, %15, %7\n\t" "s_setprio 0\n\t" "s_cbranch_scc1 ag_tr_b\n\t" "ag_bk_b:\n\t"); BAR;
      W0P1(1);
      WAIT_ABB(); BAR; MMAW(0, 0, B0, "s_setprio 1\n\t" "v_mfma_f32_16x16x32_bf16 %0, %16, %8, %0\n\t" "s_cmp_lg_u32 s63, 0\n\t" "s_cselect_b32 s72, 0x7fffffff, s72\n\t" "v_mfma_f32_16x16x32_bf16 %0, %17, %9, %0\n\t" "s_add_i32 s98, s70, 6\n\t" "v_mfma_f32_16x16x32_bf16 %1, %18, %8, %1\n\t" "s_add_i32 s70, s70, 10\n\t" "s_cmp_lt_u32 s74, " STR(ID1) "\n\t" "v_mfma_f32_16x16x32_bf16 %1, %19, %9, %1\n\t" "s_cselect_b32 s99, 0, (" STR(D2) "-" STR(D1) ")\n\t" "v_mfma_f32_16x16x32_bf16 %2, %16, %10, %2\n\t" "s_add_i32 s99, s99, s73\n\t" "s_addk_i32 s99, " STR(D1) "\n\t" "v_mfma_f32_16x16x32_bf16 %2, %17, %11, %2\n\t" "s_cmp_lt_u32 s69, 0x2000\n\t" "v_mfma_f32_16x16x32_bf16 %3, %18, %10, %3\n\t" "s_cselect_b32 s99, s99, 0x7fffffff\n\t" "s_cmp_lg_u32 s87, 0\n\t" "v_mfma_f32_16x16x32_bf16 %3, %19, %11, %3\n\t" "s_cselect_b32 s72, s98, s72\n\t" "v_mfma_f32_16x16x32_bf16 %4, %16, %12, %4\n\t" "s_cselect_b32 s75, s74, s75\n\t" "v_mfma_f32_16x16x32_bf16 %4, %17, %13, %4\n\t" "s_cselect_b32 s71, s70, s71\n\t" "s_cselect_b32 s73, s99, s73\n\t" "v_mfma_f32_16x16x32_bf16 %5, %18, %12, %5\n\t" "s_cselect_b32 s74, s69, s74\n\t" "v_mfma_f32_16x16x32_bf16 %5, %19, %13, %5\n\t" "s_add_i32 s86, s86, 1\n\t" "s_add_i32 s68, s68, 3\n\t" "v_mfma_f32_16x16x32_bf16 %6, %16, %14, %6\n\t" "s_add_i32 s98, s86, 7\n\t" "v_mfma_f32_16x16x32_bf16 %6, %17, %15, %6\n\t" "v_sub_u32 v224, s98, v227\n\t" "v_lshrrev_b32 v224, 3, v224\n\t" "v_mfma_f32_16x16x32_bf16 %7, %18, %14, %7\n\t" "v_lshl_add_u32 v237, v224, 11, v228\n\t" "v_mfma_f32_16x16x32_bf16 %7, %19, %15, %7\n\t" "v_lshl_add_u32 v224, v224, 4, v233\n\t"); MMAW(0, 1, B1, "v_mfma_f32_16x16x32_bf16 %0, %16, %8, %0\n\t" "s_cmp_lt_u32 s86, 128\n\t" "s_cselect_b32 s84, s68, 0\n\t" "v_mfma_f32_16x16x32_bf16 %0, %17, %9, %0\n\t" "s_and_b32 s85, s86, 7\n\t" "v_mfma_f32_16x16x32_bf16 %1, %18, %8, %1\n\t" "s_and_b32 s84, s84, s66\n\t" "s_lshl_b32 s85, s85, 2\n\t" "v_mfma_f32_16x16x32_bf16 %1, %19, %9, %1\n\t" "s_and_b32 s98, s75, 7\n\t" "v_mfma_f32_16x16x32_bf16 %2, %16, %10, %2\n\t" "s_lshl_b32 s99, s75, 2\n\t" "v_readlane_b32 s82, v234, s98\n\t" "v_mfma_f32_16x16x32_bf16 %2, %17, %11, %2\n\t" "v_readlane_b32 s83, v235, s98\n\t" "v_mfma_f32_16x16x32_bf16 %3, %18, %10, %3\n\t" "s_and_b32 s99, s99, 0x7fe0\n\t" "s_lshr_b32 s98, s75, 5\n\t" "v_mfma_f32_16x16x32_bf16 %3, %19, %11, %3\n\t" "s_add_i32 s98, s98, s88\n\t" "v_mfma_f32_16x16x32_bf16 %4, %16, %12, %4\n\t" "s_add_u32 s82, s82, s99\n\t" "s_addc_u32 s83, s83, 0\n\t" "v_mfma_f32_16x16x32_bf16 %4, %17, %13, %4\n\t" "v_mov_b32 v238, s98\n\t" "v_mfma_f32_16x16x32_bf16 %5, %18, %12, %5\n\t" "s_getreg_b32 s98, hwreg(HW_REG_IB_STS, 22, 2)\n\t" "s_getreg_b32 s99, hwreg(HW_REG_IB_STS, 0, 4)\n\t" "v_mfma_f32_16x16x32_bf16 %5, %19, %13, %5\n\t" "s_lshl4_add_u32 s98, s98, s99\n\t" "v_mfma_f32_16x16x32_bf16 %6, %16, %14, %6\n\t" "s_sub_i32 s76, s70, s98\n\t" "v_cmp_le_u32 s[100:101], s84, v239\n\t" "v_mfma_f32_16x16x32_bf16 %6, %17, %15, %6\n\t" "s_lshr_b32 s98, s100, s85\n\t" "v_mfma_f32_16x16x32_bf16 %7, %18, %14, %7\n\t" "s_andn2_b32 s98, 15, s98\n\t" "v_mfma_f32_16x16x32_bf16 %7, %19, %15, %7\n\t" "s_setprio 0\n\t" "s_cbranch_scc1 ag_tr_c\n\t" "ag_bk_c:\n\t"); BAR;
      LDA(1, 1); ENG("v_cmp_gt_u32 s[64:65], v224, v239\n\t" "s_cmp_ge_i32 s76, s72\n\t" "s_cselect_b64 exec, 1, 0\n\t" "global_store_dword v232, v238, s[82:83] sc0 sc1\n\t" "s_cselect_b32 s63, 1, 0\n\t" "s_and_b32 exec_lo, s64, s66\n\t" "global_load_dword v239, v237, s[92:93] sc0 sc1\n\t" "s_mov_b64 exec, -1\n\t");
      WAIT_A(); BAR; MMAW(1, 0, B0, "s_setprio 1\n\t" "v_mfma_f32_16x16x32_bf16 %0, %16, %8, %0\n\t" "s_cmp_lg_u32 s63, 0\n\t" "s_cselect_b32 s72, 0x7fffffff, s72\n\t" "v_mfma_f32_16x16x32_bf16 %0, %17, %9, %0\n\t" "s_add_i32 s70, s70, 2\n\t" "v_mfma_f32_16x16x32_bf16 %1, %18, %8, %1\n\t" "s_add_i32 s86, s86, 1\n\t" "v_mfma_f32_16x16x32_bf16 %1, %19, %9, %1\n\t" "s_add_i32 s68, s68, 1\n\t" "s_add_i32 s98, s86, 7\n\t" "v_mfma_f32_16x16x32_bf16 %2, %16, %10, %2\n\t" "v_sub_u32 v224, s98, v227\n\t" "v_mfma_f32_16x16x32_bf16 %2, %17, %11, %2\n\t" "v_lshrrev_b32 v224, 3, v224\n\t" "v_mfma_f32_16x16x32_bf16 %3, %18, %10, %3\n\t" "v_lshl_add_u32 v237, v224, 11, v228\n\t" "v_lshl_add_u32 v224, v224, 4, v233\n\t" "v_mfma_f32_16x16x32_bf16 %3, %19, %11, %3\n\t" "s_cmp_lt_u32 s86, 128\n\t" "v_mfma_f32_16x16x32_bf16 %4, %16, %12, %4\n\t" "s_cselect_b32 s84, s68, 0\n\t" "v_mfma_f32_16x16x32_bf16 %4, %17, %13, %4\n\t" "s_and_b32 s85, s86, 7\n\t" "s_and_b32 s84, s84, s66\n\t" "v_mfma_f32_16x16x32_bf16 %5, %18, %12, %5\n\t" "s_lshl_b32 s85, s85, 2\n\t" "v_mfma_f32_16x16x32_bf16 %5, %19, %13, %5\n\t" "s_and_b32 s98, s74, 7\n\t" "v_mfma_f32_16x16x32_bf16 %6, %16, %14, %6\n\t" "s_and_b32 s99, s74, 0x1ff8\n\t" "v_readlane_b32 s78, v234, s98\n\t" "v_mfma_f32_16x16x32_bf16 %6, %17, %15, %6\n\t" "v_readlane_b32 s79, v235, s98\n\t" "v_mfma_f32_16x16x32_bf16 %7, %18, %14, %7\n\t" "s_lshl_b32 s99, s99, 9\n\t" "v_mfma_f32_16x16x32_bf16 %7, %19, %15, %7\n\t" "s_memrealtime s[96:97]\n\t"); MMAW(1, 1, B1, "v_mfma_f32_16x16x32_bf16 %0, %16, %8, %0\n\t" "s_cmp_ge_u32 s98, 3\n\t" "s_addc_u32 s69, s74, 292\n\t" "v_mfma_f32_16x16x32_bf16 %0, %17, %9, %0\n\t" "s_add_u32 s78, s78, s99\n\t" "v_mfma_f32_16x16x32_bf16 %1, %18, %8, %1\n\t" "s_addc_u32 s79, s79, 0\n\t" "s_lshl_b32 s98, s69, 15\n\t" "v_mfma_f32_16x16x32_bf16 %1, %19, %9, %1\n\t" "s_and_b32 s99, s69, 0x1f80\n\t" "v_mfma_f32_16x16x32_bf16 %2, %16, %10, %2\n\t" "s_and_b32 s98, s98, 0x3c0000\n\t" "s_or_b32 s98, s98, s99\n\t" "v_mfma_f32_16x16x32_bf16 %2, %17, %11, %2\n\t" "s_add_u32 s80, s90, s98\n\t" "v_mfma_f32_16x16x32_bf16 %3, %18, %10, %3\n\t" "s_addc_u32 s81, s91, 0\n\t" "s_waitcnt lgkmcnt(0)\n\t" "v_mfma_f32_16x16x32_bf16 %3, %19, %11, %3\n\t" "s_sub_i32 s62, s96, s89\n\t" "v_mfma_f32_16x16x32_bf16 %4, %16, %12, %4\n\t" "s_add_i32 s62, s62, " STR(AHEAD) "\n\t" "s_getreg_b32 s98, hwreg(HW_REG_IB_STS, 22, 2)\n\t" "v_mfma_f32_16x16x32_bf16 %4, %17, %13, %4\n\t" "s_getreg_b32 s99, hwreg(HW_REG_IB_STS, 0, 4)\n\t" "v_mfma_f32_16x16x32_bf16 %5, %18, %12, %5\n\t" "s_lshl4_add_u32 s98, s98, s99\n\t" "s_sub_i32 s76, s70, s98\n\t" "v_mfma_f32_16x16x32_bf16 %5, %19, %13, %5\n\t" "s_cmp_ge_i32 s76, s71\n\t" "v_mfma_f32_16x16x32_bf16 %6, %16, %14, %6\n\t" "s_cselect_b32 s77, s73, 0x7fffffff\n\t" "v_cmp_le_u32 s[100:101], s84, v239\n\t" "v_mfma_f32_16x16x32_bf16 %6, %17, %15, %6\n\t" "s_lshr_b32 s98, s100, s85\n\t" "v_mfma_f32_16x16x32_bf16 %7, %18, %14, %7\n\t" "s_andn2_b32 s98, 15, s98\n\t" "v_mfma_f32_16x16x32_bf16 %7, %19, %15, %7\n\t" "s_setprio 0\n\t" "s_cbranch_scc1 ag_tr_d\n\t" "ag_bk_d:\n\t"); BAR;
    }
    asm volatile("s_mov_b32 s66, 0\n\t" "s_branch ag_fin_test\n\t" "ag_fin:\n\t" "s_and_b32 s98, s74, 7\n\t" "s_and_b32 s99, s74, 0x1ff8\n\t" "v_readlane_b32 s78, v234, s98\n\t" "v_readlane_b32 s79, v235, s98\n\t" "s_lshl_b32 s99, s99, 9\n\t" "s_cmp_ge_u32 s98, 3\n\t" "s_addc_u32 s69, s74, 292\n\t" "s_add_u32 s78, s78, s99\n\t" "s_addc_u32 s79, s79, 0\n\t" "s_lshl_b32 s98, s69, 15\n\t" "s_and_b32 s99, s69, 0x1f80\n\t" "s_and_b32 s98, s98, 0x3c0000\n\t" "s_or_b32 s98, s98, s99\n\t" "s_add_u32 s80, s90, s98\n\t" "s_addc_u32 s81, s91, 0\n\t" "s_mov_b32 s62, 0x7ffffffe\n\t" "s_waitcnt vmcnt(0)\n\t" "s_mov_b32 s76, s70\n\t" "s_cmp_ge_i32 s76, s71\n\t" "s_cselect_b32 s77, s73, 0x7fffffff\n\t" "v_cmp_gt_u32 s[64:65], v224, v239\n\t" "s_cmp_ge_i32 s76, s72\n\t" "s_cselect_b64 exec, 1, 0\n\t" "global_store_dword v232, v238, s[82:83] sc0 sc1\n\t" "s_cselect_b32 s63, 1, 0\n\t" "s_and_b32 exec_lo, s64, s66\n\t" "global_load_dword v239, v237, s[92:93] sc0 sc1\n\t" "s_cmp_ge_i32 s62, s77\n\t" "s_cselect_b64 exec, -1, 0\n\t" "global_store_dwordx4 v231, v[240:243], s[78:79] sc0 sc1\n\t" "global_store_dwordx4 v231, v[244:247], s[78:79] offset:1024 sc0 sc1\n\t" "global_store_dwordx4 v231, v[248:251], s[78:79] offset:2048 sc0 sc1\n\t" "global_store_dwordx4 v231, v[252:255], s[78:79] offset:3072 sc0 sc1\n\t" "global_load_dwordx4 v[240:243], v230, s[80:81]\n\t" "global_load_dwordx4 v[244:247], v229, s[80:81]\n\t" "global_load_dwordx4 v[248:251], v225, s[80:81]\n\t" "global_load_dwordx4 v[252:255], v223, s[80:81]\n\t" "s_mov_b64 exec, -1\n\t" "s_cselect_b32 s87, 1, 0\n\t" "s_cmp_lg_u32 s63, 0\n\t" "s_cselect_b32 s72, 0x7fffffff, s72\n\t" "s_add_i32 s98, s70, 6\n\t" "s_add_i32 s70, s70, 10\n\t" "s_cmp_lt_u32 s74, " STR(ID1) "\n\t" "s_cselect_b32 s99, 0, (" STR(D2) "-" STR(D1) ")\n\t" "s_add_i32 s99, s99, s73\n\t" "s_addk_i32 s99, " STR(D1) "\n\t" "s_cmp_lt_u32 s69, 0x2000\n\t" "s_cselect_b32 s99, s99, 0x7fffffff\n\t" "s_cmp_lg_u32 s87, 0\n\t" "s_cselect_b32 s72, s98, s72\n\t" "s_cselect_b32 s75, s74, s75\n\t" "s_cselect_b32 s71, s70, s71\n\t" "s_cselect_b32 s73, s99, s73\n\t" "s_cselect_b32 s74, s69, s74\n\t" "s_and_b32 s98, s75, 7\n\t" "s_lshl_b32 s99, s75, 2\n\t" "v_readlane_b32 s82, v234, s98\n\t" "v_readlane_b32 s83, v235, s98\n\t" "s_and_b32 s99, s99, 0x7fe0\n\t" "s_lshr_b32 s98, s75, 5\n\t" "s_add_i32 s98, s98, s88\n\t" "s_add_u32 s82, s82, s99\n\t" "s_addc_u32 s83, s83, 0\n\t" "v_mov_b32 v238, s98\n\t" "ag_fin_test:\n\t" "s_and_b32 s98, s72, s73\n\t" "s_cmp_lg_u32 s98, 0x7fffffff\n\t" "s_cbranch_scc1 ag_fin\n\t" ::: "memory", "scc");
    if (lane == 0) __hip_atomic_store((unsigned*)ctr, call, __ATOMIC_RELAXED, __HIP_MEMORY_SCOPE_AGENT);
    epilogue(lbase, acc, otile, wid);
  }
}

extern "C" void kernel_launch(int rank, int nranks, ncclComm_t comm, hipStream_t stream,
                              void* const* d_in, const int* in_sizes, int n_in,
                              void* d_out, int out_size, const int* in_dtypes, int out_dtype,
                              void* const* d_out_all, void* const* d_ws_all, size_t ws_bytes) {
  if (nranks != NR || (unsigned)rank >= (unsigned)NR || ws_bytes < WS_NEED || n_in < 2 ||
      in_sizes[0] != 512 * 4096 || in_sizes[1] != 4096 * 4096 || out_size != 4096 * 4096) {
    fprintf(stderr, "kernel_launch: unexpected shapes\n"); return;
  }
  static bool init = false;
  if (!init) {
    if (hipFuncSetAttribute((const void*)ag_gemm_kernel, hipFuncAttributeMaxDynamicSharedMemorySize, LDS_BYTES) != hipSuccess) {
      fprintf(stderr, "hipFuncSetAttribute failed\n"); return; }
    init = true;
  }
  Args a;
  a.x = (const char*)d_in[0]; a.w = (const char*)d_in[1]; a.out = (char*)d_out; a.rank = rank;
  for (int p = 0; p < NR; ++p) a.ws[p] = (char*)d_ws_all[p];
  ag_gemm_kernel<<<256, 512, LDS_BYTES, stream>>>(a);
}
```

```cpp
#include <hip/hip_runtime.h>
#include <rccl/rccl.h>
#include <cstdio>

#define D1 280
#define D2 294
#define ID1 1280
#define T0 90
#define NOCOMM 0
#define NOWAIT 0
#define SLP 3
#define APOS 1
#define SFIRST 0
#define WF 0xfff
#define WS 0xff
#define STAMP 0
#define DEADLINE 100000000
#define LDSBAD 131072
#define STR_(x) #x
#define STR(x) STR_(x)

namespace {
typedef short bf16x8 __attribute__((ext_vector_type(8)));
typedef short bf16x4 __attribute__((ext_vector_type(4)));
typedef float f32x4 __attribute__((ext_vector_type(4)));
typedef unsigned u32x4 __attribute__((ext_vector_type(4)));
typedef unsigned long long u64;
#define LAS __attribute__((address_space(3)))

constexpr int NR = 8;
constexpr int NT = 64;
constexpr int HT = 16384;
constexpr int LDS_A = 0, LDS_B = 4 * HT, LDS_M = 8 * HT, LDS_BYTES = 8 * HT + 64;
constexpr unsigned ROWB = 8192;
constexpr size_t XG_PAR = 32ull << 20, XG_SRC = 4ull << 20;
constexpr unsigned XG_KT = 65536;
constexpr size_t WS_FLAGS = 64ull << 20;
constexpr size_t WS_CTR = WS_FLAGS + 524288;
constexpr size_t WS_TMO = WS_CTR + 32768;
constexpr size_t WS_STAMP = WS_TMO + 4096;
constexpr size_t WS_NEED = WS_STAMP + 65536;
constexpr int NEVER = 0x7fffffff;

struct Args { const char* x; const char* w; char* out; char* ws[NR]; int rank; };

__device__ __forceinline__ unsigned cvt_pk_bf16(float lo, float hi) { unsigned r; asm volatile("v_cvt_pk_bf16_f32 %0, %1, %2" : "=v"(r) : "v"(lo), "v"(hi)); return r; }
__device__ __forceinline__ unsigned rfl(unsigned v) { return (unsigned)__builtin_amdgcn_readfirstlane((int)v); }
__device__ __forceinline__ unsigned now_sync() { u64 t; asm volatile("s_memrealtime %0\n\ts_waitcnt lgkmcnt(0)" : "=s"(t) :: "memory"); return (unsigned)t; }
__device__ __forceinline__ unsigned off_x(int pc, int lane) { const int row = 8 * pc + (lane >> 3); return (unsigned)row * ROWB + (unsigned)(((lane & 7) ^ ((row >> 1) & 7)) * 16); }
__device__ __forceinline__ unsigned off_w(int pc, int lane) { const int k = 4 * pc + (lane >> 4); const int f = (k & 3) | (((k >> 3) & 1) << 2); return (unsigned)k * ROWB + (unsigned)(((lane & 15) ^ (2 * f)) * 16); }
}

#define GLDS(voff, sbase, ldsb, imm) asm volatile("s_add_u32 m0, %2, %3\n\ts_nop 0\n\tglobal_load_lds_dwordx4 %0, %1" :: "v"(voff), "s"(sbase), "s"(ldsb), "i"(imm) : "memory", "scc")
#define WAIT_V(n) asm volatile("s_waitcnt vmcnt(%0)" :: "i"(n) : "memory")
#define WAIT_L(n) asm volatile("s_waitcnt lgkmcnt(" #n ")" ::: "memory")
#define BAR __builtin_amdgcn_s_barrier()
#define DSR128(dst, addr, off) asm volatile("ds_read_b128 %0, %1 offset:%2" : "=v"(dst) : "v"(addr), "i"(off) : "memory")
#define DSRTR(dst, addr, off) asm volatile("ds_read_b64_tr_b16 %0, %1 offset:%2" : "=v"(dst) : "v"(addr), "i"(off) : "memory")
#define LDA(bb, h) do { _Pragma("unroll") for (int m = 0; m < 4; ++m) { \
    DSR128(At[m][0], aoff0, ((bb) * 2 + (h)) * HT + m * 2048); DSR128(At[m][1], aoff1, ((bb) * 2 + (h)) * HT + m * 2048); } } while (0)
#define LDB(dst, bb, h) do { _Pragma("unroll") for (int kk = 0; kk < 2; ++kk) { \
    DSRTR(dst[0][kk][0], boff0, ((bb) * 2 + (h)) * HT + kk * 8192); DSRTR(dst[0][kk][1], boff0, ((bb) * 2 + (h)) * HT + kk * 8192 + 1024); \
    DSRTR(dst[1][kk][0], boff1, ((bb) * 2 + (h)) * HT + kk * 8192); DSRTR(dst[1][kk][1], boff1, ((bb) * 2 + (h)) * HT + kk * 8192 + 1024); } } while (0)
#define TIE_A "+v"(At[0][0]), "+v"(At[0][1]), "+v"(At[1][0]), "+v"(At[1][1]), "+v"(At[2][0]), "+v"(At[2][1]), "+v"(At[3][0]), "+v"(At[3][1])
#define TIE_B(B) "+v"(B[0][0][0]), "+v"(B[0][0][1]), "+v"(B[0][1][0]), "+v"(B[0][1][1]), "+v"(B[1][0][0]), "+v"(B[1][0][1]), "+v"(B[1][1][0]), "+v"(B[1][1][1])
#define TIE_C(ai, bj) "+v"(acc[ai][bj][0][0]), "+v"(acc[ai][bj][0][1]), "+v"(acc[ai][bj][1][0]), "+v"(acc[ai][bj][1][1]), "+v"(acc[ai][bj][2][0]), "+v"(acc[ai][bj][2][1]), "+v"(acc[ai][bj][3][0]), "+v"(acc[ai][bj][3][1])
#define WAIT_AB(B) asm volatile("s_waitcnt lgkmcnt(0)" : TIE_A, TIE_B(B) :: "memory")
#define WAIT_B(B) asm volatile("s_waitcnt lgkmcnt(0)" : TIE_B(B) :: "memory")
#define WAIT_A() asm volatile("s_waitcnt lgkmcnt(0)" : TIE_A :: "memory")
#define BFRAG(Bt, n, k) __builtin_shufflevector(Bt[n][k][0], Bt[n][k][1], 0, 1, 2, 3, 4, 5, 6, 7)
#define WAIT_ABB() asm volatile("s_waitcnt lgkmcnt(0)" : TIE_A, TIE_B(B0), TIE_B(B1) :: "memory")
#define MMAW(ai, bj, Bt, TEXT) asm volatile(TEXT : TIE_C(ai, bj) : "v"(At[0][0]), "v"(At[0][1]), "v"(At[1][0]), "v"(At[1][1]), "v"(At[2][0]), "v"(At[2][1]), "v"(At[3][0]), "v"(At[3][1]), \
    "v"(BFRAG(Bt, 0, 0)), "v"(BFRAG(Bt, 0, 1)), "v"(BFRAG(Bt, 1, 0)), "v"(BFRAG(Bt, 1, 1)) : "memory", "scc")

#define MMA2(ai) do { MMAW(ai, 0, B0, "s_setprio 1\n\t" "v_mfma_f32_16x16x32_bf16 %0, %16, %8, %0\n\t" "v_mfma_f32_16x16x32_bf16 %0, %17, %9, %0\n\t" "v_mfma_f32_16x16x32_bf16 %2, %17, %11, %2\n\t" "v_mfma_f32_16x16x32_bf16 %2, %16, %10, %2\n\t" "v_mfma_f32_16x16x32_bf16 %4, %16, %12, %4\n\t" "v_mfma_f32_16x16x32_bf16 %4, %17, %13, %4\n\t" "v_mfma_f32_16x16x32_bf16 %6, %17, %15, %6\n\t" "v_mfma_f32_16x16x32_bf16 %6, %16, %14, %6\n\t" "v_mfma_f32_16x16x32_bf16 %7, %18, %14, %7\n\t" "v_mfma_f32_16x16x32_bf16 %7, %19, %15, %7\n\t" "v_mfma_f32_16x16x32_bf16 %5, %19, %13, %5\n\t" "v_mfma_f32_16x16x32_bf16 %5, %18, %12, %5\n\t" "v_mfma_f32_16x16x32_bf16 %3, %18, %10, %3\n\t" "v_mfma_f32_16x16x32_bf16 %3, %19, %11, %3\n\t" "v_mfma_f32_16x16x32_bf16 %1, %19, %9, %1\n\t" "v_mfma_f32_16x16x32_bf16 %1, %18, %8, %1\n\t"); MMAW(ai, 1, B1, "v_mfma_f32_16x16x32_bf16 %0, %16, %8, %0\n\t" "v_mfma_f32_16x16x32_bf16 %0, %17, %9, %0\n\t" "v_mfma_f32_16x16x32_bf16 %2, %17, %11, %2\n\t" "v_mfma_f32_16x16x32_bf16 %2, %16, %10, %2\n\t" "v_mfma_f32_16x16x32_bf16 %4, %16, %12, %4\n\t" "v_mfma_f32_16x16x32_bf16 %4, %17, %13, %4\n\t" "v_mfma_f32_16x16x32_bf16 %6, %17, %15, %6\n\t" "v_mfma_f32_16x16x32_bf16 %6, %16, %14, %6\n\t" "v_mfma_f32_16x16x32_bf16 %7, %18, %14, %7\n\t" "v_mfma_f32_16x16x32_bf16 %7, %19, %15, %7\n\t" "v_mfma_f32_16x16x32_bf16 %5, %19, %13, %5\n\t" "v_mfma_f32_16x16x32_bf16 %5, %18, %12, %5\n\t" "v_mfma_f32_16x16x32_bf16 %3, %18, %10, %3\n\t" "v_mfma_f32_16x16x32_bf16 %3, %19, %11, %3\n\t" "v_mfma_f32_16x16x32_bf16 %1, %19, %9, %1\n\t" "v_mfma_f32_16x16x32_bf16 %1, %18, %8, %1\n\t" "s_setprio 0\n\t"); } while (0)
#define ROLE_SETUP() \
  int tid_ = threadIdx.x; asm volatile("" : "+v"(tid_)); const int lane = tid_ & 63, fr = lane & 15, fq = lane >> 4; \
  unsigned aoff0, aoff1, boff0, boff1; \
  { const unsigned rb = (unsigned)((wid >> 2) * 64 + fr) * 128u; const unsigned s = (unsigned)(fr >> 1); \
    aoff0 = lbase + LDS_A + rb + (((unsigned)fq ^ s) << 4); aoff1 = lbase + LDS_A + rb + (((unsigned)(4 + fq) ^ s) << 4); \
    const unsigned q = (unsigned)(lane & 15) >> 2, p = (unsigned)lane & 3u, g = (unsigned)fq, f = q | ((g & 1u) << 2); \
    const unsigned kb = (8u * g + q) * 256u + 8u * (p & 1u); \
    boff0 = lbase + LDS_B + kb + ((((unsigned)(4 * (wid & 3)) + (p >> 1)) ^ (2u * f)) << 4); \
    boff1 = lbase + LDS_B + kb + ((((unsigned)(4 * (wid & 3) + 2) + (p >> 1)) ^ (2u * f)) << 4); } \
  f32x4 acc[2][2][4][2]; \
  _Pragma("unroll") for (int i0 = 0; i0 < 2; ++i0) _Pragma("unroll") for (int i1 = 0; i1 < 2; ++i1) \
  _Pragma("unroll") for (int m = 0; m < 4; ++m) _Pragma("unroll") for (int n = 0; n < 2; ++n) acc[i0][i1][m][n] = (f32x4){0.f, 0.f, 0.f, 0.f}; \
  bf16x8 At[4][2]; bf16x4 B0[2][2][2], B1[2][2][2]

namespace {
template <int AI>
__device__ __forceinline__ void store_half(const f32x4 (&acc)[2][2][4][2], char* otile, int wid, bool bad) {
  int tz = threadIdx.x; asm volatile("" : "+v"(tz));
  const int ln = tz & 63, r16 = ln & 15, q4 = ln >> 4;
  char* op = otile + (size_t)((wid >> 2) * 64) * ROWB + (size_t)((wid & 3) * 64);
  auto ro = __builtin_amdgcn_make_buffer_rsrc(op, 0, 192 * ROWB, 0x00020000);
  const unsigned vo = (unsigned)r16 * ROWB + (unsigned)((((q4 & 1) << 4) | ((q4 >> 1) << 3)) * 2);
#pragma unroll
  for (int m = 0; m < 4; ++m)
#pragma unroll
    for (int bj = 0; bj < 2; ++bj) {
      const f32x4 v0 = acc[AI][bj][m][0], v1 = acc[AI][bj][m][1];
      unsigned a0 = cvt_pk_bf16(v0[0], v0[1]), a1 = cvt_pk_bf16(v0[2], v0[3]), b0 = cvt_pk_bf16(v1[0], v1[1]), b1 = cvt_pk_bf16(v1[2], v1[3]);
      auto s0 = __builtin_amdgcn_permlane16_swap(a0, b0, false, false);
      auto s1 = __builtin_amdgcn_permlane16_swap(a1, b1, false, false);
      u32x4 w = {s0[0], s1[0], s0[1], s1[1]};
      if (bad) w = (u32x4){0x7fc07fc0u, 0x7fc07fc0u, 0x7fc07fc0u, 0x7fc07fc0u};
      __builtin_amdgcn_raw_buffer_store_b128(w, ro, vo + bj * 256, (AI * 128 + m * 16) * ROWB, 16);
    }
}
#define READ_BAD() unsigned badv; asm volatile("ds_read_b32 %0, %1" : "=v"(badv) : "v"(lbase + LDS_M) : "memory")
#define TAKE_BAD() asm volatile("" : "+v"(badv) :: "memory"); const bool bad = badv != 0u
#define MFMA_DRAIN() asm volatile("s_nop 7\n\ts_nop 7\n\ts_nop 3" ::: "memory")

#define P1(bb, ST, WE, WL) do { if (SFIRST) { ST; } LDB(B0, bb, 0); LDA(bb, 0); LDB(B1, bb, 1); if (!SFIRST) { ST; } WE; WAIT_ABB(); BAR; MMA2(0); WL; BAR; } while (0)
#define P2(bb, ST, WE, WL) do { LDA(bb, 1); ST; WE; WAIT_A(); BAR; MMA2(1); WL; BAR; } while (0)
#define LAST_PHASE(G0) do { TAKE_BAD(); LDA(1, 1); store_half<0>(acc, otile, wid, bad); WAIT_A(); BAR; MMA2(1); if (G0) BAR; MFMA_DRAIN(); store_half<1>(acc, otile, wid, bad); } while (0)
template <int NG>
__device__ __forceinline__ void worker_a(unsigned lbase, const char* xs, const char* xg, const char* ctr, char* otile, bool local, int wid) {
  ROLE_SETUP();
  const int p0 = wid == 1 ? 0 : wid * 5 - 4;
  const unsigned vA0 = local ? off_x(p0, lane) : (unsigned)(p0 * 1024 + lane * 16), vA1 = local ? off_x(p0 + 1, lane) : vA0 + 1024u;
  const unsigned ldsw = rfl(lbase + (unsigned)p0 * 1024u);
  const size_t qA = local ? 16 * (size_t)ROWB : 2048, hA = local ? 128 * (size_t)ROWB : HT, kA = local ? 128 : XG_KT;
#define STAGE_A(bb, h, gp) do { const char* g0_ = (gp) + (h) * hA; const char* g1_ = g0_ + qA; const char* g2_ = g1_ + qA; \
    GLDS(vA0, g0_, ldsw, LDS_A + ((bb) * 2 + (h)) * HT); GLDS(vA1, g0_, ldsw, LDS_A + ((bb) * 2 + (h)) * HT + 1024); \
    GLDS(vA0, g1_, ldsw, LDS_A + ((bb) * 2 + (h)) * HT + 2048); GLDS(vA1, g1_, ldsw, LDS_A + ((bb) * 2 + (h)) * HT + 3072); \
    GLDS(vA0, g2_, ldsw, LDS_A + ((bb) * 2 + (h)) * HT + 4096); if (NG == 6) GLDS(vA1, g2_, ldsw, LDS_A + ((bb) * 2 + (h)) * HT + 5120); } while (0)
  unsigned cv; { unsigned z = 0; asm volatile("global_load_dword %0, %1, %2 sc1\n\ts_waitcnt vmcnt(0)" : "=&v"(cv) : "v"(z), "s"(ctr) : "memory"); }
  const char* pA = local ? xs : xg + ((rfl(cv) + 1u) & 1u) * XG_PAR;
  BAR; STAGE_A(0, 0, pA);
  BAR; STAGE_A(0, 1, pA);
  BAR;
  WAIT_V(NG); BAR;
#pragma unroll 1
  for (int t = 0; t < NT - 2; t += 2) {
    const char* a1 = pA + (size_t)(t + 1) * kA; const char* a2 = a1 + kA;
    P1(0, STAGE_A(1, 0, a1), , WAIT_V(NG));
    P2(0, STAGE_A(1, 1, a1), , WAIT_V(NG));
    P1(1, STAGE_A(0, 0, a2), , WAIT_V(NG));
    P2(1, STAGE_A(0, 1, a2), , WAIT_V(NG));
  }
  P1(0, STAGE_A(1, 0, pA + (size_t)(NT - 1) * kA), , WAIT_V(NG));
  P2(0, STAGE_A(1, 1, pA + (size_t)(NT - 1) * kA), , WAIT_V(NG));
  READ_BAD();
  P1(1, , , WAIT_V(0));
  LAST_PHASE(true);
#undef STAGE_A
}
__device__ __forceinline__ void worker_b(unsigned lbase, const char* pB, char* otile, int wid) {
  ROLE_SETUP();
  const unsigned vB0 = off_w(wid - 4, lane), vB1 = vB0 + 16u * ROWB, vB2 = vB0 + 32u * ROWB, vB3 = vB0 + 48u * ROWB;
  const unsigned ldsw = rfl(lbase + (unsigned)(wid - 4) * 1024u);
  constexpr size_t kB = 64 * (size_t)ROWB;
#define STAGE_BH(bb, h, g_) do { GLDS(vB0, g_, ldsw, LDS_B + ((bb) * 2 + (h)) * HT); GLDS(vB1, g_, ldsw, LDS_B + ((bb) * 2 + (h)) * HT + 4096); \
    GLDS(vB2, g_, ldsw, LDS_B + ((bb) * 2 + (h)) * HT + 8192); GLDS(vB3, g_, ldsw, LDS_B + ((bb) * 2 + (h)) * HT + 12288); } while (0)
#define STAGE_B(bb, gp) do { const char* h0_ = (gp); const char* h1_ = h0_ + 256; STAGE_BH(bb, 0, h0_); STAGE_BH(bb, 1, h1_); } while (0)
  STAGE_B(0, pB); STAGE_B(1, pB + kB);
  BAR; BAR; BAR;
  WAIT_V(8); BAR;
  BAR;
#pragma unroll 1
  for (int t = 0; t < NT - 2; t += 2) {
    const char* b2 = pB + (size_t)(t + 2) * kB;
    P1(0, , , );
    P2(0, STAGE_B(0, b2), WAIT_V(8), );
    P1(1, , , );
    P2(1, STAGE_B(1, b2 + kB), WAIT_V(8), );
  }
  P1(0, , , );
  P2(0, , WAIT_V(0), );
  READ_BAD();
  P1(1, , , );
  LAST_PHASE(false);
#undef STAGE_B
#undef STAGE_BH
}
}

__global__ __launch_bounds__(512, 2) __attribute__((amdgpu_num_vgpr(108), amdgpu_num_sgpr(62))) void ag_gemm_kernel(Args a) {
  extern __shared__ __attribute__((aligned(16))) char smem[];
  LAS char* lds = (LAS char*)smem;
  asm volatile("v_mov_b32 v239, 0\n\tv_mov_b32 v236, 0" ::: "v222", "v255", "s56", "s101");
  const int wid = __builtin_amdgcn_readfirstlane(threadIdx.x >> 6);
  const int b = blockIdx.x, rank = a.rank;
  const int xcd = b & 7, idx = b >> 3;
  const int pm = 4 * (xcd >> 1) + (idx & 3), pn = 8 * (xcd & 1) + (idx >> 2);
  const int src = pm >> 1, strip = pm & 1;
  const bool local = (src == rank);
  char* myws = a.ws[rank];
  const char* ctr = myws + WS_CTR + (size_t)b * 128;

  const unsigned lbase = (unsigned)(size_t)lds;
  char* otile = a.out + (size_t)(pm * 256) * ROWB + (size_t)pn * 512;
  const char* xs = a.x + (size_t)strip * 256 * ROWB;
  const char* xg = myws + (size_t)src * XG_SRC + (size_t)strip * 2 * HT;
  const char* pB = a.w + (size_t)pn * 512;

  if (wid >= 4) worker_b(lbase, pB, otile, wid);
  else if (wid >= 2) worker_a<5>(lbase, xs, xg, ctr, otile, local, wid);
  else if (wid == 1) worker_a<6>(lbase, xs, xg, ctr, otile, local, wid);
  else {
    const unsigned t_start = now_sync();
    ROLE_SETUP();
    const unsigned jj = (unsigned)((((((idx >> 1) ^ rank) & 1) << 4) | ((idx >> 2) << 1) | (idx & 1)) * 8 + xcd);
    const unsigned vx0 = off_x(0, lane), vx1 = off_x(1, lane);
    unsigned cv; { unsigned z = 0; asm volatile("global_load_dword %0, %1, %2 sc1" : "=v"(cv) : "v"(z), "s"(ctr) : "memory"); }
    const unsigned idA = (jj / 7u) * 8u + jj % 7u;
    { const char* sa = a.x + (size_t)(((idA << 15) & 0x3c0000u) | (idA & 0x1f80u));
      asm volatile("v_mov_b32 v230, %0\n\tv_mov_b32 v229, %1\n\tv_add_u32 v225, 0x20000, %0\n\tv_add_u32 v223, 0x20000, %1\n\t"
                   "global_load_dwordx4 v[240:243], v230, %2\n\tglobal_load_dwordx4 v[244:247], v229, %2\n\tglobal_load_dwordx4 v[248:251], v225, %2\n\tglobal_load_dwordx4 v[252:255], v223, %2"
                   :: "v"(vx0), "v"(vx1), "s"(sa) : "memory"); }
    char* pw = a.ws[(rank + 1) & 7];
#pragma unroll
    for (int q = 1; q < 7; ++q) { char* c = a.ws[(rank + 1 + q) & 7]; asm volatile("" : "+s"(c)); if (lane == q) pw = c; }
    *(LAS unsigned*)(lds + LDS_M) = 0u;
    const unsigned foff = (unsigned)(src * 65536 + strip * 256);
    const char* flags = myws + WS_FLAGS + foff;
    const unsigned g = ((unsigned)lane >> 2) & 7u, gcode = (g >> 1) * 4u + (g & 1u);
    { const unsigned pbase = gcode * 128u + ((unsigned)lane & 3u) * 32u;
      asm volatile("v_mov_b32 v231, %0\n\tv_mov_b32 v228, %1\n\tv_mov_b32 v237, %1\n\tv_mov_b32 v227, %2\n\tv_mov_b32 v226, %3"
                   :: "v"((unsigned)lane * 16u), "v"(pbase), "v"(g), "s"((unsigned)(WS_TMO - WS_FLAGS) - foff) : "memory"); }
    { const int due0 = T0 + (int)((jj * D1) >> 8);
      asm volatile("s_mov_b32 s70, 0\n\ts_mov_b32 s71, 0\n\ts_mov_b32 s72, 0x7fffffff\n\ts_mov_b32 s75, 0\n\t"
                   "s_mov_b32 s73, %0\n\ts_mov_b32 s74, %1\n\ts_mov_b32 s86, 7\n\ts_sub_u32 s66, 0, %2\n\t"
                   "s_mov_b32 s89, %3\n\ts_mov_b64 s[90:91], %4\n\ts_mov_b64 s[92:93], %5\n\ts_mov_b32 s67, 0"
                   :: "s"(NOCOMM ? NEVER : due0), "s"(idA), "s"(rfl((local || NOCOMM || NOWAIT) ? 0u : 1u)),
                      "s"(t_start), "s"(a.x), "s"(flags) : "memory", "scc"); }
    asm volatile("" : TIE_C(0, 0), TIE_C(0, 1)); asm volatile("" : TIE_C(1, 0), TIE_C(1, 1));
    asm volatile("s_waitcnt vmcnt(0)" : "+v"(cv) :: "memory");
    const unsigned call = rfl(cv) + 1u;
    { const unsigned xgo = (call & 1u) * (unsigned)XG_PAR + (unsigned)rank * (unsigned)XG_SRC; const u64 pd = (u64)(pw + xgo);
      asm volatile("v_mov_b32 v234, %0\n\tv_mov_b32 v235, %1\n\ts_mov_b32 s88, %2\n\ts_mov_b32 s68, %3\n\tv_mov_b32 v233, %4\n\tv_mov_b32 v224, %4\n\tv_mov_b32 v232, %5\n\ts_nop 1\n\t" "s_mov_b32 s87, 0\n\t" "s_mov_b32 s85, 28\n\t" "s_mov_b32 s69, 0\n\t" "s_add_i32 s98, s70, 6\n\t" "s_add_i32 s70, s70, 10\n\t" "s_cmp_lt_u32 s74, " STR(ID1) "\n\t" "s_cselect_b32 s99, 0, (" STR(D2) "-" STR(D1) ")\n\t" "s_add_i32 s99, s99, s73\n\t" "s_addk_i32 s99, " STR(D1) "\n\t" "s_cmp_lt_u32 s69, 0x2000\n\t" "s_cselect_b32 s99, s99, 0x7fffffff\n\t" "s_cmp_lg_u32 s87, 0\n\t" "s_cselect_b32 s72, s98, s72\n\t" "s_cselect_b32 s75, s74, s75\n\t" "s_cselect_b32 s71, s70, s71\n\t" "s_cselect_b32 s73, s99, s73\n\t" "s_cselect_b32 s74, s69, s74\n\t" "s_and_b32 s98, s75, 7\n\t" "s_lshl_b32 s99, s75, 2\n\t" "v_readlane_b32 s82, v234, s98\n\t" "v_readlane_b32 s83, v235, s98\n\t" "s_and_b32 s99, s99, 0x7fe0\n\t" "s_lshr_b32 s98, s75, 5\n\t" "s_add_i32 s98, s98, s88\n\t" "s_add_u32 s82, s82, s99\n\t" "s_addc_u32 s83, s83, 0\n\t" "v_mov_b32 v238, s98\n\t" "s_mov_b32 s70, 0\n\t" "s_add_i32 s85, s85, 4\n\t" "s_and_b32 s84, s68, s66\n\t" "s_and_b32 s85, s85, 28\n\t" "s_mov_b32 s56, " STR(WF) "\n\t" "s_mov_b32 s57, " STR(WF) "\n\t" "s_lshl_b64 s[56:57], s[56:57], s85\n\t" "s_and_b32 s59, s57, s66\n\t" "s_memrealtime s[96:97]\n\t" "s_and_b32 s98, s74, 7\n\t" "s_and_b32 s99, s74, 0x1ff8\n\t" "v_readlane_b32 s78, v234, s98\n\t" "v_readlane_b32 s79, v235, s98\n\t" "s_lshl_b32 s99, s99, 9\n\t" "s_cmp_ge_u32 s98, 3\n\t" "s_addc_u32 s69, s74, 292\n\t" "s_add_u32 s78, s78, s99\n\t" "s_addc_u32 s79, s79, 0\n\t" "s_lshl_b32 s98, s69, 15\n\t" "s_and_b32 s99, s69, 0x1f80\n\t" "s_and_b32 s98, s98, 0x3c0000\n\t" "s_or_b32 s98, s98, s99\n\t" "s_add_u32 s80, s90, s98\n\t" "s_addc_u32 s81, s91, 0\n\t" "s_waitcnt lgkmcnt(0)\n\t" "s_sub_i32 s62, s96, s89\n\t" "s_getreg_b32 s98, hwreg(HW_REG_IB_STS, 22, 2)\n\t" "s_getreg_b32 s99, hwreg(HW_REG_IB_STS, 0, 4)\n\t" "s_lshl4_add_u32 s98, s98, s99\n\t" "s_sub_i32 s76, s70, s98\n\t" "s_cmp_ge_i32 s76, s71\n\t" "s_cselect_b32 s77, s73, 0x7fffffff\n\t"
                   :: "v"((unsigned)pd), "v"((unsigned)(pd >> 32)), "s"(call * 256u + 1u), "s"(call * 256u + 1u + 2u * (unsigned)strip),
                      "v"(call * 256u + 1u + 2u * (unsigned)strip + gcode), "s"((unsigned)WS_FLAGS + (unsigned)rank * 65536u - xgo) : "memory", "scc"); }
    asm volatile("s_branch ag_slow_end\n\t" "ag_tr_a:\n\t" "s_call_b64 s[94:95], ag_slow_sub\n\t" "s_branch ag_bk_a\n\t" "ag_tr_b:\n\t" "s_call_b64 s[94:95], ag_slow_sub\n\t" "s_branch ag_bk_b\n\t" "ag_tr_c:\n\t" "s_call_b64 s[94:95], ag_slow_sub\n\t" "s_branch ag_bk_c\n\t" "ag_tr_d:\n\t" "s_call_b64 s[94:95], ag_slow_sub\n\t" "s_branch ag_bk_d\n\t" "ag_tr_e:\n\t" "s_call_b64 s[94:95], ag_slow_sub\n\t" "s_branch ag_bk_e\n\t" "ag_tr_p0:\n\t" "s_call_b64 s[94:95], ag_slow_sub\n\t" "s_branch ag_bk_p0\n\t" "ag_tr_p1:\n\t" "s_call_b64 s[94:95], ag_slow_sub\n\t" "s_branch ag_bk_p1\n\t" "ag_tr_p2:\n\t" "s_call_b64 s[94:95], ag_slow_sub\n\t" "s_branch ag_bk_p2\n\t" "ag_slow_sub:\n\t" "s_mov_b32 s56, " STR(WS) "\n\t" "s_mov_b32 s57, " STR(WS) "\n\t" "s_lshl_b64 s[56:57], s[56:57], s85\n\t" "s_and_b32 s59, s57, s66\n\t" "s_memrealtime s[96:97]\n\t" "s_and_b32 s98, s74, 7\n\t" "s_and_b32 s99, s74, 0x1ff8\n\t" "v_readlane_b32 s78, v234, s98\n\t" "v_readlane_b32 s79, v235, s98\n\t" "s_lshl_b32 s99, s99, 9\n\t" "s_cmp_ge_u32 s98, 3\n\t" "s_addc_u32 s69, s74, 292\n\t" "s_add_u32 s78, s78, s99\n\t" "s_addc_u32 s79, s79, 0\n\t" "s_lshl_b32 s98, s69, 15\n\t" "s_and_b32 s99, s69, 0x1f80\n\t" "s_and_b32 s98, s98, 0x3c0000\n\t" "s_or_b32 s98, s98, s99\n\t" "s_add_u32 s80, s90, s98\n\t" "s_addc_u32 s81, s91, 0\n\t" "s_waitcnt lgkmcnt(0)\n\t" "s_sub_i32 s62, s96, s89\n\t" "s_getreg_b32 s98, hwreg(HW_REG_IB_STS, 22, 2)\n\t" "s_getreg_b32 s99, hwreg(HW_REG_IB_STS, 0, 4)\n\t" "s_lshl4_add_u32 s98, s98, s99\n\t" "s_sub_i32 s76, s70, s98\n\t" "s_cmp_ge_i32 s76, s71\n\t" "s_cselect_b32 s77, s73, 0x7fffffff\n\t" "ag_slow_loop:\n\t" "s_add_i32 s67, s67, 1\n\t" "v_cmp_gt_u32 s[64:65], v224, v239\n\t" "s_cmp_ge_i32 s76, s72\n\t" "s_cselect_b64 exec, 1, 0\n\t" "global_store_dword v232, v238, s[82:83] sc0 sc1\n\t" "s_cselect_b32 s72, 0x7fffffff, s72\n\t" "s_and_b32 exec_lo, s64, s59\n\t" "global_load_dword v239, v237, s[92:93] sc0 sc1\n\t" "s_cmp_ge_i32 s62, s77\n\t" "s_cselect_b64 exec, -1, 0\n\t" "global_store_dwordx4 v231, v[240:243], s[78:79] sc0 sc1\n\t" "global_store_dwordx4 v231, v[244:247], s[78:79] offset:1024 sc0 sc1\n\t" "global_store_dwordx4 v231, v[248:251], s[78:79] offset:2048 sc0 sc1\n\t" "global_store_dwordx4 v231, v[252:255], s[78:79] offset:3072 sc0 sc1\n\t" "global_load_dwordx4 v[240:243], v230, s[80:81]\n\t" "global_load_dwordx4 v[244:247], v229, s[80:81]\n\t" "global_load_dwordx4 v[248:251], v225, s[80:81]\n\t" "global_load_dwordx4 v[252:255], v223, s[80:81]\n\t" "s_mov_b64 exec, -1\n\t" "s_cselect_b32 s87, 1, 0\n\t" "global_load_dword v236, v226, s[92:93] sc1\n\t" "s_add_i32 s98, s70, 6\n\t" "s_add_i32 s70, s70, 10\n\t" "s_cmp_lt_u32 s74, " STR(ID1) "\n\t" "s_cselect_b32 s99, 0, (" STR(D2) "-" STR(D1) ")\n\t" "s_add_i32 s99, s99, s73\n\t" "s_addk_i32 s99, " STR(D1) "\n\t" "s_cmp_lt_u32 s69, 0x2000\n\t" "s_cselect_b32 s99, s99, 0x7fffffff\n\t" "s_cmp_lg_u32 s87, 0\n\t" "s_cselect_b32 s72, s98, s72\n\t" "s_cselect_b32 s75, s74, s75\n\t" "s_cselect_b32 s71, s70, s71\n\t" "s_cselect_b32 s73, s99, s73\n\t" "s_cselect_b32 s74, s69, s74\n\t" "s_add_i32 s70, s70, 1\n\t" "s_and_b32 s98, s75, 7\n\t" "s_lshl_b32 s99, s75, 2\n\t" "v_readlane_b32 s82, v234, s98\n\t" "v_readlane_b32 s83, v235, s98\n\t" "s_and_b32 s99, s99, 0x7fe0\n\t" "s_lshr_b32 s98, s75, 5\n\t" "s_add_i32 s98, s98, s88\n\t" "s_add_u32 s82, s82, s99\n\t" "s_addc_u32 s83, s83, 0\n\t" "v_mov_b32 v238, s98\n\t" "s_sleep " STR(SLP) "\n\t" "s_memrealtime s[96:97]\n\t" "s_and_b32 s98, s74, 7\n\t" "s_and_b32 s99, s74, 0x1ff8\n\t" "v_readlane_b32 s78, v234, s98\n\t" "v_readlane_b32 s79, v235, s98\n\t" "s_lshl_b32 s99, s99, 9\n\t" "s_cmp_ge_u32 s98, 3\n\t" "s_addc_u32 s69, s74, 292\n\t" "s_add_u32 s78, s78, s99\n\t" "s_addc_u32 s79, s79, 0\n\t" "s_lshl_b32 s98, s69, 15\n\t" "s_and_b32 s99, s69, 0x1f80\n\t" "s_and_b32 s98, s98, 0x3c0000\n\t" "s_or_b32 s98, s98, s99\n\t" "s_add_u32 s80, s90, s98\n\t" "s_addc_u32 s81, s91, 0\n\t" "s_waitcnt lgkmcnt(0)\n\t" "s_sub_i32 s62, s96, s89\n\t" "s_getreg_b32 s98, hwreg(HW_REG_IB_STS, 22, 2)\n\t" "s_getreg_b32 s99, hwreg(HW_REG_IB_STS, 0, 4)\n\t" "s_lshl4_add_u32 s98, s98, s99\n\t" "s_sub_i32 s76, s70, s98\n\t" "s_cmp_ge_i32 s76, s71\n\t" "s_cselect_b32 s77, s73, 0x7fffffff\n\t" "v_readfirstlane_b32 s98, v236\n\t" "s_cmp_gt_u32 s62, " STR(DEADLINE) "\n\t" "s_cselect_b32 s98, s88, s98\n\t" "s_cmp_eq_u32 s98, s88\n\t" "s_cbranch_scc1 ag_slow_dead\n\t" "v_cmp_le_u32 s[100:101], s84, v239\n\t" "s_lshr_b32 s98, s100, s85\n\t" "s_andn2_b32 s98, 15, s98\n\t" "s_cbranch_scc1 ag_slow_loop\n\t" "s_mov_b32 s56, " STR(WF) "\n\t" "s_mov_b32 s57, " STR(WF) "\n\t" "s_lshl_b64 s[56:57], s[56:57], s85\n\t" "s_and_b32 s59, s57, s66\n\t" "s_setpc_b64 s[94:95]\n\t" "ag_slow_dead:\n\t" "s_mov_b32 s66, 0\n\t" "s_mov_b32 s59, 0\n\t" "s_mov_b32 s84, 0\n\t" "v_mov_b32 v224, s88\n\t" "v_mov_b32 v237, " STR(LDSBAD) "\n\t" "global_store_dword v226, v224, s[92:93] sc1\n\t" "ds_write_b32 v237, v224\n\t" "s_add_i32 s70, s70, 1\n\t" "s_waitcnt lgkmcnt(0)\n\t" "s_setpc_b64 s[94:95]\n\t" "ag_slow_end:\n\t" ::: "memory", "scc");
#define ENG(TEXT) asm volatile(TEXT ::: "memory", "scc")
#define PST(x)
    PST(st_pa);
    ENG("s_cmp_lg_u32 s66, 0\n\t" "s_cbranch_scc1 ag_pre_end\n\t" "s_cmp_eq_u32 s73, 0x7fffffff\n\t" "s_cbranch_scc1 ag_pre_end\n\t" "s_mov_b32 s61, s74\n\t" "ag_pre:\n\t" "s_call_b64 s[94:95], ag_slow_sub\n\t" "s_cmp_eq_u32 s74, s61\n\t" "s_cbranch_scc1 ag_pre\n\t" "s_cmp_lg_u32 s72, 0x7fffffff\n\t" "s_cbranch_scc1 ag_pre\n\t" "ag_pre_end:\n\t");
    ENG("v_cmp_le_u32 s[100:101], s84, v239\n\t" "s_lshr_b32 s98, s100, s85\n\t" "s_andn2_b32 s98, 15, s98\n\t" "s_cbranch_scc1 ag_tr_p0\n\t" "ag_bk_p0:\n\t"); PST(st_p0); BAR; ENG("s_add_i32 s86, s86, 1\n\t" "s_add_i32 s68, s68, 1\n\t" "v_sub_u32 v224, s86, v227\n\t" "v_lshrrev_b32 v224, 3, v224\n\t" "v_lshl_add_u32 v237, v224, 11, v228\n\t" "v_lshl_add_u32 v224, v224, 4, v233\n\t" "s_add_i32 s85, s85, 4\n\t" "s_and_b32 s84, s68, s66\n\t" "s_and_b32 s85, s85, 28\n\t" "s_mov_b32 s56, " STR(WF) "\n\t" "s_mov_b32 s57, " STR(WF) "\n\t" "s_lshl_b64 s[56:57], s[56:57], s85\n\t" "s_and_b32 s59, s57, s66\n\t");
    ENG("v_cmp_le_u32 s[100:101], s84, v239\n\t" "s_lshr_b32 s98, s100, s85\n\t" "s_andn2_b32 s98, 15, s98\n\t" "s_cbranch_scc1 ag_tr_p1\n\t" "ag_bk_p1:\n\t"); PST(st_p1); BAR; ENG("s_add_i32 s86, s86, 1\n\t" "s_add_i32 s68, s68, 3\n\t" "v_sub_u32 v224, s86, v227\n\t" "v_lshrrev_b32 v224, 3, v224\n\t" "v_lshl_add_u32 v237, v224, 11, v228\n\t" "v_lshl_add_u32 v224, v224, 4, v233\n\t" "s_add_i32 s85, s85, 4\n\t" "s_and_b32 s84, s68, s66\n\t" "s_and_b32 s85, s85, 28\n\t" "s_mov_b32 s56, " STR(WF) "\n\t" "s_mov_b32 s57, " STR(WF) "\n\t" "s_lshl_b64 s[56:57], s[56:57], s85\n\t" "s_and_b32 s59, s57, s66\n\t");
    BAR;
    ENG("v_cmp_le_u32 s[100:101], s84, v239\n\t" "s_lshr_b32 s98, s100, s85\n\t" "s_andn2_b32 s98, 15, s98\n\t" "s_cbranch_scc1 ag_tr_p2\n\t" "ag_bk_p2:\n\t"); PST(st_p2); PST(st_p3);
    BAR;
#define W0P1(bb) do { if (APOS == 0) ENG("v_cmp_gt_u32 s[64:65], v224, v239\n\t" "s_cmp_ge_i32 s76, s72\n\t" "s_cselect_b64 exec, 1, 0\n\t" "global_store_dword v232, v238, s[82:83] sc0 sc1\n\t" "s_cselect_b32 s72, 0x7fffffff, s72\n\t" "s_and_b32 exec_lo, s64, s59\n\t" "global_load_dword v239, v237, s[92:93] sc0 sc1\n\t" "s_cmp_ge_i32 s62, s77\n\t" "s_cselect_b64 exec, -1, 0\n\t" "global_store_dwordx4 v231, v[240:243], s[78:79] sc0 sc1\n\t" "global_store_dwordx4 v231, v[244:247], s[78:79] offset:1024 sc0 sc1\n\t" "global_store_dwordx4 v231, v[248:251], s[78:79] offset:2048 sc0 sc1\n\t" "global_store_dwordx4 v231, v[252:255], s[78:79] offset:3072 sc0 sc1\n\t" "global_load_dwordx4 v[240:243], v230, s[80:81]\n\t" "global_load_dwordx4 v[244:247], v229, s[80:81]\n\t" "global_load_dwordx4 v[248:251], v225, s[80:81]\n\t" "global_load_dwordx4 v[252:255], v223, s[80:81]\n\t" "s_mov_b64 exec, -1\n\t" "s_cselect_b32 s87, 1, 0\n\t"); LDB(B0, bb, 0); if (APOS == 1) ENG("v_cmp_gt_u32 s[64:65], v224, v239\n\t" "s_cmp_ge_i32 s76, s72\n\t" "s_cselect_b64 exec, 1, 0\n\t" "global_store_dword v232, v238, s[82:83] sc0 sc1\n\t" "s_cselect_b32 s72, 0x7fffffff, s72\n\t" "s_and_b32 exec_lo, s64, s59\n\t" "global_load_dword v239, v237, s[92:93] sc0 sc1\n\t" "s_cmp_ge_i32 s62, s77\n\t" "s_cselect_b64 exec, -1, 0\n\t" "global_store_dwordx4 v231, v[240:243], s[78:79] sc0 sc1\n\t" "global_store_dwordx4 v231, v[244:247], s[78:79] offset:1024 sc0 sc1\n\t" "global_store_dwordx4 v231, v[248:251], s[78:79] offset:2048 sc0 sc1\n\t" "global_store_dwordx4 v231, v[252:255], s[78:79] offset:3072 sc0 sc1\n\t" "global_load_dwordx4 v[240:243], v230, s[80:81]\n\t" "global_load_dwordx4 v[244:247], v229, s[80:81]\n\t" "global_load_dwordx4 v[248:251], v225, s[80:81]\n\t" "global_load_dwordx4 v[252:255], v223, s[80:81]\n\t" "s_mov_b64 exec, -1\n\t" "s_cselect_b32 s87, 1, 0\n\t"); LDA(bb, 0); if (APOS == 2) ENG("v_cmp_gt_u32 s[64:65], v224, v239\n\t" "s_cmp_ge_i32 s76, s72\n\t" "s_cselect_b64 exec, 1, 0\n\t" "global_store_dword v232, v238, s[82:83] sc0 sc1\n\t" "s_cselect_b32 s72, 0x7fffffff, s72\n\t" "s_and_b32 exec_lo, s64, s59\n\t" "global_load_dword v239, v237, s[92:93] sc0 sc1\n\t" "s_cmp_ge_i32 s62, s77\n\t" "s_cselect_b64 exec, -1, 0\n\t" "global_store_dwordx4 v231, v[240:243], s[78:79] sc0 sc1\n\t" "global_store_dwordx4 v231, v[244:247], s[78:79] offset:1024 sc0 sc1\n\t" "global_store_dwordx4 v231, v[248:251], s[78:79] offset:2048 sc0 sc1\n\t" "global_store_dwordx4 v231, v[252:255], s[78:79] offset:3072 sc0 sc1\n\t" "global_load_dwordx4 v[240:243], v230, s[80:81]\n\t" "global_load_dwordx4 v[244:247], v229, s[80:81]\n\t" "global_load_dwordx4 v[248:251], v225, s[80:81]\n\t" "global_load_dwordx4 v[252:255], v223, s[80:81]\n\t" "s_mov_b64 exec, -1\n\t" "s_cselect_b32 s87, 1, 0\n\t"); LDB(B1, bb, 1); if (APOS == 3) ENG("v_cmp_gt_u32 s[64:65], v224, v239\n\t" "s_cmp_ge_i32 s76, s72\n\t" "s_cselect_b64 exec, 1, 0\n\t" "global_store_dword v232, v238, s[82:83] sc0 sc1\n\t" "s_cselect_b32 s72, 0x7fffffff, s72\n\t" "s_and_b32 exec_lo, s64, s59\n\t" "global_load_dword v239, v237, s[92:93] sc0 sc1\n\t" "s_cmp_ge_i32 s62, s77\n\t" "s_cselect_b64 exec, -1, 0\n\t" "global_store_dwordx4 v231, v[240:243], s[78:79] sc0 sc1\n\t" "global_store_dwordx4 v231, v[244:247], s[78:79] offset:1024 sc0 sc1\n\t" "global_store_dwordx4 v231, v[248:251], s[78:79] offset:2048 sc0 sc1\n\t" "global_store_dwordx4 v231, v[252:255], s[78:79] offset:3072 sc0 sc1\n\t" "global_load_dwordx4 v[240:243], v230, s[80:81]\n\t" "global_load_dwordx4 v[244:247], v229, s[80:81]\n\t" "global_load_dwordx4 v[248:251], v225, s[80:81]\n\t" "global_load_dwordx4 v[252:255], v223, s[80:81]\n\t" "s_mov_b64 exec, -1\n\t" "s_cselect_b32 s87, 1, 0\n\t"); } while (0)
#pragma unroll 1
    for (int t = 0; t < NT - 2; t += 2) {
      W0P1(0);
      WAIT_ABB(); BAR; MMAW(0, 0, B0, "s_setprio 1\n\t" "v_mfma_f32_16x16x32_bf16 %0, %16, %8, %0\n\t" "s_add_i32 s98, s70, 6\n\t" "s_add_i32 s70, s70, 10\n\t" "v_mfma_f32_16x16x32_bf16 %0, %17, %9, %0\n\t" "s_cmp_lt_u32 s74, " STR(ID1) "\n\t" "v_mfma_f32_16x16x32_bf16 %2, %17, %11, %2\n\t" "s_cselect_b32 s99, 0, (" STR(D2) "-" STR(D1) ")\n\t" "v_mfma_f32_16x16x32_bf16 %2, %16, %10, %2\n\t" "s_add_i32 s99, s99, s73\n\t" "v_mfma_f32_16x16x32_bf16 %4, %16, %12, %4\n\t" "s_addk_i32 s99, " STR(D1) "\n\t" "s_cmp_lt_u32 s69, 0x2000\n\t" "v_mfma_f32_16x16x32_bf16 %4, %17, %13, %4\n\t" "s_cselect_b32 s99, s99, 0x7fffffff\n\t" "v_mfma_f32_16x16x32_bf16 %6, %17, %15, %6\n\t" "s_cmp_lg_u32 s87, 0\n\t" "v_mfma_f32_16x16x32_bf16 %6, %16, %14, %6\n\t" "s_cselect_b32 s72, s98, s72\n\t" "v_mfma_f32_16x16x32_bf16 %7, %18, %14, %7\n\t" "s_cselect_b32 s75, s74, s75\n\t" "s_cselect_b32 s71, s70, s71\n\t" "v_mfma_f32_16x16x32_bf16 %7, %19, %15, %7\n\t" "s_cselect_b32 s73, s99, s73\n\t" "v_mfma_f32_16x16x32_bf16 %5, %19, %13, %5\n\t" "s_cselect_b32 s74, s69, s74\n\t" "v_mfma_f32_16x16x32_bf16 %5, %18, %12, %5\n\t" "s_add_i32 s86, s86, 1\n\t" "v_mfma_f32_16x16x32_bf16 %3, %18, %10, %3\n\t" "s_add_i32 s68, s68, 1\n\t" "v_sub_u32 v224, s86, v227\n\t" "v_mfma_f32_16x16x32_bf16 %3, %19, %11, %3\n\t" "v_lshrrev_b32 v224, 3, v224\n\t" "v_mfma_f32_16x16x32_bf16 %1, %19, %9, %1\n\t" "v_lshl_add_u32 v237, v224, 11, v228\n\t" "v_mfma_f32_16x16x32_bf16 %1, %18, %8, %1\n\t" "v_lshl_add_u32 v224, v224, 4, v233\n\t"); MMAW(0, 1, B1, "v_mfma_f32_16x16x32_bf16 %0, %16, %8, %0\n\t" "s_add_i32 s85, s85, 4\n\t" "s_and_b32 s84, s68, s66\n\t" "v_mfma_f32_16x16x32_bf16 %0, %17, %9, %0\n\t" "s_and_b32 s85, s85, 28\n\t" "v_mfma_f32_16x16x32_bf16 %2, %17, %11, %2\n\t" "s_mov_b32 s56, " STR(WF) "\n\t" "v_mfma_f32_16x16x32_bf16 %2, %16, %10, %2\n\t" "s_mov_b32 s57, " STR(WF) "\n\t" "s_lshl_b64 s[56:57], s[56:57], s85\n\t" "v_mfma_f32_16x16x32_bf16 %4, %16, %12, %4\n\t" "s_and_b32 s59, s57, s66\n\t" "v_mfma_f32_16x16x32_bf16 %4, %17, %13, %4\n\t" "s_and_b32 s98, s75, 7\n\t" "v_mfma_f32_16x16x32_bf16 %6, %17, %15, %6\n\t" "s_lshl_b32 s99, s75, 2\n\t" "v_readlane_b32 s82, v234, s98\n\t" "v_mfma_f32_16x16x32_bf16 %6, %16, %14, %6\n\t" "v_readlane_b32 s83, v235, s98\n\t" "v_mfma_f32_16x16x32_bf16 %7, %18, %14, %7\n\t" "s_and_b32 s99, s99, 0x7fe0\n\t" "v_mfma_f32_16x16x32_bf16 %7, %19, %15, %7\n\t" "s_lshr_b32 s98, s75, 5\n\t" "s_add_i32 s98, s98, s88\n\t" "v_mfma_f32_16x16x32_bf16 %5, %19, %13, %5\n\t" "s_add_u32 s82, s82, s99\n\t" "v_mfma_f32_16x16x32_bf16 %5, %18, %12, %5\n\t" "s_addc_u32 s83, s83, 0\n\t" "v_mfma_f32_16x16x32_bf16 %3, %18, %10, %3\n\t" "v_mov_b32 v238, s98\n\t" "v_cmp_le_u32 s[100:101], s84, v239\n\t" "v_mfma_f32_16x16x32_bf16 %3, %19, %11, %3\n\t" "s_lshr_b32 s98, s100, s85\n\t" "v_mfma_f32_16x16x32_bf16 %1, %19, %9, %1\n\t" "s_andn2_b32 s98, 15, s98\n\t" "v_mfma_f32_16x16x32_bf16 %1, %18, %8, %1\n\t" "s_setprio 0\n\t" "s_cbranch_scc1 ag_tr_a\n\t" "ag_bk_a:\n\t"); BAR;
      LDA(0, 1); ENG("v_cmp_gt_u32 s[64:65], v224, v239\n\t" "s_mov_b32 exec_hi, 0\n\t" "s_and_b32 exec_lo, s64, s59\n\t" "global_load_dword v239, v237, s[92:93] sc0 sc1\n\t" "s_mov_b64 exec, -1\n\t");
      WAIT_A(); BAR; MMAW(1, 0, B0, "s_setprio 1\n\t" "v_mfma_f32_16x16x32_bf16 %0, %16, %8, %0\n\t" "s_add_i32 s70, s70, 1\n\t" "s_add_i32 s86, s86, 1\n\t" "v_mfma_f32_16x16x32_bf16 %0, %17, %9, %0\n\t" "s_add_i32 s68, s68, 3\n\t" "v_mfma_f32_16x16x32_bf16 %2, %17, %11, %2\n\t" "v_sub_u32 v224, s86, v227\n\t" "v_mfma_f32_16x16x32_bf16 %2, %16, %10, %2\n\t" "v_lshrrev_b32 v224, 3, v224\n\t" "v_mfma_f32_16x16x32_bf16 %4, %16, %12, %4\n\t" "v_lshl_add_u32 v237, v224, 11, v228\n\t" "v_lshl_add_u32 v224, v224, 4, v233\n\t" "v_mfma_f32_16x16x32_bf16 %4, %17, %13, %4\n\t" "s_add_i32 s85, s85, 4\n\t" "v_mfma_f32_16x16x32_bf16 %6, %17, %15, %6\n\t" "s_and_b32 s84, s68, s66\n\t" "v_mfma_f32_16x16x32_bf16 %6, %16, %14, %6\n\t" "s_and_b32 s85, s85, 28\n\t" "v_mfma_f32_16x16x32_bf16 %7, %18, %14, %7\n\t" "s_mov_b32 s56, " STR(WF) "\n\t" "s_mov_b32 s57, " STR(WF) "\n\t" "v_mfma_f32_16x16x32_bf16 %7, %19, %15, %7\n\t" "s_lshl_b64 s[56:57], s[56:57], s85\n\t" "v_mfma_f32_16x16x32_bf16 %5, %19, %13, %5\n\t" "s_and_b32 s59, s57, s66\n\t" "v_mfma_f32_16x16x32_bf16 %5, %18, %12, %5\n\t" "s_and_b32 s98, s74, 7\n\t" "v_mfma_f32_16x16x32_bf16 %3, %18, %10, %3\n\t" "s_and_b32 s99, s74, 0x1ff8\n\t" "v_readlane_b32 s78, v234, s98\n\t" "v_mfma_f32_16x16x32_bf16 %3, %19, %11, %3\n\t" "v_readlane_b32 s79, v235, s98\n\t" "v_mfma_f32_16x16x32_bf16 %1, %19, %9, %1\n\t" "s_lshl_b32 s99, s99, 9\n\t" "v_mfma_f32_16x16x32_bf16 %1, %18, %8, %1\n\t" "s_memrealtime s[96:97]\n\t"); MMAW(1, 1, B1, "v_mfma_f32_16x16x32_bf16 %0, %16, %8, %0\n\t" "s_cmp_ge_u32 s98, 3\n\t" "s_addc_u32 s69, s74, 292\n\t" "v_mfma_f32_16x16x32_bf16 %0, %17, %9, %0\n\t" "s_add_u32 s78, s78, s99\n\t" "v_mfma_f32_16x16x32_bf16 %2, %17, %11, %2\n\t" "s_addc_u32 s79, s79, 0\n\t" "s_lshl_b32 s98, s69, 15\n\t" "v_mfma_f32_16x16x32_bf16 %2, %16, %10, %2\n\t" "s_and_b32 s99, s69, 0x1f80\n\t" "v_mfma_f32_16x16x32_bf16 %4, %16, %12, %4\n\t" "s_and_b32 s98, s98, 0x3c0000\n\t" "v_mfma_f32_16x16x32_bf16 %4, %17, %13, %4\n\t" "s_or_b32 s98, s98, s99\n\t" "s_add_u32 s80, s90, s98\n\t" "v_mfma_f32_16x16x32_bf16 %6, %17, %15, %6\n\t" "s_addc_u32 s81, s91, 0\n\t" "v_mfma_f32_16x16x32_bf16 %6, %16, %14, %6\n\t" "s_waitcnt lgkmcnt(0)\n\t" "s_sub_i32 s62, s96, s89\n\t" "v_mfma_f32_16x16x32_bf16 %7, %18, %14, %7\n\t" "s_getreg_b32 s98, hwreg(HW_REG_IB_STS, 22, 2)\n\t" "v_mfma_f32_16x16x32_bf16 %7, %19, %15, %7\n\t" "s_getreg_b32 s99, hwreg(HW_REG_IB_STS, 0, 4)\n\t" "v_mfma_f32_16x16x32_bf16 %5, %19, %13, %5\n\t" "s_lshl4_add_u32 s98, s98, s99\n\t" "s_sub_i32 s76, s70, s98\n\t" "v_mfma_f32_16x16x32_bf16 %5, %18, %12, %5\n\t" "s_cmp_ge_i32 s76, s71\n\t" "v_mfma_f32_16x16x32_bf16 %3, %18, %10, %3\n\t" "s_cselect_b32 s77, s73, 0x7fffffff\n\t" "v_cmp_le_u32 s[100:101], s84, v239\n\t" "v_mfma_f32_16x16x32_bf16 %3, %19, %11, %3\n\t" "s_lshr_b32 s98, s100, s85\n\t" "v_mfma_f32_16x16x32_bf16 %1, %19, %9, %1\n\t" "s_andn2_b32 s98, 15, s98\n\t" "v_mfma_f32_16x16x32_bf16 %1, %18, %8, %1\n\t" "s_setprio 0\n\t" "s_cbranch_scc1 ag_tr_b\n\t" "ag_bk_b:\n\t"); BAR;
      W0P1(1);
      WAIT_ABB(); BAR; MMAW(0, 0, B0, "s_setprio 1\n\t" "v_mfma_f32_16x16x32_bf16 %0, %16, %8, %0\n\t" "s_add_i32 s98, s70, 6\n\t" "s_add_i32 s70, s70, 10\n\t" "v_mfma_f32_16x16x32_bf16 %0, %17, %9, %0\n\t" "s_cmp_lt_u32 s74, " STR(ID1) "\n\t" "v_mfma_f32_16x16x32_bf16 %2, %17, %11, %2\n\t" "s_cselect_b32 s99, 0, (" STR(D2) "-" STR(D1) ")\n\t" "v_mfma_f32_16x16x32_bf16 %2, %16, %10, %2\n\t" "s_add_i32 s99, s99, s73\n\t" "v_mfma_f32_16x16x32_bf16 %4, %16, %12, %4\n\t" "s_addk_i32 s99, " STR(D1) "\n\t" "s_cmp_lt_u32 s69, 0x2000\n\t" "v_mfma_f32_16x16x32_bf16 %4, %17, %13, %4\n\t" "s_cselect_b32 s99, s99, 0x7fffffff\n\t" "v_mfma_f32_16x16x32_bf16 %6, %17, %15, %6\n\t" "s_cmp_lg_u32 s87, 0\n\t" "v_mfma_f32_16x16x32_bf16 %6, %16, %14, %6\n\t" "s_cselect_b32 s72, s98, s72\n\t" "v_mfma_f32_16x16x32_bf16 %7, %18, %14, %7\n\t" "s_cselect_b32 s75, s74, s75\n\t" "s_cselect_b32 s71, s70, s71\n\t" "v_mfma_f32_16x16x32_bf16 %7, %19, %15, %7\n\t" "s_cselect_b32 s73, s99, s73\n\t" "v_mfma_f32_16x16x32_bf16 %5, %19, %13, %5\n\t" "s_cselect_b32 s74, s69, s74\n\t" "v_mfma_f32_16x16x32_bf16 %5, %18, %12, %5\n\t" "s_add_i32 s86, s86, 1\n\t" "v_mfma_f32_16x16x32_bf16 %3, %18, %10, %3\n\t" "s_add_i32 s68, s68, 1\n\t" "v_sub_u32 v224, s86, v227\n\t" "v_mfma_f32_16x16x32_bf16 %3, %19, %11, %3\n\t" "v_lshrrev_b32 v224, 3, v224\n\t" "v_mfma_f32_16x16x32_bf16 %1, %19, %9, %1\n\t" "v_lshl_add_u32 v237, v224, 11, v228\n\t" "v_mfma_f32_16x16x32_bf16 %1, %18, %8, %1\n\t" "v_lshl_add_u32 v224, v224, 4, v233\n\t"); MMAW(0, 1, B1, "v_mfma_f32_16x16x32_bf16 %0, %16, %8, %0\n\t" "s_add_i32 s85, s85, 4\n\t" "s_and_b32 s84, s68, s66\n\t" "v_mfma_f32_16x16x32_bf16 %0, %17, %9, %0\n\t" "s_and_b32 s85, s85, 28\n\t" "v_mfma_f32_16x16x32_bf16 %2, %17, %11, %2\n\t" "s_mov_b32 s56, " STR(WF) "\n\t" "v_mfma_f32_16x16x32_bf16 %2, %16, %10, %2\n\t" "s_mov_b32 s57, " STR(WF) "\n\t" "s_lshl_b64 s[56:57], s[56:57], s85\n\t" "v_mfma_f32_16x16x32_bf16 %4, %16, %12, %4\n\t" "s_and_b32 s59, s57, s66\n\t" "v_mfma_f32_16x16x32_bf16 %4, %17, %13, %4\n\t" "s_and_b32 s98, s75, 7\n\t" "v_mfma_f32_16x16x32_bf16 %6, %17, %15, %6\n\t" "s_lshl_b32 s99, s75, 2\n\t" "v_readlane_b32 s82, v234, s98\n\t" "v_mfma_f32_16x16x32_bf16 %6, %16, %14, %6\n\t" "v_readlane_b32 s83, v235, s98\n\t" "v_mfma_f32_16x16x32_bf16 %7, %18, %14, %7\n\t" "s_and_b32 s99, s99, 0x7fe0\n\t" "v_mfma_f32_16x16x32_bf16 %7, %19, %15, %7\n\t" "s_lshr_b32 s98, s75, 5\n\t" "s_add_i32 s98, s98, s88\n\t" "v_mfma_f32_16x16x32_bf16 %5, %19, %13, %5\n\t" "s_add_u32 s82, s82, s99\n\t" "v_mfma_f32_16x16x32_bf16 %5, %18, %12, %5\n\t" "s_addc_u32 s83, s83, 0\n\t" "v_mfma_f32_16x16x32_bf16 %3, %18, %10, %3\n\t" "v_mov_b32 v238, s98\n\t" "v_cmp_le_u32 s[100:101], s84, v239\n\t" "v_mfma_f32_16x16x32_bf16 %3, %19, %11, %3\n\t" "s_lshr_b32 s98, s100, s85\n\t" "v_mfma_f32_16x16x32_bf16 %1, %19, %9, %1\n\t" "s_andn2_b32 s98, 15, s98\n\t" "v_mfma_f32_16x16x32_bf16 %1, %18, %8, %1\n\t" "s_setprio 0\n\t" "s_cbranch_scc1 ag_tr_c\n\t" "ag_bk_c:\n\t"); BAR;
      LDA(1, 1); ENG("v_cmp_gt_u32 s[64:65], v224, v239\n\t" "s_mov_b32 exec_hi, 0\n\t" "s_and_b32 exec_lo, s64, s59\n\t" "global_load_dword v239, v237, s[92:93] sc0 sc1\n\t" "s_mov_b64 exec, -1\n\t");
      WAIT_A(); BAR; MMAW(1, 0, B0, "s_setprio 1\n\t" "v_mfma_f32_16x16x32_bf16 %0, %16, %8, %0\n\t" "s_add_i32 s70, s70, 1\n\t" "s_add_i32 s86, s86, 1\n\t" "v_mfma_f32_16x16x32_bf16 %0, %17, %9, %0\n\t" "s_add_i32 s68, s68, 3\n\t" "v_mfma_f32_16x16x32_bf16 %2, %17, %11, %2\n\t" "v_sub_u32 v224, s86, v227\n\t" "v_mfma_f32_16x16x32_bf16 %2, %16, %10, %2\n\t" "v_lshrrev_b32 v224, 3, v224\n\t" "v_mfma_f32_16x16x32_bf16 %4, %16, %12, %4\n\t" "v_lshl_add_u32 v237, v224, 11, v228\n\t" "v_lshl_add_u32 v224, v224, 4, v233\n\t" "v_mfma_f32_16x16x32_bf16 %4, %17, %13, %4\n\t" "s_add_i32 s85, s85, 4\n\t" "v_mfma_f32_16x16x32_bf16 %6, %17, %15, %6\n\t" "s_and_b32 s84, s68, s66\n\t" "v_mfma_f32_16x16x32_bf16 %6, %16, %14, %6\n\t" "s_and_b32 s85, s85, 28\n\t" "v_mfma_f32_16x16x32_bf16 %7, %18, %14, %7\n\t" "s_mov_b32 s56, " STR(WF) "\n\t" "s_mov_b32 s57, " STR(WF) "\n\t" "v_mfma_f32_16x16x32_bf16 %7, %19, %15, %7\n\t" "s_lshl_b64 s[56:57], s[56:57], s85\n\t" "v_mfma_f32_16x16x32_bf16 %5, %19, %13, %5\n\t" "s_and_b32 s59, s57, s66\n\t" "v_mfma_f32_16x16x32_bf16 %5, %18, %12, %5\n\t" "s_and_b32 s98, s74, 7\n\t" "v_mfma_f32_16x16x32_bf16 %3, %18, %10, %3\n\t" "s_and_b32 s99, s74, 0x1ff8\n\t" "v_readlane_b32 s78, v234, s98\n\t" "v_mfma_f32_16x16x32_bf16 %3, %19, %11, %3\n\t" "v_readlane_b32 s79, v235, s98\n\t" "v_mfma_f32_16x16x32_bf16 %1, %19, %9, %1\n\t" "s_lshl_b32 s99, s99, 9\n\t" "v_mfma_f32_16x16x32_bf16 %1, %18, %8, %1\n\t" "s_memrealtime s[96:97]\n\t"); MMAW(1, 1, B1, "v_mfma_f32_16x16x32_bf16 %0, %16, %8, %0\n\t" "s_cmp_ge_u32 s98, 3\n\t" "s_addc_u32 s69, s74, 292\n\t" "v_mfma_f32_16x16x32_bf16 %0, %17, %9, %0\n\t" "s_add_u32 s78, s78, s99\n\t" "v_mfma_f32_16x16x32_bf16 %2, %17, %11, %2\n\t" "s_addc_u32 s79, s79, 0\n\t" "s_lshl_b32 s98, s69, 15\n\t" "v_mfma_f32_16x16x32_bf16 %2, %16, %10, %2\n\t" "s_and_b32 s99, s69, 0x1f80\n\t" "v_mfma_f32_16x16x32_bf16 %4, %16, %12, %4\n\t" "s_and_b32 s98, s98, 0x3c0000\n\t" "v_mfma_f32_16x16x32_bf16 %4, %17, %13, %4\n\t" "s_or_b32 s98, s98, s99\n\t" "s_add_u32 s80, s90, s98\n\t" "v_mfma_f32_16x16x32_bf16 %6, %17, %15, %6\n\t" "s_addc_u32 s81, s91, 0\n\t" "v_mfma_f32_16x16x32_bf16 %6, %16, %14, %6\n\t" "s_waitcnt lgkmcnt(0)\n\t" "s_sub_i32 s62, s96, s89\n\t" "v_mfma_f32_16x16x32_bf16 %7, %18, %14, %7\n\t" "s_getreg_b32 s98, hwreg(HW_REG_IB_STS, 22, 2)\n\t" "v_mfma_f32_16x16x32_bf16 %7, %19, %15, %7\n\t" "s_getreg_b32 s99, hwreg(HW_REG_IB_STS, 0, 4)\n\t" "v_mfma_f32_16x16x32_bf16 %5, %19, %13, %5\n\t" "s_lshl4_add_u32 s98, s98, s99\n\t" "s_sub_i32 s76, s70, s98\n\t" "v_mfma_f32_16x16x32_bf16 %5, %18, %12, %5\n\t" "s_cmp_ge_i32 s76, s71\n\t" "v_mfma_f32_16x16x32_bf16 %3, %18, %10, %3\n\t" "s_cselect_b32 s77, s73, 0x7fffffff\n\t" "v_cmp_le_u32 s[100:101], s84, v239\n\t" "v_mfma_f32_16x16x32_bf16 %3, %19, %11, %3\n\t" "s_lshr_b32 s98, s100, s85\n\t" "v_mfma_f32_16x16x32_bf16 %1, %19, %9, %1\n\t" "s_andn2_b32 s98, 15, s98\n\t" "v_mfma_f32_16x16x32_bf16 %1, %18, %8, %1\n\t" "s_setprio 0\n\t" "s_cbranch_scc1 ag_tr_d\n\t" "ag_bk_d:\n\t"); BAR;
    }
    W0P1(0);
    WAIT_ABB(); BAR; MMAW(0, 0, B0, "s_setprio 1\n\t" "v_mfma_f32_16x16x32_bf16 %0, %16, %8, %0\n\t" "s_add_i32 s98, s70, 6\n\t" "s_add_i32 s70, s70, 10\n\t" "v_mfma_f32_16x16x32_bf16 %0, %17, %9, %0\n\t" "s_cmp_lt_u32 s74, " STR(ID1) "\n\t" "v_mfma_f32_16x16x32_bf16 %2, %17, %11, %2\n\t" "s_cselect_b32 s99, 0, (" STR(D2) "-" STR(D1) ")\n\t" "v_mfma_f32_16x16x32_bf16 %2, %16, %10, %2\n\t" "s_add_i32 s99, s99, s73\n\t" "v_mfma_f32_16x16x32_bf16 %4, %16, %12, %4\n\t" "s_addk_i32 s99, " STR(D1) "\n\t" "s_cmp_lt_u32 s69, 0x2000\n\t" "v_mfma_f32_16x16x32_bf16 %4, %17, %13, %4\n\t" "s_cselect_b32 s99, s99, 0x7fffffff\n\t" "v_mfma_f32_16x16x32_bf16 %6, %17, %15, %6\n\t" "s_cmp_lg_u32 s87, 0\n\t" "v_mfma_f32_16x16x32_bf16 %6, %16, %14, %6\n\t" "s_cselect_b32 s72, s98, s72\n\t" "v_mfma_f32_16x16x32_bf16 %7, %18, %14, %7\n\t" "s_cselect_b32 s75, s74, s75\n\t" "s_cselect_b32 s71, s70, s71\n\t" "v_mfma_f32_16x16x32_bf16 %7, %19, %15, %7\n\t" "s_cselect_b32 s73, s99, s73\n\t" "v_mfma_f32_16x16x32_bf16 %5, %19, %13, %5\n\t" "s_cselect_b32 s74, s69, s74\n\t" "v_mfma_f32_16x16x32_bf16 %5, %18, %12, %5\n\t" "s_add_i32 s86, s86, 1\n\t" "v_mfma_f32_16x16x32_bf16 %3, %18, %10, %3\n\t" "s_add_i32 s68, s68, 1\n\t" "v_sub_u32 v224, s86, v227\n\t" "v_mfma_f32_16x16x32_bf16 %3, %19, %11, %3\n\t" "v_lshrrev_b32 v224, 3, v224\n\t" "v_mfma_f32_16x16x32_bf16 %1, %19, %9, %1\n\t" "v_lshl_add_u32 v237, v224, 11, v228\n\t" "v_mfma_f32_16x16x32_bf16 %1, %18, %8, %1\n\t" "v_lshl_add_u32 v224, v224, 4, v233\n\t"); MMAW(0, 1, B1, "v_mfma_f32_16x16x32_bf16 %0, %16, %8, %0\n\t" "s_add_i32 s85, s85, 4\n\t" "s_and_b32 s84, s68, s66\n\t" "v_mfma_f32_16x16x32_bf16 %0, %17, %9, %0\n\t" "s_and_b32 s85, s85, 28\n\t" "v_mfma_f32_16x16x32_bf16 %2, %17, %11, %2\n\t" "s_mov_b32 s56, " STR(WF) "\n\t" "v_mfma_f32_16x16x32_bf16 %2, %16, %10, %2\n\t" "s_mov_b32 s57, " STR(WF) "\n\t" "s_lshl_b64 s[56:57], s[56:57], s85\n\t" "v_mfma_f32_16x16x32_bf16 %4, %16, %12, %4\n\t" "s_and_b32 s59, s57, s66\n\t" "v_mfma_f32_16x16x32_bf16 %4, %17, %13, %4\n\t" "s_and_b32 s98, s75, 7\n\t" "v_mfma_f32_16x16x32_bf16 %6, %17, %15, %6\n\t" "s_lshl_b32 s99, s75, 2\n\t" "v_readlane_b32 s82, v234, s98\n\t" "v_mfma_f32_16x16x32_bf16 %6, %16, %14, %6\n\t" "v_readlane_b32 s83, v235, s98\n\t" "v_mfma_f32_16x16x32_bf16 %7, %18, %14, %7\n\t" "s_and_b32 s99, s99, 0x7fe0\n\t" "v_mfma_f32_16x16x32_bf16 %7, %19, %15, %7\n\t" "s_lshr_b32 s98, s75, 5\n\t" "s_add_i32 s98, s98, s88\n\t" "v_mfma_f32_16x16x32_bf16 %5, %19, %13, %5\n\t" "s_add_u32 s82, s82, s99\n\t" "v_mfma_f32_16x16x32_bf16 %5, %18, %12, %5\n\t" "s_addc_u32 s83, s83, 0\n\t" "v_mfma_f32_16x16x32_bf16 %3, %18, %10, %3\n\t" "v_mov_b32 v238, s98\n\t" "v_cmp_le_u32 s[100:101], s84, v239\n\t" "v_mfma_f32_16x16x32_bf16 %3, %19, %11, %3\n\t" "s_lshr_b32 s98, s100, s85\n\t" "v_mfma_f32_16x16x32_bf16 %1, %19, %9, %1\n\t" "s_andn2_b32 s98, 15, s98\n\t" "v_mfma_f32_16x16x32_bf16 %1, %18, %8, %1\n\t" "s_setprio 0\n\t" "s_cbranch_scc1 ag_tr_e\n\t" "ag_bk_e:\n\t"); BAR;
    LDA(0, 1); ENG("v_cmp_gt_u32 s[64:65], v224, v239\n\t" "s_mov_b32 exec_hi, 0\n\t" "s_and_b32 exec_lo, s64, s59\n\t" "global_load_dword v239, v237, s[92:93] sc0 sc1\n\t" "s_mov_b64 exec, -1\n\t");
    WAIT_A(); BAR; MMAW(1, 0, B0, "s_setprio 1\n\t" "v_mfma_f32_16x16x32_bf16 %0, %16, %8, %0\n\t" "s_add_i32 s70, s70, 1\n\t" "s_add_i32 s86, s86, 1\n\t" "v_mfma_f32_16x16x32_bf16 %0, %17, %9, %0\n\t" "s_add_i32 s68, s68, 3\n\t" "v_mfma_f32_16x16x32_bf16 %2, %17, %11, %2\n\t" "v_sub_u32 v224, s86, v227\n\t" "v_mfma_f32_16x16x32_bf16 %2, %16, %10, %2\n\t" "v_lshrrev_b32 v224, 3, v224\n\t" "v_mfma_f32_16x16x32_bf16 %4, %16, %12, %4\n\t" "v_lshl_add_u32 v237, v224, 11, v228\n\t" "v_lshl_add_u32 v224, v224, 4, v233\n\t" "v_mfma_f32_16x16x32_bf16 %4, %17, %13, %4\n\t" "s_add_i32 s85, s85, 4\n\t" "v_mfma_f32_16x16x32_bf16 %6, %17, %15, %6\n\t" "s_and_b32 s84, s68, s66\n\t" "v_mfma_f32_16x16x32_bf16 %6, %16, %14, %6\n\t" "s_and_b32 s85, s85, 28\n\t" "v_mfma_f32_16x16x32_bf16 %7, %18, %14, %7\n\t" "s_mov_b32 s56, " STR(WF) "\n\t" "s_mov_b32 s57, " STR(WF) "\n\t" "v_mfma_f32_16x16x32_bf16 %7, %19, %15, %7\n\t" "s_lshl_b64 s[56:57], s[56:57], s85\n\t" "v_mfma_f32_16x16x32_bf16 %5, %19, %13, %5\n\t" "s_and_b32 s59, s57, s66\n\t" "v_mfma_f32_16x16x32_bf16 %5, %18, %12, %5\n\t" "s_and_b32 s98, s74, 7\n\t" "v_mfma_f32_16x16x32_bf16 %3, %18, %10, %3\n\t" "s_and_b32 s99, s74, 0x1ff8\n\t" "v_readlane_b32 s78, v234, s98\n\t" "v_mfma_f32_16x16x32_bf16 %3, %19, %11, %3\n\t" "v_readlane_b32 s79, v235, s98\n\t" "v_mfma_f32_16x16x32_bf16 %1, %19, %9, %1\n\t" "s_lshl_b32 s99, s99, 9\n\t" "v_mfma_f32_16x16x32_bf16 %1, %18, %8, %1\n\t" "s_memrealtime s[96:97]\n\t"); MMAW(1, 1, B1, "v_mfma_f32_16x16x32_bf16 %0, %16, %8, %0\n\t" "s_cmp_ge_u32 s98, 3\n\t" "s_addc_u32 s69, s74, 292\n\t" "v_mfma_f32_16x16x32_bf16 %0, %17, %9, %0\n\t" "s_add_u32 s78, s78, s99\n\t" "v_mfma_f32_16x16x32_bf16 %2, %17, %11, %2\n\t" "s_addc_u32 s79, s79, 0\n\t" "v_mfma_f32_16x16x32_bf16 %2, %16, %10, %2\n\t" "s_lshl_b32 s98, s69, 15\n\t" "v_mfma_f32_16x16x32_bf16 %4, %16, %12, %4\n\t" "s_and_b32 s99, s69, 0x1f80\n\t" "v_mfma_f32_16x16x32_bf16 %4, %17, %13, %4\n\t" "s_and_b32 s98, s98, 0x3c0000\n\t" "s_or_b32 s98, s98, s99\n\t" "v_mfma_f32_16x16x32_bf16 %6, %17, %15, %6\n\t" "s_add_u32 s80, s90, s98\n\t" "v_mfma_f32_16x16x32_bf16 %6, %16, %14, %6\n\t" "s_addc_u32 s81, s91, 0\n\t" "v_mfma_f32_16x16x32_bf16 %7, %18, %14, %7\n\t" "s_waitcnt lgkmcnt(0)\n\t" "v_mfma_f32_16x16x32_bf16 %7, %19, %15, %7\n\t" "s_sub_i32 s62, s96, s89\n\t" "v_mfma_f32_16x16x32_bf16 %5, %19, %13, %5\n\t" "s_getreg_b32 s98, hwreg(HW_REG_IB_STS, 22, 2)\n\t" "s_getreg_b32 s99, hwreg(HW_REG_IB_STS, 0, 4)\n\t" "v_mfma_f32_16x16x32_bf16 %5, %18, %12, %5\n\t" "s_lshl4_add_u32 s98, s98, s99\n\t" "v_mfma_f32_16x16x32_bf16 %3, %18, %10, %3\n\t" "s_sub_i32 s76, s70, s98\n\t" "v_mfma_f32_16x16x32_bf16 %3, %19, %11, %3\n\t" "s_cmp_ge_i32 s76, s71\n\t" "v_mfma_f32_16x16x32_bf16 %1, %19, %9, %1\n\t" "s_cselect_b32 s77, s73, 0x7fffffff\n\t" "v_mfma_f32_16x16x32_bf16 %1, %18, %8, %1\n\t" "s_setprio 0\n\t"); BAR;
    READ_BAD();
    W0P1(1);
    WAIT_ABB(); BAR; MMAW(0, 0, B0, "s_setprio 1\n\t" "v_mfma_f32_16x16x32_bf16 %0, %16, %8, %0\n\t" "s_add_i32 s98, s70, 6\n\t" "s_add_i32 s70, s70, 10\n\t" "v_mfma_f32_16x16x32_bf16 %0, %17, %9, %0\n\t" "s_cmp_lt_u32 s74, " STR(ID1) "\n\t" "v_mfma_f32_16x16x32_bf16 %2, %17, %11, %2\n\t" "s_cselect_b32 s99, 0, (" STR(D2) "-" STR(D1) ")\n\t" "v_mfma_f32_16x16x32_bf16 %2, %16, %10, %2\n\t" "s_add_i32 s99, s99, s73\n\t" "v_mfma_f32_16x16x32_bf16 %4, %16, %12, %4\n\t" "s_addk_i32 s99, " STR(D1) "\n\t" "s_cmp_lt_u32 s69, 0x2000\n\t" "v_mfma_f32_16x16x32_bf16 %4, %17, %13, %4\n\t" "s_cselect_b32 s99, s99, 0x7fffffff\n\t" "v_mfma_f32_16x16x32_bf16 %6, %17, %15, %6\n\t" "s_cmp_lg_u32 s87, 0\n\t" "v_mfma_f32_16x16x32_bf16 %6, %16, %14, %6\n\t" "s_cselect_b32 s72, s98, s72\n\t" "v_mfma_f32_16x16x32_bf16 %7, %18, %14, %7\n\t" "s_cselect_b32 s75, s74, s75\n\t" "s_cselect_b32 s71, s70, s71\n\t" "v_mfma_f32_16x16x32_bf16 %7, %19, %15, %7\n\t" "s_cselect_b32 s73, s99, s73\n\t" "v_mfma_f32_16x16x32_bf16 %5, %19, %13, %5\n\t" "s_cselect_b32 s74, s69, s74\n\t" "v_mfma_f32_16x16x32_bf16 %5, %18, %12, %5\n\t" "s_add_i32 s86, s86, 1\n\t" "v_mfma_f32_16x16x32_bf16 %3, %18, %10, %3\n\t" "s_add_i32 s68, s68, 1\n\t" "v_sub_u32 v224, s86, v227\n\t" "v_mfma_f32_16x16x32_bf16 %3, %19, %11, %3\n\t" "v_lshrrev_b32 v224, 3, v224\n\t" "v_mfma_f32_16x16x32_bf16 %1, %19, %9, %1\n\t" "v_lshl_add_u32 v237, v224, 11, v228\n\t" "v_mfma_f32_16x16x32_bf16 %1, %18, %8, %1\n\t" "v_lshl_add_u32 v224, v224, 4, v233\n\t"); MMAW(0, 1, B1, "v_mfma_f32_16x16x32_bf16 %0, %16, %8, %0\n\t" "s_add_i32 s85, s85, 4\n\t" "s_and_b32 s84, s68, s66\n\t" "v_mfma_f32_16x16x32_bf16 %0, %17, %9, %0\n\t" "s_and_b32 s85, s85, 28\n\t" "v_mfma_f32_16x16x32_bf16 %2, %17, %11, %2\n\t" "s_mov_b32 s56, " STR(WF) "\n\t" "v_mfma_f32_16x16x32_bf16 %2, %16, %10, %2\n\t" "s_mov_b32 s57, " STR(WF) "\n\t" "v_mfma_f32_16x16x32_bf16 %4, %16, %12, %4\n\t" "s_lshl_b64 s[56:57], s[56:57], s85\n\t" "v_mfma_f32_16x16x32_bf16 %4, %17, %13, %4\n\t" "s_and_b32 s59, s57, s66\n\t" "v_mfma_f32_16x16x32_bf16 %6, %17, %15, %6\n\t" "s_and_b32 s98, s75, 7\n\t" "v_mfma_f32_16x16x32_bf16 %6, %16, %14, %6\n\t" "s_lshl_b32 s99, s75, 2\n\t" "v_readlane_b32 s82, v234, s98\n\t" "v_mfma_f32_16x16x32_bf16 %7, %18, %14, %7\n\t" "v_readlane_b32 s83, v235, s98\n\t" "v_mfma_f32_16x16x32_bf16 %7, %19, %15, %7\n\t" "s_and_b32 s99, s99, 0x7fe0\n\t" "v_mfma_f32_16x16x32_bf16 %5, %19, %13, %5\n\t" "s_lshr_b32 s98, s75, 5\n\t" "v_mfma_f32_16x16x32_bf16 %5, %18, %12, %5\n\t" "s_add_i32 s98, s98, s88\n\t" "v_mfma_f32_16x16x32_bf16 %3, %18, %10, %3\n\t" "s_add_u32 s82, s82, s99\n\t" "v_mfma_f32_16x16x32_bf16 %3, %19, %11, %3\n\t" "s_addc_u32 s83, s83, 0\n\t" "v_mfma_f32_16x16x32_bf16 %1, %19, %9, %1\n\t" "v_mov_b32 v238, s98\n\t" "v_mfma_f32_16x16x32_bf16 %1, %18, %8, %1\n\t" "s_setprio 0\n\t"); BAR;
    TAKE_BAD();
    LDA(1, 1); ENG("v_cmp_gt_u32 s[64:65], v224, v239\n\t" "s_mov_b32 exec_hi, 0\n\t" "s_and_b32 exec_lo, s64, s59\n\t" "global_load_dword v239, v237, s[92:93] sc0 sc1\n\t" "s_mov_b64 exec, -1\n\t"); store_half<0>(acc, otile, wid, bad);
    WAIT_A(); BAR; MMAW(1, 0, B0, "s_setprio 1\n\t" "v_mfma_f32_16x16x32_bf16 %0, %16, %8, %0\n\t" "s_add_i32 s70, s70, 1\n\t" "s_add_i32 s86, s86, 1\n\t" "v_mfma_f32_16x16x32_bf16 %0, %17, %9, %0\n\t" "s_add_i32 s68, s68, 3\n\t" "v_mfma_f32_16x16x32_bf16 %2, %17, %11, %2\n\t" "v_sub_u32 v224, s86, v227\n\t" "v_mfma_f32_16x16x32_bf16 %2, %16, %10, %2\n\t" "v_lshrrev_b32 v224, 3, v224\n\t" "v_mfma_f32_16x16x32_bf16 %4, %16, %12, %4\n\t" "v_lshl_add_u32 v237, v224, 11, v228\n\t" "v_lshl_add_u32 v224, v224, 4, v233\n\t" "v_mfma_f32_16x16x32_bf16 %4, %17, %13, %4\n\t" "s_add_i32 s85, s85, 4\n\t" "v_mfma_f32_16x16x32_bf16 %6, %17, %15, %6\n\t" "s_and_b32 s84, s68, s66\n\t" "v_mfma_f32_16x16x32_bf16 %6, %16, %14, %6\n\t" "s_and_b32 s85, s85, 28\n\t" "v_mfma_f32_16x16x32_bf16 %7, %18, %14, %7\n\t" "s_mov_b32 s56, " STR(WF) "\n\t" "s_mov_b32 s57, " STR(WF) "\n\t" "v_mfma_f32_16x16x32_bf16 %7, %19, %15, %7\n\t" "s_lshl_b64 s[56:57], s[56:57], s85\n\t" "v_mfma_f32_16x16x32_bf16 %5, %19, %13, %5\n\t" "s_and_b32 s59, s57, s66\n\t" "v_mfma_f32_16x16x32_bf16 %5, %18, %12, %5\n\t" "s_and_b32 s98, s74, 7\n\t" "v_mfma_f32_16x16x32_bf16 %3, %18, %10, %3\n\t" "s_and_b32 s99, s74, 0x1ff8\n\t" "v_readlane_b32 s78, v234, s98\n\t" "v_mfma_f32_16x16x32_bf16 %3, %19, %11, %3\n\t" "v_readlane_b32 s79, v235, s98\n\t" "v_mfma_f32_16x16x32_bf16 %1, %19, %9, %1\n\t" "s_lshl_b32 s99, s99, 9\n\t" "v_mfma_f32_16x16x32_bf16 %1, %18, %8, %1\n\t" "s_memrealtime s[96:97]\n\t"); MMAW(1, 1, B1, "v_mfma_f32_16x16x32_bf16 %0, %16, %8, %0\n\t" "s_cmp_ge_u32 s98, 3\n\t" "s_addc_u32 s69, s74, 292\n\t" "v_mfma_f32_16x16x32_bf16 %0, %17, %9, %0\n\t" "s_add_u32 s78, s78, s99\n\t" "v_mfma_f32_16x16x32_bf16 %2, %17, %11, %2\n\t" "s_addc_u32 s79, s79, 0\n\t" "v_mfma_f32_16x16x32_bf16 %2, %16, %10, %2\n\t" "s_lshl_b32 s98, s69, 15\n\t" "v_mfma_f32_16x16x32_bf16 %4, %16, %12, %4\n\t" "s_and_b32 s99, s69, 0x1f80\n\t" "v_mfma_f32_16x16x32_bf16 %4, %17, %13, %4\n\t" "s_and_b32 s98, s98, 0x3c0000\n\t" "s_or_b32 s98, s98, s99\n\t" "v_mfma_f32_16x16x32_bf16 %6, %17, %15, %6\n\t" "s_add_u32 s80, s90, s98\n\t" "v_mfma_f32_16x16x32_bf16 %6, %16, %14, %6\n\t" "s_addc_u32 s81, s91, 0\n\t" "v_mfma_f32_16x16x32_bf16 %7, %18, %14, %7\n\t" "s_waitcnt lgkmcnt(0)\n\t" "v_mfma_f32_16x16x32_bf16 %7, %19, %15, %7\n\t" "s_sub_i32 s62, s96, s89\n\t" "v_mfma_f32_16x16x32_bf16 %5, %19, %13, %5\n\t" "s_getreg_b32 s98, hwreg(HW_REG_IB_STS, 22, 2)\n\t" "s_getreg_b32 s99, hwreg(HW_REG_IB_STS, 0, 4)\n\t" "v_mfma_f32_16x16x32_bf16 %5, %18, %12, %5\n\t" "s_lshl4_add_u32 s98, s98, s99\n\t" "v_mfma_f32_16x16x32_bf16 %3, %18, %10, %3\n\t" "s_sub_i32 s76, s70, s98\n\t" "v_mfma_f32_16x16x32_bf16 %3, %19, %11, %3\n\t" "s_cmp_ge_i32 s76, s71\n\t" "v_mfma_f32_16x16x32_bf16 %1, %19, %9, %1\n\t" "s_cselect_b32 s77, s73, 0x7fffffff\n\t" "v_mfma_f32_16x16x32_bf16 %1, %18, %8, %1\n\t" "s_setprio 0\n\t"); BAR;
    MFMA_DRAIN();
    store_half<1>(acc, otile, wid, bad);
    asm volatile("s_mov_b32 s66, 0\n\t" "s_mov_b32 s59, 0\n\t" "s_branch ag_fin_test\n\t" "ag_fin:\n\t" "s_and_b32 s98, s74, 7\n\t" "s_and_b32 s99, s74, 0x1ff8\n\t" "v_readlane_b32 s78, v234, s98\n\t" "v_readlane_b32 s79, v235, s98\n\t" "s_lshl_b32 s99, s99, 9\n\t" "s_cmp_ge_u32 s98, 3\n\t" "s_addc_u32 s69, s74, 292\n\t" "s_add_u32 s78, s78, s99\n\t" "s_addc_u32 s79, s79, 0\n\t" "s_lshl_b32 s98, s69, 15\n\t" "s_and_b32 s99, s69, 0x1f80\n\t" "s_and_b32 s98, s98, 0x3c0000\n\t" "s_or_b32 s98, s98, s99\n\t" "s_add_u32 s80, s90, s98\n\t" "s_addc_u32 s81, s91, 0\n\t" "s_mov_b32 s62, 0x7ffffffe\n\t" "s_waitcnt vmcnt(0)\n\t" "s_mov_b32 s76, s70\n\t" "s_cmp_ge_i32 s76, s71\n\t" "s_cselect_b32 s77, s73, 0x7fffffff\n\t" "v_cmp_gt_u32 s[64:65], v224, v239\n\t" "s_cmp_ge_i32 s76, s72\n\t" "s_cselect_b64 exec, 1, 0\n\t" "global_store_dword v232, v238, s[82:83] sc0 sc1\n\t" "s_cselect_b32 s72, 0x7fffffff, s72\n\t" "s_and_b32 exec_lo, s64, s59\n\t" "global_load_dword v239, v237, s[92:93] sc0 sc1\n\t" "s_cmp_ge_i32 s62, s77\n\t" "s_cselect_b64 exec, -1, 0\n\t" "global_store_dwordx4 v231, v[240:243], s[78:79] sc0 sc1\n\t" "global_store_dwordx4 v231, v[244:247], s[78:79] offset:1024 sc0 sc1\n\t" "global_store_dwordx4 v231, v[248:251], s[78:79] offset:2048 sc0 sc1\n\t" "global_store_dwordx4 v231, v[252:255], s[78:79] offset:3072 sc0 sc1\n\t" "global_load_dwordx4 v[240:243], v230, s[80:81]\n\t" "global_load_dwordx4 v[244:247], v229, s[80:81]\n\t" "global_load_dwordx4 v[248:251], v225, s[80:81]\n\t" "global_load_dwordx4 v[252:255], v223, s[80:81]\n\t" "s_mov_b64 exec, -1\n\t" "s_cselect_b32 s87, 1, 0\n\t" "s_add_i32 s98, s70, 6\n\t" "s_add_i32 s70, s70, 10\n\t" "s_cmp_lt_u32 s74, " STR(ID1) "\n\t" "s_cselect_b32 s99, 0, (" STR(D2) "-" STR(D1) ")\n\t" "s_add_i32 s99, s99, s73\n\t" "s_addk_i32 s99, " STR(D1) "\n\t" "s_cmp_lt_u32 s69, 0x2000\n\t" "s_cselect_b32 s99, s99, 0x7fffffff\n\t" "s_cmp_lg_u32 s87, 0\n\t" "s_cselect_b32 s72, s98, s72\n\t" "s_cselect_b32 s75, s74, s75\n\t" "s_cselect_b32 s71, s70, s71\n\t" "s_cselect_b32 s73, s99, s73\n\t" "s_cselect_b32 s74, s69, s74\n\t" "s_and_b32 s98, s75, 7\n\t" "s_lshl_b32 s99, s75, 2\n\t" "v_readlane_b32 s82, v234, s98\n\t" "v_readlane_b32 s83, v235, s98\n\t" "s_and_b32 s99, s99, 0x7fe0\n\t" "s_lshr_b32 s98, s75, 5\n\t" "s_add_i32 s98, s98, s88\n\t" "s_add_u32 s82, s82, s99\n\t" "s_addc_u32 s83, s83, 0\n\t" "v_mov_b32 v238, s98\n\t" "ag_fin_test:\n\t" "s_and_b32 s98, s72, s73\n\t" "s_cmp_lg_u32 s98, 0x7fffffff\n\t" "s_cbranch_scc1 ag_fin\n\t" ::: "memory", "scc");
    if (lane == 0) __hip_atomic_store((unsigned*)ctr, call, __ATOMIC_RELAXED, __HIP_MEMORY_SCOPE_AGENT);
  }
}

extern "C" void kernel_launch(int rank, int nranks, ncclComm_t comm, hipStream_t stream,
                              void* const* d_in, const int* in_sizes, int n_in,
                              void* d_out, int out_size, const int* in_dtypes, int out_dtype,
                              void* const* d_out_all, void* const* d_ws_all, size_t ws_bytes) {
  if (nranks != NR || (unsigned)rank >= (unsigned)NR || ws_bytes < WS_NEED || n_in < 2 ||
      in_sizes[0] != 512 * 4096 || in_sizes[1] != 4096 * 4096 || out_size != 4096 * 4096) {
    fprintf(stderr, "kernel_launch: unexpected shapes\n"); return;
  }
  static bool init = false;
  if (!init) {
    if (hipFuncSetAttribute((const void*)ag_gemm_kernel, hipFuncAttributeMaxDynamicSharedMemorySize, LDS_BYTES) != hipSuccess) {
      fprintf(stderr, "hipFuncSetAttribute failed\n"); return; }
    init = true;
  }
  Args a;
  a.x = (const char*)d_in[0]; a.w = (const char*)d_in[1]; a.out = (char*)d_out; a.rank = rank;
  for (int p = 0; p < NR; ++p) a.ws[p] = (char*)d_ws_all[p];
  ag_gemm_kernel<<<256, 512, LDS_BYTES, stream>>>(a);
}
```

```cpp
#include <hip/hip_runtime.h>
#include <rccl/rccl.h>
#include <cstdio>

#define D1 276
#define D2 288
#define ID1 1280
#define T0 90
#define NOCOMM 0
#define NOWAIT 0
#define SLP 3
#define APOS 1
#define SFIRST 0
#define WF 0xff
#define WS 0xff
#define STAMP 0
#define DEADLINE 100000000
#define LDSBAD 131072
#define STR_(x) #x
#define STR(x) STR_(x)

namespace {
typedef short bf16x8 __attribute__((ext_vector_type(8)));
typedef short bf16x4 __attribute__((ext_vector_type(4)));
typedef float f32x4 __attribute__((ext_vector_type(4)));
typedef unsigned u32x4 __attribute__((ext_vector_type(4)));
typedef unsigned long long u64;
#define LAS __attribute__((address_space(3)))

constexpr int NR = 8;
constexpr int NT = 64;
constexpr int HT = 16384;
constexpr int LDS_A = 0, LDS_B = 4 * HT, LDS_M = 8 * HT, LDS_BYTES = 8 * HT + 64;
constexpr unsigned ROWB = 8192;
constexpr size_t XG_PAR = 32ull << 20, XG_SRC = 4ull << 20;
constexpr unsigned XG_KT = 65536;
constexpr size_t WS_FLAGS = 64ull << 20;
constexpr size_t WS_CTR = WS_FLAGS + 524288;
constexpr size_t WS_TMO = WS_CTR + 32768;
constexpr size_t WS_STAMP = WS_TMO + 4096;
constexpr size_t WS_NEED = WS_STAMP + 65536;
constexpr int NEVER = 0x7fffffff;

struct Args { const char* x; const char* w; char* out; char* ws[NR]; int rank; };

__device__ __forceinline__ unsigned cvt_pk_bf16(float lo, float hi) { unsigned r; asm volatile("v_cvt_pk_bf16_f32 %0, %1, %2" : "=v"(r) : "v"(lo), "v"(hi)); return r; }
__device__ __forceinline__ unsigned rfl(unsigned v) { return (unsigned)__builtin_amdgcn_readfirstlane((int)v); }
__device__ __forceinline__ unsigned now_sync() { u64 t; asm volatile("s_memrealtime %0\n\ts_waitcnt lgkmcnt(0)" : "=s"(t) :: "memory"); return (unsigned)t; }
__device__ __forceinline__ unsigned off_x(int pc, int lane) { const int row = 8 * pc + (lane >> 3); return (unsigned)row * ROWB + (unsigned)(((lane & 7) ^ ((row >> 1) & 7)) * 16); }
__device__ __forceinline__ unsigned off_w(int pc, int lane) { const int k = 4 * pc + (lane >> 4); const int f = (k & 3) | (((k >> 3) & 1) << 2); return (unsigned)k * ROWB + (unsigned)(((lane & 15) ^ (2 * f)) * 16); }
}

#define GLDS(voff, sbase, ldsb, imm) asm volatile("s_add_u32 m0, %2, %3\n\ts_nop 0\n\tglobal_load_lds_dwordx4 %0, %1" :: "v"(voff), "s"(sbase), "s"(ldsb), "i"(imm) : "memory", "scc")
#define WAIT_V(n) asm volatile("s_waitcnt vmcnt(%0)" :: "i"(n) : "memory")
#define WAIT_L(n) asm volatile("s_waitcnt lgkmcnt(" #n ")" ::: "memory")
#define BAR __builtin_amdgcn_s_barrier()
#define DSR128(dst, addr, off) asm volatile("ds_read_b128 %0, %1 offset:%2" : "=v"(dst) : "v"(addr), "i"(off) : "memory")
#define DSRTR(dst, addr, off) asm volatile("ds_read_b64_tr_b16 %0, %1 offset:%2" : "=v"(dst) : "v"(addr), "i"(off) : "memory")
#define LDA(bb, h) do { _Pragma("unroll") for (int m = 0; m < 4; ++m) { \
    DSR128(At[m][0], aoff0, ((bb) * 2 + (h)) * HT + m * 2048); DSR128(At[m][1], aoff1, ((bb) * 2 + (h)) * HT + m * 2048); } } while (0)
#define LDB(dst, bb, h) do { _Pragma("unroll") for (int kk = 0; kk < 2; ++kk) { \
    DSRTR(dst[0][kk][0], boff0, ((bb) * 2 + (h)) * HT + kk * 8192); DSRTR(dst[0][kk][1], boff0, ((bb) * 2 + (h)) * HT + kk * 8192 + 1024); \
    DSRTR(dst[1][kk][0], boff1, ((bb) * 2 + (h)) * HT + kk * 8192); DSRTR(dst[1][kk][1], boff1, ((bb) * 2 + (h)) * HT + kk * 8192 + 1024); } } while (0)
#define TIE_A "+v"(At[0][0]), "+v"(At[0][1]), "+v"(At[1][0]), "+v"(At[1][1]), "+v"(At[2][0]), "+v"(At[2][1]), "+v"(At[3][0]), "+v"(At[3][1])
#define TIE_B(B) "+v"(B[0][0][0]), "+v"(B[0][0][1]), "+v"(B[0][1][0]), "+v"(B[0][1][1]), "+v"(B[1][0][0]), "+v"(B[1][0][1]), "+v"(B[1][1][0]), "+v"(B[1][1][1])
#define TIE_C(ai, bj) "+v"(acc[ai][bj][0][0]), "+v"(acc[ai][bj][0][1]), "+v"(acc[ai][bj][1][0]), "+v"(acc[ai][bj][1][1]), "+v"(acc[ai][bj][2][0]), "+v"(acc[ai][bj][2][1]), "+v"(acc[ai][bj][3][0]), "+v"(acc[ai][bj][3][1])
#define WAIT_AB(B) asm volatile("s_waitcnt lgkmcnt(0)" : TIE_A, TIE_B(B) :: "memory")
#define WAIT_B(B) asm volatile("s_waitcnt lgkmcnt(0)" : TIE_B(B) :: "memory")
#define WAIT_A() asm volatile("s_waitcnt lgkmcnt(0)" : TIE_A :: "memory")
#define BFRAG(Bt, n, k) __builtin_shufflevector(Bt[n][k][0], Bt[n][k][1], 0, 1, 2, 3, 4, 5, 6, 7)
#define WAIT_ABB() asm volatile("s_waitcnt lgkmcnt(0)" : TIE_A, TIE_B(B0), TIE_B(B1) :: "memory")
#define MMAW(ai, bj, Bt, TEXT) asm volatile(TEXT : TIE_C(ai, bj) : "v"(At[0][0]), "v"(At[0][1]), "v"(At[1][0]), "v"(At[1][1]), "v"(At[2][0]), "v"(At[2][1]), "v"(At[3][0]), "v"(At[3][1]), \
    "v"(BFRAG(Bt, 0, 0)), "v"(BFRAG(Bt, 0, 1)), "v"(BFRAG(Bt, 1, 0)), "v"(BFRAG(Bt, 1, 1)) : "memory", "scc")

#define MMA2(ai) do { MMAW(ai, 0, B0, "s_setprio 1\n\t" "v_mfma_f32_16x16x32_bf16 %0, %16, %8, %0\n\t" "v_mfma_f32_16x16x32_bf16 %0, %17, %9, %0\n\t" "v_mfma_f32_16x16x32_bf16 %2, %17, %11, %2\n\t" "v_mfma_f32_16x16x32_bf16 %2, %16, %10, %2\n\t" "v_mfma_f32_16x16x32_bf16 %4, %16, %12, %4\n\t" "v_mfma_f32_16x16x32_bf16 %4, %17, %13, %4\n\t" "v_mfma_f32_16x16x32_bf16 %6, %17, %15, %6\n\t" "v_mfma_f32_16x16x32_bf16 %6, %16, %14, %6\n\t" "v_mfma_f32_16x16x32_bf16 %7, %18, %14, %7\n\t" "v_mfma_f32_16x16x32_bf16 %7, %19, %15, %7\n\t" "v_mfma_f32_16x16x32_bf16 %5, %19, %13, %5\n\t" "v_mfma_f32_16x16x32_bf16 %5, %18, %12, %5\n\t" "v_mfma_f32_16x16x32_bf16 %3, %18, %10, %3\n\t" "v_mfma_f32_16x16x32_bf16 %3, %19, %11, %3\n\t" "v_mfma_f32_16x16x32_bf16 %1, %19, %9, %1\n\t" "v_mfma_f32_16x16x32_bf16 %1, %18, %8, %1\n\t"); MMAW(ai, 1, B1, "v_mfma_f32_16x16x32_bf16 %0, %16, %8, %0\n\t" "v_mfma_f32_16x16x32_bf16 %0, %17, %9, %0\n\t" "v_mfma_f32_16x16x32_bf16 %2, %17, %11, %2\n\t" "v_mfma_f32_16x16x32_bf16 %2, %16, %10, %2\n\t" "v_mfma_f32_16x16x32_bf16 %4, %16, %12, %4\n\t" "v_mfma_f32_16x16x32_bf16 %4, %17, %13, %4\n\t" "v_mfma_f32_16x16x32_bf16 %6, %17, %15, %6\n\t" "v_mfma_f32_16x16x32_bf16 %6, %16, %14, %6\n\t" "v_mfma_f32_16x16x32_bf16 %7, %18, %14, %7\n\t" "v_mfma_f32_16x16x32_bf16 %7, %19, %15, %7\n\t" "v_mfma_f32_16x16x32_bf16 %5, %19, %13, %5\n\t" "v_mfma_f32_16x16x32_bf16 %5, %18, %12, %5\n\t" "v_mfma_f32_16x16x32_bf16 %3, %18, %10, %3\n\t" "v_mfma_f32_16x16x32_bf16 %3, %19, %11, %3\n\t" "v_mfma_f32_16x16x32_bf16 %1, %19, %9, %1\n\t" "v_mfma_f32_16x16x32_bf16 %1, %18, %8, %1\n\t" "s_setprio 0\n\t"); } while (0)
#define ROLE_SETUP() \
  int tid_ = threadIdx.x; asm volatile("" : "+v"(tid_)); const int lane = tid_ & 63, fr = lane & 15, fq = lane >> 4; \
  unsigned aoff0, aoff1, boff0, boff1; \
  { const unsigned rb = (unsigned)((wid >> 2) * 64 + fr) * 128u; const unsigned s = (unsigned)(fr >> 1); \
    aoff0 = lbase + LDS_A + rb + (((unsigned)fq ^ s) << 4); aoff1 = lbase + LDS_A + rb + (((unsigned)(4 + fq) ^ s) << 4); \
    const unsigned q = (unsigned)(lane & 15) >> 2, p = (unsigned)lane & 3u, g = (unsigned)fq, f = q | ((g & 1u) << 2); \
    const unsigned kb = (8u * g + q) * 256u + 8u * (p & 1u); \
    boff0 = lbase + LDS_B + kb + ((((unsigned)(4 * (wid & 3)) + (p >> 1)) ^ (2u * f)) << 4); \
    boff1 = lbase + LDS_B + kb + ((((unsigned)(4 * (wid & 3) + 2) + (p >> 1)) ^ (2u * f)) << 4); } \
  f32x4 acc[2][2][4][2]; \
  _Pragma("unroll") for (int i0 = 0; i0 < 2; ++i0) _Pragma("unroll") for (int i1 = 0; i1 < 2; ++i1) \
  _Pragma("unroll") for (int m = 0; m < 4; ++m) _Pragma("unroll") for (int n = 0; n < 2; ++n) acc[i0][i1][m][n] = (f32x4){0.f, 0.f, 0.f, 0.f}; \
  bf16x8 At[4][2]; bf16x4 B0[2][2][2], B1[2][2][2]

namespace {
template <int AI>
__device__ __forceinline__ void store_half(const f32x4 (&acc)[2][2][4][2], char* otile, int wid, bool bad) {
  int tz = threadIdx.x; asm volatile("" : "+v"(tz));
  const int ln = tz & 63, r16 = ln & 15, q4 = ln >> 4;
  char* op = otile + (size_t)((wid >> 2) * 64) * ROWB + (size_t)((wid & 3) * 64);
  auto ro = __builtin_amdgcn_make_buffer_rsrc(op, 0, 192 * ROWB, 0x00020000);
  const unsigned vo = (unsigned)r16 * ROWB + (unsigned)((((q4 & 1) << 4) | ((q4 >> 1) << 3)) * 2);
#pragma unroll
  for (int m = 0; m < 4; ++m)
#pragma unroll
    for (int bj = 0; bj < 2; ++bj) {
      const f32x4 v0 = acc[AI][bj][m][0], v1 = acc[AI][bj][m][1];
      unsigned a0 = cvt_pk_bf16(v0[0], v0[1]), a1 = cvt_pk_bf16(v0[2], v0[3]), b0 = cvt_pk_bf16(v1[0], v1[1]), b1 = cvt_pk_bf16(v1[2], v1[3]);
      auto s0 = __builtin_amdgcn_permlane16_swap(a0, b0, false, false);
      auto s1 = __builtin_amdgcn_permlane16_swap(a1, b1, false, false);
      u32x4 w = {s0[0], s1[0], s0[1], s1[1]};
      if (bad) w = (u32x4){0x7fc07fc0u, 0x7fc07fc0u, 0x7fc07fc0u, 0x7fc07fc0u};
      __builtin_amdgcn_raw_buffer_store_b128(w, ro, vo + bj * 256, (AI * 128 + m * 16) * ROWB, 16);
    }
}
#define READ_BAD() unsigned badv; asm volatile("ds_read_b32 %0, %1" : "=v"(badv) : "v"(lbase + LDS_M) : "memory")
#define TAKE_BAD() asm volatile("" : "+v"(badv) :: "memory"); const bool bad = badv != 0u
#define MFMA_DRAIN() asm volatile("s_nop 7\n\ts_nop 7\n\ts_nop 3" ::: "memory")

#define P1(bb, ST, WE, WL) do { if (SFIRST) { ST; } LDB(B0, bb, 0); LDA(bb, 0); LDB(B1, bb, 1); if (!SFIRST) { ST; } WE; WAIT_ABB(); BAR; MMA2(0); WL; BAR; } while (0)
#define P2(bb, ST, WE, WL) do { LDA(bb, 1); ST; WE; WAIT_A(); BAR; MMA2(1); WL; BAR; } while (0)
#define LAST_PHASE(G0) do { TAKE_BAD(); LDA(1, 1); store_half<0>(acc, otile, wid, bad); WAIT_A(); BAR; MMA2(1); if (G0) BAR; MFMA_DRAIN(); store_half<1>(acc, otile, wid, bad); } while (0)
template <int NG>
__device__ __forceinline__ void worker_a(unsigned lbase, const char* xs, const char* xg, const char* ctr, char* otile, bool local, int wid) {
  ROLE_SETUP();
  const int p0 = wid == 1 ? 0 : wid * 5 - 4;
  const unsigned vA0 = local ? off_x(p0, lane) : (unsigned)(p0 * 1024 + lane * 16), vA1 = local ? off_x(p0 + 1, lane) : vA0 + 1024u;
  const unsigned ldsw = rfl(lbase + (unsigned)p0 * 1024u);
  const size_t qA = local ? 16 * (size_t)ROWB : 2048, hA = local ? 128 * (size_t)ROWB : HT, kA = local ? 128 : XG_KT;
#define STAGE_A(bb, h, gp) do { const char* g0_ = (gp) + (h) * hA; const char* g1_ = g0_ + qA; const char* g2_ = g1_ + qA; \
    GLDS(vA0, g0_, ldsw, LDS_A + ((bb) * 2 + (h)) * HT); GLDS(vA1, g0_, ldsw, LDS_A + ((bb) * 2 + (h)) * HT + 1024); \
    GLDS(vA0, g1_, ldsw, LDS_A + ((bb) * 2 + (h)) * HT + 2048); GLDS(vA1, g1_, ldsw, LDS_A + ((bb) * 2 + (h)) * HT + 3072); \
    GLDS(vA0, g2_, ldsw, LDS_A + ((bb) * 2 + (h)) * HT + 4096); if (NG == 6) GLDS(vA1, g2_, ldsw, LDS_A + ((bb) * 2 + (h)) * HT + 5120); } while (0)
  unsigned cv; { unsigned z = 0; asm volatile("global_load_dword %0, %1, %2 sc1\n\ts_waitcnt vmcnt(0)" : "=&v"(cv) : "v"(z), "s"(ctr) : "memory"); }
  const char* pA = local ? xs : xg + ((rfl(cv) + 1u) & 1u) * XG_PAR;
  BAR; STAGE_A(0, 0, pA);
  BAR; STAGE_A(0, 1, pA);
  BAR;
  WAIT_V(NG); BAR;
#pragma unroll 1
  for (int t = 0; t < NT - 2; t += 2) {
    const char* a1 = pA + (size_t)(t + 1) * kA; const char* a2 = a1 + kA;
    P1(0, STAGE_A(1, 0, a1), , WAIT_V(NG));
    P2(0, STAGE_A(1, 1, a1), , WAIT_V(NG));
    P1(1, STAGE_A(0, 0, a2), , WAIT_V(NG));
    P2(1, STAGE_A(0, 1, a2), , WAIT_V(NG));
  }
  P1(0, STAGE_A(1, 0, pA + (size_t)(NT - 1) * kA), , WAIT_V(NG));
  P2(0, STAGE_A(1, 1, pA + (size_t)(NT - 1) * kA), , WAIT_V(NG));
  READ_BAD();
  P1(1, , , WAIT_V(0));
  LAST_PHASE(true);
#undef STAGE_A
}
__device__ __forceinline__ void worker_b(unsigned lbase, const char* pB, char* otile, int wid) {
  ROLE_SETUP();
  const unsigned vB0 = off_w(wid - 4, lane), vB1 = vB0 + 16u * ROWB, vB2 = vB0 + 32u * ROWB, vB3 = vB0 + 48u * ROWB;
  const unsigned ldsw = rfl(lbase + (unsigned)(wid - 4) * 1024u);
  constexpr size_t kB = 64 * (size_t)ROWB;
#define STAGE_BH(bb, h, g_) do { GLDS(vB0, g_, ldsw, LDS_B + ((bb) * 2 + (h)) * HT); GLDS(vB1, g_, ldsw, LDS_B + ((bb) * 2 + (h)) * HT + 4096); \
    GLDS(vB2, g_, ldsw, LDS_B + ((bb) * 2 + (h)) * HT + 8192); GLDS(vB3, g_, ldsw, LDS_B + ((bb) * 2 + (h)) * HT + 12288); } while (0)
#define STAGE_B(bb, gp) do { const char* h0_ = (gp); const char* h1_ = h0_ + 256; STAGE_BH(bb, 0, h0_); STAGE_BH(bb, 1, h1_); } while (0)
  STAGE_B(0, pB); STAGE_B(1, pB + kB);
  BAR; BAR; BAR;
  WAIT_V(8); BAR;
  BAR;
#pragma unroll 1
  for (int t = 0; t < NT - 2; t += 2) {
    const char* b2 = pB + (size_t)(t + 2) * kB;
    P1(0, , , );
    P2(0, STAGE_B(0, b2), WAIT_V(8), );
    P1(1, , , );
    P2(1, STAGE_B(1, b2 + kB), WAIT_V(8), );
  }
  P1(0, , , );
  P2(0, , WAIT_V(0), );
  READ_BAD();
  P1(1, , , );
  LAST_PHASE(false);
#undef STAGE_B
#undef STAGE_BH
}
}

__global__ __launch_bounds__(512, 2) __attribute__((amdgpu_num_vgpr(108), amdgpu_num_sgpr(62))) void ag_gemm_kernel(Args a) {
  extern __shared__ __attribute__((aligned(16))) char smem[];
  LAS char* lds = (LAS char*)smem;
  asm volatile("v_mov_b32 v239, 0\n\tv_mov_b32 v236, 0" ::: "v222", "v255", "s56", "s101");
  const int wid = __builtin_amdgcn_readfirstlane(threadIdx.x >> 6);
  const int b = blockIdx.x, rank = a.rank;
  const int xcd = b & 7, idx = b >> 3;
  const int pm = 4 * (xcd >> 1) + (idx & 3), pn = 8 * (xcd & 1) + (idx >> 2);
  const int src = pm >> 1, strip = pm & 1;
  const bool local = (src == rank);
  char* myws = a.ws[rank];
  const char* ctr = myws + WS_CTR + (size_t)b * 128;

  const unsigned lbase = (unsigned)(size_t)lds;
  char* otile = a.out + (size_t)(pm * 256) * ROWB + (size_t)pn * 512;
  const char* xs = a.x + (size_t)strip * 256 * ROWB;
  const char* xg = myws + (size_t)src * XG_SRC + (size_t)strip * 2 * HT;
  const char* pB = a.w + (size_t)pn * 512;

  if (wid >= 4) worker_b(lbase, pB, otile, wid);
  else if (wid >= 2) worker_a<5>(lbase, xs, xg, ctr, otile, local, wid);
  else if (wid == 1) worker_a<6>(lbase, xs, xg, ctr, otile, local, wid);
  else {
    const unsigned t_start = now_sync();
    ROLE_SETUP();
    const unsigned jj = (unsigned)((((((idx >> 1) ^ rank) & 1) << 4) | ((idx >> 2) << 1) | (idx & 1)) * 8 + xcd);
    const unsigned vx0 = off_x(0, lane), vx1 = off_x(1, lane);
    unsigned cv; { unsigned z = 0; asm volatile("global_load_dword %0, %1, %2 sc1" : "=v"(cv) : "v"(z), "s"(ctr) : "memory"); }
    const unsigned idA = (jj / 7u) * 8u + jj % 7u;
    { const char* sa = a.x + (size_t)(((idA << 15) & 0x3c0000u) | (idA & 0x1f80u));
      asm volatile("v_mov_b32 v230, %0\n\tv_mov_b32 v229, %1\n\tv_add_u32 v225, 0x20000, %0\n\tv_add_u32 v223, 0x20000, %1\n\t"
                   "global_load_dwordx4 v[240:243], v230, %2\n\tglobal_load_dwordx4 v[244:247], v229, %2\n\tglobal_load_dwordx4 v[248:251], v225, %2\n\tglobal_load_dwordx4 v[252:255], v223, %2"
                   :: "v"(vx0), "v"(vx1), "s"(sa) : "memory"); }
    char* pw = a.ws[(rank + 1) & 7];
#pragma unroll
    for (int q = 1; q < 7; ++q) { char* c = a.ws[(rank + 1 + q) & 7]; asm volatile("" : "+s"(c)); if (lane == q) pw = c; }
    *(LAS unsigned*)(lds + LDS_M) = 0u;
    const unsigned foff = (unsigned)(src * 65536 + strip * 256);
    const char* flags = myws + WS_FLAGS + foff;
    const unsigned g = ((unsigned)lane >> 2) & 7u, gcode = (g >> 1) * 4u + (g & 1u);
    { const unsigned pbase = gcode * 128u + ((unsigned)lane & 3u) * 32u;
      asm volatile("v_mov_b32 v231, %0\n\tv_mov_b32 v228, %1\n\tv_mov_b32 v237, %1\n\tv_mov_b32 v227, %2\n\tv_mov_b32 v226, %3"
                   :: "v"((unsigned)lane * 16u), "v"(pbase), "v"(g), "s"((unsigned)(WS_TMO - WS_FLAGS) - foff) : "memory"); }
    { const int due0 = T0 + (int)((jj * D1) >> 8);
      asm volatile("s_mov_b32 s70, 0\n\ts_mov_b32 s71, 0\n\ts_mov_b32 s72, 0x7fffffff\n\ts_mov_b32 s75, 0\n\t"
                   "s_mov_b32 s73, %0\n\ts_mov_b32 s74, %1\n\ts_mov_b32 s86, 7\n\ts_sub_u32 s66, 0, %2\n\t"
                   "s_mov_b32 s89, %3\n\ts_mov_b64 s[90:91], %4\n\ts_mov_b64 s[92:93], %5\n\ts_mov_b32 s67, 0"
                   :: "s"(NOCOMM ? NEVER : due0), "s"(idA), "s"(rfl((local || NOCOMM || NOWAIT) ? 0u : 1u)),
                      "s"(t_start), "s"(a.x), "s"(flags) : "memory", "scc"); }
    asm volatile("" : TIE_C(0, 0), TIE_C(0, 1)); asm volatile("" : TIE_C(1, 0), TIE_C(1, 1));
    asm volatile("s_waitcnt vmcnt(0)" : "+v"(cv) :: "memory");
    const unsigned call = rfl(cv) + 1u;
    { const unsigned xgo = (call & 1u) * (unsigned)XG_PAR + (unsigned)rank * (unsigned)XG_SRC; const u64 pd = (u64)(pw + xgo);
      asm volatile("v_mov_b32 v234, %0\n\tv_mov_b32 v235, %1\n\ts_mov_b32 s88, %2\n\ts_mov_b32 s68, %3\n\tv_mov_b32 v233, %4\n\tv_mov_b32 v224, %4\n\tv_mov_b32 v232, %5\n\ts_nop 1\n\t" "s_mov_b32 s87, 0\n\t" "s_mov_b32 s63, 0\n\t" "s_mov_b32 s85, 28\n\t" "s_mov_b32 s69, 0\n\t" "s_add_i32 s98, s70, 6\n\t" "s_add_i32 s70, s70, s63\n\t" "s_mov_b32 s63, 0\n\t" "s_cmp_lt_u32 s74, " STR(ID1) "\n\t" "s_cselect_b32 s99, 0, (" STR(D2) "-" STR(D1) ")\n\t" "s_add_i32 s99, s99, s73\n\t" "s_addk_i32 s99, " STR(D1) "\n\t" "s_cmp_lt_u32 s69, 0x2000\n\t" "s_cselect_b32 s99, s99, 0x7fffffff\n\t" "s_cmp_lg_u32 s87, 0\n\t" "s_cselect_b32 s72, s98, s72\n\t" "s_cselect_b32 s75, s74, s75\n\t" "s_cselect_b32 s71, s70, s71\n\t" "s_cselect_b32 s73, s99, s73\n\t" "s_cselect_b32 s74, s69, s74\n\t" "s_mov_b32 s87, 0\n\t" "s_and_b32 s98, s75, 7\n\t" "s_lshl_b32 s99, s75, 2\n\t" "v_readlane_b32 s82, v234, s98\n\t" "v_readlane_b32 s83, v235, s98\n\t" "s_and_b32 s99, s99, 0x7fe0\n\t" "s_lshr_b32 s98, s75, 5\n\t" "s_add_i32 s98, s98, s88\n\t" "s_add_u32 s82, s82, s99\n\t" "s_addc_u32 s83, s83, 0\n\t" "v_mov_b32 v238, s98\n\t" "s_mov_b32 s70, 0\n\t" "s_add_i32 s85, s85, 4\n\t" "s_and_b32 s84, s68, s66\n\t" "s_and_b32 s85, s85, 28\n\t" "s_mov_b32 s56, " STR(WF) "\n\t" "s_mov_b32 s57, " STR(WF) "\n\t" "s_lshl_b64 s[56:57], s[56:57], s85\n\t" "s_and_b32 s59, s57, s66\n\t" "s_memrealtime s[96:97]\n\t" "s_and_b32 s98, s74, 7\n\t" "s_and_b32 s99, s74, 0x1ff8\n\t" "v_readlane_b32 s78, v234, s98\n\t" "v_readlane_b32 s79, v235, s98\n\t" "s_lshl_b32 s99, s99, 9\n\t" "s_cmp_ge_u32 s98, 3\n\t" "s_addc_u32 s69, s74, 292\n\t" "s_add_u32 s78, s78, s99\n\t" "s_addc_u32 s79, s79, 0\n\t" "s_lshl_b32 s98, s69, 15\n\t" "s_and_b32 s99, s69, 0x1f80\n\t" "s_and_b32 s98, s98, 0x3c0000\n\t" "s_or_b32 s98, s98, s99\n\t" "s_add_u32 s80, s90, s98\n\t" "s_addc_u32 s81, s91, 0\n\t" "s_waitcnt lgkmcnt(0)\n\t" "s_sub_i32 s62, s96, s89\n\t" "s_getreg_b32 s98, hwreg(HW_REG_IB_STS, 22, 2)\n\t" "s_getreg_b32 s99, hwreg(HW_REG_IB_STS, 0, 4)\n\t" "s_lshl4_add_u32 s98, s98, s99\n\t" "s_sub_i32 s76, s70, s98\n\t" "s_cmp_ge_i32 s76, s71\n\t" "s_cselect_b32 s77, s73, 0x7fffffff\n\t"
                   :: "v"((unsigned)pd), "v"((unsigned)(pd >> 32)), "s"(call * 256u + 1u), "s"(call * 256u + 1u + 2u * (unsigned)strip),
                      "v"(call * 256u + 1u + 2u * (unsigned)strip + gcode), "s"((unsigned)WS_FLAGS + (unsigned)rank * 65536u - xgo) : "memory", "scc"); }
    asm volatile("s_branch ag_slow_end\n\t" "ag_tr_a:\n\t" "s_call_b64 s[94:95], ag_slow_sub\n\t" "s_branch ag_bk_a\n\t" "ag_tr_b:\n\t" "s_call_b64 s[94:95], ag_slow_sub\n\t" "s_branch ag_bk_b\n\t" "ag_tr_c:\n\t" "s_call_b64 s[94:95], ag_slow_sub\n\t" "s_branch ag_bk_c\n\t" "ag_tr_d:\n\t" "s_call_b64 s[94:95], ag_slow_sub\n\t" "s_branch ag_bk_d\n\t" "ag_tr_e:\n\t" "s_call_b64 s[94:95], ag_slow_sub\n\t" "s_branch ag_bk_e\n\t" "ag_tr_p0:\n\t" "s_call_b64 s[94:95], ag_slow_sub\n\t" "s_branch ag_bk_p0\n\t" "ag_tr_p1:\n\t" "s_call_b64 s[94:95], ag_slow_sub\n\t" "s_branch ag_bk_p1\n\t" "ag_tr_p2:\n\t" "s_call_b64 s[94:95], ag_slow_sub\n\t" "s_branch ag_bk_p2\n\t" "ag_slow_sub:\n\t" "s_add_i32 s98, s70, 6\n\t" "s_add_i32 s70, s70, s63\n\t" "s_mov_b32 s63, 0\n\t" "s_cmp_lt_u32 s74, " STR(ID1) "\n\t" "s_cselect_b32 s99, 0, (" STR(D2) "-" STR(D1) ")\n\t" "s_add_i32 s99, s99, s73\n\t" "s_addk_i32 s99, " STR(D1) "\n\t" "s_cmp_lt_u32 s69, 0x2000\n\t" "s_cselect_b32 s99, s99, 0x7fffffff\n\t" "s_cmp_lg_u32 s87, 0\n\t" "s_cselect_b32 s72, s98, s72\n\t" "s_cselect_b32 s75, s74, s75\n\t" "s_cselect_b32 s71, s70, s71\n\t" "s_cselect_b32 s73, s99, s73\n\t" "s_cselect_b32 s74, s69, s74\n\t" "s_mov_b32 s87, 0\n\t" "s_and_b32 s98, s75, 7\n\t" "s_lshl_b32 s99, s75, 2\n\t" "v_readlane_b32 s82, v234, s98\n\t" "v_readlane_b32 s83, v235, s98\n\t" "s_and_b32 s99, s99, 0x7fe0\n\t" "s_lshr_b32 s98, s75, 5\n\t" "s_add_i32 s98, s98, s88\n\t" "s_add_u32 s82, s82, s99\n\t" "s_addc_u32 s83, s83, 0\n\t" "v_mov_b32 v238, s98\n\t" "s_mov_b32 s56, " STR(WS) "\n\t" "s_mov_b32 s57, " STR(WS) "\n\t" "s_lshl_b64 s[56:57], s[56:57], s85\n\t" "s_and_b32 s59, s57, s66\n\t" "s_memrealtime s[96:97]\n\t" "s_and_b32 s98, s74, 7\n\t" "s_and_b32 s99, s74, 0x1ff8\n\t" "v_readlane_b32 s78, v234, s98\n\t" "v_readlane_b32 s79, v235, s98\n\t" "s_lshl_b32 s99, s99, 9\n\t" "s_cmp_ge_u32 s98, 3\n\t" "s_addc_u32 s69, s74, 292\n\t" "s_add_u32 s78, s78, s99\n\t" "s_addc_u32 s79, s79, 0\n\t" "s_lshl_b32 s98, s69, 15\n\t" "s_and_b32 s99, s69, 0x1f80\n\t" "s_and_b32 s98, s98, 0x3c0000\n\t" "s_or_b32 s98, s98, s99\n\t" "s_add_u32 s80, s90, s98\n\t" "s_addc_u32 s81, s91, 0\n\t" "s_waitcnt lgkmcnt(0)\n\t" "s_sub_i32 s62, s96, s89\n\t" "s_getreg_b32 s98, hwreg(HW_REG_IB_STS, 22, 2)\n\t" "s_getreg_b32 s99, hwreg(HW_REG_IB_STS, 0, 4)\n\t" "s_lshl4_add_u32 s98, s98, s99\n\t" "s_sub_i32 s76, s70, s98\n\t" "s_cmp_ge_i32 s76, s71\n\t" "s_cselect_b32 s77, s73, 0x7fffffff\n\t" "ag_slow_loop:\n\t" "s_add_i32 s67, s67, 1\n\t" "v_cmp_gt_u32 s[64:65], v224, v239\n\t" "s_cmp_ge_i32 s76, s72\n\t" "s_cselect_b64 exec, 1, 0\n\t" "global_store_dword v232, v238, s[82:83] sc0 sc1\n\t" "s_cselect_b32 s72, 0x7fffffff, s72\n\t" "s_and_b32 exec_lo, s64, s59\n\t" "global_load_dword v239, v237, s[92:93] sc0 sc1\n\t" "s_cmp_ge_i32 s62, s77\n\t" "s_mov_b64 exec, -1\n\t" "s_cselect_b32 s87, 1, 0\n\t" "s_cselect_b32 s63, 10, 2\n\t" "s_cbranch_scc0 1f\n\t" "global_store_dwordx4 v231, v[240:243], s[78:79] sc0 sc1\n\t" "global_store_dwordx4 v231, v[244:247], s[78:79] offset:1024 sc0 sc1\n\t" "global_store_dwordx4 v231, v[248:251], s[78:79] offset:2048 sc0 sc1\n\t" "global_store_dwordx4 v231, v[252:255], s[78:79] offset:3072 sc0 sc1\n\t" "global_load_dwordx4 v[240:243], v230, s[80:81]\n\t" "global_load_dwordx4 v[244:247], v229, s[80:81]\n\t" "global_load_dwordx4 v[248:251], v225, s[80:81]\n\t" "global_load_dwordx4 v[252:255], v223, s[80:81]\n\t" "1:\n\t" "global_load_dword v236, v226, s[92:93] sc1\n\t" "s_add_i32 s98, s70, 6\n\t" "s_add_i32 s70, s70, s63\n\t" "s_mov_b32 s63, 0\n\t" "s_cmp_lt_u32 s74, " STR(ID1) "\n\t" "s_cselect_b32 s99, 0, (" STR(D2) "-" STR(D1) ")\n\t" "s_add_i32 s99, s99, s73\n\t" "s_addk_i32 s99, " STR(D1) "\n\t" "s_cmp_lt_u32 s69, 0x2000\n\t" "s_cselect_b32 s99, s99, 0x7fffffff\n\t" "s_cmp_lg_u32 s87, 0\n\t" "s_cselect_b32 s72, s98, s72\n\t" "s_cselect_b32 s75, s74, s75\n\t" "s_cselect_b32 s71, s70, s71\n\t" "s_cselect_b32 s73, s99, s73\n\t" "s_cselect_b32 s74, s69, s74\n\t" "s_mov_b32 s87, 0\n\t" "s_add_i32 s70, s70, 1\n\t" "s_and_b32 s98, s75, 7\n\t" "s_lshl_b32 s99, s75, 2\n\t" "v_readlane_b32 s82, v234, s98\n\t" "v_readlane_b32 s83, v235, s98\n\t" "s_and_b32 s99, s99, 0x7fe0\n\t" "s_lshr_b32 s98, s75, 5\n\t" "s_add_i32 s98, s98, s88\n\t" "s_add_u32 s82, s82, s99\n\t" "s_addc_u32 s83, s83, 0\n\t" "v_mov_b32 v238, s98\n\t" "s_sleep " STR(SLP) "\n\t" "s_memrealtime s[96:97]\n\t" "s_and_b32 s98, s74, 7\n\t" "s_and_b32 s99, s74, 0x1ff8\n\t" "v_readlane_b32 s78, v234, s98\n\t" "v_readlane_b32 s79, v235, s98\n\t" "s_lshl_b32 s99, s99, 9\n\t" "s_cmp_ge_u32 s98, 3\n\t" "s_addc_u32 s69, s74, 292\n\t" "s_add_u32 s78, s78, s99\n\t" "s_addc_u32 s79, s79, 0\n\t" "s_lshl_b32 s98, s69, 15\n\t" "s_and_b32 s99, s69, 0x1f80\n\t" "s_and_b32 s98, s98, 0x3c0000\n\t" "s_or_b32 s98, s98, s99\n\t" "s_add_u32 s80, s90, s98\n\t" "s_addc_u32 s81, s91, 0\n\t" "s_waitcnt lgkmcnt(0)\n\t" "s_sub_i32 s62, s96, s89\n\t" "s_getreg_b32 s98, hwreg(HW_REG_IB_STS, 22, 2)\n\t" "s_getreg_b32 s99, hwreg(HW_REG_IB_STS, 0, 4)\n\t" "s_lshl4_add_u32 s98, s98, s99\n\t" "s_sub_i32 s76, s70, s98\n\t" "s_cmp_ge_i32 s76, s71\n\t" "s_cselect_b32 s77, s73, 0x7fffffff\n\t" "v_readfirstlane_b32 s98, v236\n\t" "s_cmp_gt_u32 s62, " STR(DEADLINE) "\n\t" "s_cselect_b32 s98, s88, s98\n\t" "s_cmp_eq_u32 s98, s88\n\t" "s_cbranch_scc1 ag_slow_dead\n\t" "v_cmp_le_u32 s[100:101], s84, v239\n\t" "s_lshr_b32 s98, s100, s85\n\t" "s_andn2_b32 s98, 15, s98\n\t" "s_cbranch_scc1 ag_slow_loop\n\t" "s_setpc_b64 s[94:95]\n\t" "ag_slow_dead:\n\t" "s_mov_b32 s66, 0\n\t" "s_mov_b32 s59, 0\n\t" "s_mov_b32 s58, 0\n\t" "s_mov_b32 s84, 0\n\t" "v_mov_b32 v224, s88\n\t" "v_mov_b32 v237, " STR(LDSBAD) "\n\t" "global_store_dword v226, v224, s[92:93] sc1\n\t" "ds_write_b32 v237, v224\n\t" "s_add_i32 s70, s70, 1\n\t" "s_waitcnt lgkmcnt(0)\n\t" "s_setpc_b64 s[94:95]\n\t" "ag_slow_end:\n\t" ::: "memory", "scc");
#define ENG(TEXT) asm volatile(TEXT ::: "memory", "scc")
#define PST(x)
    PST(st_pa);
    ENG("s_cmp_lg_u32 s66, 0\n\t" "s_cbranch_scc1 ag_pre_end\n\t" "s_cmp_eq_u32 s73, 0x7fffffff\n\t" "s_cbranch_scc1 ag_pre_end\n\t" "s_mov_b32 s61, s74\n\t" "ag_pre:\n\t" "s_call_b64 s[94:95], ag_slow_sub\n\t" "s_cmp_eq_u32 s74, s61\n\t" "s_cbranch_scc1 ag_pre\n\t" "s_cmp_lg_u32 s72, 0x7fffffff\n\t" "s_cbranch_scc1 ag_pre\n\t" "ag_pre_end:\n\t");
    ENG("v_cmp_le_u32 s[100:101], s84, v239\n\t" "s_lshr_b32 s98, s100, s85\n\t" "s_andn2_b32 s98, 15, s98\n\t" "s_cbranch_scc1 ag_tr_p0\n\t" "ag_bk_p0:\n\t"); PST(st_p0); BAR; ENG("s_add_i32 s86, s86, 1\n\t" "s_add_i32 s68, s68, 1\n\t" "v_sub_u32 v224, s86, v227\n\t" "v_lshrrev_b32 v224, 3, v224\n\t" "v_lshl_add_u32 v237, v224, 11, v228\n\t" "v_lshl_add_u32 v224, v224, 4, v233\n\t" "s_add_i32 s85, s85, 4\n\t" "s_and_b32 s84, s68, s66\n\t" "s_and_b32 s85, s85, 28\n\t" "s_mov_b32 s56, " STR(WF) "\n\t" "s_mov_b32 s57, " STR(WF) "\n\t" "s_lshl_b64 s[56:57], s[56:57], s85\n\t" "s_and_b32 s59, s57, s66\n\t");
    ENG("v_cmp_le_u32 s[100:101], s84, v239\n\t" "s_lshr_b32 s98, s100, s85\n\t" "s_andn2_b32 s98, 15, s98\n\t" "s_cbranch_scc1 ag_tr_p1\n\t" "ag_bk_p1:\n\t"); PST(st_p1); BAR; ENG("s_add_i32 s86, s86, 1\n\t" "s_add_i32 s68, s68, 3\n\t" "v_sub_u32 v224, s86, v227\n\t" "v_lshrrev_b32 v224, 3, v224\n\t" "v_lshl_add_u32 v237, v224, 11, v228\n\t" "v_lshl_add_u32 v224, v224, 4, v233\n\t" "s_add_i32 s85, s85, 4\n\t" "s_and_b32 s84, s68, s66\n\t" "s_and_b32 s85, s85, 28\n\t" "s_mov_b32 s56, " STR(WF) "\n\t" "s_mov_b32 s57, " STR(WF) "\n\t" "s_lshl_b64 s[56:57], s[56:57], s85\n\t" "s_and_b32 s59, s57, s66\n\t");
    BAR;
    ENG("v_cmp_le_u32 s[100:101], s84, v239\n\t" "s_lshr_b32 s98, s100, s85\n\t" "s_andn2_b32 s98, 15, s98\n\t" "s_cbranch_scc1 ag_tr_p2\n\t" "ag_bk_p2:\n\t"); PST(st_p2); PST(st_p3);
    ENG("s_mov_b32 s56, 0xff\n\t" "s_mov_b32 s57, 0xff\n\t" "s_lshl_b64 s[56:57], s[56:57], s85\n\t" "s_and_b32 s59, s57, s66\n\t" "s_mov_b32 s56, 0xff0\n\t" "s_mov_b32 s57, 0xff0\n\t" "s_lshl_b64 s[56:57], s[56:57], s85\n\t" "s_and_b32 s58, s57, s66\n\t");
    BAR;
#define W0P1(bb) do { if (APOS == 0) ENG("v_cmp_gt_u32 s[64:65], v224, v239\n\t" "s_cmp_ge_i32 s76, s72\n\t" "s_cselect_b64 exec, 1, 0\n\t" "global_store_dword v232, v238, s[82:83] sc0 sc1\n\t" "s_cselect_b32 s72, 0x7fffffff, s72\n\t" "s_and_b32 exec_lo, s64, s58\n\t" "global_load_dword v239, v237, s[92:93] sc0 sc1\n\t" "s_cmp_ge_i32 s62, s77\n\t" "s_mov_b64 exec, -1\n\t" "s_cselect_b32 s87, 1, 0\n\t" "s_cselect_b32 s63, 10, 2\n\t" "s_cbranch_scc0 1f\n\t" "global_store_dwordx4 v231, v[240:243], s[78:79] sc0 sc1\n\t" "global_store_dwordx4 v231, v[244:247], s[78:79] offset:1024 sc0 sc1\n\t" "global_store_dwordx4 v231, v[248:251], s[78:79] offset:2048 sc0 sc1\n\t" "global_store_dwordx4 v231, v[252:255], s[78:79] offset:3072 sc0 sc1\n\t" "global_load_dwordx4 v[240:243], v230, s[80:81]\n\t" "global_load_dwordx4 v[244:247], v229, s[80:81]\n\t" "global_load_dwordx4 v[248:251], v225, s[80:81]\n\t" "global_load_dwordx4 v[252:255], v223, s[80:81]\n\t" "1:\n\t"); LDB(B0, bb, 0); if (APOS == 1) ENG("v_cmp_gt_u32 s[64:65], v224, v239\n\t" "s_cmp_ge_i32 s76, s72\n\t" "s_cselect_b64 exec, 1, 0\n\t" "global_store_dword v232, v238, s[82:83] sc0 sc1\n\t" "s_cselect_b32 s72, 0x7fffffff, s72\n\t" "s_and_b32 exec_lo, s64, s58\n\t" "global_load_dword v239, v237, s[92:93] sc0 sc1\n\t" "s_cmp_ge_i32 s62, s77\n\t" "s_mov_b64 exec, -1\n\t" "s_cselect_b32 s87, 1, 0\n\t" "s_cselect_b32 s63, 10, 2\n\t" "s_cbranch_scc0 1f\n\t" "global_store_dwordx4 v231, v[240:243], s[78:79] sc0 sc1\n\t" "global_store_dwordx4 v231, v[244:247], s[78:79] offset:1024 sc0 sc1\n\t" "global_store_dwordx4 v231, v[248:251], s[78:79] offset:2048 sc0 sc1\n\t" "global_store_dwordx4 v231, v[252:255], s[78:79] offset:3072 sc0 sc1\n\t" "global_load_dwordx4 v[240:243], v230, s[80:81]\n\t" "global_load_dwordx4 v[244:247], v229, s[80:81]\n\t" "global_load_dwordx4 v[248:251], v225, s[80:81]\n\t" "global_load_dwordx4 v[252:255], v223, s[80:81]\n\t" "1:\n\t"); LDA(bb, 0); if (APOS == 2) ENG("v_cmp_gt_u32 s[64:65], v224, v239\n\t" "s_cmp_ge_i32 s76, s72\n\t" "s_cselect_b64 exec, 1, 0\n\t" "global_store_dword v232, v238, s[82:83] sc0 sc1\n\t" "s_cselect_b32 s72, 0x7fffffff, s72\n\t" "s_and_b32 exec_lo, s64, s58\n\t" "global_load_dword v239, v237, s[92:93] sc0 sc1\n\t" "s_cmp_ge_i32 s62, s77\n\t" "s_mov_b64 exec, -1\n\t" "s_cselect_b32 s87, 1, 0\n\t" "s_cselect_b32 s63, 10, 2\n\t" "s_cbranch_scc0 1f\n\t" "global_store_dwordx4 v231, v[240:243], s[78:79] sc0 sc1\n\t" "global_store_dwordx4 v231, v[244:247], s[78:79] offset:1024 sc0 sc1\n\t" "global_store_dwordx4 v231, v[248:251], s[78:79] offset:2048 sc0 sc1\n\t" "global_store_dwordx4 v231, v[252:255], s[78:79] offset:3072 sc0 sc1\n\t" "global_load_dwordx4 v[240:243], v230, s[80:81]\n\t" "global_load_dwordx4 v[244:247], v229, s[80:81]\n\t" "global_load_dwordx4 v[248:251], v225, s[80:81]\n\t" "global_load_dwordx4 v[252:255], v223, s[80:81]\n\t" "1:\n\t"); LDB(B1, bb, 1); if (APOS == 3) ENG("v_cmp_gt_u32 s[64:65], v224, v239\n\t" "s_cmp_ge_i32 s76, s72\n\t" "s_cselect_b64 exec, 1, 0\n\t" "global_store_dword v232, v238, s[82:83] sc0 sc1\n\t" "s_cselect_b32 s72, 0x7fffffff, s72\n\t" "s_and_b32 exec_lo, s64, s58\n\t" "global_load_dword v239, v237, s[92:93] sc0 sc1\n\t" "s_cmp_ge_i32 s62, s77\n\t" "s_mov_b64 exec, -1\n\t" "s_cselect_b32 s87, 1, 0\n\t" "s_cselect_b32 s63, 10, 2\n\t" "s_cbranch_scc0 1f\n\t" "global_store_dwordx4 v231, v[240:243], s[78:79] sc0 sc1\n\t" "global_store_dwordx4 v231, v[244:247], s[78:79] offset:1024 sc0 sc1\n\t" "global_store_dwordx4 v231, v[248:251], s[78:79] offset:2048 sc0 sc1\n\t" "global_store_dwordx4 v231, v[252:255], s[78:79] offset:3072 sc0 sc1\n\t" "global_load_dwordx4 v[240:243], v230, s[80:81]\n\t" "global_load_dwordx4 v[244:247], v229, s[80:81]\n\t" "global_load_dwordx4 v[248:251], v225, s[80:81]\n\t" "global_load_dwordx4 v[252:255], v223, s[80:81]\n\t" "1:\n\t"); } while (0)
#pragma unroll 1
    for (int t = 0; t < NT - 2; t += 2) {
      W0P1(0);
      WAIT_ABB(); BAR; MMAW(0, 0, B0, "s_setprio 1\n\t" "v_mfma_f32_16x16x32_bf16 %0, %16, %8, %0\n\t" "s_add_i32 s68, s68, 1\n\t" "v_mfma_f32_16x16x32_bf16 %0, %17, %9, %0\n\t" "s_add_i32 s85, s85, 4\n\t" "v_mfma_f32_16x16x32_bf16 %2, %17, %11, %2\n\t" "s_and_b32 s84, s68, s66\n\t" "v_mfma_f32_16x16x32_bf16 %2, %16, %10, %2\n\t" "s_and_b32 s85, s85, 28\n\t" "v_mfma_f32_16x16x32_bf16 %4, %16, %12, %4\n\t" "v_mfma_f32_16x16x32_bf16 %4, %17, %13, %4\n\t" "v_mfma_f32_16x16x32_bf16 %6, %17, %15, %6\n\t" "v_mfma_f32_16x16x32_bf16 %6, %16, %14, %6\n\t" "v_mfma_f32_16x16x32_bf16 %7, %18, %14, %7\n\t" "v_mfma_f32_16x16x32_bf16 %7, %19, %15, %7\n\t" "v_mfma_f32_16x16x32_bf16 %5, %19, %13, %5\n\t" "v_mfma_f32_16x16x32_bf16 %5, %18, %12, %5\n\t" "v_mfma_f32_16x16x32_bf16 %3, %18, %10, %3\n\t" "v_mfma_f32_16x16x32_bf16 %3, %19, %11, %3\n\t" "v_mfma_f32_16x16x32_bf16 %1, %19, %9, %1\n\t" "v_mfma_f32_16x16x32_bf16 %1, %18, %8, %1\n\t"); MMAW(0, 1, B1, "v_mfma_f32_16x16x32_bf16 %0, %16, %8, %0\n\t" "v_mfma_f32_16x16x32_bf16 %0, %17, %9, %0\n\t" "v_mfma_f32_16x16x32_bf16 %2, %17, %11, %2\n\t" "v_mfma_f32_16x16x32_bf16 %2, %16, %10, %2\n\t" "v_mfma_f32_16x16x32_bf16 %4, %16, %12, %4\n\t" "v_mfma_f32_16x16x32_bf16 %4, %17, %13, %4\n\t" "v_mfma_f32_16x16x32_bf16 %6, %17, %15, %6\n\t" "v_mfma_f32_16x16x32_bf16 %6, %16, %14, %6\n\t" "v_mfma_f32_16x16x32_bf16 %7, %18, %14, %7\n\t" "v_mfma_f32_16x16x32_bf16 %7, %19, %15, %7\n\t" "v_mfma_f32_16x16x32_bf16 %5, %19, %13, %5\n\t" "v_mfma_f32_16x16x32_bf16 %5, %18, %12, %5\n\t" "v_mfma_f32_16x16x32_bf16 %3, %18, %10, %3\n\t" "v_cmp_le_u32 s[100:101], s84, v239\n\t" "v_mfma_f32_16x16x32_bf16 %3, %19, %11, %3\n\t" "s_lshr_b32 s98, s100, s85\n\t" "v_mfma_f32_16x16x32_bf16 %1, %19, %9, %1\n\t" "s_andn2_b32 s98, 15, s98\n\t" "v_mfma_f32_16x16x32_bf16 %1, %18, %8, %1\n\t" "s_setprio 0\n\t" "s_cbranch_scc1 ag_tr_a\n\t" "ag_bk_a:\n\t"); BAR;
      LDA(0, 1); ENG("s_add_i32 s68, s68, 3\n\t" "s_add_i32 s85, s85, 4\n\t" "s_and_b32 s84, s68, s66\n\t" "s_and_b32 s85, s85, 28\n\t" "s_add_i32 s86, s86, 2\n\t" "v_sub_u32 v224, s86, v227\n\t" "v_lshrrev_b32 v224, 3, v224\n\t" "v_lshl_add_u32 v237, v224, 11, v228\n\t" "v_lshl_add_u32 v224, v224, 4, v233\n\t" "s_mov_b32 s56, 0xff\n\t" "s_mov_b32 s57, 0xff\n\t" "s_lshl_b64 s[56:57], s[56:57], s85\n\t" "s_and_b32 s59, s57, s66\n\t" "s_mov_b32 s56, 0xff0\n\t" "s_mov_b32 s57, 0xff0\n\t" "s_lshl_b64 s[56:57], s[56:57], s85\n\t" "s_and_b32 s58, s57, s66\n\t" "v_cmp_gt_u32 s[64:65], v224, v239\n\t" "s_mov_b32 exec_hi, 0\n\t" "s_and_b32 exec_lo, s64, s59\n\t" "global_load_dword v239, v237, s[92:93] sc0 sc1\n\t" "s_mov_b64 exec, -1\n\t" "s_add_i32 s98, s70, 6\n\t" "s_add_i32 s70, s70, s63\n\t" "s_mov_b32 s63, 0\n\t" "s_cmp_lt_u32 s74, " STR(ID1) "\n\t" "s_cselect_b32 s99, 0, (" STR(D2) "-" STR(D1) ")\n\t" "s_add_i32 s99, s99, s73\n\t" "s_addk_i32 s99, " STR(D1) "\n\t" "s_cmp_lt_u32 s69, 0x2000\n\t" "s_cselect_b32 s99, s99, 0x7fffffff\n\t" "s_cmp_lg_u32 s87, 0\n\t" "s_cselect_b32 s72, s98, s72\n\t" "s_cselect_b32 s75, s74, s75\n\t" "s_cselect_b32 s71, s70, s71\n\t" "s_cselect_b32 s73, s99, s73\n\t" "s_cselect_b32 s74, s69, s74\n\t" "s_mov_b32 s87, 0\n\t" "s_add_i32 s70, s70, 1\n\t" "s_and_b32 s98, s75, 7\n\t" "s_lshl_b32 s99, s75, 2\n\t" "v_readlane_b32 s82, v234, s98\n\t" "v_readlane_b32 s83, v235, s98\n\t" "s_and_b32 s99, s99, 0x7fe0\n\t" "s_lshr_b32 s98, s75, 5\n\t" "s_add_i32 s98, s98, s88\n\t" "s_add_u32 s82, s82, s99\n\t" "s_addc_u32 s83, s83, 0\n\t" "v_mov_b32 v238, s98\n\t" "s_and_b32 s98, s74, 7\n\t" "s_and_b32 s99, s74, 0x1ff8\n\t" "v_readlane_b32 s78, v234, s98\n\t" "v_readlane_b32 s79, v235, s98\n\t" "s_lshl_b32 s99, s99, 9\n\t" "s_cmp_ge_u32 s98, 3\n\t" "s_addc_u32 s69, s74, 292\n\t" "s_add_u32 s78, s78, s99\n\t" "s_addc_u32 s79, s79, 0\n\t" "s_lshl_b32 s98, s69, 15\n\t" "s_and_b32 s99, s69, 0x1f80\n\t" "s_and_b32 s98, s98, 0x3c0000\n\t" "s_or_b32 s98, s98, s99\n\t" "s_add_u32 s80, s90, s98\n\t" "s_addc_u32 s81, s91, 0\n\t");
      WAIT_A(); BAR; MMAW(1, 0, B0, "s_setprio 1\n\t" "v_mfma_f32_16x16x32_bf16 %0, %16, %8, %0\n\t" "v_mfma_f32_16x16x32_bf16 %0, %17, %9, %0\n\t" "v_mfma_f32_16x16x32_bf16 %2, %17, %11, %2\n\t" "v_mfma_f32_16x16x32_bf16 %2, %16, %10, %2\n\t" "v_mfma_f32_16x16x32_bf16 %4, %16, %12, %4\n\t" "v_mfma_f32_16x16x32_bf16 %4, %17, %13, %4\n\t" "v_mfma_f32_16x16x32_bf16 %6, %17, %15, %6\n\t" "v_mfma_f32_16x16x32_bf16 %6, %16, %14, %6\n\t" "v_mfma_f32_16x16x32_bf16 %7, %18, %14, %7\n\t" "v_mfma_f32_16x16x32_bf16 %7, %19, %15, %7\n\t" "v_mfma_f32_16x16x32_bf16 %5, %19, %13, %5\n\t" "v_mfma_f32_16x16x32_bf16 %5, %18, %12, %5\n\t" "v_mfma_f32_16x16x32_bf16 %3, %18, %10, %3\n\t" "v_mfma_f32_16x16x32_bf16 %3, %19, %11, %3\n\t" "v_mfma_f32_16x16x32_bf16 %1, %19, %9, %1\n\t" "v_mfma_f32_16x16x32_bf16 %1, %18, %8, %1\n\t"); MMAW(1, 1, B1, "v_mfma_f32_16x16x32_bf16 %0, %16, %8, %0\n\t" "v_mfma_f32_16x16x32_bf16 %0, %17, %9, %0\n\t" "s_memrealtime s[96:97]\n\t" "v_mfma_f32_16x16x32_bf16 %2, %17, %11, %2\n\t" "v_mfma_f32_16x16x32_bf16 %2, %16, %10, %2\n\t" "v_mfma_f32_16x16x32_bf16 %4, %16, %12, %4\n\t" "s_getreg_b32 s98, hwreg(HW_REG_IB_STS, 22, 2)\n\t" "v_mfma_f32_16x16x32_bf16 %4, %17, %13, %4\n\t" "s_getreg_b32 s99, hwreg(HW_REG_IB_STS, 0, 4)\n\t" "v_mfma_f32_16x16x32_bf16 %6, %17, %15, %6\n\t" "s_lshl4_add_u32 s98, s98, s99\n\t" "v_mfma_f32_16x16x32_bf16 %6, %16, %14, %6\n\t" "s_sub_i32 s76, s70, s98\n\t" "v_mfma_f32_16x16x32_bf16 %7, %18, %14, %7\n\t" "s_cmp_ge_i32 s76, s71\n\t" "v_mfma_f32_16x16x32_bf16 %7, %19, %15, %7\n\t" "s_cselect_b32 s77, s73, 0x7fffffff\n\t" "v_mfma_f32_16x16x32_bf16 %5, %19, %13, %5\n\t" "s_waitcnt lgkmcnt(0)\n\t" "v_mfma_f32_16x16x32_bf16 %5, %18, %12, %5\n\t" "s_sub_i32 s62, s96, s89\n\t" "v_mfma_f32_16x16x32_bf16 %3, %18, %10, %3\n\t" "v_cmp_le_u32 s[100:101], s84, v239\n\t" "v_mfma_f32_16x16x32_bf16 %3, %19, %11, %3\n\t" "s_lshr_b32 s98, s100, s85\n\t" "v_mfma_f32_16x16x32_bf16 %1, %19, %9, %1\n\t" "s_andn2_b32 s98, 15, s98\n\t" "v_mfma_f32_16x16x32_bf16 %1, %18, %8, %1\n\t" "s_setprio 0\n\t" "s_cbranch_scc1 ag_tr_b\n\t" "ag_bk_b:\n\t"); BAR;
      W0P1(1);
      WAIT_ABB(); BAR; MMAW(0, 0, B0, "s_setprio 1\n\t" "v_mfma_f32_16x16x32_bf16 %0, %16, %8, %0\n\t" "s_add_i32 s68, s68, 1\n\t" "v_mfma_f32_16x16x32_bf16 %0, %17, %9, %0\n\t" "s_add_i32 s85, s85, 4\n\t" "v_mfma_f32_16x16x32_bf16 %2, %17, %11, %2\n\t" "s_and_b32 s84, s68, s66\n\t" "v_mfma_f32_16x16x32_bf16 %2, %16, %10, %2\n\t" "s_and_b32 s85, s85, 28\n\t" "v_mfma_f32_16x16x32_bf16 %4, %16, %12, %4\n\t" "v_mfma_f32_16x16x32_bf16 %4, %17, %13, %4\n\t" "v_mfma_f32_16x16x32_bf16 %6, %17, %15, %6\n\t" "v_mfma_f32_16x16x32_bf16 %6, %16, %14, %6\n\t" "v_mfma_f32_16x16x32_bf16 %7, %18, %14, %7\n\t" "v_mfma_f32_16x16x32_bf16 %7, %19, %15, %7\n\t" "v_mfma_f32_16x16x32_bf16 %5, %19, %13, %5\n\t" "v_mfma_f32_16x16x32_bf16 %5, %18, %12, %5\n\t" "v_mfma_f32_16x16x32_bf16 %3, %18, %10, %3\n\t" "v_mfma_f32_16x16x32_bf16 %3, %19, %11, %3\n\t" "v_mfma_f32_16x16x32_bf16 %1, %19, %9, %1\n\t" "v_mfma_f32_16x16x32_bf16 %1, %18, %8, %1\n\t"); MMAW(0, 1, B1, "v_mfma_f32_16x16x32_bf16 %0, %16, %8, %0\n\t" "v_mfma_f32_16x16x32_bf16 %0, %17, %9, %0\n\t" "v_mfma_f32_16x16x32_bf16 %2, %17, %11, %2\n\t" "v_mfma_f32_16x16x32_bf16 %2, %16, %10, %2\n\t" "v_mfma_f32_16x16x32_bf16 %4, %16, %12, %4\n\t" "v_mfma_f32_16x16x32_bf16 %4, %17, %13, %4\n\t" "v_mfma_f32_16x16x32_bf16 %6, %17, %15, %6\n\t" "v_mfma_f32_16x16x32_bf16 %6, %16, %14, %6\n\t" "v_mfma_f32_16x16x32_bf16 %7, %18, %14, %7\n\t" "v_mfma_f32_16x16x32_bf16 %7, %19, %15, %7\n\t" "v_mfma_f32_16x16x32_bf16 %5, %19, %13, %5\n\t" "v_mfma_f32_16x16x32_bf16 %5, %18, %12, %5\n\t" "v_mfma_f32_16x16x32_bf16 %3, %18, %10, %3\n\t" "v_cmp_le_u32 s[100:101], s84, v239\n\t" "v_mfma_f32_16x16x32_bf16 %3, %19, %11, %3\n\t" "s_lshr_b32 s98, s100, s85\n\t" "v_mfma_f32_16x16x32_bf16 %1, %19, %9, %1\n\t" "s_andn2_b32 s98, 15, s98\n\t" "v_mfma_f32_16x16x32_bf16 %1, %18, %8, %1\n\t" "s_setprio 0\n\t" "s_cbranch_scc1 ag_tr_c\n\t" "ag_bk_c:\n\t"); BAR;
      LDA(1, 1); ENG("s_add_i32 s68, s68, 3\n\t" "s_add_i32 s85, s85, 4\n\t" "s_and_b32 s84, s68, s66\n\t" "s_and_b32 s85, s85, 28\n\t" "s_add_i32 s86, s86, 2\n\t" "v_sub_u32 v224, s86, v227\n\t" "v_lshrrev_b32 v224, 3, v224\n\t" "v_lshl_add_u32 v237, v224, 11, v228\n\t" "v_lshl_add_u32 v224, v224, 4, v233\n\t" "s_mov_b32 s56, 0xff\n\t" "s_mov_b32 s57, 0xff\n\t" "s_lshl_b64 s[56:57], s[56:57], s85\n\t" "s_and_b32 s59, s57, s66\n\t" "s_mov_b32 s56, 0xff0\n\t" "s_mov_b32 s57, 0xff0\n\t" "s_lshl_b64 s[56:57], s[56:57], s85\n\t" "s_and_b32 s58, s57, s66\n\t" "v_cmp_gt_u32 s[64:65], v224, v239\n\t" "s_mov_b32 exec_hi, 0\n\t" "s_and_b32 exec_lo, s64, s59\n\t" "global_load_dword v239, v237, s[92:93] sc0 sc1\n\t" "s_mov_b64 exec, -1\n\t" "s_add_i32 s98, s70, 6\n\t" "s_add_i32 s70, s70, s63\n\t" "s_mov_b32 s63, 0\n\t" "s_cmp_lt_u32 s74, " STR(ID1) "\n\t" "s_cselect_b32 s99, 0, (" STR(D2) "-" STR(D1) ")\n\t" "s_add_i32 s99, s99, s73\n\t" "s_addk_i32 s99, " STR(D1) "\n\t" "s_cmp_lt_u32 s69, 0x2000\n\t" "s_cselect_b32 s99, s99, 0x7fffffff\n\t" "s_cmp_lg_u32 s87, 0\n\t" "s_cselect_b32 s72, s98, s72\n\t" "s_cselect_b32 s75, s74, s75\n\t" "s_cselect_b32 s71, s70, s71\n\t" "s_cselect_b32 s73, s99, s73\n\t" "s_cselect_b32 s74, s69, s74\n\t" "s_mov_b32 s87, 0\n\t" "s_add_i32 s70, s70, 1\n\t" "s_and_b32 s98, s75, 7\n\t" "s_lshl_b32 s99, s75, 2\n\t" "v_readlane_b32 s82, v234, s98\n\t" "v_readlane_b32 s83, v235, s98\n\t" "s_and_b32 s99, s99, 0x7fe0\n\t" "s_lshr_b32 s98, s75, 5\n\t" "s_add_i32 s98, s98, s88\n\t" "s_add_u32 s82, s82, s99\n\t" "s_addc_u32 s83, s83, 0\n\t" "v_mov_b32 v238, s98\n\t" "s_and_b32 s98, s74, 7\n\t" "s_and_b32 s99, s74, 0x1ff8\n\t" "v_readlane_b32 s78, v234, s98\n\t" "v_readlane_b32 s79, v235, s98\n\t" "s_lshl_b32 s99, s99, 9\n\t" "s_cmp_ge_u32 s98, 3\n\t" "s_addc_u32 s69, s74, 292\n\t" "s_add_u32 s78, s78, s99\n\t" "s_addc_u32 s79, s79, 0\n\t" "s_lshl_b32 s98, s69, 15\n\t" "s_and_b32 s99, s69, 0x1f80\n\t" "s_and_b32 s98, s98, 0x3c0000\n\t" "s_or_b32 s98, s98, s99\n\t" "s_add_u32 s80, s90, s98\n\t" "s_addc_u32 s81, s91, 0\n\t");
      WAIT_A(); BAR; MMAW(1, 0, B0, "s_setprio 1\n\t" "v_mfma_f32_16x16x32_bf16 %0, %16, %8, %0\n\t" "v_mfma_f32_16x16x32_bf16 %0, %17, %9, %0\n\t" "v_mfma_f32_16x16x32_bf16 %2, %17, %11, %2\n\t" "v_mfma_f32_16x16x32_bf16 %2, %16, %10, %2\n\t" "v_mfma_f32_16x16x32_bf16 %4, %16, %12, %4\n\t" "v_mfma_f32_16x16x32_bf16 %4, %17, %13, %4\n\t" "v_mfma_f32_16x16x32_bf16 %6, %17, %15, %6\n\t" "v_mfma_f32_16x16x32_bf16 %6, %16, %14, %6\n\t" "v_mfma_f32_16x16x32_bf16 %7, %18, %14, %7\n\t" "v_mfma_f32_16x16x32_bf16 %7, %19, %15, %7\n\t" "v_mfma_f32_16x16x32_bf16 %5, %19, %13, %5\n\t" "v_mfma_f32_16x16x32_bf16 %5, %18, %12, %5\n\t" "v_mfma_f32_16x16x32_bf16 %3, %18, %10, %3\n\t" "v_mfma_f32_16x16x32_bf16 %3, %19, %11, %3\n\t" "v_mfma_f32_16x16x32_bf16 %1, %19, %9, %1\n\t" "v_mfma_f32_16x16x32_bf16 %1, %18, %8, %1\n\t"); MMAW(1, 1, B1, "v_mfma_f32_16x16x32_bf16 %0, %16, %8, %0\n\t" "v_mfma_f32_16x16x32_bf16 %0, %17, %9, %0\n\t" "s_memrealtime s[96:97]\n\t" "v_mfma_f32_16x16x32_bf16 %2, %17, %11, %2\n\t" "v_mfma_f32_16x16x32_bf16 %2, %16, %10, %2\n\t" "v_mfma_f32_16x16x32_bf16 %4, %16, %12, %4\n\t" "s_getreg_b32 s98, hwreg(HW_REG_IB_STS, 22, 2)\n\t" "v_mfma_f32_16x16x32_bf16 %4, %17, %13, %4\n\t" "s_getreg_b32 s99, hwreg(HW_REG_IB_STS, 0, 4)\n\t" "v_mfma_f32_16x16x32_bf16 %6, %17, %15, %6\n\t" "s_lshl4_add_u32 s98, s98, s99\n\t" "v_mfma_f32_16x16x32_bf16 %6, %16, %14, %6\n\t" "s_sub_i32 s76, s70, s98\n\t" "v_mfma_f32_16x16x32_bf16 %7, %18, %14, %7\n\t" "s_cmp_ge_i32 s76, s71\n\t" "v_mfma_f32_16x16x32_bf16 %7, %19, %15, %7\n\t" "s_cselect_b32 s77, s73, 0x7fffffff\n\t" "v_mfma_f32_16x16x32_bf16 %5, %19, %13, %5\n\t" "s_waitcnt lgkmcnt(0)\n\t" "v_mfma_f32_16x16x32_bf16 %5, %18, %12, %5\n\t" "s_sub_i32 s62, s96, s89\n\t" "v_mfma_f32_16x16x32_bf16 %3, %18, %10, %3\n\t" "v_cmp_le_u32 s[100:101], s84, v239\n\t" "v_mfma_f32_16x16x32_bf16 %3, %19, %11, %3\n\t" "s_lshr_b32 s98, s100, s85\n\t" "v_mfma_f32_16x16x32_bf16 %1, %19, %9, %1\n\t" "s_andn2_b32 s98, 15, s98\n\t" "v_mfma_f32_16x16x32_bf16 %1, %18, %8, %1\n\t" "s_setprio 0\n\t" "s_cbranch_scc1 ag_tr_d\n\t" "ag_bk_d:\n\t"); BAR;
    }
    W0P1(0);
    WAIT_ABB(); BAR; MMAW(0, 0, B0, "s_setprio 1\n\t" "v_mfma_f32_16x16x32_bf16 %0, %16, %8, %0\n\t" "s_add_i32 s68, s68, 1\n\t" "v_mfma_f32_16x16x32_bf16 %0, %17, %9, %0\n\t" "s_add_i32 s85, s85, 4\n\t" "v_mfma_f32_16x16x32_bf16 %2, %17, %11, %2\n\t" "s_and_b32 s84, s68, s66\n\t" "v_mfma_f32_16x16x32_bf16 %2, %16, %10, %2\n\t" "s_and_b32 s85, s85, 28\n\t" "v_mfma_f32_16x16x32_bf16 %4, %16, %12, %4\n\t" "v_mfma_f32_16x16x32_bf16 %4, %17, %13, %4\n\t" "v_mfma_f32_16x16x32_bf16 %6, %17, %15, %6\n\t" "v_mfma_f32_16x16x32_bf16 %6, %16, %14, %6\n\t" "v_mfma_f32_16x16x32_bf16 %7, %18, %14, %7\n\t" "v_mfma_f32_16x16x32_bf16 %7, %19, %15, %7\n\t" "v_mfma_f32_16x16x32_bf16 %5, %19, %13, %5\n\t" "v_mfma_f32_16x16x32_bf16 %5, %18, %12, %5\n\t" "v_mfma_f32_16x16x32_bf16 %3, %18, %10, %3\n\t" "v_mfma_f32_16x16x32_bf16 %3, %19, %11, %3\n\t" "v_mfma_f32_16x16x32_bf16 %1, %19, %9, %1\n\t" "v_mfma_f32_16x16x32_bf16 %1, %18, %8, %1\n\t"); MMAW(0, 1, B1, "v_mfma_f32_16x16x32_bf16 %0, %16, %8, %0\n\t" "v_mfma_f32_16x16x32_bf16 %0, %17, %9, %0\n\t" "v_mfma_f32_16x16x32_bf16 %2, %17, %11, %2\n\t" "v_mfma_f32_16x16x32_bf16 %2, %16, %10, %2\n\t" "v_mfma_f32_16x16x32_bf16 %4, %16, %12, %4\n\t" "v_mfma_f32_16x16x32_bf16 %4, %17, %13, %4\n\t" "v_mfma_f32_16x16x32_bf16 %6, %17, %15, %6\n\t" "v_mfma_f32_16x16x32_bf16 %6, %16, %14, %6\n\t" "v_mfma_f32_16x16x32_bf16 %7, %18, %14, %7\n\t" "v_mfma_f32_16x16x32_bf16 %7, %19, %15, %7\n\t" "v_mfma_f32_16x16x32_bf16 %5, %19, %13, %5\n\t" "v_mfma_f32_16x16x32_bf16 %5, %18, %12, %5\n\t" "v_mfma_f32_16x16x32_bf16 %3, %18, %10, %3\n\t" "v_cmp_le_u32 s[100:101], s84, v239\n\t" "v_mfma_f32_16x16x32_bf16 %3, %19, %11, %3\n\t" "s_lshr_b32 s98, s100, s85\n\t" "v_mfma_f32_16x16x32_bf16 %1, %19, %9, %1\n\t" "s_andn2_b32 s98, 15, s98\n\t" "v_mfma_f32_16x16x32_bf16 %1, %18, %8, %1\n\t" "s_setprio 0\n\t" "s_cbranch_scc1 ag_tr_e\n\t" "ag_bk_e:\n\t"); BAR;
    LDA(0, 1); ENG("s_add_i32 s68, s68, 3\n\t" "s_add_i32 s85, s85, 4\n\t" "s_and_b32 s84, s68, s66\n\t" "s_and_b32 s85, s85, 28\n\t" "s_add_i32 s86, s86, 2\n\t" "v_sub_u32 v224, s86, v227\n\t" "v_lshrrev_b32 v224, 3, v224\n\t" "v_lshl_add_u32 v237, v224, 11, v228\n\t" "v_lshl_add_u32 v224, v224, 4, v233\n\t" "s_mov_b32 s56, 0xff\n\t" "s_mov_b32 s57, 0xff\n\t" "s_lshl_b64 s[56:57], s[56:57], s85\n\t" "s_and_b32 s59, s57, s66\n\t" "s_mov_b32 s56, 0xff0\n\t" "s_mov_b32 s57, 0xff0\n\t" "s_lshl_b64 s[56:57], s[56:57], s85\n\t" "s_and_b32 s58, s57, s66\n\t" "v_cmp_gt_u32 s[64:65], v224, v239\n\t" "s_mov_b32 exec_hi, 0\n\t" "s_and_b32 exec_lo, s64, s59\n\t" "global_load_dword v239, v237, s[92:93] sc0 sc1\n\t" "s_mov_b64 exec, -1\n\t" "s_add_i32 s98, s70, 6\n\t" "s_add_i32 s70, s70, s63\n\t" "s_mov_b32 s63, 0\n\t" "s_cmp_lt_u32 s74, " STR(ID1) "\n\t" "s_cselect_b32 s99, 0, (" STR(D2) "-" STR(D1) ")\n\t" "s_add_i32 s99, s99, s73\n\t" "s_addk_i32 s99, " STR(D1) "\n\t" "s_cmp_lt_u32 s69, 0x2000\n\t" "s_cselect_b32 s99, s99, 0x7fffffff\n\t" "s_cmp_lg_u32 s87, 0\n\t" "s_cselect_b32 s72, s98, s72\n\t" "s_cselect_b32 s75, s74, s75\n\t" "s_cselect_b32 s71, s70, s71\n\t" "s_cselect_b32 s73, s99, s73\n\t" "s_cselect_b32 s74, s69, s74\n\t" "s_mov_b32 s87, 0\n\t" "s_add_i32 s70, s70, 1\n\t" "s_and_b32 s98, s75, 7\n\t" "s_lshl_b32 s99, s75, 2\n\t" "v_readlane_b32 s82, v234, s98\n\t" "v_readlane_b32 s83, v235, s98\n\t" "s_and_b32 s99, s99, 0x7fe0\n\t" "s_lshr_b32 s98, s75, 5\n\t" "s_add_i32 s98, s98, s88\n\t" "s_add_u32 s82, s82, s99\n\t" "s_addc_u32 s83, s83, 0\n\t" "v_mov_b32 v238, s98\n\t" "s_and_b32 s98, s74, 7\n\t" "s_and_b32 s99, s74, 0x1ff8\n\t" "v_readlane_b32 s78, v234, s98\n\t" "v_readlane_b32 s79, v235, s98\n\t" "s_lshl_b32 s99, s99, 9\n\t" "s_cmp_ge_u32 s98, 3\n\t" "s_addc_u32 s69, s74, 292\n\t" "s_add_u32 s78, s78, s99\n\t" "s_addc_u32 s79, s79, 0\n\t" "s_lshl_b32 s98, s69, 15\n\t" "s_and_b32 s99, s69, 0x1f80\n\t" "s_and_b32 s98, s98, 0x3c0000\n\t" "s_or_b32 s98, s98, s99\n\t" "s_add_u32 s80, s90, s98\n\t" "s_addc_u32 s81, s91, 0\n\t");
    WAIT_A(); BAR; MMAW(1, 0, B0, "s_setprio 1\n\t" "v_mfma_f32_16x16x32_bf16 %0, %16, %8, %0\n\t" "v_mfma_f32_16x16x32_bf16 %0, %17, %9, %0\n\t" "v_mfma_f32_16x16x32_bf16 %2, %17, %11, %2\n\t" "v_mfma_f32_16x16x32_bf16 %2, %16, %10, %2\n\t" "v_mfma_f32_16x16x32_bf16 %4, %16, %12, %4\n\t" "v_mfma_f32_16x16x32_bf16 %4, %17, %13, %4\n\t" "v_mfma_f32_16x16x32_bf16 %6, %17, %15, %6\n\t" "v_mfma_f32_16x16x32_bf16 %6, %16, %14, %6\n\t" "v_mfma_f32_16x16x32_bf16 %7, %18, %14, %7\n\t" "v_mfma_f32_16x16x32_bf16 %7, %19, %15, %7\n\t" "v_mfma_f32_16x16x32_bf16 %5, %19, %13, %5\n\t" "v_mfma_f32_16x16x32_bf16 %5, %18, %12, %5\n\t" "v_mfma_f32_16x16x32_bf16 %3, %18, %10, %3\n\t" "v_mfma_f32_16x16x32_bf16 %3, %19, %11, %3\n\t" "v_mfma_f32_16x16x32_bf16 %1, %19, %9, %1\n\t" "v_mfma_f32_16x16x32_bf16 %1, %18, %8, %1\n\t"); MMAW(1, 1, B1, "v_mfma_f32_16x16x32_bf16 %0, %16, %8, %0\n\t" "v_mfma_f32_16x16x32_bf16 %0, %17, %9, %0\n\t" "v_mfma_f32_16x16x32_bf16 %2, %17, %11, %2\n\t" "v_mfma_f32_16x16x32_bf16 %2, %16, %10, %2\n\t" "v_mfma_f32_16x16x32_bf16 %4, %16, %12, %4\n\t" "s_memrealtime s[96:97]\n\t" "v_mfma_f32_16x16x32_bf16 %4, %17, %13, %4\n\t" "v_mfma_f32_16x16x32_bf16 %6, %17, %15, %6\n\t" "v_mfma_f32_16x16x32_bf16 %6, %16, %14, %6\n\t" "s_getreg_b32 s98, hwreg(HW_REG_IB_STS, 22, 2)\n\t" "v_mfma_f32_16x16x32_bf16 %7, %18, %14, %7\n\t" "s_getreg_b32 s99, hwreg(HW_REG_IB_STS, 0, 4)\n\t" "v_mfma_f32_16x16x32_bf16 %7, %19, %15, %7\n\t" "s_lshl4_add_u32 s98, s98, s99\n\t" "v_mfma_f32_16x16x32_bf16 %5, %19, %13, %5\n\t" "s_sub_i32 s76, s70, s98\n\t" "v_mfma_f32_16x16x32_bf16 %5, %18, %12, %5\n\t" "s_cmp_ge_i32 s76, s71\n\t" "v_mfma_f32_16x16x32_bf16 %3, %18, %10, %3\n\t" "s_cselect_b32 s77, s73, 0x7fffffff\n\t" "v_mfma_f32_16x16x32_bf16 %3, %19, %11, %3\n\t" "s_waitcnt lgkmcnt(0)\n\t" "v_mfma_f32_16x16x32_bf16 %1, %19, %9, %1\n\t" "s_sub_i32 s62, s96, s89\n\t" "v_mfma_f32_16x16x32_bf16 %1, %18, %8, %1\n\t" "s_setprio 0\n\t"); BAR;
    READ_BAD();
    W0P1(1);
    WAIT_ABB(); BAR; MMAW(0, 0, B0, "s_setprio 1\n\t" "v_mfma_f32_16x16x32_bf16 %0, %16, %8, %0\n\t" "s_add_i32 s68, s68, 1\n\t" "v_mfma_f32_16x16x32_bf16 %0, %17, %9, %0\n\t" "s_add_i32 s85, s85, 4\n\t" "v_mfma_f32_16x16x32_bf16 %2, %17, %11, %2\n\t" "s_and_b32 s84, s68, s66\n\t" "v_mfma_f32_16x16x32_bf16 %2, %16, %10, %2\n\t" "s_and_b32 s85, s85, 28\n\t" "v_mfma_f32_16x16x32_bf16 %4, %16, %12, %4\n\t" "v_mfma_f32_16x16x32_bf16 %4, %17, %13, %4\n\t" "v_mfma_f32_16x16x32_bf16 %6, %17, %15, %6\n\t" "v_mfma_f32_16x16x32_bf16 %6, %16, %14, %6\n\t" "v_mfma_f32_16x16x32_bf16 %7, %18, %14, %7\n\t" "v_mfma_f32_16x16x32_bf16 %7, %19, %15, %7\n\t" "v_mfma_f32_16x16x32_bf16 %5, %19, %13, %5\n\t" "v_mfma_f32_16x16x32_bf16 %5, %18, %12, %5\n\t" "v_mfma_f32_16x16x32_bf16 %3, %18, %10, %3\n\t" "v_mfma_f32_16x16x32_bf16 %3, %19, %11, %3\n\t" "v_mfma_f32_16x16x32_bf16 %1, %19, %9, %1\n\t" "v_mfma_f32_16x16x32_bf16 %1, %18, %8, %1\n\t"); MMAW(0, 1, B1, "v_mfma_f32_16x16x32_bf16 %0, %16, %8, %0\n\t" "v_mfma_f32_16x16x32_bf16 %0, %17, %9, %0\n\t" "v_mfma_f32_16x16x32_bf16 %2, %17, %11, %2\n\t" "v_mfma_f32_16x16x32_bf16 %2, %16, %10, %2\n\t" "v_mfma_f32_16x16x32_bf16 %4, %16, %12, %4\n\t" "v_mfma_f32_16x16x32_bf16 %4, %17, %13, %4\n\t" "v_mfma_f32_16x16x32_bf16 %6, %17, %15, %6\n\t" "v_mfma_f32_16x16x32_bf16 %6, %16, %14, %6\n\t" "v_mfma_f32_16x16x32_bf16 %7, %18, %14, %7\n\t" "v_mfma_f32_16x16x32_bf16 %7, %19, %15, %7\n\t" "v_mfma_f32_16x16x32_bf16 %5, %19, %13, %5\n\t" "v_mfma_f32_16x16x32_bf16 %5, %18, %12, %5\n\t" "v_mfma_f32_16x16x32_bf16 %3, %18, %10, %3\n\t" "v_mfma_f32_16x16x32_bf16 %3, %19, %11, %3\n\t" "v_mfma_f32_16x16x32_bf16 %1, %19, %9, %1\n\t" "v_mfma_f32_16x16x32_bf16 %1, %18, %8, %1\n\t" "s_setprio 0\n\t"); BAR;
    TAKE_BAD();
    LDA(1, 1); ENG("s_add_i32 s68, s68, 3\n\t" "s_add_i32 s85, s85, 4\n\t" "s_and_b32 s84, s68, s66\n\t" "s_and_b32 s85, s85, 28\n\t" "s_add_i32 s86, s86, 2\n\t" "v_sub_u32 v224, s86, v227\n\t" "v_lshrrev_b32 v224, 3, v224\n\t" "v_lshl_add_u32 v237, v224, 11, v228\n\t" "v_lshl_add_u32 v224, v224, 4, v233\n\t" "s_mov_b32 s56, 0xff\n\t" "s_mov_b32 s57, 0xff\n\t" "s_lshl_b64 s[56:57], s[56:57], s85\n\t" "s_and_b32 s59, s57, s66\n\t" "s_mov_b32 s56, 0xff0\n\t" "s_mov_b32 s57, 0xff0\n\t" "s_lshl_b64 s[56:57], s[56:57], s85\n\t" "s_and_b32 s58, s57, s66\n\t" "v_cmp_gt_u32 s[64:65], v224, v239\n\t" "s_mov_b32 exec_hi, 0\n\t" "s_and_b32 exec_lo, s64, s59\n\t" "global_load_dword v239, v237, s[92:93] sc0 sc1\n\t" "s_mov_b64 exec, -1\n\t" "s_add_i32 s98, s70, 6\n\t" "s_add_i32 s70, s70, s63\n\t" "s_mov_b32 s63, 0\n\t" "s_cmp_lt_u32 s74, " STR(ID1) "\n\t" "s_cselect_b32 s99, 0, (" STR(D2) "-" STR(D1) ")\n\t" "s_add_i32 s99, s99, s73\n\t" "s_addk_i32 s99, " STR(D1) "\n\t" "s_cmp_lt_u32 s69, 0x2000\n\t" "s_cselect_b32 s99, s99, 0x7fffffff\n\t" "s_cmp_lg_u32 s87, 0\n\t" "s_cselect_b32 s72, s98, s72\n\t" "s_cselect_b32 s75, s74, s75\n\t" "s_cselect_b32 s71, s70, s71\n\t" "s_cselect_b32 s73, s99, s73\n\t" "s_cselect_b32 s74, s69, s74\n\t" "s_mov_b32 s87, 0\n\t" "s_add_i32 s70, s70, 1\n\t" "s_and_b32 s98, s75, 7\n\t" "s_lshl_b32 s99, s75, 2\n\t" "v_readlane_b32 s82, v234, s98\n\t" "v_readlane_b32 s83, v235, s98\n\t" "s_and_b32 s99, s99, 0x7fe0\n\t" "s_lshr_b32 s98, s75, 5\n\t" "s_add_i32 s98, s98, s88\n\t" "s_add_u32 s82, s82, s99\n\t" "s_addc_u32 s83, s83, 0\n\t" "v_mov_b32 v238, s98\n\t" "s_and_b32 s98, s74, 7\n\t" "s_and_b32 s99, s74, 0x1ff8\n\t" "v_readlane_b32 s78, v234, s98\n\t" "v_readlane_b32 s79, v235, s98\n\t" "s_lshl_b32 s99, s99, 9\n\t" "s_cmp_ge_u32 s98, 3\n\t" "s_addc_u32 s69, s74, 292\n\t" "s_add_u32 s78, s78, s99\n\t" "s_addc_u32 s79, s79, 0\n\t" "s_lshl_b32 s98, s69, 15\n\t" "s_and_b32 s99, s69, 0x1f80\n\t" "s_and_b32 s98, s98, 0x3c0000\n\t" "s_or_b32 s98, s98, s99\n\t" "s_add_u32 s80, s90, s98\n\t" "s_addc_u32 s81, s91, 0\n\t"); store_half<0>(acc, otile, wid, bad);
    WAIT_A(); BAR; MMAW(1, 0, B0, "s_setprio 1\n\t" "v_mfma_f32_16x16x32_bf16 %0, %16, %8, %0\n\t" "v_mfma_f32_16x16x32_bf16 %0, %17, %9, %0\n\t" "v_mfma_f32_16x16x32_bf16 %2, %17, %11, %2\n\t" "v_mfma_f32_16x16x32_bf16 %2, %16, %10, %2\n\t" "v_mfma_f32_16x16x32_bf16 %4, %16, %12, %4\n\t" "v_mfma_f32_16x16x32_bf16 %4, %17, %13, %4\n\t" "v_mfma_f32_16x16x32_bf16 %6, %17, %15, %6\n\t" "v_mfma_f32_16x16x32_bf16 %6, %16, %14, %6\n\t" "v_mfma_f32_16x16x32_bf16 %7, %18, %14, %7\n\t" "v_mfma_f32_16x16x32_bf16 %7, %19, %15, %7\n\t" "v_mfma_f32_16x16x32_bf16 %5, %19, %13, %5\n\t" "v_mfma_f32_16x16x32_bf16 %5, %18, %12, %5\n\t" "v_mfma_f32_16x16x32_bf16 %3, %18, %10, %3\n\t" "v_mfma_f32_16x16x32_bf16 %3, %19, %11, %3\n\t" "v_mfma_f32_16x16x32_bf16 %1, %19, %9, %1\n\t" "v_mfma_f32_16x16x32_bf16 %1, %18, %8, %1\n\t"); MMAW(1, 1, B1, "v_mfma_f32_16x16x32_bf16 %0, %16, %8, %0\n\t" "v_mfma_f32_16x16x32_bf16 %0, %17, %9, %0\n\t" "v_mfma_f32_16x16x32_bf16 %2, %17, %11, %2\n\t" "v_mfma_f32_16x16x32_bf16 %2, %16, %10, %2\n\t" "v_mfma_f32_16x16x32_bf16 %4, %16, %12, %4\n\t" "s_memrealtime s[96:97]\n\t" "v_mfma_f32_16x16x32_bf16 %4, %17, %13, %4\n\t" "v_mfma_f32_16x16x32_bf16 %6, %17, %15, %6\n\t" "v_mfma_f32_16x16x32_bf16 %6, %16, %14, %6\n\t" "s_getreg_b32 s98, hwreg(HW_REG_IB_STS, 22, 2)\n\t" "v_mfma_f32_16x16x32_bf16 %7, %18, %14, %7\n\t" "s_getreg_b32 s99, hwreg(HW_REG_IB_STS, 0, 4)\n\t" "v_mfma_f32_16x16x32_bf16 %7, %19, %15, %7\n\t" "s_lshl4_add_u32 s98, s98, s99\n\t" "v_mfma_f32_16x16x32_bf16 %5, %19, %13, %5\n\t" "s_sub_i32 s76, s70, s98\n\t" "v_mfma_f32_16x16x32_bf16 %5, %18, %12, %5\n\t" "s_cmp_ge_i32 s76, s71\n\t" "v_mfma_f32_16x16x32_bf16 %3, %18, %10, %3\n\t" "s_cselect_b32 s77, s73, 0x7fffffff\n\t" "v_mfma_f32_16x16x32_bf16 %3, %19, %11, %3\n\t" "s_waitcnt lgkmcnt(0)\n\t" "v_mfma_f32_16x16x32_bf16 %1, %19, %9, %1\n\t" "s_sub_i32 s62, s96, s89\n\t" "v_mfma_f32_16x16x32_bf16 %1, %18, %8, %1\n\t" "s_setprio 0\n\t"); BAR;
    MFMA_DRAIN();
    store_half<1>(acc, otile, wid, bad);
    asm volatile("s_mov_b32 s66, 0\n\t" "s_mov_b32 s59, 0\n\t" "s_mov_b32 s58, 0\n\t" "s_add_i32 s98, s70, 6\n\t" "s_add_i32 s70, s70, s63\n\t" "s_mov_b32 s63, 0\n\t" "s_cmp_lt_u32 s74, " STR(ID1) "\n\t" "s_cselect_b32 s99, 0, (" STR(D2) "-" STR(D1) ")\n\t" "s_add_i32 s99, s99, s73\n\t" "s_addk_i32 s99, " STR(D1) "\n\t" "s_cmp_lt_u32 s69, 0x2000\n\t" "s_cselect_b32 s99, s99, 0x7fffffff\n\t" "s_cmp_lg_u32 s87, 0\n\t" "s_cselect_b32 s72, s98, s72\n\t" "s_cselect_b32 s75, s74, s75\n\t" "s_cselect_b32 s71, s70, s71\n\t" "s_cselect_b32 s73, s99, s73\n\t" "s_cselect_b32 s74, s69, s74\n\t" "s_mov_b32 s87, 0\n\t" "s_and_b32 s98, s75, 7\n\t" "s_lshl_b32 s99, s75, 2\n\t" "v_readlane_b32 s82, v234, s98\n\t" "v_readlane_b32 s83, v235, s98\n\t" "s_and_b32 s99, s99, 0x7fe0\n\t" "s_lshr_b32 s98, s75, 5\n\t" "s_add_i32 s98, s98, s88\n\t" "s_add_u32 s82, s82, s99\n\t" "s_addc_u32 s83, s83, 0\n\t" "v_mov_b32 v238, s98\n\t" "s_branch ag_fin_test\n\t" "ag_fin:\n\t" "s_and_b32 s98, s74, 7\n\t" "s_and_b32 s99, s74, 0x1ff8\n\t" "v_readlane_b32 s78, v234, s98\n\t" "v_readlane_b32 s79, v235, s98\n\t" "s_lshl_b32 s99, s99, 9\n\t" "s_cmp_ge_u32 s98, 3\n\t" "s_addc_u32 s69, s74, 292\n\t" "s_add_u32 s78, s78, s99\n\t" "s_addc_u32 s79, s79, 0\n\t" "s_lshl_b32 s98, s69, 15\n\t" "s_and_b32 s99, s69, 0x1f80\n\t" "s_and_b32 s98, s98, 0x3c0000\n\t" "s_or_b32 s98, s98, s99\n\t" "s_add_u32 s80, s90, s98\n\t" "s_addc_u32 s81, s91, 0\n\t" "s_mov_b32 s62, 0x7ffffffe\n\t" "s_waitcnt vmcnt(0)\n\t" "s_mov_b32 s76, s70\n\t" "s_cmp_ge_i32 s76, s71\n\t" "s_cselect_b32 s77, s73, 0x7fffffff\n\t" "v_cmp_gt_u32 s[64:65], v224, v239\n\t" "s_cmp_ge_i32 s76, s72\n\t" "s_cselect_b64 exec, 1, 0\n\t" "global_store_dword v232, v238, s[82:83] sc0 sc1\n\t" "s_cselect_b32 s72, 0x7fffffff, s72\n\t" "s_and_b32 exec_lo, s64, s59\n\t" "global_load_dword v239, v237, s[92:93] sc0 sc1\n\t" "s_cmp_ge_i32 s62, s77\n\t" "s_mov_b64 exec, -1\n\t" "s_cselect_b32 s87, 1, 0\n\t" "s_cselect_b32 s63, 10, 2\n\t" "s_cbranch_scc0 1f\n\t" "global_store_dwordx4 v231, v[240:243], s[78:79] sc0 sc1\n\t" "global_store_dwordx4 v231, v[244:247], s[78:79] offset:1024 sc0 sc1\n\t" "global_store_dwordx4 v231, v[248:251], s[78:79] offset:2048 sc0 sc1\n\t" "global_store_dwordx4 v231, v[252:255], s[78:79] offset:3072 sc0 sc1\n\t" "global_load_dwordx4 v[240:243], v230, s[80:81]\n\t" "global_load_dwordx4 v[244:247], v229, s[80:81]\n\t" "global_load_dwordx4 v[248:251], v225, s[80:81]\n\t" "global_load_dwordx4 v[252:255], v223, s[80:81]\n\t" "1:\n\t" "s_add_i32 s98, s70, 6\n\t" "s_add_i32 s70, s70, s63\n\t" "s_mov_b32 s63, 0\n\t" "s_cmp_lt_u32 s74, " STR(ID1) "\n\t" "s_cselect_b32 s99, 0, (" STR(D2) "-" STR(D1) ")\n\t" "s_add_i32 s99, s99, s73\n\t" "s_addk_i32 s99, " STR(D1) "\n\t" "s_cmp_lt_u32 s69, 0x2000\n\t" "s_cselect_b32 s99, s99, 0x7fffffff\n\t" "s_cmp_lg_u32 s87, 0\n\t" "s_cselect_b32 s72, s98, s72\n\t" "s_cselect_b32 s75, s74, s75\n\t" "s_cselect_b32 s71, s70, s71\n\t" "s_cselect_b32 s73, s99, s73\n\t" "s_cselect_b32 s74, s69, s74\n\t" "s_mov_b32 s87, 0\n\t" "s_and_b32 s98, s75, 7\n\t" "s_lshl_b32 s99, s75, 2\n\t" "v_readlane_b32 s82, v234, s98\n\t" "v_readlane_b32 s83, v235, s98\n\t" "s_and_b32 s99, s99, 0x7fe0\n\t" "s_lshr_b32 s98, s75, 5\n\t" "s_add_i32 s98, s98, s88\n\t" "s_add_u32 s82, s82, s99\n\t" "s_addc_u32 s83, s83, 0\n\t" "v_mov_b32 v238, s98\n\t" "ag_fin_test:\n\t" "s_and_b32 s98, s72, s73\n\t" "s_cmp_lg_u32 s98, 0x7fffffff\n\t" "s_cbranch_scc1 ag_fin\n\t" ::: "memory", "scc");
    if (lane == 0) __hip_atomic_store((unsigned*)ctr, call, __ATOMIC_RELAXED, __HIP_MEMORY_SCOPE_AGENT);
  }
}

extern "C" void kernel_launch(int rank, int nranks, ncclComm_t comm, hipStream_t stream,
                              void* const* d_in, const int* in_sizes, int n_in,
                              void* d_out, int out_size, const int* in_dtypes, int out_dtype,
                              void* const* d_out_all, void* const* d_ws_all, size_t ws_bytes) {
  if (nranks != NR || (unsigned)rank >= (unsigned)NR || ws_bytes < WS_NEED || n_in < 2 ||
      in_sizes[0] != 512 * 4096 || in_sizes[1] != 4096 * 4096 || out_size != 4096 * 4096) {
    fprintf(stderr, "kernel_launch: unexpected shapes\n"); return;
  }
  static bool init = false;
  if (!init) {
    if (hipFuncSetAttribute((const void*)ag_gemm_kernel, hipFuncAttributeMaxDynamicSharedMemorySize, LDS_BYTES) != hipSuccess) {
      fprintf(stderr, "hipFuncSetAttribute failed\n"); return; }
    init = true;
  }
  Args a;
  a.x = (const char*)d_in[0]; a.w = (const char*)d_in[1]; a.out = (char*)d_out; a.rank = rank;
  for (int p = 0; p < NR; ++p) a.ws[p] = (char*)d_ws_all[p];
  ag_gemm_kernel<<<256, 512, LDS_BYTES, stream>>>(a);
}
```

```cpp
#include <hip/hip_runtime.h>
#include <rccl/rccl.h>
#include <cstdio>

#define D1 286
#define D2 294
#define ID1 1280
#define T0 90
#define SLP 3
#define WF 0xff
#define WS 0xff
#define NPRE 3
#define PFB sc1
#define DEADLINE 100000000
#define LDSBAD 131072
#define STR_(x) #x
#define STR(x) STR_(x)

namespace {
typedef short bf16x8 __attribute__((ext_vector_type(8)));
typedef short bf16x4 __attribute__((ext_vector_type(4)));
typedef float f32x4 __attribute__((ext_vector_type(4)));
typedef unsigned u32x4 __attribute__((ext_vector_type(4)));
typedef unsigned long long u64;
#define LAS __attribute__((address_space(3)))

constexpr int NR = 8;
constexpr int NT = 64;
constexpr int HT = 16384;
constexpr int LDS_A = 0, LDS_B = 4 * HT, LDS_M = 8 * HT, LDS_BYTES = 8 * HT + 64;
constexpr unsigned ROWB = 8192;
constexpr size_t XG_PAR = 32ull << 20, XG_SRC = 4ull << 20;
constexpr unsigned XG_KT = 65536;
constexpr size_t WS_FLAGS = 64ull << 20;
constexpr size_t WS_CTR = WS_FLAGS + 524288;
constexpr size_t WS_TMO = WS_CTR + 32768;
constexpr size_t WS_NEED = WS_TMO + 4096;

struct Args { const char* x; const char* w; char* out; char* ws[NR]; int rank; };

__device__ __forceinline__ unsigned cvt_pk_bf16(float lo, float hi) { unsigned r; asm volatile("v_cvt_pk_bf16_f32 %0, %1, %2" : "=v"(r) : "v"(lo), "v"(hi)); return r; }
__device__ __forceinline__ unsigned rfl(unsigned v) { return (unsigned)__builtin_amdgcn_readfirstlane((int)v); }
__device__ __forceinline__ unsigned now_sync() { u64 t; asm volatile("s_memrealtime %0\n\ts_waitcnt lgkmcnt(0)" : "=s"(t) :: "memory"); return (unsigned)t; }
__device__ __forceinline__ unsigned off_x(int pc, int lane) { const int row = 8 * pc + (lane >> 3); return (unsigned)row * ROWB + (unsigned)(((lane & 7) ^ ((row >> 1) & 7)) * 16); }
__device__ __forceinline__ unsigned off_w(int pc, int lane) { const int k = 4 * pc + (lane >> 4); const int f = (k & 3) | (((k >> 3) & 1) << 2); return (unsigned)k * ROWB + (unsigned)(((lane & 15) ^ (2 * f)) * 16); }
}

#define GLDS(voff, sbase, ldsb, imm) asm volatile("s_add_u32 m0, %2, %3\n\ts_nop 0\n\tglobal_load_lds_dwordx4 %0, %1" :: "v"(voff), "s"(sbase), "s"(ldsb), "i"(imm) : "memory", "scc")
#define GLDS_SYS(voff, sbase, ldsb, imm) asm volatile("s_add_u32 m0, %2, %3\n\ts_nop 0\n\tglobal_load_lds_dwordx4 %0, %1 sc0 sc1" :: "v"(voff), "s"(sbase), "s"(ldsb), "i"(imm) : "memory", "scc")
#define WAIT_V(n) asm volatile("s_waitcnt vmcnt(%0)" :: "i"(n) : "memory")
#define BAR __builtin_amdgcn_s_barrier()
#define DSR128(dst, addr, off) asm volatile("ds_read_b128 %0, %1 offset:%2" : "=v"(dst) : "v"(addr), "i"(off) : "memory")
#define DSRTR(dst, addr, off) asm volatile("ds_read_b64_tr_b16 %0, %1 offset:%2" : "=v"(dst) : "v"(addr), "i"(off) : "memory")
#define LDA(bb, h) do { _Pragma("unroll") for (int m = 0; m < 4; ++m) { \
    DSR128(At[m][0], aoff0, ((bb) * 2 + (h)) * HT + m * 2048); DSR128(At[m][1], aoff1, ((bb) * 2 + (h)) * HT + m * 2048); } } while (0)
#define LDB(dst, bb, h) do { _Pragma("unroll") for (int kk = 0; kk < 2; ++kk) { \
    DSRTR(dst[0][kk][0], boff0, ((bb) * 2 + (h)) * HT + kk * 8192); DSRTR(dst[0][kk][1], boff0, ((bb) * 2 + (h)) * HT + kk * 8192 + 1024); \
    DSRTR(dst[1][kk][0], boff1, ((bb) * 2 + (h)) * HT + kk * 8192); DSRTR(dst[1][kk][1], boff1, ((bb) * 2 + (h)) * HT + kk * 8192 + 1024); } } while (0)
#define TIE_A "+v"(At[0][0]), "+v"(At[0][1]), "+v"(At[1][0]), "+v"(At[1][1]), "+v"(At[2][0]), "+v"(At[2][1]), "+v"(At[3][0]), "+v"(At[3][1])
#define TIE_B(B) "+v"(B[0][0][0]), "+v"(B[0][0][1]), "+v"(B[0][1][0]), "+v"(B[0][1][1]), "+v"(B[1][0][0]), "+v"(B[1][0][1]), "+v"(B[1][1][0]), "+v"(B[1][1][1])
#define TIE_C(ai, bj) "+v"(acc[ai][bj][0][0]), "+v"(acc[ai][bj][0][1]), "+v"(acc[ai][bj][1][0]), "+v"(acc[ai][bj][1][1]), "+v"(acc[ai][bj][2][0]), "+v"(acc[ai][bj][2][1]), "+v"(acc[ai][bj][3][0]), "+v"(acc[ai][bj][3][1])
#define WAIT_A() asm volatile("s_waitcnt lgkmcnt(0)" : TIE_A :: "memory")
#define BFRAG(Bt, n, k) __builtin_shufflevector(Bt[n][k][0], Bt[n][k][1], 0, 1, 2, 3, 4, 5, 6, 7)
#define WAIT_ABB() asm volatile("s_waitcnt lgkmcnt(0)" : TIE_A, TIE_B(B0), TIE_B(B1) :: "memory")
#define MMAW(ai, bj, Bt, TEXT) asm volatile(TEXT : TIE_C(ai, bj) : "v"(At[0][0]), "v"(At[0][1]), "v"(At[1][0]), "v"(At[1][1]), "v"(At[2][0]), "v"(At[2][1]), "v"(At[3][0]), "v"(At[3][1]), \
    "v"(BFRAG(Bt, 0, 0)), "v"(BFRAG(Bt, 0, 1)), "v"(BFRAG(Bt, 1, 0)), "v"(BFRAG(Bt, 1, 1)) : "memory", "scc")

#define TXT_W1 "s_setprio 1\n\t" "v_mfma_f32_16x16x32_bf16 %0, %16, %8, %0\n\t" "v_mfma_f32_16x16x32_bf16 %0, %17, %9, %0\n\t" "v_mfma_f32_16x16x32_bf16 %2, %17, %11, %2\n\t" "v_mfma_f32_16x16x32_bf16 %2, %16, %10, %2\n\t" "v_mfma_f32_16x16x32_bf16 %4, %16, %12, %4\n\t" "v_mfma_f32_16x16x32_bf16 %4, %17, %13, %4\n\t" "v_mfma_f32_16x16x32_bf16 %6, %17, %15, %6\n\t" "v_mfma_f32_16x16x32_bf16 %6, %16, %14, %6\n\t" "v_mfma_f32_16x16x32_bf16 %7, %18, %14, %7\n\t" "v_mfma_f32_16x16x32_bf16 %7, %19, %15, %7\n\t" "v_mfma_f32_16x16x32_bf16 %5, %19, %13, %5\n\t" "v_mfma_f32_16x16x32_bf16 %5, %18, %12, %5\n\t" "v_mfma_f32_16x16x32_bf16 %3, %18, %10, %3\n\t" "v_mfma_f32_16x16x32_bf16 %3, %19, %11, %3\n\t" "v_mfma_f32_16x16x32_bf16 %1, %19, %9, %1\n\t" "v_mfma_f32_16x16x32_bf16 %1, %18, %8, %1\n\t"
#define TXT_W2 "v_mfma_f32_16x16x32_bf16 %0, %16, %8, %0\n\t" "v_mfma_f32_16x16x32_bf16 %0, %17, %9, %0\n\t" "v_mfma_f32_16x16x32_bf16 %2, %17, %11, %2\n\t" "v_mfma_f32_16x16x32_bf16 %2, %16, %10, %2\n\t" "v_mfma_f32_16x16x32_bf16 %4, %16, %12, %4\n\t" "v_mfma_f32_16x16x32_bf16 %4, %17, %13, %4\n\t" "v_mfma_f32_16x16x32_bf16 %6, %17, %15, %6\n\t" "v_mfma_f32_16x16x32_bf16 %6, %16, %14, %6\n\t" "v_mfma_f32_16x16x32_bf16 %7, %18, %14, %7\n\t" "v_mfma_f32_16x16x32_bf16 %7, %19, %15, %7\n\t" "v_mfma_f32_16x16x32_bf16 %5, %19, %13, %5\n\t" "v_mfma_f32_16x16x32_bf16 %5, %18, %12, %5\n\t" "v_mfma_f32_16x16x32_bf16 %3, %18, %10, %3\n\t" "v_mfma_f32_16x16x32_bf16 %3, %19, %11, %3\n\t" "v_mfma_f32_16x16x32_bf16 %1, %19, %9, %1\n\t" "v_mfma_f32_16x16x32_bf16 %1, %18, %8, %1\n\t" "s_setprio 0\n\t"
#define MMA2(ai) do { MMAW(ai, 0, B0, TXT_W1); MMAW(ai, 1, B1, TXT_W2); } while (0)
#define ROLE_SETUP() \
  int tid_ = threadIdx.x; asm volatile("" : "+v"(tid_)); const int lane = tid_ & 63, fr = lane & 15, fq = lane >> 4; \
  unsigned aoff0, aoff1, boff0, boff1; \
  { const unsigned rb = (unsigned)((wid >> 2) * 64 + fr) * 128u; const unsigned s = (unsigned)(fr >> 1); \
    aoff0 = lbase + LDS_A + rb + (((unsigned)fq ^ s) << 4); aoff1 = lbase + LDS_A + rb + (((unsigned)(4 + fq) ^ s) << 4); \
    const unsigned q = (unsigned)(lane & 15) >> 2, p = (unsigned)lane & 3u, g = (unsigned)fq, f = q | ((g & 1u) << 2); \
    const unsigned kb = (8u * g + q) * 256u + 8u * (p & 1u); \
    boff0 = lbase + LDS_B + kb + ((((unsigned)(4 * (wid & 3)) + (p >> 1)) ^ (2u * f)) << 4); \
    boff1 = lbase + LDS_B + kb + ((((unsigned)(4 * (wid & 3) + 2) + (p >> 1)) ^ (2u * f)) << 4); } \
  f32x4 acc[2][2][4][2]; \
  _Pragma("unroll") for (int i0 = 0; i0 < 2; ++i0) _Pragma("unroll") for (int i1 = 0; i1 < 2; ++i1) \
  _Pragma("unroll") for (int m = 0; m < 4; ++m) _Pragma("unroll") for (int n = 0; n < 2; ++n) acc[i0][i1][m][n] = (f32x4){0.f, 0.f, 0.f, 0.f}; \
  bf16x8 At[4][2]; bf16x4 B0[2][2][2], B1[2][2][2]

namespace {
template <int AI>
__device__ __forceinline__ void store_half(const f32x4 (&acc)[2][2][4][2], char* otile, int wid, bool bad) {
  int tz = threadIdx.x; asm volatile("" : "+v"(tz));
  const int ln = tz & 63, r16 = ln & 15, q4 = ln >> 4;
  char* op = otile + (size_t)((wid >> 2) * 64) * ROWB + (size_t)((wid & 3) * 64);
  auto ro = __builtin_amdgcn_make_buffer_rsrc(op, 0, 192 * ROWB, 0x00020000);
  const unsigned vo = (unsigned)r16 * ROWB + (unsigned)((((q4 & 1) << 4) | ((q4 >> 1) << 3)) * 2);
#pragma unroll
  for (int m = 0; m < 4; ++m)
#pragma unroll
    for (int bj = 0; bj < 2; ++bj) {
      const f32x4 v0 = acc[AI][bj][m][0], v1 = acc[AI][bj][m][1];
      unsigned a0 = cvt_pk_bf16(v0[0], v0[1]), a1 = cvt_pk_bf16(v0[2], v0[3]), b0 = cvt_pk_bf16(v1[0], v1[1]), b1 = cvt_pk_bf16(v1[2], v1[3]);
      auto s0 = __builtin_amdgcn_permlane16_swap(a0, b0, false, false);
      auto s1 = __builtin_amdgcn_permlane16_swap(a1, b1, false, false);
      u32x4 w = {s0[0], s1[0], s0[1], s1[1]};
      if (bad) w = (u32x4){0x7fc07fc0u, 0x7fc07fc0u, 0x7fc07fc0u, 0x7fc07fc0u};
      __builtin_amdgcn_raw_buffer_store_b128(w, ro, vo + bj * 256, (AI * 128 + m * 16) * ROWB, 16);
    }
}
#define READ_BAD() unsigned badv; asm volatile("ds_read_b32 %0, %1" : "=v"(badv) : "v"(lbase + LDS_M) : "memory")
#define TAKE_BAD() asm volatile("" : "+v"(badv) :: "memory"); const bool bad = badv != 0u
#define MFMA_DRAIN() asm volatile("s_nop 7\n\ts_nop 7\n\ts_nop 3" ::: "memory")

#define P1(bb, ST, WE, WL) do { LDB(B0, bb, 0); LDA(bb, 0); LDB(B1, bb, 1); ST; WE; WAIT_ABB(); BAR; MMA2(0); WL; BAR; } while (0)
#define P2(bb, ST, WE, WL) do { LDA(bb, 1); ST; WE; WAIT_A(); BAR; MMA2(1); WL; BAR; } while (0)
#define LAST_PHASE(G0) do { TAKE_BAD(); LDA(1, 1); store_half<0>(acc, otile, wid, bad); WAIT_A(); BAR; MMA2(1); if (G0) BAR; MFMA_DRAIN(); store_half<1>(acc, otile, wid, bad); } while (0)
template <int NG>
__device__ __forceinline__ void worker_a(unsigned lbase, const char* xs, const char* xg, const char* ctr, char* otile, bool local, int wid) {
  ROLE_SETUP();
  const int p0 = wid == 1 ? 0 : wid * 5 - 4;
  const unsigned vA0 = local ? off_x(p0, lane) : (unsigned)(p0 * 1024 + lane * 16), vA1 = local ? off_x(p0 + 1, lane) : vA0 + 1024u;
  const unsigned ldsw = rfl(lbase + (unsigned)p0 * 1024u);
  const size_t qA = local ? 16 * (size_t)ROWB : 2048, hA = local ? 128 * (size_t)ROWB : HT, kA = local ? 128 : XG_KT;
#define STAGE_A(bb, h, gp) do { const char* g0_ = (gp) + (h) * hA; const char* g1_ = g0_ + qA; const char* g2_ = g1_ + qA; \
    GLDS_SYS(vA0, g0_, ldsw, LDS_A + ((bb) * 2 + (h)) * HT); GLDS_SYS(vA1, g0_, ldsw, LDS_A + ((bb) * 2 + (h)) * HT + 1024); \
    GLDS_SYS(vA0, g1_, ldsw, LDS_A + ((bb) * 2 + (h)) * HT + 2048); GLDS_SYS(vA1, g1_, ldsw, LDS_A + ((bb) * 2 + (h)) * HT + 3072); \
    GLDS_SYS(vA0, g2_, ldsw, LDS_A + ((bb) * 2 + (h)) * HT + 4096); if (NG == 6) GLDS_SYS(vA1, g2_, ldsw, LDS_A + ((bb) * 2 + (h)) * HT + 5120); } while (0)
  unsigned cv; { unsigned z = 0; asm volatile("global_load_dword %0, %1, %2 sc1\n\ts_waitcnt vmcnt(0)" : "=&v"(cv) : "v"(z), "s"(ctr) : "memory"); }
  const char* pA = local ? xs : xg + ((rfl(cv) + 1u) & 1u) * XG_PAR;
  BAR; STAGE_A(0, 0, pA);
  BAR; STAGE_A(0, 1, pA);
  BAR;
  WAIT_V(NG); BAR;
#pragma unroll 1
  for (int t = 0; t < NT - 2; t += 2) {
    const char* a1 = pA + (size_t)(t + 1) * kA; const char* a2 = a1 + kA;
    P1(0, STAGE_A(1, 0, a1), , WAIT_V(NG));
    P2(0, STAGE_A(1, 1, a1), , WAIT_V(NG));
    P1(1, STAGE_A(0, 0, a2), , WAIT_V(NG));
    P2(1, STAGE_A(0, 1, a2), , WAIT_V(NG));
  }
  P1(0, STAGE_A(1, 0, pA + (size_t)(NT - 1) * kA), , WAIT_V(NG));
  P2(0, STAGE_A(1, 1, pA + (size_t)(NT - 1) * kA), , WAIT_V(NG));
  READ_BAD();
  P1(1, , , WAIT_V(0));
  LAST_PHASE(true);
#undef STAGE_A
}
__device__ __forceinline__ void worker_b(unsigned lbase, const char* pB, char* otile, int wid) {
  ROLE_SETUP();
  const unsigned vB0 = off_w(wid - 4, lane), vB1 = vB0 + 16u * ROWB, vB2 = vB0 + 32u * ROWB, vB3 = vB0 + 48u * ROWB;
  const unsigned ldsw = rfl(lbase + (unsigned)(wid - 4) * 1024u);
  constexpr size_t kB = 64 * (size_t)ROWB;
#define STAGE_BH(bb, h, g_) do { GLDS(vB0, g_, ldsw, LDS_B + ((bb) * 2 + (h)) * HT); GLDS(vB1, g_, ldsw, LDS_B + ((bb) * 2 + (h)) * HT + 4096); \
    GLDS(vB2, g_, ldsw, LDS_B + ((bb) * 2 + (h)) * HT + 8192); GLDS(vB3, g_, ldsw, LDS_B + ((bb) * 2 + (h)) * HT + 12288); } while (0)
#define STAGE_B(bb, gp) do { const char* h0_ = (gp); const char* h1_ = h0_ + 256; STAGE_BH(bb, 0, h0_); STAGE_BH(bb, 1, h1_); } while (0)
  STAGE_B(0, pB); STAGE_B(1, pB + kB);
  BAR; BAR; BAR;
  WAIT_V(8); BAR;
  BAR;
#pragma unroll 1
  for (int t = 0; t < NT - 2; t += 2) {
    const char* b2 = pB + (size_t)(t + 2) * kB;
    P1(0, , , );
    P2(0, STAGE_B(0, b2), WAIT_V(8), );
    P1(1, , , );
    P2(1, STAGE_B(1, b2 + kB), WAIT_V(8), );
  }
  P1(0, , , );
  P2(0, , WAIT_V(0), );
  READ_BAD();
  P1(1, , , );
  LAST_PHASE(false);
#undef STAGE_B
#undef STAGE_BH
}
}

__global__ __launch_bounds__(512, 2) __attribute__((amdgpu_num_vgpr(108), amdgpu_num_sgpr(60))) void ag_gemm_kernel(Args a) {
  extern __shared__ __attribute__((aligned(16))) char smem[];
  LAS char* lds = (LAS char*)smem;
  asm volatile("v_mov_b32 v239, 0\n\tv_mov_b32 v236, 0" ::: "v220", "v255", "s54", "s101");
  const int wid = __builtin_amdgcn_readfirstlane(threadIdx.x >> 6);
  const int b = blockIdx.x, rank = a.rank;
  const int xcd = b & 7, idx = b >> 3;
  const int pm = 4 * (xcd >> 1) + (idx & 3), pn = 8 * (xcd & 1) + (idx >> 2);
  const int src = pm >> 1, strip = pm & 1;
  const bool local = (src == rank);
  char* myws = a.ws[rank];
  const char* ctr = myws + WS_CTR + (size_t)b * 128;

  const unsigned lbase = (unsigned)(size_t)lds;
  char* otile = a.out + (size_t)(pm * 256) * ROWB + (size_t)pn * 512;
  const char* xs = a.x + (size_t)strip * 256 * ROWB;
  const char* xg = myws + (size_t)src * XG_SRC + (size_t)strip * 2 * HT;
  const char* pB = a.w + (size_t)pn * 512;

  if (wid >= 4) worker_b(lbase, pB, otile, wid);
  else if (wid >= 2) worker_a<5>(lbase, xs, xg, ctr, otile, local, wid);
  else if (wid == 1) worker_a<6>(lbase, xs, xg, ctr, otile, local, wid);
  else {
    const unsigned t_start = now_sync();
    ROLE_SETUP();
    const unsigned jj = (unsigned)((((((idx >> 1) ^ rank) & 1) << 4) | ((idx >> 2) << 1) | (idx & 1)) * 8 + xcd);
    const unsigned vx0 = off_x(0, lane), vx1 = off_x(1, lane);
    unsigned cv; { unsigned z = 0; asm volatile("global_load_dword %0, %1, %2 sc1" : "=v"(cv) : "v"(z), "s"(ctr) : "memory"); }
    const unsigned idA = (jj / 7u) * 8u + jj % 7u;
    { const char* sa = a.x + (size_t)(((idA << 15) & 0x3c0000u) | (idA & 0x1f80u));
      asm volatile("v_mov_b32 v230, %0\n\tv_mov_b32 v229, %1\n\tv_add_u32 v225, 0x20000, %0\n\tv_add_u32 v223, 0x20000, %1\n\t"
                   "global_load_dwordx4 v[240:243], v230, %2\n\tglobal_load_dwordx4 v[244:247], v229, %2\n\tglobal_load_dwordx4 v[248:251], v225, %2\n\tglobal_load_dwordx4 v[252:255], v223, %2"
                   :: "v"(vx0), "v"(vx1), "s"(sa) : "memory"); }
    { const unsigned l128 = (unsigned)lane * 128u;
      asm volatile("global_load_dword v222, %0, %1 sc1\n\tglobal_load_dword v222, %2, %1 sc1\n\ts_mov_b64 s[54:55], %3\n\tv_mov_b32 v221, %4\n\tv_mov_b32 v220, %5"
                   :: "v"(l128 + jj * ROWB), "s"(a.x), "v"(l128 + (256u + jj) * ROWB), "s"(a.w), "v"(l128 + (128u + jj) * ROWB), "v"(l128 + (3840u + jj) * ROWB) : "memory");
      asm volatile("global_load_dword v222, v221, s[54:55] sc1\n\tv_add_u32 v221, 0x200000, v221" ::: "memory"); }
    char* pw = a.ws[(rank + 1) & 7];
#pragma unroll
    for (int q = 1; q < 7; ++q) { char* c = a.ws[(rank + 1 + q) & 7]; asm volatile("" : "+s"(c)); if (lane == q) pw = c; }
    *(LAS unsigned*)(lds + LDS_M) = 0u;
    const unsigned foff = (unsigned)(src * 65536 + strip * 256);
    const char* flags = myws + WS_FLAGS + foff;
    const unsigned g = ((unsigned)lane >> 2) & 7u, gcode = (g >> 1) * 4u + (g & 1u);
    { const unsigned pbase = gcode * 128u + ((unsigned)lane & 3u) * 32u;
      asm volatile("v_mov_b32 v231, %0\n\tv_mov_b32 v228, %1\n\tv_mov_b32 v237, %1\n\tv_mov_b32 v227, %2\n\tv_mov_b32 v226, %3"
                   :: "v"((unsigned)lane * 16u), "v"(pbase), "v"(g), "s"((unsigned)(WS_TMO - WS_FLAGS) - foff) : "memory"); }
    { const int due0 = T0 + (int)((jj * D1) >> 8);
      asm volatile("s_mov_b32 s70, 0\n\ts_mov_b32 s71, -4\n\ts_mov_b32 s72, 0x7fffffff\n\ts_mov_b32 s75, 0\n\t"
                   "s_mov_b32 s73, %0\n\ts_mov_b32 s74, %1\n\ts_mov_b32 s86, 7\n\ts_sub_u32 s66, 0, %2\n\t"
                   "s_mov_b32 s89, %3\n\ts_mov_b64 s[90:91], %4\n\ts_mov_b64 s[92:93], %5"
                   :: "s"(due0), "s"(idA), "s"(rfl(local ? 0u : 1u)),
                      "s"(t_start), "s"(a.x), "s"(flags) : "memory", "scc"); }
    asm volatile("" : TIE_C(0, 0), TIE_C(0, 1)); asm volatile("" : TIE_C(1, 0), TIE_C(1, 1));
    asm volatile("s_waitcnt vmcnt(" STR(NPRE) ")" : "+v"(cv) :: "memory");
    const unsigned call = rfl(cv) + 1u;
    { const int late = max((int)(now_sync() - t_start) + 10 - T0, 0); asm volatile("s_add_i32 s73, s73, %0" :: "s"(late) : "memory", "scc"); }
    { const unsigned xgo = (call & 1u) * (unsigned)XG_PAR + (unsigned)rank * (unsigned)XG_SRC; const u64 pd = (u64)(pw + xgo);
      asm volatile("v_mov_b32 v234, %0\n\tv_mov_b32 v235, %1\n\ts_mov_b32 s88, %2\n\ts_mov_b32 s68, %3\n\tv_mov_b32 v233, %4\n\tv_mov_b32 v224, %4\n\tv_mov_b32 v232, %5\n\ts_nop 1\n\t" "s_mov_b32 s87, 0\n\t" "s_mov_b32 s63, 0\n\t" "s_mov_b32 s85, 28\n\t" "s_mov_b32 s69, 0\n\t" "s_add_i32 s98, s70, 6\n\t" "s_add_i32 s70, s70, s63\n\t" "s_mov_b32 s63, 0\n\t" "s_cmp_lt_u32 s74, " STR(ID1) "\n\t" "s_cselect_b32 s99, 0, (" STR(D2) "-" STR(D1) ")\n\t" "s_add_i32 s99, s99, s73\n\t" "s_addk_i32 s99, " STR(D1) "\n\t" "s_cmp_lt_u32 s69, 0x2000\n\t" "s_cselect_b32 s99, s99, 0x7fffffff\n\t" "s_cmp_lg_u32 s87, 0\n\t" "s_cselect_b32 s72, s98, s72\n\t" "s_cselect_b32 s75, s74, s75\n\t" "s_cselect_b32 s71, s98, s71\n\t" "s_cselect_b32 s73, s99, s73\n\t" "s_cselect_b32 s74, s69, s74\n\t" "s_mov_b32 s87, 0\n\t" "s_and_b32 s98, s75, 7\n\t" "s_lshl_b32 s99, s75, 2\n\t" "v_readlane_b32 s82, v234, s98\n\t" "v_readlane_b32 s83, v235, s98\n\t" "s_and_b32 s99, s99, 0x7fe0\n\t" "s_lshr_b32 s98, s75, 5\n\t" "s_add_i32 s98, s98, s88\n\t" "s_add_u32 s82, s82, s99\n\t" "s_addc_u32 s83, s83, 0\n\t" "v_mov_b32 v238, s98\n\t" "s_mov_b32 s70, " STR(NPRE) "\n\t" "s_add_i32 s85, s85, 4\n\t" "s_and_b32 s84, s68, s66\n\t" "s_and_b32 s85, s85, 28\n\t" "s_mov_b32 s56, " STR(WF) "\n\t" "s_mov_b32 s57, " STR(WF) "\n\t" "s_lshl_b64 s[56:57], s[56:57], s85\n\t" "s_and_b32 s59, s57, s66\n\t" "s_memrealtime s[96:97]\n\t" "s_and_b32 s98, s74, 7\n\t" "s_and_b32 s99, s74, 0x1ff8\n\t" "v_readlane_b32 s78, v234, s98\n\t" "v_readlane_b32 s79, v235, s98\n\t" "s_lshl_b32 s99, s99, 9\n\t" "s_cmp_ge_u32 s98, 3\n\t" "s_addc_u32 s69, s74, 292\n\t" "s_add_u32 s78, s78, s99\n\t" "s_addc_u32 s79, s79, 0\n\t" "s_lshl_b32 s98, s69, 15\n\t" "s_and_b32 s99, s69, 0x1f80\n\t" "s_and_b32 s98, s98, 0x3c0000\n\t" "s_or_b32 s98, s98, s99\n\t" "s_add_u32 s80, s90, s98\n\t" "s_addc_u32 s81, s91, 0\n\t" "s_waitcnt lgkmcnt(0)\n\t" "s_sub_i32 s62, s96, s89\n\t" "s_getreg_b32 s98, hwreg(HW_REG_IB_STS, 22, 2)\n\t" "s_getreg_b32 s99, hwreg(HW_REG_IB_STS, 0, 4)\n\t" "s_lshl4_add_u32 s98, s98, s99\n\t" "s_sub_i32 s76, s70, s98\n\t" "s_add_i32 s99, s71, 4\n\t" "s_cmp_ge_i32 s76, s99\n\t" "s_cselect_b32 s77, s73, 0x7fffffff\n\t"
                   :: "v"((unsigned)pd), "v"((unsigned)(pd >> 32)), "s"(call * 256u + 1u), "s"(call * 256u + 1u + 2u * (unsigned)strip),
                      "v"(call * 256u + 1u + 2u * (unsigned)strip + gcode), "s"((unsigned)WS_FLAGS + (unsigned)rank * 65536u - xgo) : "memory", "scc"); }
    asm volatile("s_branch ag_slow_end\n\t" "ag_tr_a:\n\t" "s_call_b64 s[94:95], ag_slow_sub\n\t" "s_branch ag_bk_a\n\t" "ag_tr_b:\n\t" "s_call_b64 s[94:95], ag_slow_sub\n\t" "s_branch ag_bk_b\n\t" "ag_tr_c:\n\t" "s_call_b64 s[94:95], ag_slow_sub\n\t" "s_branch ag_bk_c\n\t" "ag_tr_d:\n\t" "s_call_b64 s[94:95], ag_slow_sub\n\t" "s_branch ag_bk_d\n\t" "ag_tr_e:\n\t" "s_call_b64 s[94:95], ag_slow_sub\n\t" "s_branch ag_bk_e\n\t" "ag_tr_p0:\n\t" "s_call_b64 s[94:95], ag_slow_sub\n\t" "s_branch ag_bk_p0\n\t" "ag_tr_p1:\n\t" "s_call_b64 s[94:95], ag_slow_sub\n\t" "s_branch ag_bk_p1\n\t" "ag_tr_p2:\n\t" "s_call_b64 s[94:95], ag_slow_sub\n\t" "s_branch ag_bk_p2\n\t" "ag_slow_sub:\n\t" "s_add_i32 s98, s70, 6\n\t" "s_add_i32 s70, s70, s63\n\t" "s_mov_b32 s63, 0\n\t" "s_cmp_lt_u32 s74, " STR(ID1) "\n\t" "s_cselect_b32 s99, 0, (" STR(D2) "-" STR(D1) ")\n\t" "s_add_i32 s99, s99, s73\n\t" "s_addk_i32 s99, " STR(D1) "\n\t" "s_cmp_lt_u32 s69, 0x2000\n\t" "s_cselect_b32 s99, s99, 0x7fffffff\n\t" "s_cmp_lg_u32 s87, 0\n\t" "s_cselect_b32 s72, s98, s72\n\t" "s_cselect_b32 s75, s74, s75\n\t" "s_cselect_b32 s71, s98, s71\n\t" "s_cselect_b32 s73, s99, s73\n\t" "s_cselect_b32 s74, s69, s74\n\t" "s_mov_b32 s87, 0\n\t" "s_and_b32 s98, s75, 7\n\t" "s_lshl_b32 s99, s75, 2\n\t" "v_readlane_b32 s82, v234, s98\n\t" "v_readlane_b32 s83, v235, s98\n\t" "s_and_b32 s99, s99, 0x7fe0\n\t" "s_lshr_b32 s98, s75, 5\n\t" "s_add_i32 s98, s98, s88\n\t" "s_add_u32 s82, s82, s99\n\t" "s_addc_u32 s83, s83, 0\n\t" "v_mov_b32 v238, s98\n\t" "s_mov_b32 s56, " STR(WS) "\n\t" "s_mov_b32 s57, " STR(WS) "\n\t" "s_lshl_b64 s[56:57], s[56:57], s85\n\t" "s_and_b32 s59, s57, s66\n\t" "s_memrealtime s[96:97]\n\t" "s_and_b32 s98, s74, 7\n\t" "s_and_b32 s99, s74, 0x1ff8\n\t" "v_readlane_b32 s78, v234, s98\n\t" "v_readlane_b32 s79, v235, s98\n\t" "s_lshl_b32 s99, s99, 9\n\t" "s_cmp_ge_u32 s98, 3\n\t" "s_addc_u32 s69, s74, 292\n\t" "s_add_u32 s78, s78, s99\n\t" "s_addc_u32 s79, s79, 0\n\t" "s_lshl_b32 s98, s69, 15\n\t" "s_and_b32 s99, s69, 0x1f80\n\t" "s_and_b32 s98, s98, 0x3c0000\n\t" "s_or_b32 s98, s98, s99\n\t" "s_add_u32 s80, s90, s98\n\t" "s_addc_u32 s81, s91, 0\n\t" "s_waitcnt lgkmcnt(0)\n\t" "s_sub_i32 s62, s96, s89\n\t" "s_getreg_b32 s98, hwreg(HW_REG_IB_STS, 22, 2)\n\t" "s_getreg_b32 s99, hwreg(HW_REG_IB_STS, 0, 4)\n\t" "s_lshl4_add_u32 s98, s98, s99\n\t" "s_sub_i32 s76, s70, s98\n\t" "s_add_i32 s99, s71, 4\n\t" "s_cmp_ge_i32 s76, s99\n\t" "s_cselect_b32 s77, s73, 0x7fffffff\n\t" "ag_slow_loop:\n\t" "v_cmp_gt_u32 s[64:65], v224, v239\n\t" "s_cmp_ge_i32 s76, s72\n\t" "s_cselect_b64 exec, 1, 0\n\t" "global_store_dword v232, v238, s[82:83] sc0 sc1\n\t" "s_cselect_b32 s72, 0x7fffffff, s72\n\t" "s_and_b32 exec_lo, s64, s59\n\t" "global_load_dword v239, v237, s[92:93] sc0 sc1\n\t" "s_cmp_ge_i32 s62, s77\n\t" "s_mov_b64 exec, -1\n\t" "s_cselect_b32 s87, 1, 0\n\t" "s_cselect_b32 s63, 11, 2\n\t" "s_cbranch_scc0 1f\n\t" "global_store_dwordx4 v231, v[240:243], s[78:79] sc0 sc1\n\t" "global_store_dwordx4 v231, v[244:247], s[78:79] offset:1024 sc0 sc1\n\t" "global_store_dwordx4 v231, v[248:251], s[78:79] offset:2048 sc0 sc1\n\t" "global_store_dwordx4 v231, v[252:255], s[78:79] offset:3072 sc0 sc1\n\t" "global_load_dwordx4 v[240:243], v230, s[80:81]\n\t" "global_load_dwordx4 v[244:247], v229, s[80:81]\n\t" "global_load_dwordx4 v[248:251], v225, s[80:81]\n\t" "global_load_dwordx4 v[252:255], v223, s[80:81]\n\t" "global_load_dword v222, v221, s[54:55] " STR(PFB) "\n\t" "v_add_u32 v221, 0x200000, v221\n\t" "v_min_u32 v221, v221, v220\n\t" "1:\n\t" "global_load_dword v236, v226, s[92:93] sc1\n\t" "s_add_i32 s98, s70, 6\n\t" "s_add_i32 s70, s70, s63\n\t" "s_mov_b32 s63, 0\n\t" "s_cmp_lt_u32 s74, " STR(ID1) "\n\t" "s_cselect_b32 s99, 0, (" STR(D2) "-" STR(D1) ")\n\t" "s_add_i32 s99, s99, s73\n\t" "s_addk_i32 s99, " STR(D1) "\n\t" "s_cmp_lt_u32 s69, 0x2000\n\t" "s_cselect_b32 s99, s99, 0x7fffffff\n\t" "s_cmp_lg_u32 s87, 0\n\t" "s_cselect_b32 s72, s98, s72\n\t" "s_cselect_b32 s75, s74, s75\n\t" "s_cselect_b32 s71, s98, s71\n\t" "s_cselect_b32 s73, s99, s73\n\t" "s_cselect_b32 s74, s69, s74\n\t" "s_mov_b32 s87, 0\n\t" "s_add_i32 s70, s70, 1\n\t" "s_and_b32 s98, s75, 7\n\t" "s_lshl_b32 s99, s75, 2\n\t" "v_readlane_b32 s82, v234, s98\n\t" "v_readlane_b32 s83, v235, s98\n\t" "s_and_b32 s99, s99, 0x7fe0\n\t" "s_lshr_b32 s98, s75, 5\n\t" "s_add_i32 s98, s98, s88\n\t" "s_add_u32 s82, s82, s99\n\t" "s_addc_u32 s83, s83, 0\n\t" "v_mov_b32 v238, s98\n\t" "s_sleep " STR(SLP) "\n\t" "s_memrealtime s[96:97]\n\t" "s_and_b32 s98, s74, 7\n\t" "s_and_b32 s99, s74, 0x1ff8\n\t" "v_readlane_b32 s78, v234, s98\n\t" "v_readlane_b32 s79, v235, s98\n\t" "s_lshl_b32 s99, s99, 9\n\t" "s_cmp_ge_u32 s98, 3\n\t" "s_addc_u32 s69, s74, 292\n\t" "s_add_u32 s78, s78, s99\n\t" "s_addc_u32 s79, s79, 0\n\t" "s_lshl_b32 s98, s69, 15\n\t" "s_and_b32 s99, s69, 0x1f80\n\t" "s_and_b32 s98, s98, 0x3c0000\n\t" "s_or_b32 s98, s98, s99\n\t" "s_add_u32 s80, s90, s98\n\t" "s_addc_u32 s81, s91, 0\n\t" "s_waitcnt lgkmcnt(0)\n\t" "s_sub_i32 s62, s96, s89\n\t" "s_getreg_b32 s98, hwreg(HW_REG_IB_STS, 22, 2)\n\t" "s_getreg_b32 s99, hwreg(HW_REG_IB_STS, 0, 4)\n\t" "s_lshl4_add_u32 s98, s98, s99\n\t" "s_sub_i32 s76, s70, s98\n\t" "s_add_i32 s99, s71, 4\n\t" "s_cmp_ge_i32 s76, s99\n\t" "s_cselect_b32 s77, s73, 0x7fffffff\n\t" "v_readfirstlane_b32 s98, v236\n\t" "s_cmp_gt_u32 s62, " STR(DEADLINE) "\n\t" "s_cselect_b32 s98, s88, s98\n\t" "s_cmp_eq_u32 s98, s88\n\t" "s_cbranch_scc1 ag_slow_dead\n\t" "v_cmp_le_u32 s[100:101], s84, v239\n\t" "s_lshr_b32 s98, s100, s85\n\t" "s_andn2_b32 s98, 15, s98\n\t" "s_cbranch_scc1 ag_slow_loop\n\t" "s_setpc_b64 s[94:95]\n\t" "ag_slow_dead:\n\t" "s_mov_b32 s66, 0\n\t" "s_mov_b32 s59, 0\n\t" "s_mov_b32 s58, 0\n\t" "s_mov_b32 s84, 0\n\t" "v_mov_b32 v224, s88\n\t" "v_mov_b32 v237, " STR(LDSBAD) "\n\t" "global_store_dword v226, v224, s[92:93] sc1\n\t" "ds_write_b32 v237, v224\n\t" "s_add_i32 s70, s70, 1\n\t" "s_waitcnt lgkmcnt(0)\n\t" "s_setpc_b64 s[94:95]\n\t" "ag_slow_end:\n\t" ::: "memory", "scc");
#define ENG(TEXT) asm volatile(TEXT ::: "memory", "scc")
#define TXT_ACT "v_cmp_gt_u32 s[64:65], v224, v239\n\t" "s_cmp_ge_i32 s76, s72\n\t" "s_cselect_b64 exec, 1, 0\n\t" "global_store_dword v232, v238, s[82:83] sc0 sc1\n\t" "s_cselect_b32 s72, 0x7fffffff, s72\n\t" "s_and_b32 exec_lo, s64, s58\n\t" "global_load_dword v239, v237, s[92:93] sc0 sc1\n\t" "s_cmp_ge_i32 s62, s77\n\t" "s_mov_b64 exec, -1\n\t" "s_cselect_b32 s87, 1, 0\n\t" "s_cselect_b32 s63, 11, 2\n\t" "s_cbranch_scc0 1f\n\t" "global_store_dwordx4 v231, v[240:243], s[78:79] sc0 sc1\n\t" "global_store_dwordx4 v231, v[244:247], s[78:79] offset:1024 sc0 sc1\n\t" "global_store_dwordx4 v231, v[248:251], s[78:79] offset:2048 sc0 sc1\n\t" "global_store_dwordx4 v231, v[252:255], s[78:79] offset:3072 sc0 sc1\n\t" "global_load_dwordx4 v[240:243], v230, s[80:81]\n\t" "global_load_dwordx4 v[244:247], v229, s[80:81]\n\t" "global_load_dwordx4 v[248:251], v225, s[80:81]\n\t" "global_load_dwordx4 v[252:255], v223, s[80:81]\n\t" "global_load_dword v222, v221, s[54:55] " STR(PFB) "\n\t" "v_add_u32 v221, 0x200000, v221\n\t" "v_min_u32 v221, v221, v220\n\t" "1:\n\t"
#define TXT_SEC2 "s_add_i32 s68, s68, 3\n\t" "s_add_i32 s85, s85, 4\n\t" "s_and_b32 s84, s68, s66\n\t" "s_and_b32 s85, s85, 28\n\t" "s_add_i32 s86, s86, 2\n\t" "v_sub_u32 v224, s86, v227\n\t" "v_lshrrev_b32 v224, 3, v224\n\t" "v_lshl_add_u32 v237, v224, 11, v228\n\t" "v_lshl_add_u32 v224, v224, 4, v233\n\t" "s_mov_b32 s56, 0xff\n\t" "s_mov_b32 s57, 0xff\n\t" "s_lshl_b64 s[56:57], s[56:57], s85\n\t" "s_and_b32 s59, s57, s66\n\t" "s_mov_b32 s56, 0xff0\n\t" "s_mov_b32 s57, 0xff0\n\t" "s_lshl_b64 s[56:57], s[56:57], s85\n\t" "s_and_b32 s58, s57, s66\n\t" "v_cmp_gt_u32 s[64:65], v224, v239\n\t" "s_mov_b32 exec_hi, 0\n\t" "s_and_b32 exec_lo, s64, s59\n\t" "global_load_dword v239, v237, s[92:93] sc0 sc1\n\t" "s_mov_b64 exec, -1\n\t" "s_add_i32 s98, s70, 6\n\t" "s_add_i32 s70, s70, s63\n\t" "s_mov_b32 s63, 0\n\t" "s_cmp_lt_u32 s74, " STR(ID1) "\n\t" "s_cselect_b32 s99, 0, (" STR(D2) "-" STR(D1) ")\n\t" "s_add_i32 s99, s99, s73\n\t" "s_addk_i32 s99, " STR(D1) "\n\t" "s_cmp_lt_u32 s69, 0x2000\n\t" "s_cselect_b32 s99, s99, 0x7fffffff\n\t" "s_cmp_lg_u32 s87, 0\n\t" "s_cselect_b32 s72, s98, s72\n\t" "s_cselect_b32 s75, s74, s75\n\t" "s_cselect_b32 s71, s98, s71\n\t" "s_cselect_b32 s73, s99, s73\n\t" "s_cselect_b32 s74, s69, s74\n\t" "s_mov_b32 s87, 0\n\t" "s_add_i32 s70, s70, 1\n\t" "s_and_b32 s98, s75, 7\n\t" "s_lshl_b32 s99, s75, 2\n\t" "v_readlane_b32 s82, v234, s98\n\t" "v_readlane_b32 s83, v235, s98\n\t" "s_and_b32 s99, s99, 0x7fe0\n\t" "s_lshr_b32 s98, s75, 5\n\t" "s_add_i32 s98, s98, s88\n\t" "s_add_u32 s82, s82, s99\n\t" "s_addc_u32 s83, s83, 0\n\t" "v_mov_b32 v238, s98\n\t" "s_and_b32 s98, s74, 7\n\t" "s_and_b32 s99, s74, 0x1ff8\n\t" "v_readlane_b32 s78, v234, s98\n\t" "v_readlane_b32 s79, v235, s98\n\t" "s_lshl_b32 s99, s99, 9\n\t" "s_cmp_ge_u32 s98, 3\n\t" "s_addc_u32 s69, s74, 292\n\t" "s_add_u32 s78, s78, s99\n\t" "s_addc_u32 s79, s79, 0\n\t" "s_lshl_b32 s98, s69, 15\n\t" "s_and_b32 s99, s69, 0x1f80\n\t" "s_and_b32 s98, s98, 0x3c0000\n\t" "s_or_b32 s98, s98, s99\n\t" "s_add_u32 s80, s90, s98\n\t" "s_addc_u32 s81, s91, 0\n\t"
#define TXT_P1A "s_setprio 1\n\t" "v_mfma_f32_16x16x32_bf16 %0, %16, %8, %0\n\t" "s_add_i32 s68, s68, 1\n\t" "v_mfma_f32_16x16x32_bf16 %0, %17, %9, %0\n\t" "s_add_i32 s85, s85, 4\n\t" "v_mfma_f32_16x16x32_bf16 %2, %17, %11, %2\n\t" "s_and_b32 s84, s68, s66\n\t" "v_mfma_f32_16x16x32_bf16 %2, %16, %10, %2\n\t" "s_and_b32 s85, s85, 28\n\t" "v_mfma_f32_16x16x32_bf16 %4, %16, %12, %4\n\t" "v_mfma_f32_16x16x32_bf16 %4, %17, %13, %4\n\t" "v_mfma_f32_16x16x32_bf16 %6, %17, %15, %6\n\t" "v_mfma_f32_16x16x32_bf16 %6, %16, %14, %6\n\t" "v_mfma_f32_16x16x32_bf16 %7, %18, %14, %7\n\t" "v_mfma_f32_16x16x32_bf16 %7, %19, %15, %7\n\t" "v_mfma_f32_16x16x32_bf16 %5, %19, %13, %5\n\t" "v_mfma_f32_16x16x32_bf16 %5, %18, %12, %5\n\t" "v_mfma_f32_16x16x32_bf16 %3, %18, %10, %3\n\t" "v_mfma_f32_16x16x32_bf16 %3, %19, %11, %3\n\t" "v_mfma_f32_16x16x32_bf16 %1, %19, %9, %1\n\t" "v_mfma_f32_16x16x32_bf16 %1, %18, %8, %1\n\t"
#define TXT_P2BN "v_mfma_f32_16x16x32_bf16 %0, %16, %8, %0\n\t" "v_mfma_f32_16x16x32_bf16 %0, %17, %9, %0\n\t" "v_mfma_f32_16x16x32_bf16 %2, %17, %11, %2\n\t" "v_mfma_f32_16x16x32_bf16 %2, %16, %10, %2\n\t" "v_mfma_f32_16x16x32_bf16 %4, %16, %12, %4\n\t" "s_memrealtime s[96:97]\n\t" "v_mfma_f32_16x16x32_bf16 %4, %17, %13, %4\n\t" "v_mfma_f32_16x16x32_bf16 %6, %17, %15, %6\n\t" "s_getreg_b32 s98, hwreg(HW_REG_IB_STS, 22, 2)\n\t" "v_mfma_f32_16x16x32_bf16 %6, %16, %14, %6\n\t" "s_getreg_b32 s99, hwreg(HW_REG_IB_STS, 0, 4)\n\t" "v_mfma_f32_16x16x32_bf16 %7, %18, %14, %7\n\t" "s_lshl4_add_u32 s98, s98, s99\n\t" "v_mfma_f32_16x16x32_bf16 %7, %19, %15, %7\n\t" "s_sub_i32 s76, s70, s98\n\t" "v_mfma_f32_16x16x32_bf16 %5, %19, %13, %5\n\t" "s_add_i32 s99, s71, 4\n\t" "v_mfma_f32_16x16x32_bf16 %5, %18, %12, %5\n\t" "s_cmp_ge_i32 s76, s99\n\t" "v_mfma_f32_16x16x32_bf16 %3, %18, %10, %3\n\t" "s_cselect_b32 s77, s73, 0x7fffffff\n\t" "v_mfma_f32_16x16x32_bf16 %3, %19, %11, %3\n\t" "s_waitcnt lgkmcnt(0)\n\t" "v_mfma_f32_16x16x32_bf16 %1, %19, %9, %1\n\t" "s_sub_i32 s62, s96, s89\n\t" "v_mfma_f32_16x16x32_bf16 %1, %18, %8, %1\n\t" "s_setprio 0\n\t"
    ENG("s_cmp_lg_u32 s66, 0\n\t" "s_cbranch_scc1 ag_pre_end\n\t" "s_cmp_eq_u32 s73, 0x7fffffff\n\t" "s_cbranch_scc1 ag_pre_end\n\t" "s_mov_b32 s61, s74\n\t" "ag_pre:\n\t" "s_call_b64 s[94:95], ag_slow_sub\n\t" "s_cmp_eq_u32 s74, s61\n\t" "s_cbranch_scc1 ag_pre\n\t" "s_cmp_lg_u32 s72, 0x7fffffff\n\t" "s_cbranch_scc1 ag_pre\n\t" "ag_pre_end:\n\t");
    ENG("v_cmp_le_u32 s[100:101], s84, v239\n\t" "s_lshr_b32 s98, s100, s85\n\t" "s_andn2_b32 s98, 15, s98\n\t" "s_cbranch_scc1 ag_tr_p0\n\t" "ag_bk_p0:\n\t"); BAR; ENG("s_add_i32 s86, s86, 1\n\t" "s_add_i32 s68, s68, 1\n\t" "v_sub_u32 v224, s86, v227\n\t" "v_lshrrev_b32 v224, 3, v224\n\t" "v_lshl_add_u32 v237, v224, 11, v228\n\t" "v_lshl_add_u32 v224, v224, 4, v233\n\t" "s_add_i32 s85, s85, 4\n\t" "s_and_b32 s84, s68, s66\n\t" "s_and_b32 s85, s85, 28\n\t" "s_mov_b32 s56, " STR(WF) "\n\t" "s_mov_b32 s57, " STR(WF) "\n\t" "s_lshl_b64 s[56:57], s[56:57], s85\n\t" "s_and_b32 s59, s57, s66\n\t");
    ENG("v_cmp_le_u32 s[100:101], s84, v239\n\t" "s_lshr_b32 s98, s100, s85\n\t" "s_andn2_b32 s98, 15, s98\n\t" "s_cbranch_scc1 ag_tr_p1\n\t" "ag_bk_p1:\n\t"); BAR; ENG("s_add_i32 s86, s86, 1\n\t" "s_add_i32 s68, s68, 3\n\t" "v_sub_u32 v224, s86, v227\n\t" "v_lshrrev_b32 v224, 3, v224\n\t" "v_lshl_add_u32 v237, v224, 11, v228\n\t" "v_lshl_add_u32 v224, v224, 4, v233\n\t" "s_add_i32 s85, s85, 4\n\t" "s_and_b32 s84, s68, s66\n\t" "s_and_b32 s85, s85, 28\n\t" "s_mov_b32 s56, " STR(WF) "\n\t" "s_mov_b32 s57, " STR(WF) "\n\t" "s_lshl_b64 s[56:57], s[56:57], s85\n\t" "s_and_b32 s59, s57, s66\n\t");
    BAR;
    ENG("v_cmp_le_u32 s[100:101], s84, v239\n\t" "s_lshr_b32 s98, s100, s85\n\t" "s_andn2_b32 s98, 15, s98\n\t" "s_cbranch_scc1 ag_tr_p2\n\t" "ag_bk_p2:\n\t");
    ENG("s_mov_b32 s56, 0xff\n\t" "s_mov_b32 s57, 0xff\n\t" "s_lshl_b64 s[56:57], s[56:57], s85\n\t" "s_and_b32 s59, s57, s66\n\t" "s_mov_b32 s56, 0xff0\n\t" "s_mov_b32 s57, 0xff0\n\t" "s_lshl_b64 s[56:57], s[56:57], s85\n\t" "s_and_b32 s58, s57, s66\n\t");
    BAR;
#define W0P1(bb) do { ENG(TXT_ACT); LDB(B0, bb, 0); LDA(bb, 0); LDB(B1, bb, 1); } while (0)
#pragma unroll 1
    for (int t = 0; t < NT - 2; t += 2) {
      W0P1(0);
      WAIT_ABB(); BAR; MMAW(0, 0, B0, TXT_P1A); MMAW(0, 1, B1, "v_mfma_f32_16x16x32_bf16 %0, %16, %8, %0\n\t" "v_mfma_f32_16x16x32_bf16 %0, %17, %9, %0\n\t" "v_mfma_f32_16x16x32_bf16 %2, %17, %11, %2\n\t" "v_mfma_f32_16x16x32_bf16 %2, %16, %10, %2\n\t" "v_mfma_f32_16x16x32_bf16 %4, %16, %12, %4\n\t" "v_mfma_f32_16x16x32_bf16 %4, %17, %13, %4\n\t" "v_mfma_f32_16x16x32_bf16 %6, %17, %15, %6\n\t" "v_mfma_f32_16x16x32_bf16 %6, %16, %14, %6\n\t" "v_mfma_f32_16x16x32_bf16 %7, %18, %14, %7\n\t" "v_mfma_f32_16x16x32_bf16 %7, %19, %15, %7\n\t" "v_mfma_f32_16x16x32_bf16 %5, %19, %13, %5\n\t" "v_mfma_f32_16x16x32_bf16 %5, %18, %12, %5\n\t" "v_mfma_f32_16x16x32_bf16 %3, %18, %10, %3\n\t" "v_cmp_le_u32 s[100:101], s84, v239\n\t" "v_mfma_f32_16x16x32_bf16 %3, %19, %11, %3\n\t" "s_lshr_b32 s98, s100, s85\n\t" "v_mfma_f32_16x16x32_bf16 %1, %19, %9, %1\n\t" "s_andn2_b32 s98, 15, s98\n\t" "v_mfma_f32_16x16x32_bf16 %1, %18, %8, %1\n\t" "s_setprio 0\n\t" "s_cbranch_scc1 ag_tr_a\n\t" "ag_bk_a:\n\t"); BAR;
      LDA(0, 1); ENG(TXT_SEC2);
      WAIT_A(); BAR; MMAW(1, 0, B0, TXT_W1); MMAW(1, 1, B1, "v_mfma_f32_16x16x32_bf16 %0, %16, %8, %0\n\t" "v_mfma_f32_16x16x32_bf16 %0, %17, %9, %0\n\t" "s_memrealtime s[96:97]\n\t" "v_mfma_f32_16x16x32_bf16 %2, %17, %11, %2\n\t" "v_mfma_f32_16x16x32_bf16 %2, %16, %10, %2\n\t" "s_getreg_b32 s98, hwreg(HW_REG_IB_STS, 22, 2)\n\t" "v_mfma_f32_16x16x32_bf16 %4, %16, %12, %4\n\t" "s_getreg_b32 s99, hwreg(HW_REG_IB_STS, 0, 4)\n\t" "v_mfma_f32_16x16x32_bf16 %4, %17, %13, %4\n\t" "s_lshl4_add_u32 s98, s98, s99\n\t" "v_mfma_f32_16x16x32_bf16 %6, %17, %15, %6\n\t" "s_sub_i32 s76, s70, s98\n\t" "v_mfma_f32_16x16x32_bf16 %6, %16, %14, %6\n\t" "s_add_i32 s99, s71, 4\n\t" "v_mfma_f32_16x16x32_bf16 %7, %18, %14, %7\n\t" "s_cmp_ge_i32 s76, s99\n\t" "v_mfma_f32_16x16x32_bf16 %7, %19, %15, %7\n\t" "s_cselect_b32 s77, s73, 0x7fffffff\n\t" "v_mfma_f32_16x16x32_bf16 %5, %19, %13, %5\n\t" "s_waitcnt lgkmcnt(0)\n\t" "v_mfma_f32_16x16x32_bf16 %5, %18, %12, %5\n\t" "s_sub_i32 s62, s96, s89\n\t" "v_mfma_f32_16x16x32_bf16 %3, %18, %10, %3\n\t" "v_cmp_le_u32 s[100:101], s84, v239\n\t" "v_mfma_f32_16x16x32_bf16 %3, %19, %11, %3\n\t" "s_lshr_b32 s98, s100, s85\n\t" "v_mfma_f32_16x16x32_bf16 %1, %19, %9, %1\n\t" "s_andn2_b32 s98, 15, s98\n\t" "v_mfma_f32_16x16x32_bf16 %1, %18, %8, %1\n\t" "s_setprio 0\n\t" "s_cbranch_scc1 ag_tr_b\n\t" "ag_bk_b:\n\t"); BAR;
      W0P1(1);
      WAIT_ABB(); BAR; MMAW(0, 0, B0, TXT_P1A); MMAW(0, 1, B1, "v_mfma_f32_16x16x32_bf16 %0, %16, %8, %0\n\t" "v_mfma_f32_16x16x32_bf16 %0, %17, %9, %0\n\t" "v_mfma_f32_16x16x32_bf16 %2, %17, %11, %2\n\t" "v_mfma_f32_16x16x32_bf16 %2, %16, %10, %2\n\t" "v_mfma_f32_16x16x32_bf16 %4, %16, %12, %4\n\t" "v_mfma_f32_16x16x32_bf16 %4, %17, %13, %4\n\t" "v_mfma_f32_16x16x32_bf16 %6, %17, %15, %6\n\t" "v_mfma_f32_16x16x32_bf16 %6, %16, %14, %6\n\t" "v_mfma_f32_16x16x32_bf16 %7, %18, %14, %7\n\t" "v_mfma_f32_16x16x32_bf16 %7, %19, %15, %7\n\t" "v_mfma_f32_16x16x32_bf16 %5, %19, %13, %5\n\t" "v_mfma_f32_16x16x32_bf16 %5, %18, %12, %5\n\t" "v_mfma_f32_16x16x32_bf16 %3, %18, %10, %3\n\t" "v_cmp_le_u32 s[100:101], s84, v239\n\t" "v_mfma_f32_16x16x32_bf16 %3, %19, %11, %3\n\t" "s_lshr_b32 s98, s100, s85\n\t" "v_mfma_f32_16x16x32_bf16 %1, %19, %9, %1\n\t" "s_andn2_b32 s98, 15, s98\n\t" "v_mfma_f32_16x16x32_bf16 %1, %18, %8, %1\n\t" "s_setprio 0\n\t" "s_cbranch_scc1 ag_tr_c\n\t" "ag_bk_c:\n\t"); BAR;
      LDA(1, 1); ENG(TXT_SEC2);
      WAIT_A(); BAR; MMAW(1, 0, B0, TXT_W1); MMAW(1, 1, B1, "v_mfma_f32_16x16x32_bf16 %0, %16, %8, %0\n\t" "v_mfma_f32_16x16x32_bf16 %0, %17, %9, %0\n\t" "s_memrealtime s[96:97]\n\t" "v_mfma_f32_16x16x32_bf16 %2, %17, %11, %2\n\t" "v_mfma_f32_16x16x32_bf16 %2, %16, %10, %2\n\t" "s_getreg_b32 s98, hwreg(HW_REG_IB_STS, 22, 2)\n\t" "v_mfma_f32_16x16x32_bf16 %4, %16, %12, %4\n\t" "s_getreg_b32 s99, hwreg(HW_REG_IB_STS, 0, 4)\n\t" "v_mfma_f32_16x16x32_bf16 %4, %17, %13, %4\n\t" "s_lshl4_add_u32 s98, s98, s99\n\t" "v_mfma_f32_16x16x32_bf16 %6, %17, %15, %6\n\t" "s_sub_i32 s76, s70, s98\n\t" "v_mfma_f32_16x16x32_bf16 %6, %16, %14, %6\n\t" "s_add_i32 s99, s71, 4\n\t" "v_mfma_f32_16x16x32_bf16 %7, %18, %14, %7\n\t" "s_cmp_ge_i32 s76, s99\n\t" "v_mfma_f32_16x16x32_bf16 %7, %19, %15, %7\n\t" "s_cselect_b32 s77, s73, 0x7fffffff\n\t" "v_mfma_f32_16x16x32_bf16 %5, %19, %13, %5\n\t" "s_waitcnt lgkmcnt(0)\n\t" "v_mfma_f32_16x16x32_bf16 %5, %18, %12, %5\n\t" "s_sub_i32 s62, s96, s89\n\t" "v_mfma_f32_16x16x32_bf16 %3, %18, %10, %3\n\t" "v_cmp_le_u32 s[100:101], s84, v239\n\t" "v_mfma_f32_16x16x32_bf16 %3, %19, %11, %3\n\t" "s_lshr_b32 s98, s100, s85\n\t" "v_mfma_f32_16x16x32_bf16 %1, %19, %9, %1\n\t" "s_andn2_b32 s98, 15, s98\n\t" "v_mfma_f32_16x16x32_bf16 %1, %18, %8, %1\n\t" "s_setprio 0\n\t" "s_cbranch_scc1 ag_tr_d\n\t" "ag_bk_d:\n\t"); BAR;
    }
    W0P1(0);
    WAIT_ABB(); BAR; MMAW(0, 0, B0, TXT_P1A); MMAW(0, 1, B1, "v_mfma_f32_16x16x32_bf16 %0, %16, %8, %0\n\t" "v_mfma_f32_16x16x32_bf16 %0, %17, %9, %0\n\t" "v_mfma_f32_16x16x32_bf16 %2, %17, %11, %2\n\t" "v_mfma_f32_16x16x32_bf16 %2, %16, %10, %2\n\t" "v_mfma_f32_16x16x32_bf16 %4, %16, %12, %4\n\t" "v_mfma_f32_16x16x32_bf16 %4, %17, %13, %4\n\t" "v_mfma_f32_16x16x32_bf16 %6, %17, %15, %6\n\t" "v_mfma_f32_16x16x32_bf16 %6, %16, %14, %6\n\t" "v_mfma_f32_16x16x32_bf16 %7, %18, %14, %7\n\t" "v_mfma_f32_16x16x32_bf16 %7, %19, %15, %7\n\t" "v_mfma_f32_16x16x32_bf16 %5, %19, %13, %5\n\t" "v_mfma_f32_16x16x32_bf16 %5, %18, %12, %5\n\t" "v_mfma_f32_16x16x32_bf16 %3, %18, %10, %3\n\t" "v_cmp_le_u32 s[100:101], s84, v239\n\t" "v_mfma_f32_16x16x32_bf16 %3, %19, %11, %3\n\t" "s_lshr_b32 s98, s100, s85\n\t" "v_mfma_f32_16x16x32_bf16 %1, %19, %9, %1\n\t" "s_andn2_b32 s98, 15, s98\n\t" "v_mfma_f32_16x16x32_bf16 %1, %18, %8, %1\n\t" "s_setprio 0\n\t" "s_cbranch_scc1 ag_tr_e\n\t" "ag_bk_e:\n\t"); BAR;
    LDA(0, 1); ENG(TXT_SEC2);
    WAIT_A(); BAR; MMAW(1, 0, B0, TXT_W1); MMAW(1, 1, B1, TXT_P2BN); BAR;
    READ_BAD();
    W0P1(1);
    WAIT_ABB(); BAR; MMAW(0, 0, B0, TXT_P1A); MMAW(0, 1, B1, "v_mfma_f32_16x16x32_bf16 %0, %16, %8, %0\n\t" "v_mfma_f32_16x16x32_bf16 %0, %17, %9, %0\n\t" "v_mfma_f32_16x16x32_bf16 %2, %17, %11, %2\n\t" "v_mfma_f32_16x16x32_bf16 %2, %16, %10, %2\n\t" "v_mfma_f32_16x16x32_bf16 %4, %16, %12, %4\n\t" "v_mfma_f32_16x16x32_bf16 %4, %17, %13, %4\n\t" "v_mfma_f32_16x16x32_bf16 %6, %17, %15, %6\n\t" "v_mfma_f32_16x16x32_bf16 %6, %16, %14, %6\n\t" "v_mfma_f32_16x16x32_bf16 %7, %18, %14, %7\n\t" "v_mfma_f32_16x16x32_bf16 %7, %19, %15, %7\n\t" "v_mfma_f32_16x16x32_bf16 %5, %19, %13, %5\n\t" "v_mfma_f32_16x16x32_bf16 %5, %18, %12, %5\n\t" "v_mfma_f32_16x16x32_bf16 %3, %18, %10, %3\n\t" "v_mfma_f32_16x16x32_bf16 %3, %19, %11, %3\n\t" "v_mfma_f32_16x16x32_bf16 %1, %19, %9, %1\n\t" "v_mfma_f32_16x16x32_bf16 %1, %18, %8, %1\n\t" "s_setprio 0\n\t"); BAR;
    TAKE_BAD();
    LDA(1, 1); ENG(TXT_SEC2); store_half<0>(acc, otile, wid, bad);
    WAIT_A(); BAR; MMAW(1, 0, B0, TXT_W1); MMAW(1, 1, B1, TXT_P2BN); BAR;
    MFMA_DRAIN();
    store_half<1>(acc, otile, wid, bad);
    asm volatile("s_mov_b32 s66, 0\n\t" "s_mov_b32 s59, 0\n\t" "s_mov_b32 s58, 0\n\t" "s_add_i32 s98, s70, 6\n\t" "s_add_i32 s70, s70, s63\n\t" "s_mov_b32 s63, 0\n\t" "s_cmp_lt_u32 s74, " STR(ID1) "\n\t" "s_cselect_b32 s99, 0, (" STR(D2) "-" STR(D1) ")\n\t" "s_add_i32 s99, s99, s73\n\t" "s_addk_i32 s99, " STR(D1) "\n\t" "s_cmp_lt_u32 s69, 0x2000\n\t" "s_cselect_b32 s99, s99, 0x7fffffff\n\t" "s_cmp_lg_u32 s87, 0\n\t" "s_cselect_b32 s72, s98, s72\n\t" "s_cselect_b32 s75, s74, s75\n\t" "s_cselect_b32 s71, s98, s71\n\t" "s_cselect_b32 s73, s99, s73\n\t" "s_cselect_b32 s74, s69, s74\n\t" "s_mov_b32 s87, 0\n\t" "s_and_b32 s98, s75, 7\n\t" "s_lshl_b32 s99, s75, 2\n\t" "v_readlane_b32 s82, v234, s98\n\t" "v_readlane_b32 s83, v235, s98\n\t" "s_and_b32 s99, s99, 0x7fe0\n\t" "s_lshr_b32 s98, s75, 5\n\t" "s_add_i32 s98, s98, s88\n\t" "s_add_u32 s82, s82, s99\n\t" "s_addc_u32 s83, s83, 0\n\t" "v_mov_b32 v238, s98\n\t" "s_branch ag_fin_test\n\t" "ag_fin:\n\t" "s_and_b32 s98, s74, 7\n\t" "s_and_b32 s99, s74, 0x1ff8\n\t" "v_readlane_b32 s78, v234, s98\n\t" "v_readlane_b32 s79, v235, s98\n\t" "s_lshl_b32 s99, s99, 9\n\t" "s_cmp_ge_u32 s98, 3\n\t" "s_addc_u32 s69, s74, 292\n\t" "s_add_u32 s78, s78, s99\n\t" "s_addc_u32 s79, s79, 0\n\t" "s_lshl_b32 s98, s69, 15\n\t" "s_and_b32 s99, s69, 0x1f80\n\t" "s_and_b32 s98, s98, 0x3c0000\n\t" "s_or_b32 s98, s98, s99\n\t" "s_add_u32 s80, s90, s98\n\t" "s_addc_u32 s81, s91, 0\n\t" "s_mov_b32 s62, 0x7ffffffe\n\t" "s_waitcnt vmcnt(0)\n\t" "s_mov_b32 s76, s70\n\t" "s_add_i32 s99, s71, 4\n\t" "s_cmp_ge_i32 s76, s99\n\t" "s_cselect_b32 s77, s73, 0x7fffffff\n\t" "v_cmp_gt_u32 s[64:65], v224, v239\n\t" "s_cmp_ge_i32 s76, s72\n\t" "s_cselect_b64 exec, 1, 0\n\t" "global_store_dword v232, v238, s[82:83] sc0 sc1\n\t" "s_cselect_b32 s72, 0x7fffffff, s72\n\t" "s_and_b32 exec_lo, s64, s59\n\t" "global_load_dword v239, v237, s[92:93] sc0 sc1\n\t" "s_cmp_ge_i32 s62, s77\n\t" "s_mov_b64 exec, -1\n\t" "s_cselect_b32 s87, 1, 0\n\t" "s_cselect_b32 s63, 11, 2\n\t" "s_cbranch_scc0 1f\n\t" "global_store_dwordx4 v231, v[240:243], s[78:79] sc0 sc1\n\t" "global_store_dwordx4 v231, v[244:247], s[78:79] offset:1024 sc0 sc1\n\t" "global_store_dwordx4 v231, v[248:251], s[78:79] offset:2048 sc0 sc1\n\t" "global_store_dwordx4 v231, v[252:255], s[78:79] offset:3072 sc0 sc1\n\t" "global_load_dwordx4 v[240:243], v230, s[80:81]\n\t" "global_load_dwordx4 v[244:247], v229, s[80:81]\n\t" "global_load_dwordx4 v[248:251], v225, s[80:81]\n\t" "global_load_dwordx4 v[252:255], v223, s[80:81]\n\t" "global_load_dword v222, v221, s[54:55] " STR(PFB) "\n\t" "v_add_u32 v221, 0x200000, v221\n\t" "v_min_u32 v221, v221, v220\n\t" "1:\n\t" "s_add_i32 s98, s70, 6\n\t" "s_add_i32 s70, s70, s63\n\t" "s_mov_b32 s63, 0\n\t" "s_cmp_lt_u32 s74, " STR(ID1) "\n\t" "s_cselect_b32 s99, 0, (" STR(D2) "-" STR(D1) ")\n\t" "s_add_i32 s99, s99, s73\n\t" "s_addk_i32 s99, " STR(D1) "\n\t" "s_cmp_lt_u32 s69, 0x2000\n\t" "s_cselect_b32 s99, s99, 0x7fffffff\n\t" "s_cmp_lg_u32 s87, 0\n\t" "s_cselect_b32 s72, s98, s72\n\t" "s_cselect_b32 s75, s74, s75\n\t" "s_cselect_b32 s71, s98, s71\n\t" "s_cselect_b32 s73, s99, s73\n\t" "s_cselect_b32 s74, s69, s74\n\t" "s_mov_b32 s87, 0\n\t" "s_and_b32 s98, s75, 7\n\t" "s_lshl_b32 s99, s75, 2\n\t" "v_readlane_b32 s82, v234, s98\n\t" "v_readlane_b32 s83, v235, s98\n\t" "s_and_b32 s99, s99, 0x7fe0\n\t" "s_lshr_b32 s98, s75, 5\n\t" "s_add_i32 s98, s98, s88\n\t" "s_add_u32 s82, s82, s99\n\t" "s_addc_u32 s83, s83, 0\n\t" "v_mov_b32 v238, s98\n\t" "ag_fin_test:\n\t" "s_and_b32 s98, s72, s73\n\t" "s_cmp_lg_u32 s98, 0x7fffffff\n\t" "s_cbranch_scc1 ag_fin\n\t" ::: "memory", "scc");
    if (lane == 0) __hip_atomic_store((unsigned*)ctr, call, __ATOMIC_RELAXED, __HIP_MEMORY_SCOPE_AGENT);
  }
}

extern "C" void kernel_launch(int rank, int nranks, ncclComm_t comm, hipStream_t stream,
                              void* const* d_in, const int* in_sizes, int n_in,
                              void* d_out, int out_size, const int* in_dtypes, int out_dtype,
                              void* const* d_out_all, void* const* d_ws_all, size_t ws_bytes) {
  if (nranks != NR || (unsigned)rank >= (unsigned)NR || ws_bytes < WS_NEED || n_in < 2 ||
      in_sizes[0] != 512 * 4096 || in_sizes[1] != 4096 * 4096 || out_size != 4096 * 4096) {
    fprintf(stderr, "kernel_launch: unexpected shapes\n"); return;
  }
  static bool init = false;
  if (!init) {
    if (hipFuncSetAttribute((const void*)ag_gemm_kernel, hipFuncAttributeMaxDynamicSharedMemorySize, LDS_BYTES) != hipSuccess) {
      fprintf(stderr, "hipFuncSetAttribute failed\n"); return; }
    init = true;
  }
  Args a;
  a.x = (const char*)d_in[0]; a.w = (const char*)d_in[1]; a.out = (char*)d_out; a.rank = rank;
  for (int p = 0; p < NR; ++p) a.ws[p] = (char*)d_ws_all[p];
  ag_gemm_kernel<<<256, 512, LDS_BYTES, stream>>>(a);
}
```
